```python
import numpy as np

kernel_name = "gpu_gemm_ar_m4096_k4096_n4096_bf16_reluqe4m3"


ABSMAX_RTOL = 0.09

COMPUTE_DTYPE = "bfloat16"
EPILOGUE = "relu_quant_e4m3"
IN_QUANT = None
OUT_QUANT = "e4m3"
IN_DTYPES = {"x": "bfloat16", "w_mat": "bfloat16"}
WORLD_SIZE = 8

def _to_bf16(x):
    u = np.ascontiguousarray(x, dtype=np.float32).view(np.uint32)
    return ((u + 0x7FFF + ((u >> 16) & 1)) & 0xFFFF0000).view(np.float32)

def setup_inputs():

    x = _to_bf16(np.random.randn(4096, 4096).astype(np.float32))
    w_mat = _to_bf16((np.random.randn(4096, 4096) / np.sqrt(4096)).astype(np.float32))
    return {"x": x, "w_mat": w_mat}

def _snap_fp8(x, mant_bits, max_abs, min_normal):
    a = np.abs(x.astype(np.float64))
    _m, _e = np.frexp(a)
    q = np.where(a >= min_normal, np.ldexp(1.0, _e - 1 - mant_bits),
                 min_normal * 2.0 ** -mant_bits)
    snapped = np.minimum(np.round(a / q) * q, max_abs)
    return (np.sign(x) * snapped).astype(np.float32)
def _quant_dequant(y, amax):
    scale = amax / 448.0
    q = _snap_fp8(y / scale, 3, 448.0, 2.0 ** -6)
    return (q * scale).astype(np.float32)
def _epilogue(y):
    y = np.maximum(y, 0.0)
    return _quant_dequant(y, np.abs(y).max())

def reference(x, w_mat):
    y = (x @ w_mat).astype(np.float32)
    return _epilogue(y)
```

```cpp
#include <hip/hip_runtime.h>
#include <rccl/rccl.h>
#include <cstdio>
#include <type_traits>

#define HIP_CHECK(expr) do { hipError_t _e = (expr); if (_e != hipSuccess) { \
    fprintf(stderr, "HIP error %s at %s:%d\n", hipGetErrorString(_e), __FILE__, __LINE__); \
    return; } } while (0)

typedef unsigned int u32;
typedef unsigned long long u64;
typedef u32 u32x8 __attribute__((ext_vector_type(8)));
typedef u32 u32x4 __attribute__((ext_vector_type(4)));
typedef u32 u32x2 __attribute__((ext_vector_type(2)));
typedef unsigned short u16x2 __attribute__((ext_vector_type(2)));
typedef float f32x16 __attribute__((ext_vector_type(16)));
typedef __bf16 bf16x8 __attribute__((ext_vector_type(8)));
#define GAS __attribute__((address_space(1)))
#define LAS __attribute__((address_space(3)))
typedef GAS u32 gu32;
typedef GAS u64 gu64;
typedef GAS u32x4 gu32x4;
typedef GAS u32x2 gu32x2;

constexpr int NR = 8;
constexpr int MM = 4096, NN = 4096;
constexpr int NGRID = 256, NTHR = 512;
constexpr u64 DEADLINE_TICKS = 50000000ull;

constexpr u32 OFF_CNT   = 0;
constexpr u32 OFF_TMO   = 32768;
constexpr u32 OFF_TICK  = 33024;
constexpr u32 OFF_AMAXL = 36864;
constexpr u32 OFF_AMAXG = 40960;
constexpr int NREP      = 32;
constexpr u32 OFF_FLAGA = 65536;
constexpr u32 OFF_FLAGB = 98304;
constexpr u32 OFF_FLAGO = 131072;
constexpr u32 OFF_DIAG  = 262144;
constexpr u32 PIECE     = 16384;
constexpr u32 OFF_A     = 1u << 20;
constexpr u32 OFF_B     = OFF_A + 64u * 16u * PIECE;
constexpr u32 MSG       = 4096;
constexpr u32 OFF_MSG   = OFF_B + 64u * 8u * PIECE;
constexpr u32 WS_USED   = OFF_MSG + 8u * 512u * MSG;

constexpr int LDS_W    = 0;
constexpr int LDS_STR  = 32768;
constexpr int LDS_MSK  = 65536;
constexpr int LDS_LUT  = 67584;
constexpr int LDS_RED  = 67712;
constexpr int LDS_SIZE = 98304;

struct Params {
    const char* x;
    const char* w;
    char* ws[NR];
    char* out[NR];
    int rank;
    int pad;
};

#define RLX_SYS __ATOMIC_RELAXED, __HIP_MEMORY_SCOPE_SYSTEM
#define RLX_AGT __ATOMIC_RELAXED, __HIP_MEMORY_SCOPE_AGENT
#define WAIT_VM0() asm volatile("s_waitcnt vmcnt(0)" ::: "memory")
#define CBAR() asm volatile("" ::: "memory")
#define WAIT_LGKM0() do { CBAR(); __builtin_amdgcn_s_waitcnt(0xC07F); CBAR(); } while (0)
#define LDS_BARRIER() do { WAIT_LGKM0(); __builtin_amdgcn_s_barrier(); CBAR(); } while (0)
constexpr int AUX_SYS = 17;
#define PWAVES 4
#define SWIN 2
constexpr int NBLK = 768;
constexpr u32 SEQ_SH = 14;

__device__ __forceinline__ u64 now() { return __builtin_amdgcn_s_memrealtime(); }
__device__ __forceinline__ u32 rfl(u32 v) { return (u32)__builtin_amdgcn_readfirstlane((int)v); }
__device__ __forceinline__ u32 rdl(u32 v, int l) { return (u32)__builtin_amdgcn_readlane((int)v, l); }
__device__ __forceinline__ u32 perm(u32 a, u32 b, u32 s) { return __builtin_amdgcn_perm(a, b, s); }
__device__ __forceinline__ u32 bfi(u32 m, u32 a, u32 b) { return (a & m) | (b & ~m); }
__device__ __forceinline__ __amdgpu_buffer_rsrc_t mk_rsrc(const char* base, u32 bytes) {
    return __builtin_amdgcn_make_buffer_rsrc((void*)base, 0, bytes, 0x00020000);
}
template <int CTRL, int RMASK> __device__ __forceinline__ u32 dpp0(u32 v) {
    return (u32)__builtin_amdgcn_update_dpp(0, (int)v, CTRL, RMASK, 0xf, false);
}
__device__ __forceinline__ u32 wave_max(u32 v) {
    v = max(v, dpp0<0x111, 0xf>(v)); v = max(v, dpp0<0x112, 0xf>(v));
    v = max(v, dpp0<0x114, 0xf>(v)); v = max(v, dpp0<0x118, 0xf>(v));
    v = max(v, dpp0<0x142, 0xa>(v)); v = max(v, dpp0<0x143, 0xc>(v));
    return rdl(v, 63);
}
__device__ __forceinline__ u32 wave_scan(u32 v) {
    v += dpp0<0x111, 0xf>(v); v += dpp0<0x112, 0xf>(v);
    v += dpp0<0x114, 0xf>(v); v += dpp0<0x118, 0xf>(v);
    v += dpp0<0x142, 0xa>(v); v += dpp0<0x143, 0xc>(v);
    return v;
}

#define STAMP(i) do { } while (0)

__device__ __forceinline__ bool spin_fail(u64 t0, gu32* tmo, u32 call, u32& spins) {
    __builtin_amdgcn_s_sleep(1);
    ++spins;
    if ((spins & 63u) == 0u) {
        if (rfl(__hip_atomic_load(tmo, RLX_AGT)) == call) return true;
        if (now() - t0 > DEADLINE_TICKS) { __hip_atomic_store(tmo, call, RLX_AGT); return true; }
    }
    return false;
}

__device__ __forceinline__ void fix_one(u32 (&H)[8], u32 ex) {
    const u32 pl = (ex >> 9) & 7u, mask = 0x7fu << ((ex >> 4) & 24u);
    u32 val = (ex & 127u) * 0x01010101u, t;
    u64 sv;
#define FIXP(i, r) "s_cmp_eq_u32 %[pl], " #i "\n\ts_cselect_b32 %[t], %[mk], 0\n\tv_bfi_b32 " r ", %[t], %[vl], " r "\n\t"
    asm volatile("s_mov_b64 %[sv], exec\n\ts_lshl_b64 exec, 1, %[ln]\n\t"
                 FIXP(0, "%0") FIXP(1, "%1") FIXP(2, "%2") FIXP(3, "%3") FIXP(4, "%4") FIXP(5, "%5") FIXP(6, "%6") FIXP(7, "%7")
                 "s_mov_b64 exec, %[sv]"
                 : "+v"(H[0]), "+v"(H[1]), "+v"(H[2]), "+v"(H[3]), "+v"(H[4]), "+v"(H[5]), "+v"(H[6]), "+v"(H[7]),
                   [t] "=&s"(t), [sv] "=&s"(sv)
                 : [pl] "s"(pl), [mk] "s"(mask), [vl] "v"(val), [ln] "s"((ex >> 12) & 63u) : "scc");
#undef FIXP
}
__device__ __forceinline__ void fix_planes(u32 (&H)[8], const u32x4 nb, u32 flo, u32 fhi, const char* chunk3) {
    if (__builtin_expect((fhi & 0x3000u) != 0x3000u, 1)) {
        fix_one(H, flo);
        if (__builtin_expect((fhi & 0x2000u) != 0u, 0)) fix_one(H, rfl((flo >> 18) | (fhi << 14)));
    } else {
        u32x4 t;
        asm volatile("global_load_dwordx4 %0, %1, off sc0 sc1\n\ts_waitcnt vmcnt(0)" : "=&v"(t) : "v"(chunk3) : "memory");
        asm volatile("v_mov_b32 %0, %8\n\tv_mov_b32 %1, %9\n\tv_mov_b32 %2, %10\n\tv_mov_b32 %3, %11\n\t"
                     "v_mov_b32 %4, %12\n\tv_mov_b32 %5, %13\n\tv_mov_b32 %6, %14\n\tv_mov_b32 %7, %15"
                     : "+v"(H[0]), "+v"(H[1]), "+v"(H[2]), "+v"(H[3]), "+v"(H[4]), "+v"(H[5]), "+v"(H[6]), "+v"(H[7])
                     : "v"(nb.x), "v"(nb.y), "v"(nb.z), "v"(nb.w), "v"(t.x), "v"(t.y), "v"(t.z), "v"(t.w));
    }
}

template <int K> __device__ __forceinline__ void mark_msg(u32 foff, u32x2 fv, char* dst) {
    constexpr int last = K + SWIN - 1 < 6 ? K + SWIN - 1 : 6;
    constexpr int f0 = K - SWIN + 1 > 0 ? K - SWIN + 1 : 0;
    constexpr int N = 2 * (last - K) + (K - f0);
    asm volatile("s_waitcnt vmcnt(%3)\n\ts_mov_b64 exec, 1\n\tglobal_store_dwordx2 %0, %1, %2 sc0 sc1\n\t"
                 "s_mov_b64 exec, -1\n\ts_nop 1"
                 :: "v"(foff), "v"(fv), "s"(dst), "n"(N) : "memory");
}

__global__ void __launch_bounds__(NTHR, 2) fused_kernel(Params P) {
    __shared__ __attribute__((aligned(1024))) char smem[LDS_SIZE];
    LAS char* lds = (LAS char*)smem;
    const int tid = threadIdx.x, lane = tid & 63, wave = (int)rfl((u32)tid >> 6);
    const int b = blockIdx.x, xcd = b & 7, slot = b >> 3;
    const int rank = P.rank;
    char* myws = P.ws[rank];
    gu32* tmo = (gu32*)(myws + OFF_TMO);
    const u64 tstart = now();
    STAMP(0);
    u32 ticket = 0;
    if (lane == 0) ticket = __hip_atomic_fetch_add((gu32*)(myws + OFF_CNT + b * 128 + wave * 16), 1u, RLX_AGT);

    if (slot < 16) {
        const int d = slot & 7;
        const int sw = (slot >> 3) * 8 + xcd;
        const bool producer = wave < PWAVES;
        const int p = (rank + 1 + d) & 7;
        const int r = (rank - p) & 7;
        char* dws = P.ws[p];
        __amdgpu_buffer_rsrc_t rs = mk_rsrc(dws, WS_USED);
        const int row0 = (p >> 2) * 2048, col0 = (p & 3) * 1024;
        const u32 xlane = (u32)(lane & 31) * 1024u + (u32)(lane >> 5) * 16u;
        const u32 wlane = (u32)((lane >> 5) * 8) * 8192u + (u32)(lane & 31) * 8u;
        u32 v[16];
        auto load_block = [&](int nb) {
            const int n = nb >> 2, wb = nb & 3;
            const int c = n / 24, e = n - c * 24;
            if (e < 8) {
                const int j = (e + r) & 7;
                const char* src = P.w + (size_t)(c * 64 + wb * 16) * 8192 + (size_t)(col0 + j * 128) * 2 + wlane;
#pragma unroll
                for (int k = 0; k < 8; ++k) {
                    const u32x2 t = *(const gu32x2*)(src + k * 8192);
                    v[4 * (k >> 1) + (k & 1)] = t.x;
                    v[4 * (k >> 1) + 2 + (k & 1)] = t.y;
                }
            } else {
                const int i = (e - 8 + 2 * r) & 15;
                const char* src = P.x + (size_t)(row0 + i * 128 + wb * 32) * 1024 + c * 128 + xlane;
#pragma unroll
                for (int k = 0; k < 4; ++k) {
                    const u32x4 t = *(const gu32x4*)(src + k * 32);
                    v[4 * k] = t.x; v[4 * k + 1] = t.y; v[4 * k + 2] = t.z; v[4 * k + 3] = t.w;
                }
            }
        };
        int nb = sw * PWAVES + wave;
        if (producer) load_block(nb);
        if (tid < 16) {
            u32 sel = 0, k = 0;
            for (int j = 0; j < 4; ++j) { const u32 on = (tid >> j) & 1; sel |= (on ? k : 0x0cu) << (8 * j); k += on; }
            *(LAS u32*)(lds + LDS_LUT + tid * 4) = sel;
        }
        const u32 call = rfl(ticket) + 1u;
        gu32* tick = (gu32*)(myws + OFF_TICK + d * 128);
        const u32 tbase = (call - 1u) * (u32)NBLK - 16u * PWAVES;
#pragma unroll 1
        while (producer) {
            u32 tk = 0;
            if (lane == 0) tk = __hip_atomic_fetch_add(tick, 1u, RLX_AGT);
            const int n = nb >> 2, wb = nb & 3;
            const int c = n / 24, e = n - c * 24;
            const int step = rank * 8 + c;
            const u32 voff = (u32)wb * 4096u + (u32)lane * 16u;
            u32 soff, foff, selL, selH;
            if (e < 8) { const int j = (e + r) & 7;
                soff = OFF_B + (u32)(step * 8 + j) * PIECE;  foff = OFF_FLAGB + (u32)((j * 4 + wb) * 64 + step) * 8u;
                selL = 0x06020400u; selH = 0x07030501u;
            } else { const int i = (e - 8 + 2 * r) & 15;
                soff = OFF_A + (u32)(step * 16 + i) * PIECE; foff = OFF_FLAGA + (u32)((i * 4 + wb) * 64 + step) * 8u;
                selL = 0x06040200u; selH = 0x07050301u;
            }
            u32 L[8], H[8], E[8];
            u16x2 mx = {0, 0};
#pragma unroll
            for (int i = 0; i < 16; ++i) mx = __builtin_elementwise_max(mx, __builtin_bit_cast(u16x2, v[i] & 0x7fff7fffu));
#pragma unroll
            for (int i = 0; i < 8; ++i) { L[i] = perm(v[2 * i + 1], v[2 * i], selL); H[i] = perm(v[2 * i + 1], v[2 * i], selH); }
            __builtin_amdgcn_raw_buffer_store_b128(u32x4{L[0], L[1], L[2], L[3]}, rs, voff, soff, AUX_SYS);
            __builtin_amdgcn_raw_buffer_store_b128(u32x4{L[4], L[5], L[6], L[7]}, rs, voff + 1024, soff, AUX_SYS);
            const u32 top = wave_max((u32)max(mx.x, mx.y) >> 8);
            const u32 base = top > 7u ? top - 7u : 0u;
            const u32 base4 = base * 0x01010101u;
            u32 bad = 0;
#pragma unroll
            for (int i = 0; i < 8; ++i) { E[i] = H[i] - base4; bad |= E[i]; }
            u32 mode = 0;
            u64 exc = 0;
            if (__ballot((bad & 0x78787878u) != 0u) != 0ull) {
                u32 pm = 0;
#pragma unroll
                for (int i = 0; i < 8; ++i) {
                    const u32 dd = (H[i] | 0x80808080u) - base4;
                    const u32 inv = ((dd ^ 0x80808080u) >> 1) & 0x7c7c7c7cu;
                    const u32 f = ((inv + 0x7f7f7f7fu) >> 7) & 0x01010101u;
                    pm |= __builtin_amdgcn_udot4(f, 0x08040201u, 0u, false) << (4 * i);
                    E[i] = (H[i] & 0x80808080u) | (dd & 0x07070707u);
                }
                const u32 total = rdl(wave_scan((u32)__popc(pm)), 63);
                if (total > 2u) mode = 3;
                else {
                    mode = total;
                    const u32 p0 = (u32)__ffs((int)pm) - 1u, p1 = (u32)__ffs((int)(pm & (pm - 1u))) - 1u;
                    u32 h0 = 0, h1 = 0;
#pragma unroll
                    for (int i = 0; i < 8; ++i) { if ((p0 >> 2) == (u32)i) h0 = H[i]; if ((p1 >> 2) == (u32)i) h1 = H[i]; }
                    const u32 e0 = ((u32)lane << 12) | ((p0 & 31u) << 7) | ((h0 >> (8 * (p0 & 3u))) & 127u);
                    const u32 e1 = ((u32)lane << 12) | ((p1 & 31u) << 7) | ((h1 >> (8 * (p1 & 3u))) & 127u);
                    const u64 lanes = __ballot(pm != 0u);
                    const int l0 = __ffsll((long long)lanes) - 1;
                    exc = rdl(e0, l0);
                    if (total == 2u) {
                        const u64 rest = lanes & (lanes - 1ull);
                        const u32 x1 = rest ? rdl(e0, __ffsll((long long)rest) - 1) : rdl(e1, l0);
                        exc |= (u64)x1 << 18;
                    }
                }
            }
            if (mode == 3) {
                __builtin_amdgcn_raw_buffer_store_b128(u32x4{H[0], H[1], H[2], H[3]}, rs, voff + 2048, soff, AUX_SYS);
                __builtin_amdgcn_raw_buffer_store_b128(u32x4{H[4], H[5], H[6], H[7]}, rs, voff + 3072, soff, AUX_SYS);
            } else {
                u32x4 nb;
#pragma unroll
                for (int q = 0; q < 4; ++q) nb[q] = E[2 * q] | __builtin_amdgcn_alignbit(E[2 * q + 1], E[2 * q + 1], 28);
                __builtin_amdgcn_raw_buffer_store_b128(nb, rs, voff + 2048, soff, AUX_SYS);
            }
            CBAR();
            const u32 nxt = rfl(tk) - tbase;
            if (nxt < (u32)NBLK) load_block((int)nxt);
            WAIT_VM0();
            const u64 flag = ((u64)((call << SEQ_SH) | (mode << 12) | (base << 4)) << 32) | exc;
            if (lane == 0) __hip_atomic_store((gu64*)(dws + foff), flag, RLX_SYS);
            if (nxt >= (u32)NBLK) break;
            nb = (int)nxt;
        }
        STAMP(1);
        LDS_BARRIER();
        bool dead = false;
        if (b == 56 && wave == 0) {
            u32 spins = 0, m = 0;
            for (;;) {
                const u64 g0 = __hip_atomic_load((gu64*)(myws + OFF_AMAXL) + lane, RLX_AGT);
                const u64 g1 = __hip_atomic_load((gu64*)(myws + OFF_AMAXL) + 64 + lane, RLX_AGT);
                m = max((u32)g0, (u32)g1);
                if (__all((u32)(g0 >> 32) == call && (u32)(g1 >> 32) == call)) break;
                if (spin_fail(tstart, tmo, call, spins)) { dead = true; break; }
            }
            if (!dead) {
                m = wave_max(m);
                char* dst = P.ws[lane & 7] + OFF_AMAXG + (lane >> 3) * 128 + rank * 8;
#pragma unroll
                for (int k = 0; k < NREP / 8; ++k)
                    __hip_atomic_store((gu64*)(dst + k * 1024), ((u64)call << 32) | m, RLX_SYS);
            }
            STAMP(2);
        }
        const int tile = slot * 8 + xcd, ti = tile >> 3, tj = tile & 7;
        const int wq = wave & 3, hf = wave >> 2;
        const int mw = tile * 4 + wq, rot = mw % 7;
        const int el = (lane & 1) * 32 + (lane >> 1);
        __amdgpu_buffer_rsrc_t rm = mk_rsrc(myws, WS_USED);
        __amdgpu_buffer_rsrc_t ro = mk_rsrc(P.out[rank], (u32)MM * NN);
        gu64* fo = (gu64*)(myws + OFF_FLAGO + mw * 128) + (lane & 7);
        LAS char* str = lds + LDS_STR + wave * 4096;
        u64 fl = 0;
#pragma unroll 1
        for (int k = 0; k < 7; ++k) {
            int dd = k + rot; if (dd >= 7) dd -= 7;
            const int q = (rank - 1 - dd) & 7;
            if (!dead && rdl((u32)(fl >> 32), q) != call) {
                u32 spins = 0;
                for (;;) {
                    fl = __hip_atomic_load(fo, RLX_SYS);
                    if (rdl((u32)(fl >> 32), q) == call) break;
                    if (spin_fail(tstart, tmo, call, spins)) { dead = true; break; }
                }
            }
            CBAR();
            const u32 ooff = (u32)((q >> 2) * 2048 + ti * 128 + wq * 32 + (el & 31)) * 4096u
                           + (u32)((q & 3) * 1024 + tj * 128 + (el >> 5) * 16 + hf * 64);
            if (dead) {
#pragma unroll
                for (int bb = 0; bb < 2; ++bb)
                    __builtin_amdgcn_raw_buffer_store_b128(u32x4{0x7f7f7f7fu, 0x7f7f7f7fu, 0x7f7f7f7fu, 0x7f7f7f7fu}, ro, ooff + bb * 32, 0, 0);
                continue;
            }
            const u32 nch = min(rdl((u32)fl, q), 224u);
            const u32 moff = OFF_MSG + (u32)(q * 512 + mw) * MSG;
            const u32x2 mk = __builtin_amdgcn_raw_buffer_load_b64(rm, (u32)lane * 8u, moff, AUX_SYS);
            const u32x4 c0 = __builtin_amdgcn_raw_buffer_load_b128(rm, 512u + (u32)lane * 16u, moff, AUX_SYS);
            const u32x4 c1 = __builtin_amdgcn_raw_buffer_load_b128(rm, 1536u + (u32)lane * 16u, moff, AUX_SYS);
            const u32 c_lo = (u32)__popc(mk.x), cnt = c_lo + (u32)__popc(mk.y);
            const u32 pexc = (wave_scan(cnt) - (hf ? cnt - c_lo : cnt)) * 7u;
            const u32 mm = (u32)__builtin_amdgcn_ds_bpermute(el * 4, (int)(hf ? mk.y : mk.x));
            u32 pos = (u32)__builtin_amdgcn_ds_bpermute(el * 4, (int)pexc);
            *(LAS u32x4*)(str + lane * 16) = c0;
            *(LAS u32x4*)(str + 1024 + lane * 16) = c1;
            if (nch > 128u) {
                const u32x4 c2 = __builtin_amdgcn_raw_buffer_load_b128(rm, 2560u + (u32)lane * 16u, moff, AUX_SYS);
                *(LAS u32x4*)(str + 2048 + lane * 16) = c2;
                if (lane < 32) {
                    const u32x4 c3 = __builtin_amdgcn_raw_buffer_load_b128(rm, 3584u + (u32)lane * 16u, moff, AUX_SYS);
                    *(LAS u32x4*)(str + 3072 + lane * 16) = c3;
                }
            }
            WAIT_LGKM0();
            u32x4 o[2];
#pragma unroll
            for (int i = 0; i < 8; ++i) {
                const int sh = 4 * i;
                const u32 n4 = (sh ? (mm >> (sh - 2)) : (mm << 2)) & 0x3cu;
                const u32 sel = *(LAS u32*)(lds + LDS_LUT + n4);
                const LAS u32* wp = (const LAS u32*)(str + ((pos >> 5) << 2));
                const u32 z = __builtin_amdgcn_alignbit(wp[1], wp[0], pos);
                pos += 7u * (u32)__popc(n4);
                const u32 y = bfi(0x00003fffu, z, z << 2);
                const u32 x = bfi(0x007f007fu, y, y << 1) & 0x7f7f7f7fu;
                o[i >> 2][i & 3] = perm(x, x, sel);
            }
#pragma unroll
            for (int bb = 0; bb < 2; ++bb) __builtin_amdgcn_raw_buffer_store_b128(o[bb], ro, ooff + bb * 32, 0, 0);
        }
        STAMP(6);
        return;
    }

    const int sl = slot - 16;
    const int ti = (xcd >> 1) * 4 + (sl >> 2), tj = (xcd & 1) * 4 + (sl & 3), tile = ti * 8 + tj;
    const bool helper = wave >= 4;
    const int wq = wave & 3;
    if (tid < 16) {
        u32 sel = 0x0c0c0c0cu, k = 0;
        for (int j = 0; j < 4; ++j) if ((tid >> j) & 1) { sel = (sel & ~(0xffu << (8 * k))) | ((u32)j << (8 * k)); ++k; }
        *(LAS u32*)(lds + LDS_LUT + tid * 4) = sel;
    }
    if (tid < 8) *(LAS u32*)(lds + LDS_RED + tid * 4) = 0;
    LAS char* str = lds + LDS_STR + wave * 4096;
#pragma unroll
    for (int u = 0; u < 4; ++u) *(LAS u32x4*)(str + u * 1024 + lane * 16) = u32x4{0, 0, 0, 0};
    u32 stepoff;
    {
        const int c = lane >> 3, k = (lane & 7) - 1, r0 = ti >> 1;
        const int r = k < 0 ? 0 : ((r0 + 13 - k) % 7) + 1;
        stepoff = (u32)(((rank + r) & 7) * 8 + c);
    }
    gu64* fptr = (gu64*)(myws + (helper ? OFF_FLAGB + (u32)((tj * 4 + wq) * 64) * 8u : OFF_FLAGA + (u32)((ti * 4 + wq) * 64) * 8u)) + stepoff;
    stepoff *= helper ? 8u * PIECE : 16u * PIECE;
    __amdgpu_buffer_rsrc_t rs = mk_rsrc(myws, WS_USED);
    const u32 voff = (helper ? OFF_B + (u32)tj * PIECE : OFF_A + (u32)ti * PIECE) + (u32)wq * 4096u + (u32)lane * 16u;
    const u32 call = rfl(ticket) + 1u;
    const u32 seq = call & 0x3ffffu;
    u64 ready = 0, fl = 0;
    bool dead = false;
    auto wait_ready = [&](int s) {
        if (__builtin_expect(!((ready >> s) & 1), 0)) {
            u32 spins = 0;
            for (;;) {
                fl = __hip_atomic_load(fptr, RLX_SYS);
                ready = __ballot((u32)(fl >> (32 + SEQ_SH)) == seq);
                if ((ready >> s) & 1) break;
                if (spin_fail(tstart, tmo, call, spins)) { dead = true; fl = 0; ready = ~0ull; break; }
            }
        }
        CBAR();
    };
#define SB() __builtin_amdgcn_sched_barrier(0)
    u32x4 rg[2][3];
    auto issue = [&](int s, u32x4 (&r)[3]) {
        const u32 so = rdl(stepoff, s);
#pragma unroll
        for (int u = 2; u < 5; ++u) { r[u % 3] = __builtin_amdgcn_raw_buffer_load_b128(rs, voff + (u % 3) * 1024, so, AUX_SYS); SB(); }
    };
    auto plane = [&](u32 n, u32 base4, int q, u32 (&H)[8]) {
        H[2 * q] = (n & 0x87878787u) + base4;
        H[2 * q + 1] = (__builtin_amdgcn_alignbit(n, n, 4) & 0x87878787u) + base4;
    };
    auto fix = [&](int s, u32 fhi, u32 (&H)[8], const u32x4 nb) {
        if (__builtin_expect((fhi & 0x3000u) != 0u, 0))
            fix_planes(H, nb, rdl((u32)fl, s), fhi, myws + (voff + rdl(stepoff, s) + 3072u));
    };
    using T = std::true_type; using F = std::false_type;
    using C0 = std::integral_constant<int, 0>; using C1 = std::integral_constant<int, 1>;

    wait_ready(0); issue(0, rg[0]);
    wait_ready(1); issue(1, rg[1]);
    if (helper) {
        auto putW = [&](int s, u32x4 (&r)[3], int buf) {
            const u32 fhi = rdl((u32)(fl >> 32), s);
            const u32 base4 = ((fhi >> 4) & 127u) * 0x01010101u;
            u32 H[8];
#pragma unroll
            for (int q = 0; q < 4; ++q) plane(r[2][q], base4, q, H);
            fix(s, fhi, H, r[2]);
            LAS char* pw = lds + LDS_W + buf * 16384 + wq * 4096 + lane * 16;
#pragma unroll
            for (int q = 0; q < 4; ++q) {
                const u32 sl_ = (q & 1) ? 0x07030602u : 0x05010400u;
                const int xy = q >> 1;
                *(LAS u32x4*)(pw + q * 1024) = u32x4{perm(H[xy], r[0][xy], sl_), perm(H[2 + xy], r[0][2 + xy], sl_),
                                                     perm(H[4 + xy], r[1][xy], sl_), perm(H[6 + xy], r[1][2 + xy], sl_)};
            }
        };
        putW(0, rg[0], 0);
        wait_ready(2); issue(2, rg[0]);
        LDS_BARRIER();
#pragma unroll 1
        for (int s = 0; s < 60; s += 2) {
            putW(s + 1, rg[1], 1); SB(); wait_ready(s + 3); issue(s + 3, rg[1]); LDS_BARRIER();
            putW(s + 2, rg[0], 0); SB(); wait_ready(s + 4); issue(s + 4, rg[0]); LDS_BARRIER();
        }
        putW(61, rg[1], 1); SB(); wait_ready(63); issue(63, rg[1]); LDS_BARRIER();
        putW(62, rg[0], 0); LDS_BARRIER();
        putW(63, rg[1], 1);
        if (dead && lane == 0) *(LAS u32*)(lds + LDS_RED + 20) = 1;
        LDS_BARRIER();
        return;
    }
    bf16x8 xf[2][4], wfA[8], wfB[8];
    f32x16 acc[4];
#pragma unroll
    for (int j = 0; j < 4; ++j)
#pragma unroll
        for (int i = 0; i < 16; ++i) acc[j][i] = 0.f;
    auto putX = [&](int q, u32 (&H)[8], u32x4 (&r)[3], bf16x8 (&xo)[4]) {
        const u32 ha = H[2 * q], hb = H[2 * q + 1];
        const u32 la = r[q >> 1][(2 * q) & 3], lb = r[q >> 1][(2 * q + 1) & 3];
        u32x4 c = u32x4{perm(ha, la, 0x05010400u), perm(ha, la, 0x07030602u), perm(hb, lb, 0x05010400u), perm(hb, lb, 0x07030602u)};
        asm volatile("" : "+v"(c));
        xo[q] = __builtin_bit_cast(bf16x8, c);
    };
    auto step = [&](int s, auto cur_c, auto dec_c, auto iss_c) {
        constexpr int cur = decltype(cur_c)::value, nxt = cur ^ 1;
        constexpr bool DEC = decltype(dec_c)::value, ISS = decltype(iss_c)::value;
        u32 H[8];
        u32 fhi = 0, base4 = 0;
        if constexpr (DEC) {
            fhi = rdl((u32)(fl >> 32), s + 1);
            base4 = ((fhi >> 4) & 127u) * 0x01010101u;
        }
        LAS char* pr = lds + LDS_W + cur * 16384 + 8192 + lane * 16;
#pragma unroll
        for (int q = 0; q < 8; ++q) {
            acc[q & 3] = __builtin_amdgcn_mfma_f32_32x32x16_bf16(wfA[q], xf[cur][q >> 2], acc[q & 3], 0, 0, 0);
            SB();
            if (q < 4) { wfB[2 * q] = *(LAS bf16x8*)(pr + (2 * q) * 1024); wfB[2 * q + 1] = *(LAS bf16x8*)(pr + (2 * q + 1) * 1024); }
            else if constexpr (DEC) plane(rg[nxt][2][q - 4], base4, q - 4, H);
            SB();
        }
        u32 so = 0;
        if constexpr (DEC) { fix(s + 1, fhi, H, rg[nxt][2]); LDS_BARRIER(); }
        if constexpr (ISS) { wait_ready(s + 3); so = rdl(stepoff, s + 3); }
        SB();
        pr = lds + LDS_W + nxt * 16384 + lane * 16;
#pragma unroll
        for (int q = 0; q < 8; ++q) {
            acc[q & 3] = __builtin_amdgcn_mfma_f32_32x32x16_bf16(wfB[q], xf[cur][2 + (q >> 2)], acc[q & 3], 0, 0, 0);
            SB();
            if constexpr (DEC) wfA[q] = *(LAS bf16x8*)(pr + q * 1024);
            if constexpr (ISS) {
                if (q == 0) rg[nxt][2] = __builtin_amdgcn_raw_buffer_load_b128(rs, voff + 2048, so, AUX_SYS);
                if (q == 4) rg[nxt][0] = __builtin_amdgcn_raw_buffer_load_b128(rs, voff, so, AUX_SYS);
                if (q == 7) rg[nxt][1] = __builtin_amdgcn_raw_buffer_load_b128(rs, voff + 1024, so, AUX_SYS);
            }
            if constexpr (DEC) { if (q == 1 || q == 3 || q == 5 || q == 6) putX(q == 6 ? 3 : q >> 1, H, rg[nxt], xf[nxt]); }
            SB();
        }
    };
    {
        const u32 fhi = rdl((u32)(fl >> 32), 0);
        const u32 base4 = ((fhi >> 4) & 127u) * 0x01010101u;
        u32 H[8];
#pragma unroll
        for (int q = 0; q < 4; ++q) plane(rg[0][2][q], base4, q, H);
        fix(0, fhi, H, rg[0][2]);
#pragma unroll
        for (int q = 0; q < 4; ++q) putX(q, H, rg[0], xf[0]);
    }
    wait_ready(2); issue(2, rg[0]);
    LDS_BARRIER();
#pragma unroll
    for (int i = 0; i < 8; ++i) wfA[i] = *(LAS bf16x8*)(lds + LDS_W + lane * 16 + i * 1024);
    STAMP(1);
#pragma unroll 1
    for (int s = 0; s < 60; s += 2) {
        step(s, C0{}, T{}, T{});
        step(s + 1, C1{}, T{}, T{});
    }
    step(60, C0{}, T{}, T{});
    step(61, C1{}, T{}, F{});
    step(62, C0{}, T{}, F{});
    step(63, C1{}, F{}, F{});
    STAMP(5);

    float mx = 0.f;
#pragma unroll
    for (int j = 0; j < 4; ++j)
#pragma unroll
        for (int i = 0; i < 16; ++i) mx = fmaxf(mx, acc[j][i]);
    {
        const u32 mb = wave_max(__float_as_uint(mx));
        if (lane == 0) *(LAS u32*)(lds + LDS_RED + wave * 4) = mb;
        if (dead && lane == 0) *(LAS u32*)(lds + LDS_RED + 20) = 1;
        LDS_BARRIER();
        if (wave == 0 && lane == 0) {
            const u32x4 t = *(LAS u32x4*)(lds + LDS_RED);
            __hip_atomic_store((gu64*)(myws + OFF_AMAXL) + tile, ((u64)call << 32) | max(max(t.x, t.y), max(t.z, t.w)), RLX_AGT);
        }
    }
    u32 m0 = 0, m1 = 0;
#pragma unroll
    for (int i = 7; i >= 0; --i)
#pragma unroll
        for (int j = 3; j >= 0; --j) {
            m0 = __builtin_amdgcn_alignbit(m0, __float_as_uint(acc[j][i]), 31);
            m1 = __builtin_amdgcn_alignbit(m1, __float_as_uint(acc[j][i + 8]), 31);
        }
    m0 = ~m0; m1 = ~m1;
    const int mw = tile * 4 + wave, rot = mw % 7;
    const u32 moff = OFF_MSG + (u32)(rank * 512 + mw) * MSG;
    *(LAS u32x2*)(lds + LDS_MSK + wave * 512 + lane * 8) = u32x2{m0, m1};
    WAIT_LGKM0();
    if (lane < 32) {
        const u32x4 mk = *(LAS u32x4*)(lds + LDS_MSK + wave * 512 + lane * 16);
#pragma unroll 1
        for (int k = 0; k < 7; ++k) {
            int dd = k + rot; if (dd >= 7) dd -= 7;
            __builtin_amdgcn_raw_buffer_store_b128(mk, mk_rsrc(P.ws[(rank + 1 + dd) & 7], WS_USED), (u32)lane * 16u, moff, AUX_SYS);
        }
    }
    const u32 cnt = (u32)(__popc(m0) + __popc(m1));
    const u32 incl = wave_scan(cnt);
    const u32 nch = (rdl(incl, 63) * 7u + 127u) >> 7;
    u32 sel[16], pos[16];
    {
        u32 pp = (incl - cnt) * 7u;
#pragma unroll
        for (int i = 0; i < 16; ++i) {
            const u32 mm = i < 8 ? m0 : m1;
            const int sh = 4 * (i & 7);
            const u32 n4 = (sh ? (mm >> (sh - 2)) : (mm << 2)) & 0x3cu;
            sel[i] = *(LAS u32*)(lds + LDS_LUT + n4);
            pos[i] = pp;
            pp += 7u * (u32)__popc(n4);
        }
    }
#pragma unroll
    for (int j = 0; j < 4; ++j)
#pragma unroll
        for (int i = 0; i < 16; ++i) acc[j][i] = fmaxf(acc[j][i], 0.f);
    if (wave == 0) {
        u32 gb = 0;
        if (!dead) {
            gu64* g = lane == 8 ? (gu64*)tmo : (gu64*)(myws + OFF_AMAXG + (tile & (NREP - 1)) * 128) + (lane & 7);
            u32 spins = 0;
            for (;;) {
                const u64 a = __hip_atomic_load(g, RLX_SYS);
                gb = (u32)a;
                if ((__ballot((u32)(a >> 32) == call) & 0xffull) == 0xffull) break;
                if (spin_fail(tstart, tmo, call, spins)) { dead = true; break; }
            }
            if (rdl(gb, 8) == call) dead = true;
            gb = wave_max(lane == 8 ? 0u : gb);
        }
        if (lane == 0) { *(LAS u32*)(lds + LDS_RED + 16) = gb; if (dead) *(LAS u32*)(lds + LDS_RED + 20) = 1; }
    }
    LDS_BARRIER();
    const u32x2 gd = *(LAS u32x2*)(lds + LDS_RED + 16);
    const u32 gbits = gd.x;
    dead = rfl(gd.y) != 0u;
    STAMP(6);

    const float scale = __uint_as_float(gbits) / 448.0f;
    const float inv = 1.0f / scale;
    u32x4 q[4];
#pragma unroll
    for (int i = 0; i < 16; ++i) {
        int w0 = __builtin_amdgcn_cvt_pk_fp8_f32(acc[0][i] * inv, acc[1][i] * inv, 0, false);
        w0 = __builtin_amdgcn_cvt_pk_fp8_f32(acc[2][i] * inv, acc[3][i] * inv, w0, true);
        const u32 qq = (u32)w0 & 0x7f7f7f7fu;
        q[i >> 2][i & 3] = qq;
        const u32 c = perm(qq, qq, sel[i]);
        const u32 y = bfi(0x007f007fu, c, c >> 1);
        const u32 z = bfi(0x00003fffu, y, y >> 2);
        const u64 sh = (u64)z << (pos[i] & 31u);
        LAS u32* wp = (LAS u32*)(str + ((pos[i] >> 5) << 2));
        __hip_atomic_fetch_or(wp, (u32)sh, __ATOMIC_RELAXED, __HIP_MEMORY_SCOPE_WAVEFRONT);
        __hip_atomic_fetch_or(wp + 1, (u32)(sh >> 32), __ATOMIC_RELAXED, __HIP_MEMORY_SCOPE_WAVEFRONT);
    }
    WAIT_LGKM0();
    u32x4 cs[4];
#pragma unroll
    for (int u = 0; u < 4; ++u) cs[u] = *(LAS u32x4*)(str + u * 1024 + lane * 16);
    STAMP(7);
    if (!dead) {
        const u64 flag = ((u64)call << 32) | nch;
        const u32 foff = OFF_FLAGO + (u32)mw * 128u + (u32)rank * 8u;
        if (__builtin_expect(nch <= 128u, 1)) {
            char* dst[7];
#pragma unroll
            for (int k = 0; k < 7; ++k) { int dd = k + rot; if (dd >= 7) dd -= 7; dst[k] = P.ws[(rank + 1 + dd) & 7]; }
            const u64 e0 = __ballot((u32)lane < max(nch, 1u)), e1 = __ballot((u32)lane + 64u < max(nch, 65u));
            const u32 vo = moff + 512u + (u32)lane * 16u;
            const u32x2 fv = u32x2{(u32)flag, (u32)(flag >> 32)};
            auto send = [&](int k) {
                asm volatile("s_mov_b64 exec, %4\n\tglobal_store_dwordx4 %0, %1, %3 sc0 sc1\n\t"
                             "s_mov_b64 exec, %5\n\tglobal_store_dwordx4 %0, %2, %3 offset:1024 sc0 sc1\n\t"
                             "s_mov_b64 exec, -1\n\ts_nop 1"
                             :: "v"(vo), "v"(cs[0]), "v"(cs[1]), "s"(dst[k]), "s"(e0), "s"(e1) : "memory");
            };
#pragma unroll
            for (int k = 0; k < SWIN; ++k) send(k);
            mark_msg<0>(foff, fv, dst[0]); if (SWIN + 0 < 7) send(SWIN + 0);
            mark_msg<1>(foff, fv, dst[1]); if (SWIN + 1 < 7) send(SWIN + 1);
            mark_msg<2>(foff, fv, dst[2]); if (SWIN + 2 < 7) send(SWIN + 2);
            mark_msg<3>(foff, fv, dst[3]); if (SWIN + 3 < 7) send(SWIN + 3);
            mark_msg<4>(foff, fv, dst[4]); if (SWIN + 4 < 7) send(SWIN + 4);
            mark_msg<5>(foff, fv, dst[5]); if (SWIN + 5 < 7) send(SWIN + 5);
            mark_msg<6>(foff, fv, dst[6]);
        } else {
#pragma unroll 1
            for (int k = 0; k < 7; ++k) {
                int dd = k + rot; if (dd >= 7) dd -= 7;
                char* w0 = P.ws[(rank + 1 + dd) & 7];
#pragma unroll
                for (int u = 0; u < 4; ++u)
                    if ((u32)(u * 64 + lane) < nch) __builtin_amdgcn_raw_buffer_store_b128(cs[u], mk_rsrc(w0, WS_USED), 512u + u * 1024u + (u32)lane * 16u, moff, AUX_SYS);
                WAIT_VM0();
                if (lane == 0) __hip_atomic_store((gu64*)(w0 + foff), flag, RLX_SYS);
            }
        }
    }
    STAMP(8);
    {
        const u32 ooff = (u32)((rank >> 2) * 2048 + ti * 128 + wave * 32 + (lane & 31)) * 4096u
                       + (u32)((rank & 3) * 1024 + tj * 128 + (lane >> 5) * 16);
        __amdgpu_buffer_rsrc_t ro = mk_rsrc(P.out[rank], (u32)MM * NN);
#pragma unroll
        for (int bb = 0; bb < 4; ++bb)
            __builtin_amdgcn_raw_buffer_store_b128(dead ? u32x4{0x7f7f7f7fu, 0x7f7f7f7fu, 0x7f7f7f7fu, 0x7f7f7f7fu} : q[bb], ro, ooff + bb * 32, 0, 0);
    }
    STAMP(9);
}

extern "C" void kernel_launch(int rank, int nranks, ncclComm_t comm,
                              hipStream_t stream,
                              void* const* d_in, const int* in_sizes, int n_in,
                              void* d_out, int out_size,
                              const int* in_dtypes, int out_dtype,
                              void* const* d_out_all,
                              void* const* d_ws_all, size_t ws_bytes) {
    (void)comm; (void)in_sizes; (void)in_dtypes; (void)out_dtype; (void)d_out;
    if (nranks != NR || n_in < 2 || out_size != MM * NN || ws_bytes < WS_USED || (unsigned)rank >= (unsigned)NR) {
        fprintf(stderr, "unexpected problem shape\n");
        return;
    }
    Params P;
    P.x = (const char*)d_in[0];
    P.w = (const char*)d_in[1];
    for (int p = 0; p < NR; ++p) { P.ws[p] = (char*)d_ws_all[p]; P.out[p] = (char*)d_out_all[p]; }
    P.rank = rank;
    P.pad = 0;
    fused_kernel<<<dim3(NGRID), dim3(NTHR), 0, stream>>>(P);
    HIP_CHECK(hipGetLastError());
}
```

```cpp
#include <hip/hip_runtime.h>
#include <rccl/rccl.h>
#include <cstdio>
#include <type_traits>

#define HIP_CHECK(expr) do { hipError_t _e = (expr); if (_e != hipSuccess) { \
    fprintf(stderr, "HIP error %s at %s:%d\n", hipGetErrorString(_e), __FILE__, __LINE__); \
    return; } } while (0)

typedef unsigned int u32;
typedef unsigned long long u64;
typedef u32 u32x8 __attribute__((ext_vector_type(8)));
typedef u32 u32x4 __attribute__((ext_vector_type(4)));
typedef u32 u32x2 __attribute__((ext_vector_type(2)));
typedef unsigned short u16x2 __attribute__((ext_vector_type(2)));
typedef float f32x16 __attribute__((ext_vector_type(16)));
typedef __bf16 bf16x8 __attribute__((ext_vector_type(8)));
#define GAS __attribute__((address_space(1)))
#define LAS __attribute__((address_space(3)))
typedef GAS u32 gu32;
typedef GAS u64 gu64;
typedef GAS u32x4 gu32x4;
typedef GAS u32x2 gu32x2;

constexpr int NR = 8;
constexpr int MM = 4096, NN = 4096;
constexpr int NGRID = 256, NTHR = 512;
constexpr u64 DEADLINE_TICKS = 50000000ull;

constexpr u32 OFF_CNT   = 0;
constexpr u32 OFF_TMO   = 32768;
constexpr u32 OFF_TICK  = 33024;
constexpr u32 OFF_DUMMY = 34816;
constexpr u32 OFF_AMAXL = 36864;
constexpr u32 OFF_AMAXG = 40960;
#define NREP_ 32
constexpr int NREP      = NREP_;
constexpr u32 OFF_FLAGA = 65536;
constexpr u32 OFF_FLAGB = 98304;
constexpr u32 OFF_FLAGO = 131072;
constexpr u32 OFF_DIAG  = 262144;
constexpr u32 PIECE     = 16384;
constexpr u32 OFF_A     = 1u << 20;
constexpr u32 OFF_B     = OFF_A + 64u * 16u * PIECE;
constexpr u32 MSG       = 4096;
constexpr u32 OFF_MSG   = OFF_B + 64u * 8u * PIECE;
constexpr u32 WS_USED   = OFF_MSG + 8u * 512u * MSG;

constexpr int LDS_W    = 0;
constexpr int LDS_STR  = 32768;
constexpr int LDS_MSK  = 65536;
constexpr int LDS_LUT  = 67584;
constexpr int LDS_RED  = 67712;
constexpr int LDS_SIZE = 98304;

struct Params {
    const char* x;
    const char* w;
    char* ws[NR];
    char* out[NR];
    int rank;
    int pad;
};

#define RLX_SYS __ATOMIC_RELAXED, __HIP_MEMORY_SCOPE_SYSTEM
#define RLX_AGT __ATOMIC_RELAXED, __HIP_MEMORY_SCOPE_AGENT
#define WAIT_VM0() asm volatile("s_waitcnt vmcnt(0)" ::: "memory")
#define CBAR() asm volatile("" ::: "memory")
#define WAIT_LGKM0() do { CBAR(); __builtin_amdgcn_s_waitcnt(0xC07F); CBAR(); } while (0)
#define LDS_BARRIER() do { WAIT_LGKM0(); __builtin_amdgcn_s_barrier(); CBAR(); } while (0)
constexpr int AUX_SYS = 17;
#define NWL 24
#define TRACE_ON 0
#define THINK 0
#define SWIN 1
#define PACE 0x7f
#define PROT 0
#define MEARLY 5
#define MWIN 2
#define AUX_OUT 0
constexpr int NBLK = 768;
constexpr u32 SEQ_SH = 17, SEQ_MASK = 0x7fffu;

__device__ __forceinline__ u64 now() { return __builtin_amdgcn_s_memrealtime(); }
__device__ __forceinline__ u32 rfl(u32 v) { return (u32)__builtin_amdgcn_readfirstlane((int)v); }
__device__ __forceinline__ u32 rdl(u32 v, int l) { return (u32)__builtin_amdgcn_readlane((int)v, l); }
__device__ __forceinline__ u32 perm(u32 a, u32 b, u32 s) { return __builtin_amdgcn_perm(a, b, s); }
__device__ __forceinline__ u32 bfi(u32 m, u32 a, u32 b) { return (a & m) | (b & ~m); }
__device__ __forceinline__ __amdgpu_buffer_rsrc_t mk_rsrc(const char* base, u32 bytes) {
    return __builtin_amdgcn_make_buffer_rsrc((void*)base, 0, bytes, 0x00020000);
}
template <int CTRL, int RMASK> __device__ __forceinline__ u32 dpp0(u32 v) {
    return (u32)__builtin_amdgcn_update_dpp(0, (int)v, CTRL, RMASK, 0xf, false);
}
__device__ __forceinline__ u32 wave_max(u32 v) {
    v = max(v, dpp0<0x111, 0xf>(v)); v = max(v, dpp0<0x112, 0xf>(v));
    v = max(v, dpp0<0x114, 0xf>(v)); v = max(v, dpp0<0x118, 0xf>(v));
    v = max(v, dpp0<0x142, 0xa>(v)); v = max(v, dpp0<0x143, 0xc>(v));
    return rdl(v, 63);
}
__device__ __forceinline__ u32 wave_scan(u32 v) {
    v += dpp0<0x111, 0xf>(v); v += dpp0<0x112, 0xf>(v);
    v += dpp0<0x114, 0xf>(v); v += dpp0<0x118, 0xf>(v);
    v += dpp0<0x142, 0xa>(v); v += dpp0<0x143, 0xc>(v);
    return v;
}

#define STAMP(i) do { } while (0)
#define TRACE(s) do { } while (0)

__device__ __forceinline__ bool spin_fail(u64 t0, gu32* tmo, u32 call, u32& spins) {
    __builtin_amdgcn_s_sleep(1);
    ++spins;
    if ((spins & 63u) == 0u) {
        if (rfl(__hip_atomic_load(tmo, RLX_AGT)) == call) return true;
        if (now() - t0 > DEADLINE_TICKS) { __hip_atomic_store(tmo, call, RLX_AGT); return true; }
    }
    return false;
}

#define PLANE(R0, R1) asm("v_and_b32 %2, 0x87878787, %4\n\tv_alignbit_b32 %3, %4, %4, 4\n\tv_add_u32 " #R0 ", %5, %2\n\t" \
                          "v_and_b32 %3, 0x87878787, %3\n\tv_add_u32 " #R1 ", %5, %3" \
                          : "={" #R0 "}"(h0), "={" #R1 "}"(h1), "=&v"(t0), "=&v"(t1) : "v"(n), "s"(base4))
template <int Q> __device__ __forceinline__ void plane(u32 n, u32 base4, u32 (&H)[8]) {
    u32 h0, h1, t0, t1;
    if constexpr (Q == 0) PLANE(v200, v201);
    if constexpr (Q == 1) PLANE(v202, v203);
    if constexpr (Q == 2) PLANE(v204, v205);
    if constexpr (Q == 3) PLANE(v206, v207);
    H[2 * Q] = h0; H[2 * Q + 1] = h1;
}
#undef PLANE
#define HREGS "+{v200}"(H[0]), "+{v201}"(H[1]), "+{v202}"(H[2]), "+{v203}"(H[3]), "+{v204}"(H[4]), "+{v205}"(H[5]), "+{v206}"(H[6]), "+{v207}"(H[7])
__device__ __forceinline__ void fix_one(u32 (&H)[8], u32 ln, u32 pl, u32 x) {
    asm volatile("s_lshl_b64 exec, 1, %8\n\ts_set_gpr_idx_on %9, gpr_idx(SRC1,DST)\n\tv_xor_b32 v200, %10, v200\n\t"
                 "s_set_gpr_idx_off\n\ts_mov_b64 exec, -1"
                 : HREGS : "s"(ln), "s"(pl), "s"(x) : "scc");
}
__device__ __forceinline__ void fix_rare(u32 (&H)[8], const u32x4 nb, u32 flo, u32 fhi, const char* chunk3) {
    if ((fhi & 0x8000u) == 0u) {
        const u32 e = rfl((flo >> 21) | (fhi << 11));
        fix_one(H, e, (e >> 6) & 7u, ((e >> 9) & 127u) << ((e >> 13) & 24u));
    } else {
        u32x4 t;
        asm volatile("global_load_dwordx4 %0, %1, off sc0 sc1\n\ts_waitcnt vmcnt(0)" : "=&v"(t) : "v"(chunk3) : "memory");
        asm volatile("v_mov_b32 v200, %8\n\tv_mov_b32 v201, %9\n\tv_mov_b32 v202, %10\n\tv_mov_b32 v203, %11\n\t"
                     "v_mov_b32 v204, %12\n\tv_mov_b32 v205, %13\n\tv_mov_b32 v206, %14\n\tv_mov_b32 v207, %15"
                     : HREGS : "v"(nb.x), "v"(nb.y), "v"(nb.z), "v"(nb.w), "v"(t.x), "v"(t.y), "v"(t.z), "v"(t.w));
    }
}
#undef HREGS

template <int K> __device__ __forceinline__ void mark_msg(u32 foff, u32x2 fv, char* dst) {
    constexpr int last = K + SWIN - 1 < 6 ? K + SWIN - 1 : 6;
    constexpr int f0 = K - SWIN + 1 > 0 ? K - SWIN + 1 : 0;
    constexpr int N = 2 * (last - K) + (K - f0);
    asm volatile("s_waitcnt vmcnt(%3)\n\ts_mov_b64 exec, 1\n\tglobal_store_dwordx2 %0, %1, %2 sc0 sc1\n\t"
                 "s_mov_b64 exec, -1\n\ts_nop 1"
                 :: "v"(foff), "v"(fv), "s"(dst), "n"(N) : "memory");
}

__global__ void __launch_bounds__(NTHR, 2) fused_kernel(Params P) {
    __shared__ __attribute__((aligned(1024))) char smem[LDS_SIZE];
    LAS char* lds = (LAS char*)smem;
    const int tid = threadIdx.x, lane = tid & 63, wave = (int)rfl((u32)tid >> 6);
    const int b = blockIdx.x, xcd = b & 7, slot = b >> 3;
    const int rank = P.rank;
    char* myws = P.ws[rank];
    gu32* tmo = (gu32*)(myws + OFF_TMO);
    const u64 tstart = now();
    STAMP(0);
    u32 ticket = 0;
    if (lane == 0) ticket = __hip_atomic_fetch_add((gu32*)(myws + OFF_CNT + b * 128 + wave * 16), 1u, RLX_AGT);

    if (slot < 16) {
        const int d = slot & 7;
        const int sw = (slot >> 3) * 8 + xcd;
        const int wl = wave * 16 + sw;
        const bool producer = wl < NWL;
        const int p = (rank + 1 + d) & 7;
        const int r = (rank - p) & 7;
        char* dws = P.ws[p];
        __amdgpu_buffer_rsrc_t rs = mk_rsrc(dws, WS_USED);
        const int row0 = (p >> 2) * 2048, col0 = (p & 3) * 1024;
        const u32 xlane = (u32)(lane & 31) * 1024u + (u32)(lane >> 5) * 16u;
        const u32 wlane = (u32)((lane >> 5) * 8) * 8192u + (u32)(lane & 31) * 8u;
        u32 v[16];
        auto load_block = [&](int nb) {
            const int n = nb >> 2, wb = nb & 3;
            const int c = n / 24, e = n - c * 24;
            if (e < 8) {
                const int j = (e + r) & 7;
                const char* src = P.w + (size_t)(c * 64 + wb * 16) * 8192 + (size_t)(col0 + j * 128) * 2 + wlane;
#pragma unroll
                for (int k = 0; k < 8; ++k) {
                    const u32x2 t = *(const gu32x2*)(src + k * 8192);
                    v[4 * (k >> 1) + (k & 1)] = t.x;
                    v[4 * (k >> 1) + 2 + (k & 1)] = t.y;
                }
            } else {
                const int i = (e - 8 + 2 * r) & 15;
                const char* src = P.x + (size_t)(row0 + i * 128 + wb * 32) * 1024 + c * 128 + xlane;
#pragma unroll
                for (int k = 0; k < 4; ++k) {
                    const u32x4 t = *(const gu32x4*)(src + k * 32);
                    v[4 * k] = t.x; v[4 * k + 1] = t.y; v[4 * k + 2] = t.z; v[4 * k + 3] = t.w;
                }
            }
        };
        if (producer) load_block(wl);
        if (tid < 16) {
            u32 sel = 0, k = 0;
            for (int j = 0; j < 4; ++j) { const u32 on = (tid >> j) & 1; sel |= (on ? k : 0x0cu) << (8 * j); k += on; }
            *(LAS u32*)(lds + LDS_LUT + tid * 4) = sel;
        }
        const u32 call = rfl(ticket) + 1u;
        gu32* tick = (gu32*)(myws + OFF_TICK + d * 128);
        const u32 tbase = (call - 1u) * (u32)(NBLK - NWL) - 2u * NWL;
        u32x4 pk[4];
        u32 p_off = 0, p_foff = 0, p_mode = 0;
        u64 p_flag = 0;
        auto pack = [&](int nb) {
            const int n = nb >> 2, wb = nb & 3;
            const int c = n / 24, e = n - c * 24;
            const int step = rank * 8 + c;
            u32 selL, selH;
            if (e < 8) { const int j = (e + r) & 7;
                p_off = OFF_B + (u32)(step * 8 + j) * PIECE;  p_foff = OFF_FLAGB + (u32)((j * 4 + wb) * 64 + step) * 8u;
                selL = 0x06020400u; selH = 0x07030501u;
            } else { const int i = (e - 8 + 2 * r) & 15;
                p_off = OFF_A + (u32)(step * 16 + i) * PIECE; p_foff = OFF_FLAGA + (u32)((i * 4 + wb) * 64 + step) * 8u;
                selL = 0x06040200u; selH = 0x07050301u;
            }
            p_off += (u32)wb * 4096u;
            u32 H[8], E[8];
            u16x2 mx = {0, 0};
#pragma unroll
            for (int i = 0; i < 16; ++i) mx = __builtin_elementwise_max(mx, __builtin_bit_cast(u16x2, v[i] & 0x7fff7fffu));
#pragma unroll
            for (int i = 0; i < 8; ++i) { pk[i >> 2][i & 3] = perm(v[2 * i + 1], v[2 * i], selL); H[i] = perm(v[2 * i + 1], v[2 * i], selH); }
            const u32 top = wave_max((u32)max(mx.x, mx.y) >> 8);
            const u32 base = top > 7u ? top - 7u : 0u;
            const u32 base4 = base * 0x01010101u;
            u32 bad = 0;
#pragma unroll
            for (int i = 0; i < 8; ++i) { E[i] = H[i] - base4; bad |= E[i]; }
            u32 mode = 0;
            u64 exc = 0;
            if (__ballot((bad & 0x78787878u) != 0u) != 0ull) {
                u32 pm = 0;
#pragma unroll
                for (int i = 0; i < 8; ++i) {
                    const u32 dd = (H[i] | 0x80808080u) - base4;
                    const u32 inv = ((dd ^ 0x80808080u) >> 1) & 0x7c7c7c7cu;
                    const u32 f = ((inv + 0x7f7f7f7fu) >> 7) & 0x01010101u;
                    pm |= __builtin_amdgcn_udot4(f, 0x08040201u, 0u, false) << (4 * i);
                    E[i] = (H[i] & 0x80808080u) | (dd & 0x07070707u);
                }
                const u32 total = rdl(wave_scan((u32)__popc(pm)), 63);
                if (total > 2u) mode = 3;
                else {
                    mode = total & 2u;
                    const u32 p0 = (u32)__ffs((int)pm) - 1u, p1 = (u32)__ffs((int)(pm & (pm - 1u))) - 1u;
                    u32 h0 = 0, h1 = 0;
#pragma unroll
                    for (int i = 0; i < 8; ++i) {
                        const u32 x = H[i] ^ (E[i] + base4);
                        if ((p0 >> 2) == (u32)i) h0 = x;
                        if ((p1 >> 2) == (u32)i) h1 = x;
                    }
                    const u32 e0 = (u32)lane | (((p0 >> 2) & 7u) << 6) | (((h0 >> (8 * (p0 & 3u))) & 127u) << 9) | ((p0 & 3u) << 16);
                    const u32 e1 = (u32)lane | (((p1 >> 2) & 7u) << 6) | (((h1 >> (8 * (p1 & 3u))) & 127u) << 9) | ((p1 & 3u) << 16);
                    const u64 lanes = __ballot(pm != 0u);
                    const int l0 = __ffsll((long long)lanes) - 1;
                    const u32 x0 = rdl(e0, l0);
                    exc = (x0 & 0xffffu) | ((x0 >> 16) << 19);
                    if (total == 2u) {
                        const u64 rest = lanes & (lanes - 1ull);
                        const u32 x1 = rest ? rdl(e0, __ffsll((long long)rest) - 1) : rdl(e1, l0);
                        exc |= (u64)x1 << 21;
                    }
                }
            }
            if (mode == 3) pk[2] = u32x4{H[0], H[1], H[2], H[3]};
            else {
#pragma unroll
                for (int q = 0; q < 4; ++q) pk[2][q] = E[2 * q] | __builtin_amdgcn_alignbit(E[2 * q + 1], E[2 * q + 1], 28);
            }
            pk[3] = u32x4{H[4], H[5], H[6], H[7]};
            p_mode = mode;
            p_flag = ((u64)((call << SEQ_SH) | (mode << 15) | (base << 8)) << 32) | exc;
        };
        if (producer) {
            pack(wl);
            u32 nB = (u32)(wl + NWL);
            load_block((int)nB);
            const u32 vo = (u32)lane * 16u;
            char* fdst = nullptr;
            u64 flag = 0;
            const u32 zero = 0, one = 1;
            auto iter = [&](bool first) {
                const bool more = nB < (u32)NBLK;
                u32 tk = 0;
                if (more) asm volatile("s_mov_b64 exec, 1\n\tglobal_atomic_add %0, %1, %2, %3 sc0\n\ts_mov_b64 exec, -1"
                                       : "=&v"(tk) : "v"(zero), "v"(one), "s"(tick) : "memory");
                if (!first && lane == 0) __hip_atomic_store((gu64*)fdst, flag, RLX_SYS);
                CBAR();
                if (first) STAMP(7);
                __builtin_amdgcn_raw_buffer_store_b128(pk[0], rs, vo, p_off, AUX_SYS);
                __builtin_amdgcn_raw_buffer_store_b128(pk[1], rs, vo + 1024, p_off, AUX_SYS);
                __builtin_amdgcn_raw_buffer_store_b128(pk[2], rs, vo + 2048, p_off, AUX_SYS);
                if (p_mode == 3) __builtin_amdgcn_raw_buffer_store_b128(pk[3], rs, vo + 3072, p_off, AUX_SYS);
                CBAR();
                fdst = dws + p_foff;
                flag = p_flag;
                if (more) {
                    pack((int)nB);
                    if (first) asm volatile("s_waitcnt vmcnt(3)" : "+v"(tk) :: "memory");
                    else       asm volatile("s_waitcnt vmcnt(4)" : "+v"(tk) :: "memory");
                    nB = rfl(tk) - tbase;
                    if (nB < (u32)NBLK) load_block((int)nB);
                }
                __builtin_amdgcn_s_waitcnt(0x0F70);
                WAIT_VM0();
                if (THINK) __builtin_amdgcn_s_sleep(THINK);
                return more;
            };
            iter(true);
#pragma unroll 1
            while (iter(false)) { }
            if (lane == 0) __hip_atomic_store((gu64*)fdst, flag, RLX_SYS);
        }
        STAMP(1);
        LDS_BARRIER();
        bool dead = false;
        if (b == 56 && wave == 0) {
            u32 spins = 0, m = 0;
            for (;;) {
                const u64 g0 = __hip_atomic_load((gu64*)(myws + OFF_AMAXL) + lane, RLX_AGT);
                const u64 g1 = __hip_atomic_load((gu64*)(myws + OFF_AMAXL) + 64 + lane, RLX_AGT);
                m = max((u32)g0, (u32)g1);
                if (__all((u32)(g0 >> 32) == call && (u32)(g1 >> 32) == call)) break;
                if (spin_fail(tstart, tmo, call, spins)) { dead = true; break; }
            }
            if (!dead) {
                m = wave_max(m);
                char* dst = P.ws[lane & 7] + OFF_AMAXG + (lane >> 3) * 128 + rank * 8;
#pragma unroll
                for (int k = 0; k < NREP / 8; ++k)
                    __hip_atomic_store((gu64*)(dst + k * 1024), ((u64)call << 32) | m, RLX_SYS);
            }
            STAMP(2);
        }
        const int tile = slot * 8 + xcd, ti = tile >> 3, tj = tile & 7;
        const int wq = wave & 3, hf = wave >> 2;
        const int mw = tile * 4 + wq, rot = mw % 7;
        __amdgpu_buffer_rsrc_t rm = mk_rsrc(myws, WS_USED);
        __amdgpu_buffer_rsrc_t ro = mk_rsrc(P.out[rank], (u32)MM * NN);
        gu64* fo = (gu64*)(myws + OFF_FLAGO + mw * 128) + (lane & 7);
        LAS char* str = lds + LDS_STR + wave * 4096;
        LAS char* ot = lds + LDS_W + wave * 2048;
        u64 fl = 0;
#pragma unroll 1
        for (int k = 0; k < 7; ++k) {
            int dd = k + rot; if (dd >= 7) dd -= 7;
            const int q = (rank - 1 - dd) & 7;
            if (!dead && rdl((u32)(fl >> 32), q) != call) {
                u32 spins = 0;
                for (;;) {
                    fl = __hip_atomic_load(fo, RLX_SYS);
                    if (rdl((u32)(fl >> 32), q) == call) break;
                    if (spin_fail(tstart, tmo, call, spins)) { dead = true; break; }
                }
            }
            CBAR();
            const u32 ooff = (u32)((q >> 2) * 2048 + ti * 128 + wq * 32 + (lane >> 2)) * 4096u
                           + (u32)((q & 3) * 1024 + tj * 128 + hf * 64 + (lane & 3) * 16);
            if (dead) {
#pragma unroll
                for (int bb = 0; bb < 2; ++bb)
                    __builtin_amdgcn_raw_buffer_store_b128(u32x4{0x7f7f7f7fu, 0x7f7f7f7fu, 0x7f7f7f7fu, 0x7f7f7f7fu}, ro, ooff + bb * 65536u, 0, 0);
                continue;
            }
            const u32 fw = rdl((u32)fl, q);
            const u32 nch = fw & 255u;
            const u32 moff = OFF_MSG + (u32)(q * 512 + mw) * MSG;
            const u32x2 mk = __builtin_amdgcn_raw_buffer_load_b64(rm, (u32)lane * 8u, moff, AUX_SYS);
            const u32x4 c0 = __builtin_amdgcn_raw_buffer_load_b128(rm, 512u + (u32)lane * 16u, moff, AUX_SYS);
            const u32x4 c1 = __builtin_amdgcn_raw_buffer_load_b128(rm, 1536u + (u32)lane * 16u, moff, AUX_SYS);
            const u32 c_lo = (u32)__popc(mk.x), cnt = c_lo + (u32)__popc(mk.y);
            const u32 first = wave_scan(cnt) - (hf ? cnt - c_lo : cnt);
            const u32 mm = hf ? mk.y : mk.x;
            *(LAS u32x4*)(str + lane * 16) = c0;
            *(LAS u32x4*)(str + 1024 + lane * 16) = c1;
            if (nch > 128u) {
                const u32x4 c2 = __builtin_amdgcn_raw_buffer_load_b128(rm, 2560u + (u32)lane * 16u, moff, AUX_SYS);
                *(LAS u32x4*)(str + 2048 + lane * 16) = c2;
                if (lane < 32) {
                    const u32x4 c3 = __builtin_amdgcn_raw_buffer_load_b128(rm, 3584u + (u32)lane * 16u, moff, AUX_SYS);
                    *(LAS u32x4*)(str + 3072 + lane * 16) = c3;
                }
            }
            WAIT_LGKM0();
            u32x4 o[2];
            u32 ex = 0;
            if (__builtin_expect((fw & (1u << 27)) == 0u, 1)) {
                ex = *(LAS unsigned short*)(str + ((fw >> 8) & 255u) * 16u + lane * 2);
                const u32 b8 = ((fw >> 23) & 15u) * 0x08080808u;
                u32 pos = first * 6u;
#pragma unroll
                for (int i = 0; i < 8; ++i) {
                    const int sh = 4 * i;
                    const u32 n4 = (sh ? (mm >> (sh - 2)) : (mm << 2)) & 0x3cu;
                    const u32 sel = *(LAS u32*)(lds + LDS_LUT + n4);
                    const LAS u32* wp = (const LAS u32*)(str + ((pos >> 5) << 2));
                    const u32 z = __builtin_amdgcn_alignbit(wp[1], wp[0], pos);
                    pos += 6u * (u32)__popc(n4);
                    const u32 y = bfi(0x00000fffu, z, z << 4);
                    const u32 x = (bfi(0x003f003fu, y, y << 2) & 0x3f3f3f3fu) + b8;
                    o[i >> 2][i & 3] = perm(x, x, sel);
                }
            } else {
                u32 pos = first * 7u;
#pragma unroll
                for (int i = 0; i < 8; ++i) {
                    const int sh = 4 * i;
                    const u32 n4 = (sh ? (mm >> (sh - 2)) : (mm << 2)) & 0x3cu;
                    const u32 sel = *(LAS u32*)(lds + LDS_LUT + n4);
                    const LAS u32* wp = (const LAS u32*)(str + ((pos >> 5) << 2));
                    const u32 z = __builtin_amdgcn_alignbit(wp[1], wp[0], pos);
                    pos += 7u * (u32)__popc(n4);
                    const u32 y = bfi(0x00003fffu, z, z << 2);
                    const u32 x = bfi(0x007f007fu, y, y << 1) & 0x7f7f7f7fu;
                    o[i >> 2][i & 3] = perm(x, x, sel);
                }
            }
            *(LAS u32x4*)(ot + (lane & 31) * 64 + (lane >> 5) * 16) = o[0];
            *(LAS u32x4*)(ot + (lane & 31) * 64 + (lane >> 5) * 16 + 32) = o[1];
            if ((u32)lane < ((fw >> 16) & 127u) && ((ex >> 6) & 1u) == (u32)hf)
                __hip_atomic_fetch_xor((LAS u32*)(ot + ((ex >> 7) & 31u) * 64u + (ex & 60u)), 0x40u << (8u * (ex & 3u)), __ATOMIC_RELAXED, __HIP_MEMORY_SCOPE_WAVEFRONT);
            CBAR();
#pragma unroll
            for (int bb = 0; bb < 2; ++bb)
                __builtin_amdgcn_raw_buffer_store_b128(*(LAS u32x4*)(ot + bb * 1024 + lane * 16), ro, ooff + bb * 65536u, 0, AUX_OUT);
        }
        STAMP(6);
        return;
    }

    const int sl = slot - 16;
    const int ti = (xcd >> 1) * 4 + (sl >> 2), tj = (xcd & 1) * 4 + (sl & 3), tile = ti * 8 + tj;
    const bool helper = wave >= 4;
    const int wq = wave & 3;
    if (tid < 16) {
        u32 sel = 0x0c0c0c0cu, k = 0;
        for (int j = 0; j < 4; ++j) if ((tid >> j) & 1) { sel = (sel & ~(0xffu << (8 * k))) | ((u32)j << (8 * k)); ++k; }
        *(LAS u32*)(lds + LDS_LUT + tid * 4) = sel;
    }
    if (tid < 8) *(LAS u32*)(lds + LDS_RED + tid * 4) = 0;
    LAS char* str = lds + LDS_STR + wave * 4096;
#pragma unroll
    for (int u = 0; u < 4; ++u) *(LAS u32x4*)(str + u * 1024 + lane * 16) = u32x4{0, 0, 0, 0};
    u32 stepoff;
    {
        const int c = lane >> 3, k = (lane & 7) - 1, r0 = ti >> 1;
        const int r = k < 0 ? 0 : ((r0 + 13 - k) % 7) + 1;
        stepoff = (u32)(((rank + r) & 7) * 8 + c);
    }
    gu64* fptr = (gu64*)(myws + (helper ? OFF_FLAGB + (u32)((tj * 4 + wq) * 64) * 8u : OFF_FLAGA + (u32)((ti * 4 + wq) * 64) * 8u)) + stepoff;
    stepoff *= helper ? 8u * PIECE : 16u * PIECE;
    __amdgpu_buffer_rsrc_t rs = mk_rsrc(myws, WS_USED);
    const u32 voff = (helper ? OFF_B + (u32)tj * PIECE : OFF_A + (u32)ti * PIECE) + (u32)wq * 4096u + (u32)lane * 16u;
    const u32 call = rfl(ticket) + 1u;
    const u32 seq = call & SEQ_MASK;
    u64 ready = 0, fl = 0;
    bool dead = false;
    auto wait_ready = [&](int s) {
        if (__builtin_expect(!((ready >> s) & 1), 0)) {
            u32 spins = 0;
            for (;;) {
                fl = __hip_atomic_load(fptr, RLX_SYS);
                ready = __ballot((u32)(fl >> (32 + SEQ_SH)) == seq);
                if ((ready >> s) & 1) break;
                if (spin_fail(tstart, tmo, call, spins)) { dead = true; fl = 0; ready = ~0ull; break; }
            }
        }
        CBAR();
    };
#define SB() __builtin_amdgcn_sched_barrier(0)
    u32x4 rg[2][3];
    auto issue = [&](int s, u32x4 (&r)[3]) {
        const u32 so = rdl(stepoff, s);
#pragma unroll
        for (int u = 2; u < 5; ++u) { r[u % 3] = __builtin_amdgcn_raw_buffer_load_b128(rs, voff + (u % 3) * 1024, so, AUX_SYS); SB(); }
    };
    auto fix = [&](int s, u32 flo, u32 fhi, u32 (&H)[8], const u32x4 nb) {
        fix_one(H, flo, (flo >> 6) & 7u, ((flo >> 9) & 127u) << ((flo >> 16) & 31u));
        if (__builtin_expect((fhi & 0x10000u) != 0u, 0))
            fix_rare(H, nb, flo, fhi, myws + (voff + rdl(stepoff, s) + 3072u));
    };
    using T = std::true_type; using F = std::false_type;
    using C0 = std::integral_constant<int, 0>; using C1 = std::integral_constant<int, 1>;

    wait_ready(0); issue(0, rg[0]);
    wait_ready(1); issue(1, rg[1]);
    if (helper) {
        u32x4 r3[3];
        u32 pw = (u32)(LDS_W + wq * 4096 + lane * 16);
        auto putW = [&](int s, u32x4 (&r)[3], u32x4 (&rn)[3], bool iss) {
            u32 so = 0;
            if (iss) {
                wait_ready(s + 2); so = rdl(stepoff, s + 2);
                rn[2] = __builtin_amdgcn_raw_buffer_load_b128(rs, voff + 2048, so, AUX_SYS);
            }
            const u32 fhi = rdl((u32)(fl >> 32), s), flo = rdl((u32)fl, s);
            const u32 base4 = ((fhi >> 8) & 127u) * 0x01010101u;
            SB();
            u32 H[8];
            plane<0>(r[2][0], base4, H); plane<1>(r[2][1], base4, H);
            SB();
            if (iss) rn[0] = __builtin_amdgcn_raw_buffer_load_b128(rs, voff, so, AUX_SYS);
            plane<2>(r[2][2], base4, H); plane<3>(r[2][3], base4, H);
            fix(s, flo, fhi, H, r[2]);
            u32x4 c[4];
#pragma unroll
            for (int q = 0; q < 4; ++q) {
                const u32 sl_ = (q & 1) ? 0x07030602u : 0x05010400u;
                const int xy = q >> 1;
                c[q] = u32x4{perm(H[xy], r[0][xy], sl_), perm(H[2 + xy], r[0][2 + xy], sl_),
                             perm(H[4 + xy], r[1][xy], sl_), perm(H[6 + xy], r[1][2 + xy], sl_)};
                if (q) *(LAS u32x4*)(lds + pw + (q - 1) * 1024) = c[q - 1];
                SB();
            }
            *(LAS u32x4*)(lds + pw + 3 * 1024) = c[3];
            SB();
            if (iss) rn[1] = __builtin_amdgcn_raw_buffer_load_b128(rs, voff + 1024, so, AUX_SYS);
            pw ^= 16384u;
        };
        putW(0, rg[0], r3, true);
        LDS_BARRIER();
#pragma unroll 1
        for (int j = 0; j < 60; j += 3) {
            putW(j + 1, rg[1], rg[0], true); LDS_BARRIER();
            putW(j + 2, r3, rg[1], true);    LDS_BARRIER();
            putW(j + 3, rg[0], r3, true);    LDS_BARRIER();
        }
        putW(61, rg[1], rg[0], true); LDS_BARRIER();
        putW(62, r3, rg[1], false);   LDS_BARRIER();
        putW(63, rg[0], r3, false);
        if (dead && lane == 0) *(LAS u32*)(lds + LDS_RED + 20) = 1;
        LDS_BARRIER();
        LDS_BARRIER();
        if (wave == 4 && lane == 0) {
            const u32x4 t = *(LAS u32x4*)(lds + LDS_RED);
            __hip_atomic_store((gu64*)(myws + OFF_AMAXL) + tile, ((u64)call << 32) | max(max(t.x, t.y), max(t.z, t.w)), RLX_AGT);
        }
        LDS_BARRIER();
        if (wave != 4) {
#pragma unroll 1
            for (int u = wave - 5; u < 28; u += 3) {
                if (u >= 4 * MEARLY)
                    while (__hip_atomic_load((LAS u64*)(lds + LDS_RED + 24), __ATOMIC_RELAXED, __HIP_MEMORY_SCOPE_WORKGROUP) == 0ull) __builtin_amdgcn_s_sleep(1);
                const int mwu = tile * 4 + (u & 3);
                int dd = (u >> 2) + mwu % 7; if (dd >= 7) dd -= 7;
                if (lane < 32) {
                    const u32x4 mk = *(LAS u32x4*)(lds + LDS_MSK + (u & 3) * 512 + lane * 16);
                    __builtin_amdgcn_raw_buffer_store_b128(mk, mk_rsrc(P.ws[(rank + 1 + dd) & 7], WS_USED), (u32)lane * 16u,
                                                           OFF_MSG + (u32)(rank * 512 + mwu) * MSG, AUX_SYS);
                }
                asm volatile("s_waitcnt vmcnt(%0)" :: "n"(MWIN - 1) : "memory");
            }
            WAIT_VM0();
            __builtin_amdgcn_s_barrier();
            return;
        }
        u32 gb = 0;
        if (!dead) {
            gu64* g = lane == 8 ? (gu64*)tmo : (gu64*)(myws + OFF_AMAXG + (tile & (NREP - 1)) * 128) + (lane & 7);
            u32 spins = 0;
            for (;;) {
                const u64 a = __hip_atomic_load(g, RLX_SYS);
                gb = (u32)a;
                if ((__ballot((u32)(a >> 32) == call) & 0xffull) == 0xffull) break;
                if (spin_fail(tstart, tmo, call, spins)) { dead = true; break; }
            }
            if (rdl(gb, 8) == call) dead = true;
            gb = wave_max(lane == 8 ? 0u : gb);
        }
        if (*(LAS u32*)(lds + LDS_RED + 20) != 0u) dead = true;
        if (lane == 0) __hip_atomic_store((LAS u64*)(lds + LDS_RED + 24), ((u64)(dead ? 3u : 1u) << 32) | __float_as_uint(448.0f / __uint_as_float(gb)), __ATOMIC_RELEASE, __HIP_MEMORY_SCOPE_WORKGROUP);
        return;
    }
    bf16x8 xf[2][4], wfA[8], wfB[8];
    f32x16 acc[4];
#pragma unroll
    for (int j = 0; j < 4; ++j)
#pragma unroll
        for (int i = 0; i < 16; ++i) acc[j][i] = 0.f;
    auto putX = [&](int q, u32 (&H)[8], u32x4 (&r)[3], bf16x8 (&xo)[4]) {
        const u32 ha = H[2 * q], hb = H[2 * q + 1];
        const u32 la = r[q >> 1][(2 * q) & 3], lb = r[q >> 1][(2 * q + 1) & 3];
        u32x4 c = u32x4{perm(ha, la, 0x05010400u), perm(ha, la, 0x07030602u), perm(hb, lb, 0x05010400u), perm(hb, lb, 0x07030602u)};
        asm volatile("" : "+v"(c));
        xo[q] = __builtin_bit_cast(bf16x8, c);
    };
    auto step = [&](int s, auto cur_c, auto dec_c, auto iss_c) {
        constexpr int cur = decltype(cur_c)::value, nxt = cur ^ 1;
        constexpr bool DEC = decltype(dec_c)::value, ISS = decltype(iss_c)::value;
        u32 H[8];
        u32 fhi = 0, flo = 0, base4 = 0, fpl = 0, fx = 0;
        LAS char* pr = lds + LDS_W + cur * 16384 + 8192 + lane * 16;
#pragma unroll
        for (int q = 0; q < 8; ++q) {
            acc[q & 3] = __builtin_amdgcn_mfma_f32_32x32x16_bf16(wfA[q], xf[cur][q >> 2], acc[q & 3], 0, 0, 0);
            SB();
            if (q < 4) { wfB[2 * q] = *(LAS bf16x8*)(pr + (2 * q) * 1024); wfB[2 * q + 1] = *(LAS bf16x8*)(pr + (2 * q + 1) * 1024); }
            if constexpr (DEC) {
                if (q == 0) { fhi = rdl((u32)(fl >> 32), s + 1); flo = rdl((u32)fl, s + 1); asm volatile("" : "+s"(fhi), "+s"(flo)); }
                if (q == 1) { base4 = ((fhi >> 8) & 127u) * 0x01010101u; fpl = (flo >> 6) & 7u; asm volatile("" : "+s"(base4), "+s"(fpl)); }
                if (q == 2) { fx = ((flo >> 9) & 127u) << ((flo >> 16) & 31u); asm volatile("" : "+s"(fx)); }
                if (q == 3) plane<0>(rg[nxt][2][0], base4, H);
                if (q == 4) plane<1>(rg[nxt][2][1], base4, H);
                if (q == 5) plane<2>(rg[nxt][2][2], base4, H);
                if (q == 6) plane<3>(rg[nxt][2][3], base4, H);
            }
            SB();
        }
        u32 so = 0;
        if constexpr (DEC) {
            fix_one(H, flo, fpl, fx);
            if (__builtin_expect((fhi & 0x10000u) != 0u, 0))
                fix_rare(H, rg[nxt][2], flo, fhi, myws + (voff + rdl(stepoff, s + 1) + 3072u));
            LDS_BARRIER();
        }
        SB();
        pr = lds + LDS_W + nxt * 16384 + lane * 16;
#pragma unroll
        for (int q = 0; q < 8; ++q) {
            acc[q & 3] = __builtin_amdgcn_mfma_f32_32x32x16_bf16(wfB[q], xf[cur][2 + (q >> 2)], acc[q & 3], 0, 0, 0);
            SB();
            if constexpr (DEC) { if (q < 4) { wfA[2 * q] = *(LAS bf16x8*)(pr + (2 * q) * 1024); wfA[2 * q + 1] = *(LAS bf16x8*)(pr + (2 * q + 1) * 1024); } }
            if constexpr (ISS) {
                if (q == 0) { wait_ready(s + 3); so = rdl(stepoff, s + 3); asm volatile("" : "+s"(so)); }
                if (q == 2) rg[nxt][2] = __builtin_amdgcn_raw_buffer_load_b128(rs, voff + 2048, so, AUX_SYS);
                if (q == 4) rg[nxt][0] = __builtin_amdgcn_raw_buffer_load_b128(rs, voff, so, AUX_SYS);
                if (q == 7) rg[nxt][1] = __builtin_amdgcn_raw_buffer_load_b128(rs, voff + 1024, so, AUX_SYS);
            }
            if constexpr (DEC) { if (q == 1 || q == 3 || q == 5 || q == 6) putX(q == 6 ? 3 : q >> 1, H, rg[nxt], xf[nxt]); }
            if (q == 6) __builtin_amdgcn_s_waitcnt(0xC07F);
            SB();
        }
        TRACE(s);
    };
    {
        const u32 fhi = rdl((u32)(fl >> 32), 0);
        const u32 base4 = ((fhi >> 8) & 127u) * 0x01010101u;
        u32 H[8];
        plane<0>(rg[0][2][0], base4, H); plane<1>(rg[0][2][1], base4, H);
        plane<2>(rg[0][2][2], base4, H); plane<3>(rg[0][2][3], base4, H);
        fix(0, rdl((u32)fl, 0), fhi, H, rg[0][2]);
#pragma unroll
        for (int q = 0; q < 4; ++q) putX(q, H, rg[0], xf[0]);
    }
    wait_ready(2); issue(2, rg[0]);
    LDS_BARRIER();
#pragma unroll
    for (int i = 0; i < 8; ++i) wfA[i] = *(LAS bf16x8*)(lds + LDS_W + lane * 16 + i * 1024);
    WAIT_LGKM0();
    STAMP(1);
#pragma unroll 1
    for (int s = 0; s < 60; s += 2) {
        step(s, C0{}, T{}, T{});
        step(s + 1, C1{}, T{}, T{});
    }
    step(60, C0{}, T{}, T{});
    step(61, C1{}, T{}, F{});
    step(62, C0{}, T{}, F{});
    step(63, C1{}, F{}, F{});
    STAMP(5);

    float mx = 0.f;
#pragma unroll
    for (int j = 0; j < 4; ++j)
#pragma unroll
        for (int i = 0; i < 16; ++i) mx = fmaxf(mx, acc[j][i]);
    const u32 mb = wave_max(__float_as_uint(mx));
    if (lane == 0) *(LAS u32*)(lds + LDS_RED + wave * 4) = mb;
    if (dead && lane == 0) *(LAS u32*)(lds + LDS_RED + 20) = 1;
    LDS_BARRIER();
    u32 m0 = 0, m1 = 0;
#pragma unroll
    for (int i = 7; i >= 0; --i)
#pragma unroll
        for (int j = 3; j >= 0; --j) {
            m0 = __builtin_amdgcn_alignbit(m0, __float_as_uint(acc[j][i]), 31);
            m1 = __builtin_amdgcn_alignbit(m1, __float_as_uint(acc[j][i + 8]), 31);
        }
    m0 = ~m0; m1 = ~m1;
    const int mw = tile * 4 + wave, rot = mw % 7;
    const u32 moff = OFF_MSG + (u32)(rank * 512 + mw) * MSG;
    *(LAS u32x2*)(lds + LDS_MSK + wave * 512 + lane * 8) = u32x2{m0, m1};
    LDS_BARRIER();
    const u32 cnt = (u32)(__popc(m0) + __popc(m1));
    const u32 incl = wave_scan(cnt);
    const u32 total = rdl(incl, 63);
    u32 sel[16], pa[16], ps[16], bm[16];
    {
        u32 pp = (incl - cnt) * 6u;
#pragma unroll
        for (int i = 0; i < 16; ++i) {
            const u32 mm = i < 8 ? m0 : m1;
            const int sh = 4 * (i & 7);
            const u32 n4 = (sh ? (mm >> (sh - 2)) : (mm << 2)) & 0x3cu;
            sel[i] = *(LAS u32*)(lds + LDS_LUT + n4);
            bm[i] = __umul24(n4 << 4, 0x00204081u) & 0x40404040u;
            pa[i] = (pp >> 5) << 2; ps[i] = pp & 31u;
            pp += 6u * (u32)__popc(n4);
            asm volatile("" : "+v"(bm[i]), "+v"(pa[i]), "+v"(ps[i]));
        }
    }
#pragma unroll
    for (int j = 0; j < 4; ++j)
#pragma unroll
        for (int i = 0; i < 16; ++i) acc[j][i] = fmaxf(acc[j][i], 0.f);
#pragma unroll
    for (int j = 0; j < 4; ++j) asm volatile("" : "+v"(acc[j]));
    u64 gd;
    while ((gd = __hip_atomic_load((LAS u64*)(lds + LDS_RED + 24), __ATOMIC_ACQUIRE, __HIP_MEMORY_SCOPE_WORKGROUP)) == 0ull) __builtin_amdgcn_s_sleep(1);
    const float inv = __uint_as_float(rfl((u32)gd));
    dead = rfl((u32)(gd >> 32)) != 1u;
    STAMP(6);

#pragma unroll
    for (int j = 0; j < 4; ++j) acc[j] *= inv;
    const u32 etop = (rfl((u32)__builtin_amdgcn_cvt_pk_fp8_f32(__uint_as_float(mb) * inv, 0.f, 0, false)) >> 3) & 15u;
    u32 base = etop > 8u ? etop - 8u : 0u;
    u32x4 q[4];
    u32 bad0 = 0, bad1 = 0;
    {
        const u32 b8 = base * 0x08080808u;
#pragma unroll
        for (int i = 0; i < 16; ++i) {
            int w0 = __builtin_amdgcn_cvt_pk_fp8_f32(acc[0][i], acc[1][i], 0, false);
            w0 = __builtin_amdgcn_cvt_pk_fp8_f32(acc[2][i], acc[3][i], w0, true);
            q[i >> 2][i & 3] = (u32)w0 & 0x7f7f7f7fu;
            const u32 t = ((u32)w0 | 0x80808080u) - b8;
            const u32 n64 = __builtin_amdgcn_udot4(t & bm[i], 0x08040201u, 0u, false);
            const int sh = 4 * (i & 7) - 6;
            if (i < 8) bad0 |= sh < 0 ? n64 >> -sh : n64 << sh; else bad1 |= sh < 0 ? n64 >> -sh : n64 << sh;
            const u32 c = perm(t, t, sel[i]) & 0x3f3f3f3fu;
            const u32 y = bfi(0x003f003fu, c, c >> 2);
            const u32 z = bfi(0x00000fffu, y, y >> 4);
            const u64 sh64 = (u64)z << ps[i];
            LAS u32* wp = (LAS u32*)(str + pa[i]);
            __hip_atomic_fetch_or(wp, (u32)sh64, __ATOMIC_RELAXED, __HIP_MEMORY_SCOPE_WAVEFRONT);
            __hip_atomic_fetch_or(wp + 1, (u32)(sh64 >> 32), __ATOMIC_RELAXED, __HIP_MEMORY_SCOPE_WAVEFRONT);
        }
    }
    u32 nst = (total * 6u + 127u) >> 7;
    const u32 nb = (u32)(__popc(bad0) + __popc(bad1));
    const u32 nbi = wave_scan(nb);
    u32 nexc = rdl(nbi, 63), seven = 0;
    if (__builtin_expect(nexc <= 64u, 1)) {
        LAS unsigned short* lp = (LAS unsigned short*)(str + nst * 16u) + (nbi - nb);
        const u32 lb = (u32)(lane & 31) * 128u + (u32)(lane >> 5) * 16u;
        while (__ballot((bad0 | bad1) != 0u) != 0ull) {
            if ((bad0 | bad1) != 0u) {
                const u32 pb = bad0 ? (u32)__ffs((int)bad0) - 1u : (u32)__ffs((int)bad1) + 31u;
                if (bad0) bad0 &= bad0 - 1u; else bad1 &= bad1 - 1u;
                *lp++ = (unsigned short)(lb + ((pb >> 4) << 5) + (pb & 15u));
            }
        }
    } else {
        seven = 1; base = 0; nexc = 0;
        nst = (total * 7u + 127u) >> 7;
        WAIT_LGKM0();
#pragma unroll
        for (int u = 0; u < 4; ++u) *(LAS u32x4*)(str + u * 1024 + lane * 16) = u32x4{0, 0, 0, 0};
        u32 pp = (incl - cnt) * 7u;
#pragma unroll
        for (int i = 0; i < 16; ++i) {
            const u32 qq = q[i >> 2][i & 3];
            const u32 c = perm(qq, qq, sel[i]);
            const u32 y = bfi(0x007f007fu, c, c >> 1);
            const u32 z = bfi(0x00003fffu, y, y >> 2);
            const u64 sh64 = (u64)z << (pp & 31u);
            LAS u32* wp = (LAS u32*)(str + ((pp >> 5) << 2));
            __hip_atomic_fetch_or(wp, (u32)sh64, __ATOMIC_RELAXED, __HIP_MEMORY_SCOPE_WAVEFRONT);
            __hip_atomic_fetch_or(wp + 1, (u32)(sh64 >> 32), __ATOMIC_RELAXED, __HIP_MEMORY_SCOPE_WAVEFRONT);
            pp += 7u * (u32)__popc(bm[i]);
        }
    }
    const u32 nch = nst + ((nexc + 7u) >> 3);
    WAIT_LGKM0();
    u32x4 cs[4];
#pragma unroll
    for (int u = 0; u < 4; ++u) cs[u] = *(LAS u32x4*)(str + u * 1024 + lane * 16);
    STAMP(7);
    if (!dead) {
        const u64 flag = ((u64)call << 32) | nch | (nst << 8) | (nexc << 16) | (base << 23) | (seven << 27);
        const u32 foff = OFF_FLAGO + (u32)mw * 128u + (u32)rank * 8u;
        if (__builtin_expect(nch <= 128u, 1)) {
            char* dst[7];
#pragma unroll
            for (int k = 0; k < 7; ++k) { int dd = k + rot; if (dd >= 7) dd -= 7; dst[k] = P.ws[(rank + 1 + dd) & 7]; }
            const u64 e0 = __ballot((u32)lane < max(nch, 1u)), e1 = __ballot((u32)lane + 64u < max(nch, 65u));
            const u32 vo = moff + 512u + (u32)lane * 16u;
            const u32x2 fv = u32x2{(u32)flag, (u32)(flag >> 32)};
            auto send = [&](int k) {
                asm volatile("s_mov_b64 exec, %3\n\tglobal_store_dwordx4 %0, %1, %2 sc0 sc1\n\ts_mov_b64 exec, -1\n\ts_nop 1"
                             :: "v"(vo), "v"(cs[0]), "s"(dst[k]), "s"(e0) : "memory");
                if ((PACE >> ((k + (PROT ? rot : 0)) % 7)) & 1) WAIT_VM0();
                asm volatile("s_mov_b64 exec, %3\n\tglobal_store_dwordx4 %0, %1, %2 offset:1024 sc0 sc1\n\ts_mov_b64 exec, -1\n\ts_nop 1"
                             :: "v"(vo), "v"(cs[1]), "s"(dst[k]), "s"(e1) : "memory");
            };
#pragma unroll
            for (int k = 0; k < SWIN; ++k) send(k);
            __builtin_amdgcn_s_barrier();
            mark_msg<0>(foff, fv, dst[0]); if (SWIN + 0 < 7) send(SWIN + 0);
            mark_msg<1>(foff, fv, dst[1]); if (SWIN + 1 < 7) send(SWIN + 1);
            mark_msg<2>(foff, fv, dst[2]); if (SWIN + 2 < 7) send(SWIN + 2);
            mark_msg<3>(foff, fv, dst[3]); if (SWIN + 3 < 7) send(SWIN + 3);
            mark_msg<4>(foff, fv, dst[4]); if (SWIN + 4 < 7) send(SWIN + 4);
            mark_msg<5>(foff, fv, dst[5]); if (SWIN + 5 < 7) send(SWIN + 5);
            mark_msg<6>(foff, fv, dst[6]);
        } else {
            __builtin_amdgcn_s_barrier();
#pragma unroll 1
            for (int k = 0; k < 7; ++k) {
                int dd = k + rot; if (dd >= 7) dd -= 7;
                char* w0 = P.ws[(rank + 1 + dd) & 7];
#pragma unroll
                for (int u = 0; u < 4; ++u)
                    if ((u32)(u * 64 + lane) < nch) __builtin_amdgcn_raw_buffer_store_b128(cs[u], mk_rsrc(w0, WS_USED), 512u + u * 1024u + (u32)lane * 16u, moff, AUX_SYS);
                WAIT_VM0();
                if (lane == 0) __hip_atomic_store((gu64*)(w0 + foff), flag, RLX_SYS);
            }
        }
    } else {
        __builtin_amdgcn_s_barrier();
    }
    STAMP(8);
    {
        const u32 ooff = (u32)((rank >> 2) * 2048 + ti * 128 + wave * 32 + (lane & 31)) * 4096u
                       + (u32)((rank & 3) * 1024 + tj * 128 + (lane >> 5) * 16);
        __amdgpu_buffer_rsrc_t ro = mk_rsrc(P.out[rank], (u32)MM * NN);
#pragma unroll
        for (int bb = 0; bb < 4; ++bb)
            __builtin_amdgcn_raw_buffer_store_b128(dead ? u32x4{0x7f7f7f7fu, 0x7f7f7f7fu, 0x7f7f7f7fu, 0x7f7f7f7fu} : q[bb], ro, ooff + bb * 32, 0, 0);
    }
    STAMP(9);
}

extern "C" void kernel_launch(int rank, int nranks, ncclComm_t comm,
                              hipStream_t stream,
                              void* const* d_in, const int* in_sizes, int n_in,
                              void* d_out, int out_size,
                              const int* in_dtypes, int out_dtype,
                              void* const* d_out_all,
                              void* const* d_ws_all, size_t ws_bytes) {
    (void)comm; (void)in_sizes; (void)in_dtypes; (void)out_dtype; (void)d_out;
    if (nranks != NR || n_in < 2 || out_size != MM * NN || ws_bytes < WS_USED || (unsigned)rank >= (unsigned)NR) {
        fprintf(stderr, "unexpected problem shape\n");
        return;
    }
    Params P;
    P.x = (const char*)d_in[0];
    P.w = (const char*)d_in[1];
    for (int p = 0; p < NR; ++p) { P.ws[p] = (char*)d_ws_all[p]; P.out[p] = (char*)d_out_all[p]; }
    P.rank = rank;
    P.pad = 0;
    fused_kernel<<<dim3(NGRID), dim3(NTHR), 0, stream>>>(P);
    HIP_CHECK(hipGetLastError());
}
```

```cpp
#include <hip/hip_runtime.h>
#include <rccl/rccl.h>
#include <cstdio>
#include <type_traits>

#define HIP_CHECK(expr) do { hipError_t _e = (expr); if (_e != hipSuccess) { \
    fprintf(stderr, "HIP error %s at %s:%d\n", hipGetErrorString(_e), __FILE__, __LINE__); \
    return; } } while (0)

typedef unsigned int u32;
typedef unsigned long long u64;
typedef u32 u32x8 __attribute__((ext_vector_type(8)));
typedef u32 u32x4 __attribute__((ext_vector_type(4)));
typedef u32 u32x3 __attribute__((ext_vector_type(3)));
typedef u32 u32x2 __attribute__((ext_vector_type(2)));
typedef unsigned short u16x2 __attribute__((ext_vector_type(2)));
typedef float f32x16 __attribute__((ext_vector_type(16)));
typedef __bf16 bf16x8 __attribute__((ext_vector_type(8)));
typedef __bf16 bf16x2 __attribute__((ext_vector_type(2)));
#define GAS __attribute__((address_space(1)))
#define LAS __attribute__((address_space(3)))
typedef GAS u32 gu32;
typedef GAS u64 gu64;
typedef GAS u32x4 gu32x4;
typedef GAS u32x2 gu32x2;

constexpr int NR = 8;
constexpr int MM = 4096, NN = 4096;
constexpr int NGRID = 256, NTHR = 768;
constexpr u64 DEADLINE_TICKS = 50000000ull;

constexpr u32 OFF_CNT   = 0;
constexpr u32 OFF_TMO   = 32768;
constexpr u32 OFF_TICK  = 33024;
constexpr u32 OFF_DUMMY = 34816;
constexpr u32 OFF_AMAXL = 36864;
constexpr u32 OFF_AMAXG = 40960;
#define NREP_ 32
constexpr int NREP      = NREP_;
constexpr u32 OFF_FLAGA = 65536;
constexpr u32 OFF_FLAGB = 98304;
constexpr u32 OFF_FLAGO = 131072;
constexpr u32 OFF_DIAG  = 262144;
constexpr u32 PIECE     = 16384;
constexpr u32 OFF_A     = 1u << 20;
constexpr u32 OFF_B     = OFF_A + 64u * 16u * PIECE;
constexpr u32 MSG       = 4096;
constexpr u32 OFF_MSG   = OFF_B + 64u * 8u * PIECE;
constexpr u32 WS_USED   = OFF_MSG + 8u * 512u * MSG;

constexpr int LDS_W    = 0;
constexpr int LDS_STR  = 32768;
constexpr int LDS_MSK  = 65536;
constexpr int LDS_LUT  = 67584;
constexpr int LDS_RED  = 67712;
constexpr int LDS_TAB  = 69632;
constexpr int LDS_HX   = 98304;
constexpr int LDS_SIZE = 114688;

struct Params {
    const char* x;
    const char* w;
    char* myws;
    char* out;
    int rank;
    int pad;
    char* wsr[NR];
};

#define RLX_SYS __ATOMIC_RELAXED, __HIP_MEMORY_SCOPE_SYSTEM
#define RLX_AGT __ATOMIC_RELAXED, __HIP_MEMORY_SCOPE_AGENT
#define WAIT_VM0() asm volatile("s_waitcnt vmcnt(0)" ::: "memory")
#define CBAR() asm volatile("" ::: "memory")
#define WAIT_LGKM0() do { CBAR(); __builtin_amdgcn_s_waitcnt(0xC07F); CBAR(); } while (0)
#define LDS_BARRIER() do { WAIT_LGKM0(); __builtin_amdgcn_s_barrier(); CBAR(); } while (0)
constexpr int AUX_SYS = 17;
#define NWL 32
#define TRACE_ON 0
#define THINK 0
#define SWIN 1
#define PACE 0x7f
#define PROT 0
#define TOUCH 2
#define TSLEEP 64
#define WDELAY 64
#define WFAST 0
#define MEARLY 5
#define MWIN 2
#define AUX_OUT 0
constexpr int NBLK = 768;
constexpr u32 SEQ_SH = 17, SEQ_MASK = 0x7fffu;

__device__ __forceinline__ u64 now() { return __builtin_amdgcn_s_memrealtime(); }
__device__ __forceinline__ u32 rfl(u32 v) { return (u32)__builtin_amdgcn_readfirstlane((int)v); }
__device__ __forceinline__ u32 rdl(u32 v, int l) { return (u32)__builtin_amdgcn_readlane((int)v, l); }
__device__ __forceinline__ u32 perm(u32 a, u32 b, u32 s) { return __builtin_amdgcn_perm(a, b, s); }
__device__ __forceinline__ u32 bfi(u32 m, u32 a, u32 b) { return (a & m) | (b & ~m); }
__device__ __forceinline__ __amdgpu_buffer_rsrc_t mk_rsrc(const char* base, u32 bytes) {
    return __builtin_amdgcn_make_buffer_rsrc((void*)base, 0, bytes, 0x00020000);
}
template <int CTRL, int RMASK> __device__ __forceinline__ u32 dpp0(u32 v) {
    return (u32)__builtin_amdgcn_update_dpp(0, (int)v, CTRL, RMASK, 0xf, false);
}
__device__ __forceinline__ u32 wave_max(u32 v) {
    v = max(v, dpp0<0x111, 0xf>(v)); v = max(v, dpp0<0x112, 0xf>(v));
    v = max(v, dpp0<0x114, 0xf>(v)); v = max(v, dpp0<0x118, 0xf>(v));
    v = max(v, dpp0<0x142, 0xa>(v)); v = max(v, dpp0<0x143, 0xc>(v));
    return rdl(v, 63);
}
__device__ __forceinline__ u32 wave_scan(u32 v) {
    v += dpp0<0x111, 0xf>(v); v += dpp0<0x112, 0xf>(v);
    v += dpp0<0x114, 0xf>(v); v += dpp0<0x118, 0xf>(v);
    v += dpp0<0x142, 0xa>(v); v += dpp0<0x143, 0xc>(v);
    return v;
}

#define STAMP(i) do { } while (0)
#define TRACE(s) do { } while (0)

__device__ __forceinline__ bool spin_fail(u64 t0, gu32* tmo, u32 call, u32& spins) {
    __builtin_amdgcn_s_sleep(1);
    ++spins;
    if ((spins & 63u) == 0u) {
        if (rfl(__hip_atomic_load(tmo, RLX_AGT)) == call) return true;
        if (now() - t0 > DEADLINE_TICKS) { __hip_atomic_store(tmo, call, RLX_AGT); return true; }
    }
    return false;
}

__device__ __forceinline__ u32 block_id(u32 nb) {
    const u32 n = nb >> 2, m = n - 8u, g = (m * 2731u) >> 16, q = m - g * 24u, t = (q * 11u) >> 5;
    u32 ce = q == 3u * t ? (g + 1u) * 32u + t : g * 32u + 7u + q - t;
    if (n < 8u) ce = n;
    if (n >= 176u) ce = 7u * 32u + n - 168u;
    return nb < (u32)NBLK ? (ce << 2) | (nb & 3u) : 0xffffffffu;
}
constexpr u32 M83 = 0x83838383u;
template <bool ISW> __device__ __forceinline__ void patch(u32 img, unsigned short ex, u64 hi, u32 vmask) {
    u32 ad, val, sh;
    asm volatile("v_lshrrev_b32 %0, %7, %3\n\tv_bfe_i32 %1, %3, %8, 5\n\tv_lshl_or_b32 %2, %3, %9, %10\n\t"
                 "v_and_or_b32 %0, %0, %6, %4\n\tv_lshlrev_b32 %1, %2, %1\n\t"
                 "s_mov_b64 exec, %5\n\tds_add_u32 %0, %1\n\ts_mov_b64 exec, -1"
                 : "=&v"(ad), "=&v"(val), "=&v"(sh)
                 : "v"(__builtin_bit_cast(_Float16, ex)), "s"(img), "s"(hi), "v"(vmask), "n"(ISW ? 4 : 5), "n"(ISW ? 1 : 2), "n"(ISW ? 4 : 3), "n"(ISW ? 10 : 2) : "memory");
}
__device__ __forceinline__ void raw_planes(u32 (&H)[8], const char* p) {
    u32x4 a, b;
    asm volatile("global_load_dwordx4 %0, %2, off offset:2048 sc0 sc1\n\tglobal_load_dwordx4 %1, %2, off offset:3072 sc0 sc1\n\ts_waitcnt vmcnt(0)"
                 : "=&v"(a), "=&v"(b) : "v"(p) : "memory");
    asm volatile("v_mov_b32 %0, %8\n\tv_mov_b32 %1, %9\n\tv_mov_b32 %2, %10\n\tv_mov_b32 %3, %11\n\t"
                 "v_mov_b32 %4, %12\n\tv_mov_b32 %5, %13\n\tv_mov_b32 %6, %14\n\tv_mov_b32 %7, %15"
                 : "+v"(H[0]), "+v"(H[1]), "+v"(H[2]), "+v"(H[3]), "+v"(H[4]), "+v"(H[5]), "+v"(H[6]), "+v"(H[7])
                 : "v"(a.x), "v"(a.y), "v"(a.z), "v"(a.w), "v"(b.x), "v"(b.y), "v"(b.z), "v"(b.w));
}
template <bool ISW> __device__ __forceinline__ void patch_more(u32 img, const char* p, u32 vmask) {
    unsigned short ex;
    asm volatile("global_load_ushort %0, %1, off offset:2944 sc0 sc1\n\ts_waitcnt vmcnt(0)" : "=&v"(ex) : "v"(p) : "memory");
    patch<ISW>(img, ex, ~0ull, vmask);
}

template <int K> __device__ __forceinline__ void mark_msg(u32 foff, u32x2 fv, char* dst) {
    constexpr int last = K + SWIN - 1 < 6 ? K + SWIN - 1 : 6;
    constexpr int f0 = K - SWIN + 1 > 0 ? K - SWIN + 1 : 0;
    constexpr int N = 2 * (last - K) + (K - f0);
    asm volatile("s_waitcnt vmcnt(%3)\n\ts_mov_b64 exec, 1\n\tglobal_store_dwordx2 %0, %1, %2 sc0 sc1\n\t"
                 "s_mov_b64 exec, -1\n\ts_nop 1"
                 :: "v"(foff), "v"(fv), "s"(dst), "n"(N) : "memory");
}

__global__ void __launch_bounds__(NTHR, 3) fused_kernel(Params P) {
    __shared__ __attribute__((aligned(32768))) char smem[LDS_SIZE];
    LAS char* lds = (LAS char*)smem;
    const int tid = threadIdx.x, lane = tid & 63, wave = (int)rfl((u32)tid >> 6);
    const int b = blockIdx.x, xcd = b & 7, slot = b >> 3;
    const int rank = P.rank;
    char* myws = P.myws;
    asm volatile("" :: "s"(P.x), "s"(P.w), "s"(P.wsr[slot & 7]));
    gu32* tmo = (gu32*)(myws + OFF_TMO);
    const u64 tstart = now();
    STAMP(0);
    auto get_call = [&]() __attribute__((always_inline)) {
        u32 ticket = 0;
        if (lane == 0) ticket = __hip_atomic_fetch_add((gu32*)(myws + OFF_CNT + b * 128 + wave * 8), 1u, RLX_AGT);
        return rfl(ticket) + 1u;
    };

    if (__builtin_expect(slot < 16, 1)) {
        if (wave >= 8) return;
        const int d = slot & 7;
        const int sw = (slot >> 3) * 8 + xcd;
        const int wl = wave * 16 + sw;
        const bool producer = wl < NWL;
        const int p = (rank + 1 + d) & 7;
        const int r = (rank - p) & 7;
        char* dws = P.wsr[d];
        __amdgpu_buffer_rsrc_t rs = mk_rsrc(dws, WS_USED);
        const int row0 = (p >> 2) * 2048, col0 = (p & 3) * 1024;
        const u32 xlane = (u32)(lane & 31) * 1024u + (u32)(lane >> 5) * 16u;
        const u32 wlane = (u32)((lane >> 5) * 8) * 8192u + (u32)(lane & 31) * 8u;
        u32 v[16];
        u32x2 vn[8];
        auto load_block = [&](int nb) __attribute__((always_inline)) {
            const int wb = nb & 3, c = nb >> 7, e = (nb >> 2) & 31;
            const bool isw = e < 8;
            const u32 so = isw ? (u32)((c * 64 + wb * 16) * 8192 + (col0 + ((e + r) & 7) * 128) * 2)
                               : (u32)((row0 + ((e - 8 + 2 * r) & 15) * 128 + wb * 32) * 1024 + c * 128);
            const u32 s1 = isw ? 8192u : 8u, s2 = isw ? 16384u : 32u;
            __amdgpu_buffer_rsrc_t ri = mk_rsrc(isw ? P.w : P.x, 4096u * 512u * 2u);
            const u32 vo = isw ? wlane : xlane;
#pragma unroll
            for (int k = 0; k < 8; ++k) vn[k] = __builtin_amdgcn_raw_buffer_load_b64(ri, vo, so + (k & 1) * s1 + (k >> 1) * s2, 0);
        };
        auto take_block = [&](u32 nb) __attribute__((always_inline)) {
            const bool isw = (nb & 0x60u) == 0u;
#pragma unroll
            for (int k = 0; k < 4; ++k) {
                v[4 * k] = vn[2 * k].x; v[4 * k + 1] = isw ? vn[2 * k + 1].x : vn[2 * k].y;
                v[4 * k + 2] = isw ? vn[2 * k].y : vn[2 * k + 1].x; v[4 * k + 3] = vn[2 * k + 1].y;
            }
#pragma unroll
            for (int i = 0; i < 16; ++i) asm volatile("" : "+v"(v[i]));
        };
        const u32 nA = block_id((u32)wl);
        if (producer) load_block((int)nA);
        CBAR();
        const u32 call = get_call();
        if (wave >= 4) {
            const int wi = (b << 2) | (wave & 3);
            const char* px = P.x + (size_t)(wi * 8 + (lane >> 3)) * 1024 + (lane & 7) * 16;
            const char* pw = P.w + (size_t)wi * 1024 + lane * 16;
            u32x4 t[3];
#pragma unroll 1
            for (int c = 0; c < 8; ++c) {
                t[0] = *(const gu32x4*)(pw + c * 524288);
                t[1] = *(const gu32x4*)(px + c * 128);
                asm volatile("" :: "v"(t[0]), "v"(t[1]));
                if (c >= WFAST) __builtin_amdgcn_s_sleep(WDELAY);
            }
            __amdgpu_buffer_rsrc_t rt = mk_rsrc(myws, WS_USED);
            const int mwt = (slot * 8 + xcd) * 4 + (wave & 3);
#pragma unroll 1
            for (int k = 6; k > 6 - TOUCH; --k) {
                for (int i = 0; i < 1 + (wi & 3); ++i) __builtin_amdgcn_s_sleep(TSLEEP);
                int dd = k + mwt % 7; if (dd >= 7) dd -= 7;
                const u32 off = OFF_MSG + (u32)(((rank - 1 - dd) & 7) * 512 + mwt) * MSG;
                t[0] = __builtin_amdgcn_raw_buffer_load_b128(rt, (u32)lane * 16u, off, AUX_SYS);
                t[1] = __builtin_amdgcn_raw_buffer_load_b128(rt, (u32)lane * 16u + 1024u, off, AUX_SYS);
                t[2] = __builtin_amdgcn_raw_buffer_load_b128(rt, (u32)(lane & 31) * 16u + 2048u, off, AUX_SYS);
#pragma unroll
                for (int u = 0; u < 3; ++u) asm volatile("" :: "v"(t[u]));
            }
        }
        if (tid < 16) {
            u32 sel = 0, k = 0;
            for (int j = 0; j < 4; ++j) { const u32 on = (tid >> j) & 1; sel |= (on ? k : 0x0cu) << (8 * j); k += on; }
            *(LAS u32*)(lds + LDS_LUT + tid * 4) = sel;
        }
        gu32* tick = (gu32*)(myws + OFF_TICK + d * 128);
        const u32 tbase = (call - 1u) * (u32)(NBLK - 2 * NWL) - 3u * NWL;
        u32x4 pk[4];
        LAS char* stg = lds + LDS_TAB + (wave & 3) * 3072;
        u32 p_off = 0, p_foff = 0, p_mode = 0, p_lanes = 0;
        u32 p_flag = 0;
        auto pack = [&](int nb, bool early) __attribute__((always_inline)) {
            const int wb = nb & 3, c = nb >> 7, e = (nb >> 2) & 31;
            const int step = rank * 8 + c;
            u32 selL, selH;
            if (e < 8) { const int j = (e + r) & 7;
                p_off = OFF_B + (u32)(step * 8 + j) * PIECE;  p_foff = OFF_FLAGB + (u32)((j * 4 + wb) * 64 + step) * 4u;
                selL = 0x06020400u; selH = 0x07030501u;
            } else { const int i = (e - 8 + 2 * r) & 15;
                p_off = OFF_A + (u32)(step * 16 + i) * PIECE; p_foff = OFF_FLAGA + (u32)((i * 4 + wb) * 64 + step) * 4u;
                selL = 0x06040200u; selH = 0x07050301u;
            }
            p_off += (u32)wb * 4096u;
            u32 H[8], D[8];
            u16x2 mx = {0, 0}, mn = {0xffff, 0xffff};
#pragma unroll
            for (int i = 0; i < 16; ++i) {
                const u16x2 a = __builtin_bit_cast(u16x2, v[i] & 0x7fff7fffu);
                mx = __builtin_elementwise_max(mx, a); mn = __builtin_elementwise_min(mn, a);
            }
            u32 mag = max(mx.x, mx.y);
            if (__builtin_expect(__ballot(mag >= 0x7f00u || min(mn.x, mn.y) < 0x100u) == 0ull, 1)) {
                const u32 inc = e < 8 ? 0xff7fff80u : 0x00800080u;
#pragma unroll
                for (int i = 0; i < 16; ++i) v[i] += inc;
                mag += e < 8 ? 0xffffff80u : 0x80u;
            } else {
                const float f = e < 8 ? 0.5f : 2.0f;
                mag = 0;
#pragma unroll
                for (int i = 0; i < 16; ++i) {
                    const bf16x2 t = {(__bf16)(__uint_as_float(v[i] << 16) * f), (__bf16)(__uint_as_float(v[i] & 0xffff0000u) * f)};
                    v[i] = __builtin_bit_cast(u32, t);
                    mag = max(mag, max(v[i] & 0x7fffu, (v[i] >> 16) & 0x7fffu));
                }
            }
#pragma unroll
            for (int i = 0; i < 8; ++i) { pk[i >> 2][i & 3] = perm(v[2 * i + 1], v[2 * i], selL); H[i] = perm(v[2 * i + 1], v[2 * i], selH); }
            if (early) {
                STAMP(7);
                __builtin_amdgcn_raw_buffer_store_b128(pk[0], rs, (u32)lane * 16u, p_off, AUX_SYS);
                __builtin_amdgcn_raw_buffer_store_b128(pk[1], rs, (u32)lane * 16u + 1024u, p_off, AUX_SYS);
            }
            const u32 top = wave_max(mag >> 8);
            u32 base = top > 3u ? top - 3u : 0u;
            if (base > 0u && __popcll(__ballot((mag >> 8) == top)) <= 8) --base;
            const u32 base4 = base * 0x01010101u;
            u32 pm = 0;
#pragma unroll
            for (int i = 0; i < 8; ++i) {
                D[i] = (H[i] | 0x80808080u) - base4;
                const u32 n128 = __builtin_amdgcn_udot4((((D[i] & 0x7c7c7c7cu) + 0x7c7c7c7cu) | ~D[i]) & 0x80808080u, 0x08040201u, 0u, false);
                pm |= 4 * i < 7 ? n128 >> (7 - 4 * i) : n128 << (4 * i - 7);
            }
            *(LAS u32x4*)(stg + 1024 + lane * 16) = u32x4{D[0], D[1], D[2], D[3]};
            *(LAS u32x4*)(stg + 2048 + lane * 16) = u32x4{D[4], D[5], D[6], D[7]};
            *(LAS u32*)(stg + 768 + lane * 4) = (u32)lane * 0x01000100u + 0x00800000u;
            u32 deep = 0;
            const u32 np = (u32)__popc(pm), npi = wave_scan(np), total = rdl(npi, 63);
            u32 mode = total > 64u ? 2u : 0u;
            p_lanes = total > 64u ? 64u : total > 32u ? 56u : 52u;
            if (__builtin_expect(total <= 128u, 1)) {
                u32 E[8];
#pragma unroll
                for (int i = 0; i < 8; ++i) E[i] = bfi(0x80808080u, H[i], D[i]) & M83;
                LAS u32* cw = (LAS u32*)(stg + lane * 12);
                cw[0] = E[0] | __builtin_amdgcn_alignbit(E[1], E[1], 29) | ((E[6] & 0x03030303u) << 5);
                cw[1] = E[2] | __builtin_amdgcn_alignbit(E[3], E[3], 29) | ((E[7] & 0x03030303u) << 5);
                cw[2] = E[4] | __builtin_amdgcn_alignbit(E[5], E[5], 29) | ((E[6] & 0x80808080u) >> 1) | ((E[7] & 0x80808080u) >> 2);
                LAS unsigned short* lp = (LAS unsigned short*)(stg + 768) + (npi - np);
                while (__ballot(pm != 0u) != 0ull) {
                    if (pm != 0u) {
                        const u32 pb = (u32)__ffs((int)pm) - 1u;
                        pm &= pm - 1u;
                        const u32 ad = ((pb >> 4) << 10) + (u32)lane * 16u + (pb & 15u);
                        const u32 d = *(LAS unsigned char*)(stg + 1024 + ad);
                        const u32 d5 = (d >> 2) & 31u;
                        deep |= d < 64u;
                        const u32 ew = ((((pb >> 1) & 3u) << 8 | (u32)lane << 2 | pb >> 3) << 6) | (d5 << 1) | (pb & 1u);
                        *lp++ = (unsigned short)(e < 8 ? ew : ((ad >> 2) << 7) | (d5 << 2) | (pb & 3u));
                    }
                }
                CBAR();
                pk[2] = *(LAS u32x4*)(stg + lane * 16);
            }
            if (__builtin_expect(total > 128u || __ballot(deep != 0u) != 0ull, 0)) {
                mode = 3; p_lanes = 64u;
                pk[2] = u32x4{H[0], H[1], H[2], H[3]};
            }
            pk[3] = u32x4{H[4], H[5], H[6], H[7]};
            p_mode = mode;
            p_flag = (call << SEQ_SH) | (mode << 15) | (base << 8) | (total > 32u && mode != 3u ? 1u : 0u);
        };
        if (producer) {
            u32 nB = block_id((u32)(wl + NWL)), nC = block_id((u32)(wl + 2 * NWL));
            take_block(nA);
            load_block((int)nB);
            pack((int)nA, true);
            const u32 vo = (u32)lane * 16u;
            char* fdst = nullptr;
            u32 flag = 0;
            const u32 zero = 0, one = 1;
            auto iter = [&](bool first) __attribute__((always_inline)) {
                const bool more = nB != 0xffffffffu, more2 = nC != 0xffffffffu;
                u32 tk = 0;
                if (more2) asm volatile("s_mov_b64 exec, 1\n\tglobal_atomic_add %0, %1, %2, %3 sc0\n\ts_mov_b64 exec, -1"
                                        : "=&v"(tk) : "v"(zero), "v"(one), "s"(tick) : "memory");
                if (!first && lane == 0) __hip_atomic_store((gu32*)fdst, flag, RLX_SYS);
                CBAR();
                if (!first) {
                    __builtin_amdgcn_raw_buffer_store_b128(pk[0], rs, vo, p_off, AUX_SYS);
                    __builtin_amdgcn_raw_buffer_store_b128(pk[1], rs, vo + 1024, p_off, AUX_SYS);
                }
                __builtin_amdgcn_raw_buffer_store_b128(pk[2], rs, (u32)lane >= p_lanes ? 0x80000000u : vo + 2048, p_off, AUX_SYS);
                if (p_mode == 3) __builtin_amdgcn_raw_buffer_store_b128(pk[3], rs, vo + 3072, p_off, AUX_SYS);
                CBAR();
                fdst = dws + p_foff;
                flag = p_flag;
                take_block(nB);
                if (more2) load_block((int)nC);
                CBAR();
                if (more) pack((int)nB, false);
                __builtin_amdgcn_s_waitcnt(0x0F70);
                asm volatile("s_waitcnt vmcnt(0)" : "+v"(tk) :: "memory");
                nB = nC;
                if (more2) nC = block_id(rfl(tk) - tbase);
                return more;
            };
            iter(true);
#pragma unroll 1
            while (iter(false)) { }
            if (lane == 0) __hip_atomic_store((gu32*)fdst, flag, RLX_SYS);
        }
        STAMP(1);
        LDS_BARRIER();
        bool dead = false;
        if (b == 56 && wave == 0) {
            u32 spins = 0, m = 0;
            for (;;) {
                const u64 g0 = __hip_atomic_load((gu64*)(myws + OFF_AMAXL) + lane, RLX_AGT);
                const u64 g1 = __hip_atomic_load((gu64*)(myws + OFF_AMAXL) + 64 + lane, RLX_AGT);
                m = max((u32)g0, (u32)g1);
                if (__all((u32)(g0 >> 32) == call && (u32)(g1 >> 32) == call)) break;
                if (spin_fail(tstart, tmo, call, spins)) { dead = true; break; }
            }
            if (!dead) {
                m = wave_max(m);
                char* dst = P.wsr[lane & 7] + OFF_AMAXG + (lane >> 3) * 128 + rank * 8;
#pragma unroll
                for (int k = 0; k < NREP / 8; ++k)
                    __hip_atomic_store((gu64*)(dst + k * 1024), ((u64)call << 32) | m, RLX_SYS);
            }
            STAMP(2);
        }
        const int tile = slot * 8 + xcd, ti = tile >> 3, tj = tile & 7;
        const int wq = wave & 3, hf = wave >> 2;
        const int mw = tile * 4 + wq, rot = mw % 7;
        __amdgpu_buffer_rsrc_t rm = mk_rsrc(myws, WS_USED);
        __amdgpu_buffer_rsrc_t ro = mk_rsrc(P.out, (u32)MM * NN);
        gu64* fo = (gu64*)(myws + OFF_FLAGO + mw * 128) + (lane & 7);
        LAS char* str = lds + LDS_STR + wave * 4096;
        LAS char* ot = lds + LDS_W + wave * 2048;
        u64 fl = 0;
#pragma unroll 1
        for (int k = 0; k < 7; ++k) {
            int dd = k + rot; if (dd >= 7) dd -= 7;
            const int q = (rank - 1 - dd) & 7;
            if (!dead && rdl((u32)(fl >> 32), q) != call) {
                u32 spins = 0;
                for (;;) {
                    fl = __hip_atomic_load(fo, RLX_SYS);
                    if (rdl((u32)(fl >> 32), q) == call) break;
                    if (spin_fail(tstart, tmo, call, spins)) { dead = true; break; }
                }
            }
            CBAR();
            const u32 ooff = (u32)((q >> 2) * 2048 + ti * 128 + wq * 32 + (lane >> 2)) * 4096u
                           + (u32)((q & 3) * 1024 + tj * 128 + hf * 64 + (lane & 3) * 16);
            if (dead) {
#pragma unroll
                for (int bb = 0; bb < 2; ++bb)
                    __builtin_amdgcn_raw_buffer_store_b128(u32x4{0x7f7f7f7fu, 0x7f7f7f7fu, 0x7f7f7f7fu, 0x7f7f7f7fu}, ro, ooff + bb * 65536u, 0, 0);
                continue;
            }
            const u32 fw = rdl((u32)fl, q);
            const u32 nch = fw & 255u;
            const u32 moff = OFF_MSG + (u32)(q * 512 + mw) * MSG;
            const u32x2 mk = __builtin_amdgcn_raw_buffer_load_b64(rm, (u32)lane * 8u, moff, AUX_SYS);
            const u32x4 c0 = __builtin_amdgcn_raw_buffer_load_b128(rm, 512u + (u32)lane * 16u, moff, AUX_SYS);
            const u32x4 c1 = __builtin_amdgcn_raw_buffer_load_b128(rm, 1536u + (u32)lane * 16u, moff, AUX_SYS);
            const u32 c_lo = (u32)__popc(mk.x), cnt = c_lo + (u32)__popc(mk.y);
            const u32 first = wave_scan(cnt) - (hf ? cnt - c_lo : cnt);
            const u32 mm = hf ? mk.y : mk.x;
            *(LAS u32x4*)(str + lane * 16) = c0;
            *(LAS u32x4*)(str + 1024 + lane * 16) = c1;
            if (nch > 128u) {
                const u32x4 c2 = __builtin_amdgcn_raw_buffer_load_b128(rm, 2560u + (u32)lane * 16u, moff, AUX_SYS);
                *(LAS u32x4*)(str + 2048 + lane * 16) = c2;
                if (lane < 32) {
                    const u32x4 c3 = __builtin_amdgcn_raw_buffer_load_b128(rm, 3584u + (u32)lane * 16u, moff, AUX_SYS);
                    *(LAS u32x4*)(str + 3072 + lane * 16) = c3;
                }
            }
            WAIT_LGKM0();
            u32x4 o[2];
            u32 ex = 0;
            if (__builtin_expect((fw & (1u << 27)) == 0u, 1)) {
                ex = *(LAS unsigned short*)(str + ((fw >> 8) & 255u) * 16u + lane * 2);
                const u32 b8 = ((fw >> 23) & 15u) * 0x08080808u;
                u32 pos = first * 6u;
#pragma unroll
                for (int i = 0; i < 8; ++i) {
                    const int sh = 4 * i;
                    const u32 n4 = (sh ? (mm >> (sh - 2)) : (mm << 2)) & 0x3cu;
                    const u32 sel = *(LAS u32*)(lds + LDS_LUT + n4);
                    const LAS u32* wp = (const LAS u32*)(str + ((pos >> 5) << 2));
                    const u32 z = __builtin_amdgcn_alignbit(wp[1], wp[0], pos);
                    pos += 6u * (u32)__popc(n4);
                    const u32 y = bfi(0x00000fffu, z, z << 4);
                    const u32 x = (bfi(0x003f003fu, y, y << 2) & 0x3f3f3f3fu) + b8;
                    o[i >> 2][i & 3] = perm(x, x, sel);
                }
            } else {
                u32 pos = first * 7u;
#pragma unroll
                for (int i = 0; i < 8; ++i) {
                    const int sh = 4 * i;
                    const u32 n4 = (sh ? (mm >> (sh - 2)) : (mm << 2)) & 0x3cu;
                    const u32 sel = *(LAS u32*)(lds + LDS_LUT + n4);
                    const LAS u32* wp = (const LAS u32*)(str + ((pos >> 5) << 2));
                    const u32 z = __builtin_amdgcn_alignbit(wp[1], wp[0], pos);
                    pos += 7u * (u32)__popc(n4);
                    const u32 y = bfi(0x00003fffu, z, z << 2);
                    const u32 x = bfi(0x007f007fu, y, y << 1) & 0x7f7f7f7fu;
                    o[i >> 2][i & 3] = perm(x, x, sel);
                }
            }
            *(LAS u32x4*)(ot + (lane & 31) * 64 + (lane >> 5) * 16) = o[0];
            *(LAS u32x4*)(ot + (lane & 31) * 64 + (lane >> 5) * 16 + 32) = o[1];
            if ((u32)lane < ((fw >> 16) & 127u) && ((ex >> 6) & 1u) == (u32)hf)
                __hip_atomic_fetch_xor((LAS u32*)(ot + ((ex >> 7) & 31u) * 64u + (ex & 60u)), 0x40u << (8u * (ex & 3u)), __ATOMIC_RELAXED, __HIP_MEMORY_SCOPE_WAVEFRONT);
            CBAR();
#pragma unroll
            for (int bb = 0; bb < 2; ++bb)
                __builtin_amdgcn_raw_buffer_store_b128(*(LAS u32x4*)(ot + bb * 1024 + lane * 16), ro, ooff + bb * 65536u, 0, AUX_OUT);
        }
        STAMP(6);
        return;
    }

    const int sl = slot - 16;
    const int ti = (xcd >> 1) * 4 + (sl >> 2), tj = (xcd & 1) * 4 + (sl & 3), tile = ti * 8 + tj;
    const bool helper = wave >= 4;
    const int wq = wave & 3;
    if (tid < 16) {
        u32 sel = 0x0c0c0c0cu, k = 0;
        for (int j = 0; j < 4; ++j) if ((tid >> j) & 1) { sel = (sel & ~(0xffu << (8 * k))) | ((u32)j << (8 * k)); ++k; }
        *(LAS u32*)(lds + LDS_LUT + tid * 4) = sel;
    }
    if (tid < 8) *(LAS u32*)(lds + LDS_RED + tid * 4) = 0;
    LAS char* str = lds + LDS_STR + wq * 4096;
    if (!helper) {
#pragma unroll
        for (int u = 0; u < 4; ++u) *(LAS u32x4*)(str + u * 1024 + lane * 16) = u32x4{0, 0, 0, 0};
    }
    u32 order = (u32)rank;
    for (int k = 0; k < 7; ++k) order |= (u32)((rank + ((ti >> 1) + 13 - k) % 7 + 1) & 7) << (4 * k + 4);
    order = rfl(order);
    auto step_id = [&](u32 s) __attribute__((always_inline)) { return ((order >> ((s & 7u) * 4u)) & 7u) * 8u + (s >> 3); };
    __amdgpu_buffer_rsrc_t rs = mk_rsrc(myws, WS_USED);
    const u32 call = get_call();
    const u32 seq = call & SEQ_MASK;
    bool dead = false;
    struct Flags { gu32* p; u32 v; u64 ready; };
    Flags fx{(gu32*)(myws + OFF_FLAGA + (u32)((ti * 4 + wq) * 64) * 4u) + step_id((u32)lane), 0, 0};
    const u32 xblk = OFF_A + (u32)ti * PIECE + (u32)wq * 4096u;
    auto xoff = [&](int s) __attribute__((always_inline)) { return step_id((u32)s) * (16u * PIECE); };
    auto wait_ready = [&](Flags& f, int s) __attribute__((always_inline)) {
        if (__builtin_expect(!((f.ready >> s) & 1), 0)) {
            u32 spins = 0;
            for (;;) {
                f.v = __hip_atomic_load(f.p, RLX_SYS);
                f.ready = __ballot((f.v >> SEQ_SH) == seq);
                if ((f.ready >> s) & 1) break;
                if (spin_fail(tstart, tmo, call, spins)) { dead = true; f.v = 0; f.ready = ~0ull; break; }
            }
        }
        CBAR();
    };
#define SB() __builtin_amdgcn_sched_barrier(0)
#define KBAR() LDS_BARRIER()
#define KOUT() do { } while (0)
    using T = std::true_type; using F = std::false_type;
    using C0 = std::integral_constant<int, 0>; using C1 = std::integral_constant<int, 1>;
    if (helper) {
        Flags fw{(gu32*)(myws + OFF_FLAGB + (u32)((tj * 4 + wq) * 64) * 4u) + step_id((u32)lane), 0, 0};
        const u32 wblk = OFF_B + (u32)tj * PIECE + (u32)wq * 4096u;
        auto woff = [&](int s) __attribute__((always_inline)) { return step_id((u32)s) * (8u * PIECE); };
        auto lanes = [&](u32 f) __attribute__((always_inline)) { return ((u64)((f & 1u) ? 0xffffffffu : 0u) << 32) | ((f & 0x8000u) ? 0u : 0xffffffffu); };
        const u32 wl_ = wblk + (u32)lane * 16u, wc_ = wblk + 2048u + (u32)lane * 12u, we_ = wblk + 2816u + (u32)lane * 2u;
        const u32 xc_ = xblk + 2048u + (u32)lane * 12u, xe_ = xblk + 2816u + (u32)lane * 2u;
        auto planes = [&](u32 fhi, const u32x3 c, u32 blk, u32 (&H)[8]) {
            const u32 base4 = ((fhi >> 8) & 127u) * 0x01010101u;
#pragma unroll
            for (int q = 0; q < 3; ++q) { H[2 * q] = c[q] & M83; H[2 * q + 1] = __builtin_amdgcn_alignbit(c[q], c[q], 3) & M83; }
            H[6] = bfi(0x80808080u, c[2] << 1, c[0] >> 5) & M83;
            H[7] = bfi(0x80808080u, c[2] << 2, c[1] >> 5) & M83;
            SB();
#pragma unroll
            for (int q = 0; q < 8; ++q) H[q] += base4;
            if (__builtin_expect((fhi & 0x8000u) != 0u, 0)) raw_planes(H, myws + (blk + (u32)lane * 16u));
        };
        struct XC { u32x3 c; unsigned short ex; };
        u32 vmask = wave >= 8 ? 0x7fcu : 0xffcu;
        asm volatile("" : "+v"(vmask));
        auto poll_issue = [&](const Flags& f) __attribute__((always_inline)) { return __hip_atomic_load(f.p, RLX_SYS); };
        auto poll_take = [&](Flags& f, u32 v) __attribute__((always_inline)) {
            if (!dead) { f.v = v; f.ready = __ballot((v >> SEQ_SH) == seq); }
        };
        if (wave >= 8) {
            XC x0, x1, x2;
            u32 img = (u32)(uintptr_t)(lds + LDS_HX + wq * 2048);
            auto ldx = [&](int s, XC& x) __attribute__((always_inline)) {
                wait_ready(fx, s);
                const u32 sx = xoff(s);
                x.c = __builtin_amdgcn_raw_buffer_load_b96(rs, xc_, sx, AUX_SYS);
                x.ex = __builtin_amdgcn_raw_buffer_load_b16(rs, xe_, sx, AUX_SYS);
                SB();
            };
            auto unit = [&](int s, XC& x, XC& xn, bool iss) __attribute__((always_inline)) {
                u32 pv = 0;
                const bool need = iss && !((fx.ready >> (s + 2)) & 1);
                if (need) pv = poll_issue(fx);
                SB();
                u32 H[8];
                const u32 fhi = rdl(fx.v, s);
                planes(fhi, x.c, xblk + xoff(s), H);
                *(LAS u32x4*)(uintptr_t)(img + lane * 16) = u32x4{H[0], H[1], H[2], H[3]};
                *(LAS u32x4*)(uintptr_t)(img + 1024 + lane * 16) = u32x4{H[4], H[5], H[6], H[7]};
                patch<false>(img, x.ex, lanes(fhi), vmask);
                if (__builtin_expect((fhi & 0x18000u) == 0x10000u, 0)) patch_more<false>(img, myws + (xblk + (u32)lane * 2u + xoff(s)), vmask);
                img ^= 8192u;
                SB();
                if (need) poll_take(fx, pv);
                if (iss) ldx(s + 2, xn);
            };
            ldx(0, x0); ldx(1, x1);
            LDS_BARRIER();
#pragma unroll 1
            for (int j = 0; j < 60; j += 3) {
                unit(j, x0, x2, true); KBAR();
                unit(j + 1, x1, x0, true); KBAR();
                unit(j + 2, x2, x1, true); KBAR();
            }
            unit(60, x0, x2, true); LDS_BARRIER();
            unit(61, x1, x0, true); LDS_BARRIER();
            unit(62, x2, x1, false); LDS_BARRIER();
            unit(63, x0, x2, false);
            KOUT();
            if (dead && lane == 0) *(LAS u32*)(lds + LDS_RED + 20) = 1;
            LDS_BARRIER();
            return;
        }
        u32x3 cc[2];
        u32x4 la[2], lb[2];
        unsigned short le[2];
        u32 H[8], fhi;
        u32 img = (u32)(uintptr_t)(lds + LDS_W + wq * 4096);
        auto ldc = [&](int s, u32x3& c) __attribute__((always_inline)) {
            wait_ready(fw, s);
            c = __builtin_amdgcn_raw_buffer_load_b96(rs, wc_, woff(s), AUX_SYS);
            SB();
        };
        auto unit = [&](int s, u32x4& l0, u32x4& l1, unsigned short& ex, u32x3& cn, bool nxt, bool issl, bool issc) __attribute__((always_inline)) {
            u32 pv = 0;
            const bool need = issc && ((fw.ready >> (s + 3)) & 7ull) != 7ull && fw.ready != ~0ull;
            if (need) pv = poll_issue(fw);
            SB();
            u32x4 c[4];
            LAS char* pw = (LAS char*)(uintptr_t)(img + lane * 16);
#pragma unroll
            for (int q = 0; q < 4; ++q) {
                const u32 sl_ = (q & 1) ? 0x07030602u : 0x05010400u;
                const int xy = q >> 1;
                c[q] = u32x4{perm(H[xy], l0[xy], sl_), perm(H[2 + xy], l0[2 + xy], sl_),
                             perm(H[4 + xy], l1[xy], sl_), perm(H[6 + xy], l1[2 + xy], sl_)};
                if (q) *(LAS u32x4*)(pw + (q - 1) * 1024) = c[q - 1];
                SB();
            }
            *(LAS u32x4*)(pw + 3 * 1024) = c[3];
            patch<true>(img, ex, lanes(fhi), vmask);
            if (__builtin_expect((fhi & 0x18000u) == 0x10000u, 0)) patch_more<true>(img, myws + (wblk + (u32)lane * 2u + woff(s)), vmask);
            img ^= 16384u;
            SB();
            u32 so = 0;
            if (issl) { so = woff(s + 2); l0 = __builtin_amdgcn_raw_buffer_load_b128(rs, wl_, so, AUX_SYS); }
            SB();
            if (nxt) { fhi = rdl(fw.v, s + 1); planes(fhi, cn, wblk + woff(s + 1), H); }
            SB();
            if (issl) { l1 = __builtin_amdgcn_raw_buffer_load_b128(rs, wl_ + 1024, so, AUX_SYS); ex = __builtin_amdgcn_raw_buffer_load_b16(rs, we_, so, AUX_SYS); }
            if (need) poll_take(fw, pv);
            if (issc) ldc(s + 3, cn);
        };
        LDS_BARRIER();
        ldc(0, cc[0]); ldc(1, cc[1]);
#pragma unroll
        for (int k = 0; k < 2; ++k) {
            la[k] = __builtin_amdgcn_raw_buffer_load_b128(rs, wl_, woff(k), AUX_SYS);
            lb[k] = __builtin_amdgcn_raw_buffer_load_b128(rs, wl_ + 1024, woff(k), AUX_SYS);
            le[k] = __builtin_amdgcn_raw_buffer_load_b16(rs, we_, woff(k), AUX_SYS);
        }
        fhi = rdl(fw.v, 0);
        planes(fhi, cc[0], wblk + woff(0), H);
        ldc(2, cc[0]);
#pragma unroll 1
        for (int j = 0; j < 60; j += 2) {
            unit(j, la[0], lb[0], le[0], cc[1], true, true, true);     KBAR();
            unit(j + 1, la[1], lb[1], le[1], cc[0], true, true, true); KBAR();
        }
        unit(60, la[0], lb[0], le[0], cc[1], true, true, true);    LDS_BARRIER();
        unit(61, la[1], lb[1], le[1], cc[0], true, true, false);   LDS_BARRIER();
        unit(62, la[0], lb[0], le[0], cc[1], true, false, false);  LDS_BARRIER();
        unit(63, la[1], lb[1], le[1], cc[0], false, false, false);
        KOUT();
        if (dead && lane == 0) *(LAS u32*)(lds + LDS_RED + 20) = 1;
        LDS_BARRIER();
        LDS_BARRIER();
        if (wave == 4 && lane == 0) {
            const u32x4 t = *(LAS u32x4*)(lds + LDS_RED);
            __hip_atomic_store((gu64*)(myws + OFF_AMAXL) + tile, ((u64)call << 32) | max(max(t.x, t.y), max(t.z, t.w)), RLX_AGT);
        }
        LDS_BARRIER();
        if (wave != 4) {
#pragma unroll 1
            for (int u = wave - 5; u < 28; u += 3) {
                if (u >= 4 * MEARLY)
                    while (__hip_atomic_load((LAS u64*)(lds + LDS_RED + 24), __ATOMIC_RELAXED, __HIP_MEMORY_SCOPE_WORKGROUP) == 0ull) __builtin_amdgcn_s_sleep(1);
                const int mwu = tile * 4 + (u & 3);
                int dd = (u >> 2) + mwu % 7; if (dd >= 7) dd -= 7;
                if (lane < 32) {
                    const u32x4 mk = *(LAS u32x4*)(lds + LDS_MSK + (u & 3) * 512 + lane * 16);
                    __builtin_amdgcn_raw_buffer_store_b128(mk, mk_rsrc(P.wsr[dd], WS_USED), (u32)lane * 16u,
                                                           OFF_MSG + (u32)(rank * 512 + mwu) * MSG, AUX_SYS);
                }
                asm volatile("s_waitcnt vmcnt(%0)" :: "n"(MWIN - 1) : "memory");
            }
            WAIT_VM0();
            __builtin_amdgcn_s_barrier();
            return;
        }
        u32 gb = 0;
        if (!dead) {
            gu64* g = lane == 8 ? (gu64*)tmo : (gu64*)(myws + OFF_AMAXG + (tile & (NREP - 1)) * 128) + (lane & 7);
            u32 spins = 0;
            for (;;) {
                const u64 a = __hip_atomic_load(g, RLX_SYS);
                gb = (u32)a;
                if ((__ballot((u32)(a >> 32) == call) & 0xffull) == 0xffull) break;
                if (spin_fail(tstart, tmo, call, spins)) { dead = true; break; }
            }
            if (rdl(gb, 8) == call) dead = true;
            gb = wave_max(lane == 8 ? 0u : gb);
        }
        if (*(LAS u32*)(lds + LDS_RED + 20) != 0u) dead = true;
        if (lane == 0) __hip_atomic_store((LAS u64*)(lds + LDS_RED + 24), ((u64)(dead ? 3u : 1u) << 32) | __float_as_uint(448.0f / __uint_as_float(gb)), __ATOMIC_RELEASE, __HIP_MEMORY_SCOPE_WORKGROUP);
        return;
    }
    bf16x8 xf[4], wfA[8], wfB[8];
    f32x16 acc[4];
#pragma unroll
    for (int j = 0; j < 4; ++j)
#pragma unroll
        for (int i = 0; i < 16; ++i) acc[j][i] = 0.f;
    u32x4 l0[2], l1[2], ha, hb;
    const u32 xl_ = xblk + (u32)lane * 16u;
    auto putX = [&](u32 h0, u32 h1, u32 la, u32 lb, bf16x8& xo) __attribute__((always_inline)) {
        u32x4 c = u32x4{perm(h0, la, 0x05010400u), perm(h0, la, 0x07030602u), perm(h1, lb, 0x05010400u), perm(h1, lb, 0x07030602u)};
        asm volatile("" : "+v"(c));
        xo = __builtin_bit_cast(bf16x8, c);
    };
    u32 so_l1 = 0;
    auto step = [&](int s, auto cur_c, auto dec_c, auto iss_c, auto is1_c) __attribute__((always_inline)) {
        constexpr int cur = decltype(cur_c)::value, nxt = cur ^ 1;
        constexpr bool DEC = decltype(dec_c)::value, ISS = decltype(iss_c)::value, IS1 = decltype(is1_c)::value;
        LAS char* pr = lds + LDS_W + cur * 16384 + 8192 + lane * 16;
#pragma unroll
        for (int q = 0; q < 8; ++q) {
            acc[q & 3] = __builtin_amdgcn_mfma_f32_32x32x16_bf16(wfA[q], xf[q >> 2], acc[q & 3], 0, 0, 0);
            SB();
            if (q < 4) { wfB[2 * q] = *(LAS bf16x8*)(pr + (2 * q) * 1024); wfB[2 * q + 1] = *(LAS bf16x8*)(pr + (2 * q + 1) * 1024); }
            if (q == 4) putX(hb.x, hb.y, l1[cur].x, l1[cur].y, xf[2]);
            if (q == 5) putX(hb.z, hb.w, l1[cur].z, l1[cur].w, xf[3]);
            if constexpr (IS1) { if (q == 6) l1[cur] = __builtin_amdgcn_raw_buffer_load_b128(rs, xl_ + 1024, so_l1, AUX_SYS); }
            SB();
        }
        if constexpr (DEC) KBAR(); else WAIT_LGKM0();
        SB();
        pr = lds + LDS_W + nxt * 16384 + lane * 16;
        LAS char* px = lds + LDS_HX + nxt * 8192 + wq * 2048 + lane * 16;
#pragma unroll
        for (int q = 0; q < 8; ++q) {
            acc[q & 3] = __builtin_amdgcn_mfma_f32_32x32x16_bf16(wfB[q], xf[2 + (q >> 2)], acc[q & 3], 0, 0, 0);
            SB();
            if constexpr (DEC) {
                if (q == 0) { ha = *(LAS u32x4*)px; hb = *(LAS u32x4*)(px + 1024); }
                if (q >= 1 && q < 5) { wfA[2 * q - 2] = *(LAS bf16x8*)(pr + (2 * q - 2) * 1024); wfA[2 * q - 1] = *(LAS bf16x8*)(pr + (2 * q - 1) * 1024); }
                if (q == 4) putX(ha.x, ha.y, l0[nxt].x, l0[nxt].y, xf[0]);
                if (q == 5) putX(ha.z, ha.w, l0[nxt].z, l0[nxt].w, xf[1]);
            }
            if constexpr (ISS) {
                if (q == 0) { so_l1 = xoff(s + 3); asm volatile("" : "+s"(so_l1)); }
                if (q == 6) l0[nxt] = __builtin_amdgcn_raw_buffer_load_b128(rs, xl_, so_l1, AUX_SYS);
            }
            if (q == 7) __builtin_amdgcn_s_waitcnt(0xC07F);
            SB();
        }
        TRACE(s);
    };
    LDS_BARRIER();
    l0[0] = __builtin_amdgcn_raw_buffer_load_b128(rs, xl_, xoff(0), AUX_SYS);
    l1[0] = __builtin_amdgcn_raw_buffer_load_b128(rs, xl_ + 1024, xoff(0), AUX_SYS);
    l0[1] = __builtin_amdgcn_raw_buffer_load_b128(rs, xl_, xoff(1), AUX_SYS);
    l1[1] = __builtin_amdgcn_raw_buffer_load_b128(rs, xl_ + 1024, xoff(1), AUX_SYS);
    LDS_BARRIER();
    ha = *(LAS u32x4*)(lds + LDS_HX + wq * 2048 + lane * 16); hb = *(LAS u32x4*)(lds + LDS_HX + wq * 2048 + 1024 + lane * 16);
#pragma unroll
    for (int i = 0; i < 8; ++i) wfA[i] = *(LAS bf16x8*)(lds + LDS_W + lane * 16 + i * 1024);
    putX(ha.x, ha.y, l0[0].x, l0[0].y, xf[0]);
    putX(ha.z, ha.w, l0[0].z, l0[0].w, xf[1]);
    so_l1 = xoff(2);
    l0[0] = __builtin_amdgcn_raw_buffer_load_b128(rs, xl_, so_l1, AUX_SYS);
    WAIT_LGKM0();
    STAMP(1);
#pragma unroll 1
    for (int s = 0; s < 60; s += 2) {
        step(s, C0{}, T{}, T{}, T{});
        step(s + 1, C1{}, T{}, T{}, T{});
    }
    step(60, C0{}, T{}, T{}, T{});
    step(61, C1{}, T{}, F{}, T{});
    step(62, C0{}, T{}, F{}, F{});
    step(63, C1{}, F{}, F{}, F{});
    STAMP(5);
    KOUT();

    float mx = 0.f;
#pragma unroll
    for (int j = 0; j < 4; ++j)
#pragma unroll
        for (int i = 0; i < 16; ++i) mx = fmaxf(mx, acc[j][i]);
    const u32 mb = wave_max(__float_as_uint(mx));
    if (lane == 0) *(LAS u32*)(lds + LDS_RED + wave * 4) = mb;
    if (dead && lane == 0) *(LAS u32*)(lds + LDS_RED + 20) = 1;
    LDS_BARRIER();
    u32 m0 = 0, m1 = 0;
#pragma unroll
    for (int i = 7; i >= 0; --i)
#pragma unroll
        for (int j = 3; j >= 0; --j) {
            m0 = __builtin_amdgcn_alignbit(m0, __float_as_uint(acc[j][i]), 31);
            m1 = __builtin_amdgcn_alignbit(m1, __float_as_uint(acc[j][i + 8]), 31);
        }
    m0 = ~m0; m1 = ~m1;
    const int mw = tile * 4 + wave, rot = mw % 7;
    const u32 moff = OFF_MSG + (u32)(rank * 512 + mw) * MSG;
    *(LAS u32x2*)(lds + LDS_MSK + wave * 512 + lane * 8) = u32x2{m0, m1};
    LDS_BARRIER();
    const u32 cnt = (u32)(__popc(m0) + __popc(m1));
    const u32 incl = wave_scan(cnt);
    const u32 total = rdl(incl, 63);
    u32 sel[16], pa[16], ps[16], bm[16];
    {
        u32 pp = (incl - cnt) * 6u;
#pragma unroll
        for (int i = 0; i < 16; ++i) {
            const u32 mm = i < 8 ? m0 : m1;
            const int sh = 4 * (i & 7);
            const u32 n4 = (sh ? (mm >> (sh - 2)) : (mm << 2)) & 0x3cu;
            sel[i] = *(LAS u32*)(lds + LDS_LUT + n4);
            bm[i] = __umul24(n4 << 4, 0x00204081u) & 0x40404040u;
            pa[i] = (pp >> 5) << 2; ps[i] = pp & 31u;
            pp += 6u * (u32)__popc(n4);
            asm volatile("" : "+v"(bm[i]), "+v"(pa[i]), "+v"(ps[i]));
        }
    }
#pragma unroll
    for (int j = 0; j < 4; ++j)
#pragma unroll
        for (int i = 0; i < 16; ++i) acc[j][i] = fmaxf(acc[j][i], 0.f);
#pragma unroll
    for (int j = 0; j < 4; ++j) asm volatile("" : "+v"(acc[j]));
    u64 gd;
    while ((gd = __hip_atomic_load((LAS u64*)(lds + LDS_RED + 24), __ATOMIC_ACQUIRE, __HIP_MEMORY_SCOPE_WORKGROUP)) == 0ull) __builtin_amdgcn_s_sleep(1);
    const float inv = __uint_as_float(rfl((u32)gd));
    dead = rfl((u32)(gd >> 32)) != 1u;
    STAMP(6);

#pragma unroll
    for (int j = 0; j < 4; ++j) acc[j] *= inv;
    const u32 etop = (rfl((u32)__builtin_amdgcn_cvt_pk_fp8_f32(__uint_as_float(mb) * inv, 0.f, 0, false)) >> 3) & 15u;
    u32 base = etop > 8u ? etop - 8u : 0u;
    u32x4 q[4];
    u32 bad0 = 0, bad1 = 0;
    {
        const u32 b8 = base * 0x08080808u;
#pragma unroll
        for (int i = 0; i < 16; ++i) {
            int w0 = __builtin_amdgcn_cvt_pk_fp8_f32(acc[0][i], acc[1][i], 0, false);
            w0 = __builtin_amdgcn_cvt_pk_fp8_f32(acc[2][i], acc[3][i], w0, true);
            q[i >> 2][i & 3] = (u32)w0 & 0x7f7f7f7fu;
            const u32 t = ((u32)w0 | 0x80808080u) - b8;
            const u32 n64 = __builtin_amdgcn_udot4(t & bm[i], 0x08040201u, 0u, false);
            const int sh = 4 * (i & 7) - 6;
            if (i < 8) bad0 |= sh < 0 ? n64 >> -sh : n64 << sh; else bad1 |= sh < 0 ? n64 >> -sh : n64 << sh;
            const u32 c = perm(t, t, sel[i]) & 0x3f3f3f3fu;
            const u32 y = bfi(0x003f003fu, c, c >> 2);
            const u32 z = bfi(0x00000fffu, y, y >> 4);
            const u64 sh64 = (u64)z << ps[i];
            LAS u32* wp = (LAS u32*)(str + pa[i]);
            __hip_atomic_fetch_or(wp, (u32)sh64, __ATOMIC_RELAXED, __HIP_MEMORY_SCOPE_WAVEFRONT);
            __hip_atomic_fetch_or(wp + 1, (u32)(sh64 >> 32), __ATOMIC_RELAXED, __HIP_MEMORY_SCOPE_WAVEFRONT);
        }
    }
    u32 nst = (total * 6u + 127u) >> 7;
    const u32 nb = (u32)(__popc(bad0) + __popc(bad1));
    const u32 nbi = wave_scan(nb);
    u32 nexc = rdl(nbi, 63), seven = 0;
    if (__builtin_expect(nexc <= 64u, 1)) {
        LAS unsigned short* lp = (LAS unsigned short*)(str + nst * 16u) + (nbi - nb);
        const u32 lb = (u32)(lane & 31) * 128u + (u32)(lane >> 5) * 16u;
        while (__ballot((bad0 | bad1) != 0u) != 0ull) {
            if ((bad0 | bad1) != 0u) {
                const u32 pb = bad0 ? (u32)__ffs((int)bad0) - 1u : (u32)__ffs((int)bad1) + 31u;
                if (bad0) bad0 &= bad0 - 1u; else bad1 &= bad1 - 1u;
                *lp++ = (unsigned short)(lb + ((pb >> 4) << 5) + (pb & 15u));
            }
        }
    } else {
        seven = 1; base = 0; nexc = 0;
        nst = (total * 7u + 127u) >> 7;
        WAIT_LGKM0();
#pragma unroll
        for (int u = 0; u < 4; ++u) *(LAS u32x4*)(str + u * 1024 + lane * 16) = u32x4{0, 0, 0, 0};
        u32 pp = (incl - cnt) * 7u;
#pragma unroll
        for (int i = 0; i < 16; ++i) {
            const u32 qq = q[i >> 2][i & 3];
            const u32 c = perm(qq, qq, sel[i]);
            const u32 y = bfi(0x007f007fu, c, c >> 1);
            const u32 z = bfi(0x00003fffu, y, y >> 2);
            const u64 sh64 = (u64)z << (pp & 31u);
            LAS u32* wp = (LAS u32*)(str + ((pp >> 5) << 2));
            __hip_atomic_fetch_or(wp, (u32)sh64, __ATOMIC_RELAXED, __HIP_MEMORY_SCOPE_WAVEFRONT);
            __hip_atomic_fetch_or(wp + 1, (u32)(sh64 >> 32), __ATOMIC_RELAXED, __HIP_MEMORY_SCOPE_WAVEFRONT);
            pp += 7u * (u32)__popc(bm[i]);
        }
    }
    const u32 nch = nst + ((nexc + 7u) >> 3);
    WAIT_LGKM0();
    u32x4 cs[4];
#pragma unroll
    for (int u = 0; u < 4; ++u) cs[u] = *(LAS u32x4*)(str + u * 1024 + lane * 16);
    STAMP(7);
    if (!dead) {
        const u64 flag = ((u64)call << 32) | nch | (nst << 8) | (nexc << 16) | (base << 23) | (seven << 27);
        const u32 foff = OFF_FLAGO + (u32)mw * 128u + (u32)rank * 8u;
        if (__builtin_expect(nch <= 128u, 1)) {
            char* dst[7];
#pragma unroll
            for (int k = 0; k < 7; ++k) { int dd = k + rot; if (dd >= 7) dd -= 7; dst[k] = P.wsr[dd]; }
            const u64 e0 = __ballot((u32)lane < max(nch, 1u)), e1 = __ballot((u32)lane + 64u < max(nch, 65u));
            const u32 vo = moff + 512u + (u32)lane * 16u;
            const u32x2 fv = u32x2{(u32)flag, (u32)(flag >> 32)};
            auto send = [&](int k) __attribute__((always_inline)) {
                asm volatile("s_mov_b64 exec, %3\n\tglobal_store_dwordx4 %0, %1, %2 sc0 sc1\n\ts_mov_b64 exec, -1\n\ts_nop 1"
                             :: "v"(vo), "v"(cs[0]), "s"(dst[k]), "s"(e0) : "memory");
                if ((PACE >> ((k + (PROT ? rot : 0)) % 7)) & 1) WAIT_VM0();
                asm volatile("s_mov_b64 exec, %3\n\tglobal_store_dwordx4 %0, %1, %2 offset:1024 sc0 sc1\n\ts_mov_b64 exec, -1\n\ts_nop 1"
                             :: "v"(vo), "v"(cs[1]), "s"(dst[k]), "s"(e1) : "memory");
            };
#pragma unroll
            for (int k = 0; k < SWIN; ++k) send(k);
            __builtin_amdgcn_s_barrier();
            mark_msg<0>(foff, fv, dst[0]); if (SWIN + 0 < 7) send(SWIN + 0);
            mark_msg<1>(foff, fv, dst[1]); if (SWIN + 1 < 7) send(SWIN + 1);
            mark_msg<2>(foff, fv, dst[2]); if (SWIN + 2 < 7) send(SWIN + 2);
            mark_msg<3>(foff, fv, dst[3]); if (SWIN + 3 < 7) send(SWIN + 3);
            mark_msg<4>(foff, fv, dst[4]); if (SWIN + 4 < 7) send(SWIN + 4);
            mark_msg<5>(foff, fv, dst[5]); if (SWIN + 5 < 7) send(SWIN + 5);
            mark_msg<6>(foff, fv, dst[6]);
        } else {
            __builtin_amdgcn_s_barrier();
#pragma unroll 1
            for (int k = 0; k < 7; ++k) {
                int dd = k + rot; if (dd >= 7) dd -= 7;
                char* w0 = P.wsr[dd];
#pragma unroll
                for (int u = 0; u < 4; ++u)
                    if ((u32)(u * 64 + lane) < nch) __builtin_amdgcn_raw_buffer_store_b128(cs[u], mk_rsrc(w0, WS_USED), 512u + u * 1024u + (u32)lane * 16u, moff, AUX_SYS);
                WAIT_VM0();
                if (lane == 0) __hip_atomic_store((gu64*)(w0 + foff), flag, RLX_SYS);
            }
        }
    } else {
        __builtin_amdgcn_s_barrier();
    }
    STAMP(8);
    {
        const u32 ooff = (u32)((rank >> 2) * 2048 + ti * 128 + wave * 32 + (lane & 31)) * 4096u
                       + (u32)((rank & 3) * 1024 + tj * 128 + (lane >> 5) * 16);
        __amdgpu_buffer_rsrc_t ro = mk_rsrc(P.out, (u32)MM * NN);
#pragma unroll
        for (int bb = 0; bb < 4; ++bb)
            __builtin_amdgcn_raw_buffer_store_b128(dead ? u32x4{0x7f7f7f7fu, 0x7f7f7f7fu, 0x7f7f7f7fu, 0x7f7f7f7fu} : q[bb], ro, ooff + bb * 32, 0, 0);
    }
    STAMP(9);
}

extern "C" void kernel_launch(int rank, int nranks, ncclComm_t comm,
                              hipStream_t stream,
                              void* const* d_in, const int* in_sizes, int n_in,
                              void* d_out, int out_size,
                              const int* in_dtypes, int out_dtype,
                              void* const* d_out_all,
                              void* const* d_ws_all, size_t ws_bytes) {
    (void)comm; (void)in_sizes; (void)in_dtypes; (void)out_dtype; (void)d_out_all;
    if (nranks != NR || n_in < 2 || out_size != MM * NN || ws_bytes < WS_USED || (unsigned)rank >= (unsigned)NR) {
        fprintf(stderr, "unexpected problem shape\n");
        return;
    }
    Params P;
    P.x = (const char*)d_in[0];
    P.w = (const char*)d_in[1];
    for (int d = 0; d < NR; ++d) P.wsr[d] = (char*)d_ws_all[(rank + 1 + d) & 7];
    P.out = (char*)d_out;
    P.myws = (char*)d_ws_all[rank];
    P.rank = rank;
    P.pad = 0;
    fused_kernel<<<dim3(NGRID), dim3(NTHR), 0, stream>>>(P);
    HIP_CHECK(hipGetLastError());
}
```

```cpp
#include <hip/hip_runtime.h>
#include <rccl/rccl.h>
#include <cstdio>
#include <type_traits>

#define HIP_CHECK(expr) do { hipError_t _e = (expr); if (_e != hipSuccess) { \
    fprintf(stderr, "HIP error %s at %s:%d\n", hipGetErrorString(_e), __FILE__, __LINE__); \
    return; } } while (0)

typedef unsigned int u32;
typedef unsigned long long u64;
typedef u32 u32x4 __attribute__((ext_vector_type(4)));
typedef u32 u32x3 __attribute__((ext_vector_type(3)));
typedef u32 u32x2 __attribute__((ext_vector_type(2)));
typedef unsigned short u16x2 __attribute__((ext_vector_type(2)));
typedef float f32x16 __attribute__((ext_vector_type(16)));
typedef __bf16 bf16x8 __attribute__((ext_vector_type(8)));
typedef __bf16 bf16x2 __attribute__((ext_vector_type(2)));
#define GAS __attribute__((address_space(1)))
#define LAS __attribute__((address_space(3)))
typedef GAS u32 gu32;
typedef GAS u64 gu64;
typedef GAS u32x4 gu32x4;

constexpr int NR = 8;
constexpr int MM = 4096, NN = 4096;
constexpr int NGRID = 256, NTHR = 768;
constexpr u64 DEADLINE_TICKS = 50000000ull;

constexpr u32 OFF_CNT   = 0;
constexpr u32 OFF_TMO   = 32768;
constexpr u32 OFF_TICK  = 33024;
constexpr u32 OFF_AMAXL = 36864;
constexpr u32 OFF_AMAXG = 40960;
constexpr int NREP      = 8;
constexpr u32 OFF_FLAGA = 65536;
constexpr u32 OFF_FLAGB = 98304;
constexpr u32 OFF_FLAGO = 131072;
constexpr u32 OFF_DIAG  = 262144;
constexpr u32 PIECE     = 16384;
constexpr u32 OFF_A     = 1u << 20;
constexpr u32 OFF_B     = OFF_A + 64u * 16u * PIECE;
constexpr u32 MSG       = 4096;
constexpr u32 OFF_MSG   = OFF_B + 64u * 8u * PIECE;
constexpr u32 WS_USED   = OFF_MSG + 8u * 512u * MSG;

constexpr int LDS_W    = 0;
constexpr int LDS_STR  = 32768;
constexpr int LDS_MSK  = 65536;
constexpr int LDS_LUT  = 67584;
constexpr int LDS_RED  = 67712;
constexpr int LDS_ORD  = 67840;
constexpr int LDS_TAB  = 69632;
constexpr int LDS_HX   = 98304;
constexpr int LDS_SIZE = 114688;

struct Params {
    const char* x;
    const char* w;
    char* myws;
    char* out;
    int rank;
    int pad;
    char* wsr[NR];
};

#define RLX_SYS __ATOMIC_RELAXED, __HIP_MEMORY_SCOPE_SYSTEM
#define RLX_AGT __ATOMIC_RELAXED, __HIP_MEMORY_SCOPE_AGENT
#define WAIT_VM0() asm volatile("s_waitcnt vmcnt(0)" ::: "memory")
#define CBAR() asm volatile("" ::: "memory")
#define WAIT_LGKM0() do { CBAR(); __builtin_amdgcn_s_waitcnt(0xC07F); CBAR(); } while (0)
#define LDS_BARRIER() do { WAIT_LGKM0(); __builtin_amdgcn_s_barrier(); CBAR(); } while (0)
constexpr int AUX_SYS = 17;
constexpr int AUX_OUT = 16;
constexpr int NWL = 32;
constexpr int MEARLY = 5;
constexpr int MWIN = 2;
constexpr int DECW = 5;
constexpr int NBLK = 768;
constexpr u32 SEQ_SH = 17, SEQ_MASK = 0x7fffu;

__device__ __forceinline__ u64 now() { return __builtin_amdgcn_s_memrealtime(); }
__device__ __forceinline__ u32 rfl(u32 v) { return (u32)__builtin_amdgcn_readfirstlane((int)v); }
__device__ __forceinline__ u32 rdl(u32 v, int l) { return (u32)__builtin_amdgcn_readlane((int)v, l); }
__device__ __forceinline__ u32 perm(u32 a, u32 b, u32 s) { return __builtin_amdgcn_perm(a, b, s); }
__device__ __forceinline__ u32 bfi(u32 m, u32 a, u32 b) { return (a & m) | (b & ~m); }
__device__ __forceinline__ __amdgpu_buffer_rsrc_t mk_rsrc(const char* base, u32 bytes) {
    return __builtin_amdgcn_make_buffer_rsrc((void*)base, 0, bytes, 0x00020000);
}
template <int CTRL, int RMASK> __device__ __forceinline__ u32 dpp0(u32 v) {
    return (u32)__builtin_amdgcn_update_dpp(0, (int)v, CTRL, RMASK, 0xf, false);
}
__device__ __forceinline__ u32 wave_max(u32 v) {
    v = max(v, dpp0<0x111, 0xf>(v)); v = max(v, dpp0<0x112, 0xf>(v));
    v = max(v, dpp0<0x114, 0xf>(v)); v = max(v, dpp0<0x118, 0xf>(v));
    v = max(v, dpp0<0x142, 0xa>(v)); v = max(v, dpp0<0x143, 0xc>(v));
    return rdl(v, 63);
}
__device__ __forceinline__ u32 wave_scan(u32 v) {
    v += dpp0<0x111, 0xf>(v); v += dpp0<0x112, 0xf>(v);
    v += dpp0<0x114, 0xf>(v); v += dpp0<0x118, 0xf>(v);
    v += dpp0<0x142, 0xa>(v); v += dpp0<0x143, 0xc>(v);
    return v;
}

#define STAMP(i) do { } while (0)

__device__ __forceinline__ bool spin_fail(u64 t0, gu32* tmo, u32 call, u32& spins) {
    __builtin_amdgcn_s_sleep(1);
    ++spins;
    if ((spins & 63u) == 0u) {
        if (rfl(__hip_atomic_load(tmo, RLX_AGT)) == call) return true;
        if (now() - t0 > DEADLINE_TICKS) { __hip_atomic_store(tmo, call, RLX_AGT); return true; }
    }
    return false;
}

__device__ __forceinline__ u32 block_id(u32 nb, bool plain) {
    const u32 n = nb >> 2, m = plain ? n : n - 8u, g = (m * 2731u) >> 16, q = m - g * 24u, t = (q * 11u) >> 5;
    u32 ce = q == 3u * t ? (g + 1u) * 32u + t : g * 32u + 7u + q - t;
    if (n < 32u) ce = n + (n >= 24u ? 8u : 0u);
    if (n >= 176u) ce = 7u * 32u + n - 168u;
    if (plain) ce = g * 32u + q;
    return nb < (u32)NBLK ? (ce << 2) | (nb & 3u) : 0xffffffffu;
}
constexpr u32 M83 = 0x83838383u;
template <bool ISW> __device__ __forceinline__ void patch(u32 img, unsigned short ex, u64 hi, u32 vmask) {
    u32 ad, val, sh;
    asm volatile("v_lshrrev_b32 %0, %7, %3\n\tv_bfe_i32 %1, %3, %8, 5\n\tv_lshl_or_b32 %2, %3, %9, %10\n\t"
                 "v_and_or_b32 %0, %0, %6, %4\n\tv_lshlrev_b32 %1, %2, %1\n\t"
                 "s_mov_b64 exec, %5\n\tds_add_u32 %0, %1\n\ts_mov_b64 exec, -1"
                 : "=&v"(ad), "=&v"(val), "=&v"(sh)
                 : "v"(__builtin_bit_cast(_Float16, ex)), "s"(img), "s"(hi), "v"(vmask), "n"(ISW ? 4 : 5), "n"(ISW ? 1 : 2), "n"(ISW ? 4 : 3), "n"(ISW ? 10 : 2) : "memory");
}
__device__ __forceinline__ void raw_planes(u32 (&H)[8], const char* p) {
    u32x4 a, b;
    asm volatile("global_load_dwordx4 %0, %2, off offset:2048 sc0 sc1\n\tglobal_load_dwordx4 %1, %2, off offset:3072 sc0 sc1\n\ts_waitcnt vmcnt(0)"
                 : "=&v"(a), "=&v"(b) : "v"(p) : "memory");
    asm volatile("v_mov_b32 %0, %8\n\tv_mov_b32 %1, %9\n\tv_mov_b32 %2, %10\n\tv_mov_b32 %3, %11\n\t"
                 "v_mov_b32 %4, %12\n\tv_mov_b32 %5, %13\n\tv_mov_b32 %6, %14\n\tv_mov_b32 %7, %15"
                 : "+v"(H[0]), "+v"(H[1]), "+v"(H[2]), "+v"(H[3]), "+v"(H[4]), "+v"(H[5]), "+v"(H[6]), "+v"(H[7])
                 : "v"(a.x), "v"(a.y), "v"(a.z), "v"(a.w), "v"(b.x), "v"(b.y), "v"(b.z), "v"(b.w));
}
template <bool ISW> __device__ __forceinline__ void patch_more(u32 img, const char* p, u32 vmask) {
    unsigned short ex;
    asm volatile("global_load_ushort %0, %1, off offset:2944 sc0 sc1\n\ts_waitcnt vmcnt(0)" : "=&v"(ex) : "v"(p) : "memory");
    patch<ISW>(img, ex, ~0ull, vmask);
}

__global__ void __launch_bounds__(NTHR, 3) fused_kernel(Params P) {
    __shared__ __attribute__((aligned(32768))) char smem[LDS_SIZE];
    LAS char* lds = (LAS char*)smem;
    const int tid = threadIdx.x, lane = tid & 63, wave = (int)rfl((u32)tid >> 6);
    const int b = blockIdx.x, xcd = b & 7, slot = b >> 3;
    const int rank = P.rank;
    char* myws = P.myws;
    asm volatile("" :: "s"(P.x), "s"(P.w), "s"(P.wsr[slot < 16 ? slot & 7 : 7]));
    gu32* tmo = (gu32*)(myws + OFF_TMO);
    const u64 tstart = now();
    STAMP(0);
    auto get_call = [&]() __attribute__((always_inline)) {
        u32 ticket = 0;
        if (lane == 0) ticket = __hip_atomic_fetch_add((gu32*)(myws + OFF_CNT + b * 128 + wave * 8), 1u, RLX_AGT);
        return rfl(ticket) + 1u;
    };

    const bool comm = slot < 16;
    const int d = comm ? slot & 7 : 7;
    const int sw = (slot >> 3) * 8 + xcd;
    const int wl = wave * 16 + sw;
    const bool producer = d != 7 && wl < NWL;
    const int p = (rank + 1 + d) & 7;
    const int r = (rank - p) & 7;
    char* dws = P.wsr[d];
    __amdgpu_buffer_rsrc_t rs = mk_rsrc(dws, WS_USED);
    const int row0 = (p >> 2) * 2048, col0 = (p & 3) * 1024;
    const u32 xlane = (u32)(lane & 31) * 1024u + (u32)(lane >> 5) * 16u;
    const u32 wlane = (u32)((lane >> 5) * 8) * 8192u + (u32)(lane & 31) * 8u;
    u32 v[16];
    u32x2 vn[8];
    auto load_block = [&](int nb) __attribute__((always_inline)) {
        const int wb = nb & 3, c = nb >> 7, e = (nb >> 2) & 31;
        const bool isw = e < 8;
        const u32 so = isw ? (u32)((c * 64 + wb * 16) * 8192 + (col0 + ((e + r) & 7) * 128) * 2)
                           : (u32)((row0 + ((e - 8 + 2 * r) & 15) * 128 + wb * 32) * 1024 + c * 128);
        const u32 s1 = isw ? 8192u : 8u, s2 = isw ? 16384u : 32u;
        __amdgpu_buffer_rsrc_t ri = mk_rsrc(isw ? P.w : P.x, 4096u * 512u * 2u);
        const u32 vo = isw ? wlane : xlane;
#pragma unroll
        for (int k = 0; k < 8; ++k) vn[k] = __builtin_amdgcn_raw_buffer_load_b64(ri, vo, so + (k & 1) * s1 + (k >> 1) * s2, 0);
    };
    auto take_block = [&](u32 nb) __attribute__((always_inline)) {
        const bool isw = (nb & 0x60u) == 0u;
#pragma unroll
        for (int k = 0; k < 4; ++k) {
            v[4 * k] = vn[2 * k].x; v[4 * k + 1] = isw ? vn[2 * k + 1].x : vn[2 * k].y;
            v[4 * k + 2] = isw ? vn[2 * k].y : vn[2 * k + 1].x; v[4 * k + 3] = vn[2 * k + 1].y;
        }
#pragma unroll
        for (int i = 0; i < 16; ++i) asm volatile("" : "+v"(v[i]));
    };
    u32x4 pk[4];
    LAS char* stg = lds + LDS_TAB + wave * 3072;
    u32 p_off = 0, p_foff = 0, p_mode = 0, p_lanes = 0;
    u32 p_flag = 0;
    auto pack = [&](int nb, bool early, u32 call) __attribute__((always_inline)) {
        const int wb = nb & 3, c = nb >> 7, e = (nb >> 2) & 31;
        const int step = rank * 8 + c;
        u32 selL, selH;
        if (e < 8) { const int j = (e + r) & 7;
            p_off = OFF_B + (u32)(step * 8 + j) * PIECE;  p_foff = OFF_FLAGB + (u32)((j * 64 + step) * 4 + wb) * 4u;
            selL = 0x06020400u; selH = 0x07030501u;
        } else { const int i = (e - 8 + 2 * r) & 15;
            p_off = OFF_A + (u32)(step * 16 + i) * PIECE; p_foff = OFF_FLAGA + (u32)((i * 64 + step) * 4 + wb) * 4u;
            selL = 0x06040200u; selH = 0x07050301u;
        }
        p_off += (u32)wb * 4096u;
        u32 H[8], D[8];
        u16x2 mx = {0, 0}, mn = {0xffff, 0xffff};
#pragma unroll
        for (int i = 0; i < 16; ++i) {
            const u16x2 a = __builtin_bit_cast(u16x2, v[i] & 0x7fff7fffu);
            mx = __builtin_elementwise_max(mx, a); mn = __builtin_elementwise_min(mn, a);
        }
        u32 mag = max(mx.x, mx.y);
        if (__builtin_expect(__ballot(mag >= 0x7f00u || min(mn.x, mn.y) < 0x100u) == 0ull, 1)) {
            const u32 inc = e < 8 ? 0xff7fff80u : 0x00800080u;
#pragma unroll
            for (int i = 0; i < 16; ++i) v[i] += inc;
            mag += e < 8 ? 0xffffff80u : 0x80u;
        } else {
            const float f = e < 8 ? 0.5f : 2.0f;
            mag = 0;
#pragma unroll
            for (int i = 0; i < 16; ++i) {
                const bf16x2 t = {(__bf16)(__uint_as_float(v[i] << 16) * f), (__bf16)(__uint_as_float(v[i] & 0xffff0000u) * f)};
                v[i] = __builtin_bit_cast(u32, t);
                mag = max(mag, max(v[i] & 0x7fffu, (v[i] >> 16) & 0x7fffu));
            }
        }
#pragma unroll
        for (int i = 0; i < 8; ++i) { pk[i >> 2][i & 3] = perm(v[2 * i + 1], v[2 * i], selL); H[i] = perm(v[2 * i + 1], v[2 * i], selH); }
        if (early) {
            STAMP(7);
            __builtin_amdgcn_raw_buffer_store_b128(pk[0], rs, (u32)lane * 16u, p_off, AUX_SYS);
            __builtin_amdgcn_raw_buffer_store_b128(pk[1], rs, (u32)lane * 16u + 1024u, p_off, AUX_SYS);
        }
        const u32 top = wave_max(mag >> 8);
        u32 base = top > 3u ? top - 3u : 0u;
        if (base > 0u && __popcll(__ballot((mag >> 8) == top)) <= 8) --base;
        const u32 base4 = base * 0x01010101u;
        u32 pm = 0;
#pragma unroll
        for (int i = 0; i < 8; ++i) {
            D[i] = (H[i] | 0x80808080u) - base4;
            const u32 n128 = __builtin_amdgcn_udot4((((D[i] & 0x7c7c7c7cu) + 0x7c7c7c7cu) | ~D[i]) & 0x80808080u, 0x08040201u, 0u, false);
            pm |= 4 * i < 7 ? n128 >> (7 - 4 * i) : n128 << (4 * i - 7);
        }
        *(LAS u32x4*)(stg + 1024 + lane * 16) = u32x4{D[0], D[1], D[2], D[3]};
        *(LAS u32x4*)(stg + 2048 + lane * 16) = u32x4{D[4], D[5], D[6], D[7]};
        *(LAS u32*)(stg + 768 + lane * 4) = (u32)lane * 0x01000100u + 0x00800000u;
        u32 deep = 0;
        const u32 np = (u32)__popc(pm), npi = wave_scan(np), total = rdl(npi, 63);
        u32 mode = total > 64u ? 2u : 0u;
        p_lanes = total > 64u ? 64u : total > 32u ? 56u : 52u;
        if (__builtin_expect(total <= 128u, 1)) {
            u32 E[8];
#pragma unroll
            for (int i = 0; i < 8; ++i) E[i] = bfi(0x80808080u, H[i], D[i]) & M83;
            LAS u32* cw = (LAS u32*)(stg + lane * 12);
            cw[0] = E[0] | __builtin_amdgcn_alignbit(E[1], E[1], 29) | ((E[6] & 0x03030303u) << 5);
            cw[1] = E[2] | __builtin_amdgcn_alignbit(E[3], E[3], 29) | ((E[7] & 0x03030303u) << 5);
            cw[2] = E[4] | __builtin_amdgcn_alignbit(E[5], E[5], 29) | ((E[6] & 0x80808080u) >> 1) | ((E[7] & 0x80808080u) >> 2);
            LAS unsigned short* lp = (LAS unsigned short*)(stg + 768) + (npi - np);
            while (__ballot(pm != 0u) != 0ull) {
                if (pm != 0u) {
                    const u32 pb = (u32)__ffs((int)pm) - 1u;
                    pm &= pm - 1u;
                    const u32 ad = ((pb >> 4) << 10) + (u32)lane * 16u + (pb & 15u);
                    const u32 d = *(LAS unsigned char*)(stg + 1024 + ad);
                    const u32 d5 = (d >> 2) & 31u;
                    deep |= d < 64u;
                    const u32 ew = ((((pb >> 1) & 3u) << 8 | (u32)lane << 2 | pb >> 3) << 6) | (d5 << 1) | (pb & 1u);
                    *lp++ = (unsigned short)(e < 8 ? ew : ((ad >> 2) << 7) | (d5 << 2) | (pb & 3u));
                }
            }
            CBAR();
            pk[2] = *(LAS u32x4*)(stg + lane * 16);
        }
        if (__builtin_expect(total > 128u || __ballot(deep != 0u) != 0ull, 0)) {
            mode = 3; p_lanes = 64u;
            pk[2] = u32x4{H[0], H[1], H[2], H[3]};
        }
        pk[3] = u32x4{H[4], H[5], H[6], H[7]};
        p_mode = mode;
        p_flag = (call << SEQ_SH) | (mode << 15) | (base << 8) | ((u32)step << 1) | (total > 32u && mode != 3u ? 1u : 0u);
    };

    if (__builtin_expect(slot < 16, 1)) {
        if (wave >= 8) {
            const int wi = (b << 2) | (wave & 3);
            const char* px = P.x + (size_t)(wi * 8 + (lane >> 3)) * 1024 + (lane & 7) * 16;
            const char* pw = P.w + (size_t)wi * 1024 + lane * 16;
            u32x4 t[3];
#pragma unroll 1
            for (int c = 0; c < 8; ++c) {
                t[0] = *(const gu32x4*)(pw + c * 524288);
                t[1] = *(const gu32x4*)(px + c * 128);
                asm volatile("" :: "v"(t[0]), "v"(t[1]));
                __builtin_amdgcn_s_sleep(64);
            }
            __amdgpu_buffer_rsrc_t rt = mk_rsrc(myws, WS_USED);
            const int mwt = (slot * 8 + xcd) * 4 + (wave & 3);
#pragma unroll 1
            for (int k = 6; k > 4; --k) {
                for (int i = 0; i < 1 + (wi & 3); ++i) __builtin_amdgcn_s_sleep(64);
                int dd = k + mwt % 7; if (dd >= 7) dd -= 7;
                const u32 off = OFF_MSG + (u32)(((rank - 1 - dd) & 7) * 512 + mwt) * MSG;
                t[0] = __builtin_amdgcn_raw_buffer_load_b128(rt, (u32)lane * 16u, off, AUX_SYS);
                t[1] = __builtin_amdgcn_raw_buffer_load_b128(rt, (u32)lane * 16u + 1024u, off, AUX_SYS);
                t[2] = __builtin_amdgcn_raw_buffer_load_b128(rt, (u32)(lane & 31) * 16u + 2048u, off, AUX_SYS);
#pragma unroll
                for (int u = 0; u < 3; ++u) asm volatile("" :: "v"(t[u]));
            }
            return;
        }
        const u32 nA = block_id((u32)wl, false);
        if (producer) load_block((int)nA);
        CBAR();
        const u32 call = get_call();
        if (tid < 16) {
            u32 sel = 0, k = 0;
            for (int j = 0; j < 4; ++j) { const u32 on = (tid >> j) & 1; sel |= (on ? k : 0x0cu) << (8 * j); k += on; }
            *(LAS u32*)(lds + LDS_LUT + tid * 4) = sel;
        }
        gu32* tick = (gu32*)(myws + OFF_TICK + d * 128);
        const u32 tbase = (call - 1u) * (u32)(NBLK - 2 * NWL) - (u32)(3 * NWL);
        if (producer) {
            u32 nB = block_id((u32)(wl + NWL), false), nC = block_id((u32)(wl + 2 * NWL), false);
            take_block(nA);
            load_block((int)nB);
            pack((int)nA, true, call);
            const u32 vo = (u32)lane * 16u;
            char* fdst = nullptr;
            u32 flag = 0;
            const u32 zero = 0, one = 1;
            auto iter = [&](bool first) __attribute__((always_inline)) {
                const bool more = nB != 0xffffffffu, more2 = nC != 0xffffffffu;
                u32 tk = 0;
                if (more2) asm volatile("s_mov_b64 exec, 1\n\tglobal_atomic_add %0, %1, %2, %3 sc0\n\ts_mov_b64 exec, -1"
                                        : "=&v"(tk) : "v"(zero), "v"(one), "s"(tick) : "memory");
                if (!first && lane == 0) __hip_atomic_store((gu32*)fdst, flag, RLX_SYS);
                CBAR();
                if (!first) {
                    __builtin_amdgcn_raw_buffer_store_b128(pk[0], rs, vo, p_off, AUX_SYS);
                    __builtin_amdgcn_raw_buffer_store_b128(pk[1], rs, vo + 1024, p_off, AUX_SYS);
                }
                __builtin_amdgcn_raw_buffer_store_b128(pk[2], rs, (u32)lane >= p_lanes ? 0x80000000u : vo + 2048, p_off, AUX_SYS);
                if (p_mode == 3) __builtin_amdgcn_raw_buffer_store_b128(pk[3], rs, vo + 3072, p_off, AUX_SYS);
                CBAR();
                fdst = dws + p_foff;
                flag = p_flag;
                take_block(nB);
                if (more2) load_block((int)nC);
                CBAR();
                if (more) pack((int)nB, false, call);
                __builtin_amdgcn_s_waitcnt(0x0F70);
                asm volatile("s_waitcnt vmcnt(0)" : "+v"(tk) :: "memory");
                nB = nC;
                if (more2) nC = block_id(rfl(tk) - tbase, false);
                return more;
            };
            if (iter(true)) {
#pragma unroll 1
                while (iter(false)) { }
            }
            if (lane == 0) __hip_atomic_store((gu32*)fdst, flag, RLX_SYS);
        }
        STAMP(1);
        LDS_BARRIER();
        bool dead = false;
        if (b == 56 && wave == 0) {
            u32 spins = 0, m = 0;
            for (;;) {
                const u64 g0 = __hip_atomic_load((gu64*)(myws + OFF_AMAXL) + lane, RLX_AGT);
                const u64 g1 = __hip_atomic_load((gu64*)(myws + OFF_AMAXL) + 64 + lane, RLX_AGT);
                m = max((u32)g0, (u32)g1);
                if (__all((u32)(g0 >> 32) == call && (u32)(g1 >> 32) == call)) break;
                if (spin_fail(tstart, tmo, call, spins)) { dead = true; break; }
            }
            if (!dead) {
                m = wave_max(m);
                char* dst = P.wsr[lane & 7] + OFF_AMAXG + (lane >> 3) * 128 + rank * 8;
#pragma unroll
                for (int k = 0; k < NREP / 8; ++k)
                    __hip_atomic_store((gu64*)(dst + k * 1024), ((u64)call << 32) | m, RLX_SYS);
            }
            STAMP(2);
        }
        const int tile = slot * 8 + xcd, ti = tile >> 3, tj = tile & 7;
        const int wq = wave & 3, hf = wave >> 2;
        const int mw = tile * 4 + wq, rot = mw % 7;
        __amdgpu_buffer_rsrc_t rm = mk_rsrc(myws, WS_USED);
        __amdgpu_buffer_rsrc_t ro = mk_rsrc(P.out, (u32)MM * NN);
        gu64* fo = (gu64*)(myws + OFF_FLAGO + mw * 128) + (lane & 7);
        LAS char* str = lds + LDS_STR + wave * 4096;
        LAS char* ot = lds + LDS_W + wave * 2048;
        const int el = (lane & 32) + hf * 16 + (lane & 15), uh = (lane >> 4) & 1;
        u64 fl = 0;
#pragma unroll 1
        for (int k = 0; k < 7; ++k) {
            int dd = k + rot; if (dd >= 7) dd -= 7;
            const int q = (rank - 1 - dd) & 7;
            if (!dead && rdl((u32)(fl >> 32), q) != call) {
                u32 spins = 0;
                for (;;) {
                    fl = __hip_atomic_load(fo, RLX_SYS);
                    if (rdl((u32)(fl >> 32), q) == call) break;
                    if (spin_fail(tstart, tmo, call, spins)) { dead = true; break; }
                }
            }
            CBAR();
            const u32 ooff = (u32)((q >> 2) * 2048 + ti * 128 + wq * 32 + hf * 16 + (lane >> 3)) * 4096u
                           + (u32)((q & 3) * 1024 + tj * 128 + (lane & 7) * 16);
            if (dead) {
#pragma unroll
                for (int bb = 0; bb < 2; ++bb)
                    __builtin_amdgcn_raw_buffer_store_b128(u32x4{0x7f7f7f7fu, 0x7f7f7f7fu, 0x7f7f7f7fu, 0x7f7f7f7fu}, ro, ooff + bb * 32768u, 0, 0);
                continue;
            }
            const u32 fw = rdl((u32)fl, q);
            const u32 nch = fw & 255u;
            const u32 moff = OFF_MSG + (u32)(q * 512 + mw) * MSG;
            const u32x2 mk = __builtin_amdgcn_raw_buffer_load_b64(rm, (u32)lane * 8u, moff, AUX_SYS);
            const u32x4 c0 = __builtin_amdgcn_raw_buffer_load_b128(rm, 512u + (u32)lane * 16u, moff, AUX_SYS);
            const u32x4 c1 = __builtin_amdgcn_raw_buffer_load_b128(rm, 1536u + (u32)lane * 16u, moff, AUX_SYS);
            const u32 c_lo = (u32)__popc(mk.x), cnt = c_lo + (u32)__popc(mk.y);
            const u32 f0 = wave_scan(cnt) - cnt;
            const u32 ff = (u32)__builtin_amdgcn_ds_bpermute(el * 4, (int)(f0 | ((f0 + c_lo) << 16)));
            const u32 ma = (u32)__builtin_amdgcn_ds_bpermute(el * 4, (int)mk.x), mb = (u32)__builtin_amdgcn_ds_bpermute(el * 4, (int)mk.y);
            const u32 first = uh ? ff >> 16 : ff & 0xffffu, mm = uh ? mb : ma;
            *(LAS u32x4*)(str + lane * 16) = c0;
            *(LAS u32x4*)(str + 1024 + lane * 16) = c1;
            if (nch > 128u) {
                const u32x4 c2 = __builtin_amdgcn_raw_buffer_load_b128(rm, 2560u + (u32)lane * 16u, moff, AUX_SYS);
                *(LAS u32x4*)(str + 2048 + lane * 16) = c2;
                if (lane < 32) {
                    const u32x4 c3 = __builtin_amdgcn_raw_buffer_load_b128(rm, 3584u + (u32)lane * 16u, moff, AUX_SYS);
                    *(LAS u32x4*)(str + 3072 + lane * 16) = c3;
                }
            }
            WAIT_LGKM0();
            u32x4 o[2];
            u32 ex = 0;
            if (__builtin_expect((fw & (1u << 27)) == 0u, 1)) {
                ex = *(LAS unsigned short*)(str + ((fw >> 8) & 255u) * 16u + lane * 2);
                const u32 b8 = ((fw >> 23) & 15u) * 0x08080808u;
                u32 pos = first * 6u;
#pragma unroll
                for (int i = 0; i < 8; ++i) {
                    const int sh = 4 * i;
                    const u32 n4 = (sh ? (mm >> (sh - 2)) : (mm << 2)) & 0x3cu;
                    const u32 sel = *(LAS u32*)(lds + LDS_LUT + n4);
                    const LAS u32* wp = (const LAS u32*)(str + ((pos >> 5) << 2));
                    const u32 z = __builtin_amdgcn_alignbit(wp[1], wp[0], pos);
                    pos += 6u * (u32)__popc(n4);
                    const u32 y = bfi(0x00000fffu, z, z << 4);
                    const u32 x = (bfi(0x003f003fu, y, y << 2) & 0x3f3f3f3fu) + b8;
                    o[i >> 2][i & 3] = perm(x, x, sel);
                }
            } else {
                u32 pos = first * 7u;
#pragma unroll
                for (int i = 0; i < 8; ++i) {
                    const int sh = 4 * i;
                    const u32 n4 = (sh ? (mm >> (sh - 2)) : (mm << 2)) & 0x3cu;
                    const u32 sel = *(LAS u32*)(lds + LDS_LUT + n4);
                    const LAS u32* wp = (const LAS u32*)(str + ((pos >> 5) << 2));
                    const u32 z = __builtin_amdgcn_alignbit(wp[1], wp[0], pos);
                    pos += 7u * (u32)__popc(n4);
                    const u32 y = bfi(0x00003fffu, z, z << 2);
                    const u32 x = bfi(0x007f007fu, y, y << 1) & 0x7f7f7f7fu;
                    o[i >> 2][i & 3] = perm(x, x, sel);
                }
            }
            *(LAS u32x4*)(ot + (lane & 15) * 128 + uh * 64 + (lane >> 5) * 16) = o[0];
            *(LAS u32x4*)(ot + (lane & 15) * 128 + uh * 64 + (lane >> 5) * 16 + 32) = o[1];
            if ((u32)lane < ((fw >> 16) & 127u) && ((ex >> 11) & 1u) == (u32)hf)
                __hip_atomic_fetch_xor((LAS u32*)(ot + (ex & 0x7fcu)), 0x40u << (8u * (ex & 3u)), __ATOMIC_RELAXED, __HIP_MEMORY_SCOPE_WAVEFRONT);
            CBAR();
#pragma unroll
            for (int bb = 0; bb < 2; ++bb)
                __builtin_amdgcn_raw_buffer_store_b128(*(LAS u32x4*)(ot + bb * 1024 + lane * 16), ro, ooff + bb * 32768u, 0, AUX_OUT);
        }
        STAMP(6);
        return;
    }

    const int sl = slot - 16;
    const int ti = (xcd >> 1) * 4 + (sl >> 2), tj = (xcd & 1) * 4 + (sl & 3), tile = ti * 8 + tj;
    const bool helper = wave >= 4;
    const int wq = wave & 3;
    if (tid < 16) {
        u32 sel = 0x0c0c0c0cu, k = 0;
        for (int j = 0; j < 4; ++j) if ((tid >> j) & 1) { sel = (sel & ~(0xffu << (8 * k))) | ((u32)j << (8 * k)); ++k; }
        *(LAS u32*)(lds + LDS_LUT + tid * 4) = sel;
    }
    if (tid < 8) *(LAS u32*)(lds + LDS_RED + tid * 4) = 0;
    LAS char* str = lds + LDS_STR + wq * 4096;
    if (!helper) {
#pragma unroll
        for (int u = 0; u < 4; ++u) *(LAS u32x4*)(str + u * 1024 + lane * 16) = u32x4{0, 0, 0, 0};
    }
    u32 order = 0;
    for (int k = 0; k < 7; ++k) order |= (u32)((rank + ((ti >> 1) + 13 - k) % 7 + 1) & 7) << (4 * k);
    order = rfl(order);
    auto step_id = [&](u32 s) __attribute__((always_inline)) {
        const u32 m = s - 8u, c = (m * 37u) >> 8;
        return s < 8u ? (u32)rank * 8u + s : ((order >> ((m - c * 7u) * 4u)) & 7u) * 8u + c;
    };
    const u32 stepid = step_id((u32)lane);
    const u32 nown = block_id((u32)((b - 128) * 6 + wave), true);
    if (wave < 6) load_block((int)nown);
    CBAR();
    const u32 call = get_call();
    const u32 seq = call & SEQ_MASK;
    if (wave < 6) {
        take_block(nown);
        pack((int)nown, true, call);
        __builtin_amdgcn_raw_buffer_store_b128(pk[2], rs, (u32)lane >= p_lanes ? 0x80000000u : (u32)lane * 16u + 2048u, p_off, AUX_SYS);
        if (p_mode == 3) __builtin_amdgcn_raw_buffer_store_b128(pk[3], rs, (u32)lane * 16u + 3072u, p_off, AUX_SYS);
        WAIT_VM0();
        if (lane == 0) __hip_atomic_store((gu32*)(myws + p_foff), p_flag, RLX_SYS);
    }
    bool dead = false;
    const u32 xblk = OFF_A + (u32)ti * PIECE + (u32)wq * 4096u;
    LAS char* ord = lds + LDS_ORD;
#define SB() __builtin_amdgcn_sched_barrier(0)
    using T = std::true_type; using F = std::false_type;
    using C0 = std::integral_constant<int, 0>; using C1 = std::integral_constant<int, 1>;
    if (helper) {
        auto lanes = [&](u32 f) __attribute__((always_inline)) { return ((u64)((f & 1u) ? 0xffffffffu : 0u) << 32) | ((f & 0x8000u) ? 0u : 0xffffffffu); };
        auto planes = [&](u32 fhi, const u32x3 c, u32 blk, u32 (&H)[8]) __attribute__((always_inline)) {
            const u32 base4 = ((fhi >> 8) & 127u) * 0x01010101u;
#pragma unroll
            for (int q = 0; q < 3; ++q) { H[2 * q] = c[q] & M83; H[2 * q + 1] = __builtin_amdgcn_alignbit(c[q], c[q], 3) & M83; }
            H[6] = bfi(0x80808080u, c[2] << 1, c[0] >> 5) & M83;
            H[7] = bfi(0x80808080u, c[2] << 2, c[1] >> 5) & M83;
            SB();
#pragma unroll
            for (int q = 0; q < 8; ++q) H[q] += base4;
            if (__builtin_expect((fhi & 0x8000u) != 0u, 0)) raw_planes(H, myws + (blk + (u32)lane * 16u));
        };
        u32 vmask = wave >= 8 ? 0x7fcu : 0xffcu;
        asm volatile("" : "+v"(vmask));
        const u32 fo = OFF_FLAGA + (u32)ti * 1024u + stepid * 16u, go = OFF_FLAGB + (u32)tj * 1024u + stepid * 16u;
        u32x4 fv = {0, 0, 0, 0}, gv = {0, 0, 0, 0}, pv, pg;
        u64 todo = 0, done = 0;
        const u32 want = seq << SEQ_SH;
        auto take = [&](const u32x4 v, const u32x4 g) __attribute__((always_inline)) {
            fv = v; gv = g;
            const u32 df = ((v.x ^ want) | (v.y ^ want) | (v.z ^ want) | (v.w ^ want)) | ((g.x ^ want) | (g.y ^ want) | (g.z ^ want) | (g.w ^ want));
            todo = __ballot((df >> SEQ_SH) == 0u) & ~done;
        };
        auto pick = [&](int slot_) __attribute__((always_inline)) {
            const int id = __ffsll((long long)todo) - 1;
            done |= 1ull << id; todo &= todo - 1ull;
            if (lane == id) { *(LAS u32x4*)(ord + (slot_ & 3) * 32) = fv; *(LAS u32x4*)(ord + (slot_ & 3) * 32 + 16) = gv; }
        };
        auto peek = [&](int slot_) __attribute__((always_inline)) {
            bool later = false;
            if (wave == DECW) {
                later = todo == 0ull;
                if (later) { pv = __builtin_amdgcn_raw_buffer_load_b128(rs, fo, 0, AUX_SYS); pg = __builtin_amdgcn_raw_buffer_load_b128(rs, go, 0, AUX_SYS); }
                else pick(slot_);
            }
            SB();
            return later;
        };
        auto choose = [&](int slot_, bool peeked) __attribute__((always_inline)) {
            if (peeked) take(pv, pg);
            if (todo == 0ull) {
                u32 spins = 0;
                for (;;) {
                    const u32x4 v = __builtin_amdgcn_raw_buffer_load_b128(rs, fo, 0, AUX_SYS), g = __builtin_amdgcn_raw_buffer_load_b128(rs, go, 0, AUX_SYS);
                    take(v, g);
                    CBAR();
                    if (todo != 0ull) break;
                    if (spin_fail(tstart, tmo, call, spins)) { dead = true; fv = gv = u32x4{0, 0, 0, 0}; todo = ~done; break; }
                }
            }
            pick(slot_);
        };
        if (wave == DECW) { choose(0, false); choose(1, false); }
        LDS_BARRIER();
        if (wave >= 8) {
            struct XC { u32x3 c; unsigned short ex; u32 f, so; };
            XC x0, x1;
            const u32 xc_ = xblk + 2048u + (u32)lane * 12u, xe_ = xblk + 2816u + (u32)lane * 2u;
            u32 img = (u32)(uintptr_t)(lds + LDS_HX + wq * 2048);
            auto ldx = [&](int slot_, XC& x) __attribute__((always_inline)) {
                x.f = rfl(*(LAS u32*)(ord + (slot_ & 3) * 32 + wq * 4));
                x.so = ((x.f >> 1) & 63u) * (16u * PIECE);
                x.c = __builtin_amdgcn_raw_buffer_load_b96(rs, xc_, x.so, AUX_SYS);
                x.ex = __builtin_amdgcn_raw_buffer_load_b16(rs, xe_, x.so, AUX_SYS);
                SB();
            };
            auto unit = [&](int j, XC& x, XC& xn, bool ld) __attribute__((always_inline)) {
                if (ld) ldx(j + 1, xn);
                u32 H[8];
                planes(x.f, x.c, xblk + x.so, H);
                *(LAS u32x4*)(uintptr_t)(img + lane * 16) = u32x4{H[0], H[1], H[2], H[3]};
                *(LAS u32x4*)(uintptr_t)(img + 1024 + lane * 16) = u32x4{H[4], H[5], H[6], H[7]};
                patch<false>(img, x.ex, lanes(x.f), vmask);
                if (__builtin_expect((x.f & 0x18000u) == 0x10000u, 0)) patch_more<false>(img, myws + (xblk + (u32)lane * 2u + x.so), vmask);
                img ^= 8192u;
            };
            ldx(0, x0);
#pragma unroll 1
            for (int j = 0; j < 62; j += 2) {
                unit(j, x0, x1, true); LDS_BARRIER();
                unit(j + 1, x1, x0, true); LDS_BARRIER();
            }
            unit(62, x0, x1, true); LDS_BARRIER();
            unit(63, x1, x0, false);
            if (dead && lane == 0) *(LAS u32*)(lds + LDS_RED + 20) = 1;
            LDS_BARRIER();
            return;
        }
        struct WB { u32x4 l0, l1; u32x3 c; unsigned short ex; u32 f, so; };
        WB w0, w1;
        const u32 wblk = OFF_B + (u32)tj * PIECE + (u32)wq * 4096u;
        const u32 wl_ = wblk + (u32)lane * 16u, wc_ = wblk + 2048u + (u32)lane * 12u, we_ = wblk + 2816u + (u32)lane * 2u;
        u32 img = (u32)(uintptr_t)(lds + LDS_W + wq * 4096);
        auto unit = [&](int j, WB& w, WB& wn, bool ld, bool ch) __attribute__((always_inline)) {
            const bool pk_ = ch && peek(j + 2);
            u32 fn = 0;
            if (ld) fn = *(LAS u32*)(ord + ((j + 1) & 3) * 32 + 16 + wq * 4);
            SB();
            u32 H[8];
            planes(w.f, w.c, wblk + w.so, H);
            SB();
            if (ld) {
                wn.f = rfl(fn); wn.so = ((wn.f >> 1) & 63u) * (8u * PIECE);
                wn.c = __builtin_amdgcn_raw_buffer_load_b96(rs, wc_, wn.so, AUX_SYS);
                wn.l0 = __builtin_amdgcn_raw_buffer_load_b128(rs, wl_, wn.so, AUX_SYS);
            }
            SB();
            u32x4 c[4];
            LAS char* pw = (LAS char*)(uintptr_t)(img + lane * 16);
#pragma unroll
            for (int q = 0; q < 4; ++q) {
                const u32 sl_ = (q & 1) ? 0x07030602u : 0x05010400u;
                const int xy = q >> 1;
                c[q] = u32x4{perm(H[xy], w.l0[xy], sl_), perm(H[2 + xy], w.l0[2 + xy], sl_),
                             perm(H[4 + xy], w.l1[xy], sl_), perm(H[6 + xy], w.l1[2 + xy], sl_)};
                if (q) *(LAS u32x4*)(pw + (q - 1) * 1024) = c[q - 1];
                SB();
            }
            *(LAS u32x4*)(pw + 3 * 1024) = c[3];
            patch<true>(img, w.ex, lanes(w.f), vmask);
            if (__builtin_expect((w.f & 0x18000u) == 0x10000u, 0)) patch_more<true>(img, myws + (wblk + (u32)lane * 2u + w.so), vmask);
            img ^= 16384u;
            SB();
            if (ld) {
                wn.l1 = __builtin_amdgcn_raw_buffer_load_b128(rs, wl_ + 1024, wn.so, AUX_SYS);
                wn.ex = __builtin_amdgcn_raw_buffer_load_b16(rs, we_, wn.so, AUX_SYS);
            }
            if (pk_) choose(j + 2, true);
        };
        w0.f = rfl(*(LAS u32*)(ord + 16 + wq * 4)); w0.so = ((w0.f >> 1) & 63u) * (8u * PIECE);
        w0.c = __builtin_amdgcn_raw_buffer_load_b96(rs, wc_, w0.so, AUX_SYS);
        w0.l0 = __builtin_amdgcn_raw_buffer_load_b128(rs, wl_, w0.so, AUX_SYS);
        w0.l1 = __builtin_amdgcn_raw_buffer_load_b128(rs, wl_ + 1024, w0.so, AUX_SYS);
        w0.ex = __builtin_amdgcn_raw_buffer_load_b16(rs, we_, w0.so, AUX_SYS);
#pragma unroll 1
        for (int j = 0; j < 62; j += 2) {
            unit(j, w0, w1, true, true); LDS_BARRIER();
            unit(j + 1, w1, w0, true, true); LDS_BARRIER();
        }
        unit(62, w0, w1, true, false); LDS_BARRIER();
        unit(63, w1, w0, false, false);
        if (dead && lane == 0) *(LAS u32*)(lds + LDS_RED + 20) = 1;
        LDS_BARRIER();
        LDS_BARRIER();
        if (wave == 4 && lane == 0) {
            const u32x4 t = *(LAS u32x4*)(lds + LDS_RED);
            __hip_atomic_store((gu64*)(myws + OFF_AMAXL) + tile, ((u64)call << 32) | max(max(t.x, t.y), max(t.z, t.w)), RLX_AGT);
        }
        LDS_BARRIER();
        if (wave != 4) {
#pragma unroll 1
            for (int u = wave - 5; u < 28; u += 3) {
                if (u >= 4 * MEARLY)
                    while (__hip_atomic_load((LAS u64*)(lds + LDS_RED + 24), __ATOMIC_RELAXED, __HIP_MEMORY_SCOPE_WORKGROUP) == 0ull) __builtin_amdgcn_s_sleep(1);
                const int mwu = tile * 4 + (u & 3);
                int dd = (u >> 2) + mwu % 7; if (dd >= 7) dd -= 7;
                if (lane < 32) {
                    const u32x4 mk = *(LAS u32x4*)(lds + LDS_MSK + (u & 3) * 512 + lane * 16);
                    __builtin_amdgcn_raw_buffer_store_b128(mk, mk_rsrc(P.wsr[dd], WS_USED), (u32)lane * 16u,
                                                           OFF_MSG + (u32)(rank * 512 + mwu) * MSG, AUX_SYS);
                }
                asm volatile("s_waitcnt vmcnt(%0)" :: "n"(MWIN - 1) : "memory");
            }
            WAIT_VM0();
            __builtin_amdgcn_s_barrier();
            return;
        }
        u32 gb = 0;
        if (!dead) {
            gu64* g = lane == 8 ? (gu64*)tmo : (gu64*)(myws + OFF_AMAXG + (tile & (NREP - 1)) * 128) + (lane & 7);
            u32 spins = 0;
            for (;;) {
                const u64 a = __hip_atomic_load(g, RLX_SYS);
                gb = (u32)a;
                if ((__ballot((u32)(a >> 32) == call) & 0xffull) == 0xffull) break;
                if (spin_fail(tstart, tmo, call, spins)) { dead = true; break; }
            }
            if (rdl(gb, 8) == call) dead = true;
            gb = wave_max(lane == 8 ? 0u : gb);
        }
        if (*(LAS u32*)(lds + LDS_RED + 20) != 0u) dead = true;
        if (lane == 0) __hip_atomic_store((LAS u64*)(lds + LDS_RED + 24), ((u64)(dead ? 3u : 1u) << 32) | __float_as_uint(448.0f / __uint_as_float(gb)), __ATOMIC_RELEASE, __HIP_MEMORY_SCOPE_WORKGROUP);
        return;
    }
    bf16x8 xf[4], wfA[8], wfB[8];
    f32x16 acc[4];
#pragma unroll
    for (int j = 0; j < 4; ++j)
#pragma unroll
        for (int i = 0; i < 16; ++i) acc[j][i] = 0.f;
    u32x4 l0[2], l1[2], ha, hb;
    const u32 xl_ = xblk + (u32)lane * 16u;
    auto putX = [&](u32 h0, u32 h1, u32 la, u32 lb, bf16x8& xo) __attribute__((always_inline)) {
        u32x4 c = u32x4{perm(h0, la, 0x05010400u), perm(h0, la, 0x07030602u), perm(h1, lb, 0x05010400u), perm(h1, lb, 0x07030602u)};
        asm volatile("" : "+v"(c));
        xo = __builtin_bit_cast(bf16x8, c);
    };
    u32 so_l1 = 0;
    auto step = [&](int s, auto cur_c, auto dec_c, auto iss_c, auto is1_c) __attribute__((always_inline)) {
        constexpr int cur = decltype(cur_c)::value, nxt = cur ^ 1;
        constexpr bool DEC = decltype(dec_c)::value, ISS = decltype(iss_c)::value, IS1 = decltype(is1_c)::value;
        LAS char* pr = lds + LDS_W + cur * 16384 + 8192 + lane * 16;
#pragma unroll
        for (int q = 0; q < 8; ++q) {
            acc[q & 3] = __builtin_amdgcn_mfma_f32_32x32x16_bf16(wfA[q], xf[q >> 2], acc[q & 3], 0, 0, 0);
            SB();
            if (q < 4) { wfB[2 * q] = *(LAS bf16x8*)(pr + (2 * q) * 1024); wfB[2 * q + 1] = *(LAS bf16x8*)(pr + (2 * q + 1) * 1024); }
            if (q == 4) putX(hb.x, hb.y, l1[cur].x, l1[cur].y, xf[2]);
            if (q == 5) putX(hb.z, hb.w, l1[cur].z, l1[cur].w, xf[3]);
            if constexpr (IS1) { if (q == 6) l1[cur] = __builtin_amdgcn_raw_buffer_load_b128(rs, xl_ + 1024, so_l1, AUX_SYS); }
            SB();
        }
        if constexpr (DEC) LDS_BARRIER(); else WAIT_LGKM0();
        SB();
        pr = lds + LDS_W + nxt * 16384 + lane * 16;
        LAS char* px = lds + LDS_HX + nxt * 8192 + wq * 2048 + lane * 16;
        u32 idv = 0;
#pragma unroll
        for (int q = 0; q < 8; ++q) {
            acc[q & 3] = __builtin_amdgcn_mfma_f32_32x32x16_bf16(wfB[q], xf[2 + (q >> 2)], acc[q & 3], 0, 0, 0);
            SB();
            if constexpr (DEC) {
                if (q == 0) { ha = *(LAS u32x4*)px; hb = *(LAS u32x4*)(px + 1024); }
                if (q >= 1 && q < 5) { wfA[2 * q - 2] = *(LAS bf16x8*)(pr + (2 * q - 2) * 1024); wfA[2 * q - 1] = *(LAS bf16x8*)(pr + (2 * q - 1) * 1024); }
                if (q == 4) putX(ha.x, ha.y, l0[nxt].x, l0[nxt].y, xf[0]);
                if (q == 5) putX(ha.z, ha.w, l0[nxt].z, l0[nxt].w, xf[1]);
            }
            if constexpr (ISS) {
                if (q == 0) idv = *(LAS u32*)(ord + ((s + 3) & 3) * 32);
                if (q == 5) { so_l1 = ((rfl(idv) >> 1) & 63u) * (16u * PIECE); asm volatile("" : "+s"(so_l1)); }
                if (q == 6) l0[nxt] = __builtin_amdgcn_raw_buffer_load_b128(rs, xl_, so_l1, AUX_SYS);
            }
            if (q == 7) __builtin_amdgcn_s_waitcnt(0xC07F);
            SB();
        }
    };
    LDS_BARRIER();
#pragma unroll
    for (int k = 0; k < 2; ++k) {
        const u32 so = ((rfl(*(LAS u32*)(ord + k * 32)) >> 1) & 63u) * (16u * PIECE);
        l0[k] = __builtin_amdgcn_raw_buffer_load_b128(rs, xl_, so, AUX_SYS);
        l1[k] = __builtin_amdgcn_raw_buffer_load_b128(rs, xl_ + 1024, so, AUX_SYS);
    }
    LDS_BARRIER();
    ha = *(LAS u32x4*)(lds + LDS_HX + wq * 2048 + lane * 16); hb = *(LAS u32x4*)(lds + LDS_HX + wq * 2048 + 1024 + lane * 16);
#pragma unroll
    for (int i = 0; i < 8; ++i) wfA[i] = *(LAS bf16x8*)(lds + LDS_W + lane * 16 + i * 1024);
    putX(ha.x, ha.y, l0[0].x, l0[0].y, xf[0]);
    putX(ha.z, ha.w, l0[0].z, l0[0].w, xf[1]);
    so_l1 = ((rfl(*(LAS u32*)(ord + 64)) >> 1) & 63u) * (16u * PIECE);
    l0[0] = __builtin_amdgcn_raw_buffer_load_b128(rs, xl_, so_l1, AUX_SYS);
    WAIT_LGKM0();
    STAMP(1);
#pragma unroll 1
    for (int s = 0; s < 60; s += 2) {
        step(s, C0{}, T{}, T{}, T{});
        step(s + 1, C1{}, T{}, T{}, T{});
    }
    step(60, C0{}, T{}, T{}, T{});
    step(61, C1{}, T{}, F{}, T{});
    step(62, C0{}, T{}, F{}, F{});
    step(63, C1{}, F{}, F{}, F{});
    STAMP(5);

    float mx = 0.f;
#pragma unroll
    for (int j = 0; j < 4; ++j)
#pragma unroll
        for (int i = 0; i < 16; ++i) mx = fmaxf(mx, acc[j][i]);
    const u32 mb = wave_max(__float_as_uint(mx));
    if (lane == 0) *(LAS u32*)(lds + LDS_RED + wave * 4) = mb;
    if (dead && lane == 0) *(LAS u32*)(lds + LDS_RED + 20) = 1;
    LDS_BARRIER();
    u32 m0 = 0, m1 = 0;
#pragma unroll
    for (int i = 7; i >= 0; --i)
#pragma unroll
        for (int j = 3; j >= 0; --j) {
            m0 = __builtin_amdgcn_alignbit(m0, __float_as_uint(acc[j][i]), 31);
            m1 = __builtin_amdgcn_alignbit(m1, __float_as_uint(acc[j][i + 8]), 31);
        }
    m0 = ~m0; m1 = ~m1;
    const int mw = tile * 4 + wave, rot = mw % 7;
    const u32 moff = OFF_MSG + (u32)(rank * 512 + mw) * MSG;
    *(LAS u32x2*)(lds + LDS_MSK + wave * 512 + lane * 8) = u32x2{m0, m1};
    LDS_BARRIER();
    const u32 cnt = (u32)(__popc(m0) + __popc(m1));
    const u32 incl = wave_scan(cnt);
    const u32 total = rdl(incl, 63);
    u32 sel[16], pa[16], ps[16], bm[16];
    {
        u32 pp = (incl - cnt) * 6u;
#pragma unroll
        for (int i = 0; i < 16; ++i) {
            const u32 mm = i < 8 ? m0 : m1;
            const int sh = 4 * (i & 7);
            const u32 n4 = (sh ? (mm >> (sh - 2)) : (mm << 2)) & 0x3cu;
            sel[i] = *(LAS u32*)(lds + LDS_LUT + n4);
            bm[i] = __umul24(n4 << 4, 0x00204081u) & 0x40404040u;
            pa[i] = (pp >> 5) << 2; ps[i] = pp & 31u;
            pp += 6u * (u32)__popc(n4);
            asm volatile("" : "+v"(bm[i]), "+v"(pa[i]), "+v"(ps[i]));
        }
    }
#pragma unroll
    for (int j = 0; j < 4; ++j)
#pragma unroll
        for (int i = 0; i < 16; ++i) acc[j][i] = fmaxf(acc[j][i], 0.f);
#pragma unroll
    for (int j = 0; j < 4; ++j) asm volatile("" : "+v"(acc[j]));
    u64 gd;
    while ((gd = __hip_atomic_load((LAS u64*)(lds + LDS_RED + 24), __ATOMIC_ACQUIRE, __HIP_MEMORY_SCOPE_WORKGROUP)) == 0ull) __builtin_amdgcn_s_sleep(1);
    const float inv = __uint_as_float(rfl((u32)gd));
    dead = rfl((u32)(gd >> 32)) != 1u;
    STAMP(6);

#pragma unroll
    for (int j = 0; j < 4; ++j) acc[j] *= inv;
    const u32 etop = (rfl((u32)__builtin_amdgcn_cvt_pk_fp8_f32(__uint_as_float(mb) * inv, 0.f, 0, false)) >> 3) & 15u;
    u32 base = etop > 8u ? etop - 8u : 0u;
    u32x4 q[4];
    u32 bad0 = 0, bad1 = 0;
    {
        const u32 b8 = base * 0x08080808u;
#pragma unroll
        for (int i = 0; i < 16; ++i) {
            int w0 = __builtin_amdgcn_cvt_pk_fp8_f32(acc[0][i], acc[1][i], 0, false);
            w0 = __builtin_amdgcn_cvt_pk_fp8_f32(acc[2][i], acc[3][i], w0, true);
            q[i >> 2][i & 3] = (u32)w0 & 0x7f7f7f7fu;
            const u32 t = ((u32)w0 | 0x80808080u) - b8;
            const u32 n64 = __builtin_amdgcn_udot4(t & bm[i], 0x08040201u, 0u, false);
            const int sh = 4 * (i & 7) - 6;
            if (i < 8) bad0 |= sh < 0 ? n64 >> -sh : n64 << sh; else bad1 |= sh < 0 ? n64 >> -sh : n64 << sh;
            const u32 c = perm(t, t, sel[i]) & 0x3f3f3f3fu;
            const u32 y = bfi(0x003f003fu, c, c >> 2);
            const u32 z = bfi(0x00000fffu, y, y >> 4);
            const u64 sh64 = (u64)z << ps[i];
            LAS u32* wp = (LAS u32*)(str + pa[i]);
            __hip_atomic_fetch_or(wp, (u32)sh64, __ATOMIC_RELAXED, __HIP_MEMORY_SCOPE_WAVEFRONT);
            __hip_atomic_fetch_or(wp + 1, (u32)(sh64 >> 32), __ATOMIC_RELAXED, __HIP_MEMORY_SCOPE_WAVEFRONT);
        }
    }
    u32 nst = (total * 6u + 127u) >> 7;
    const u32 nb = (u32)(__popc(bad0) + __popc(bad1));
    const u32 nbi = wave_scan(nb);
    u32 nexc = rdl(nbi, 63), seven = 0;
    if (__builtin_expect(nexc <= 64u, 1)) {
        LAS unsigned short* lp = (LAS unsigned short*)(str + nst * 16u) + (nbi - nb);
        const u32 lb = (u32)(lane & 31) * 128u + (u32)(lane >> 5) * 16u;
        while (__ballot((bad0 | bad1) != 0u) != 0ull) {
            if ((bad0 | bad1) != 0u) {
                const u32 pb = bad0 ? (u32)__ffs((int)bad0) - 1u : (u32)__ffs((int)bad1) + 31u;
                if (bad0) bad0 &= bad0 - 1u; else bad1 &= bad1 - 1u;
                *lp++ = (unsigned short)(lb + ((pb >> 4) << 5) + (pb & 15u));
            }
        }
    } else {
        seven = 1; base = 0; nexc = 0;
        nst = (total * 7u + 127u) >> 7;
        WAIT_LGKM0();
#pragma unroll
        for (int u = 0; u < 4; ++u) *(LAS u32x4*)(str + u * 1024 + lane * 16) = u32x4{0, 0, 0, 0};
        u32 pp = (incl - cnt) * 7u;
#pragma unroll
        for (int i = 0; i < 16; ++i) {
            const u32 qq = q[i >> 2][i & 3];
            const u32 c = perm(qq, qq, sel[i]);
            const u32 y = bfi(0x007f007fu, c, c >> 1);
            const u32 z = bfi(0x00003fffu, y, y >> 2);
            const u64 sh64 = (u64)z << (pp & 31u);
            LAS u32* wp = (LAS u32*)(str + ((pp >> 5) << 2));
            __hip_atomic_fetch_or(wp, (u32)sh64, __ATOMIC_RELAXED, __HIP_MEMORY_SCOPE_WAVEFRONT);
            __hip_atomic_fetch_or(wp + 1, (u32)(sh64 >> 32), __ATOMIC_RELAXED, __HIP_MEMORY_SCOPE_WAVEFRONT);
            pp += 7u * (u32)__popc(bm[i]);
        }
    }
    const u32 nch = nst + ((nexc + 7u) >> 3);
    WAIT_LGKM0();
    u32x4 cs[4];
#pragma unroll
    for (int u = 0; u < 4; ++u) cs[u] = *(LAS u32x4*)(str + u * 1024 + lane * 16);
    STAMP(7);
    auto own_tile = [&]() __attribute__((always_inline)) {
        const u32 ooff = (u32)((rank >> 2) * 2048 + ti * 128 + wave * 32 + (lane >> 3)) * 4096u + (u32)((rank & 3) * 1024 + tj * 128 + (lane & 7) * 16);
        __amdgpu_buffer_rsrc_t ro = mk_rsrc(P.out, (u32)MM * NN);
        CBAR();
#pragma unroll
        for (int bb = 0; bb < 4; ++bb)
            *(LAS u32x4*)(str + (lane & 31) * 128 + bb * 32 + (lane >> 5) * 16) = dead ? u32x4{0x7f7f7f7fu, 0x7f7f7f7fu, 0x7f7f7f7fu, 0x7f7f7f7fu} : q[bb];
        CBAR();
#pragma unroll
        for (int bb = 0; bb < 4; ++bb)
            __builtin_amdgcn_raw_buffer_store_b128(*(LAS u32x4*)(str + bb * 1024 + lane * 16), ro, ooff + bb * 32768u, 0, AUX_OUT);
        CBAR();
    };
    if (!dead) {
        const u64 flag = ((u64)call << 32) | nch | (nst << 8) | (nexc << 16) | (base << 23) | (seven << 27);
        const u32 foff = OFF_FLAGO + (u32)mw * 128u + (u32)rank * 8u;
        if (__builtin_expect(nch <= 128u, 1)) {
            const u32 v0 = (u32)lane < nch ? 512u + (u32)lane * 16u : 0x80000000u, v1 = (u32)lane + 64u < nch ? 1536u + (u32)lane * 16u : 0x80000000u;
#pragma unroll
            for (int k = 0; k < 7; ++k) {
                int dd = k + rot; if (dd >= 7) dd -= 7;
                char* w0 = P.wsr[dd];
                __builtin_amdgcn_raw_buffer_store_b128(cs[0], mk_rsrc(w0, WS_USED), v0, moff, AUX_SYS);
                WAIT_VM0();
                __builtin_amdgcn_raw_buffer_store_b128(cs[1], mk_rsrc(w0, WS_USED), v1, moff, AUX_SYS);
                if (k == 0) { own_tile(); __builtin_amdgcn_s_barrier(); }
                WAIT_VM0();
                if (lane == 0) __hip_atomic_store((gu64*)(w0 + foff), flag, RLX_SYS);
            }
        } else {
            own_tile();
            __builtin_amdgcn_s_barrier();
#pragma unroll 1
            for (int k = 0; k < 7; ++k) {
                int dd = k + rot; if (dd >= 7) dd -= 7;
                char* w0 = P.wsr[dd];
#pragma unroll
                for (int u = 0; u < 4; ++u)
                    if ((u32)(u * 64 + lane) < nch) __builtin_amdgcn_raw_buffer_store_b128(cs[u], mk_rsrc(w0, WS_USED), 512u + u * 1024u + (u32)lane * 16u, moff, AUX_SYS);
                WAIT_VM0();
                if (lane == 0) __hip_atomic_store((gu64*)(w0 + foff), flag, RLX_SYS);
            }
        }
    } else {
        own_tile();
        __builtin_amdgcn_s_barrier();
    }
    STAMP(8);
}

extern "C" void kernel_launch(int rank, int nranks, ncclComm_t comm,
                              hipStream_t stream,
                              void* const* d_in, const int* in_sizes, int n_in,
                              void* d_out, int out_size,
                              const int* in_dtypes, int out_dtype,
                              void* const* d_out_all,
                              void* const* d_ws_all, size_t ws_bytes) {
    (void)comm; (void)in_sizes; (void)in_dtypes; (void)out_dtype; (void)d_out_all;
    if (nranks != NR || n_in < 2 || out_size != MM * NN || ws_bytes < WS_USED || (unsigned)rank >= (unsigned)NR) {
        fprintf(stderr, "unexpected problem shape\n");
        return;
    }
    Params P;
    P.x = (const char*)d_in[0];
    P.w = (const char*)d_in[1];
    for (int d = 0; d < NR; ++d) P.wsr[d] = (char*)d_ws_all[(rank + 1 + d) & 7];
    P.out = (char*)d_out;
    P.myws = (char*)d_ws_all[rank];
    P.rank = rank;
    P.pad = 0;
    fused_kernel<<<dim3(NGRID), dim3(NTHR), 0, stream>>>(P);
    HIP_CHECK(hipGetLastError());
}
```

```cpp
#include <hip/hip_runtime.h>
#include <rccl/rccl.h>
#include <cstdio>
#include <type_traits>

#define HIP_CHECK(expr) do { hipError_t _e = (expr); if (_e != hipSuccess) { \
    fprintf(stderr, "HIP error %s at %s:%d\n", hipGetErrorString(_e), __FILE__, __LINE__); \
    return; } } while (0)

typedef unsigned int u32;
typedef unsigned long long u64;
typedef u32 u32x4 __attribute__((ext_vector_type(4)));
typedef u32 u32x3 __attribute__((ext_vector_type(3)));
typedef u32 u32x2 __attribute__((ext_vector_type(2)));
typedef _Float16 f16x2 __attribute__((ext_vector_type(2)));
typedef float f32x16 __attribute__((ext_vector_type(16)));
typedef __bf16 bf16x8 __attribute__((ext_vector_type(8)));
typedef __bf16 bf16x2 __attribute__((ext_vector_type(2)));
#define GAS __attribute__((address_space(1)))
#define LAS __attribute__((address_space(3)))
typedef GAS u32 gu32;
typedef GAS u64 gu64;
typedef GAS u32x4 gu32x4;

constexpr int NR = 8;
constexpr int MM = 4096, NN = 4096;
constexpr int NGRID = 256, NTHR = 768;
constexpr u64 DEADLINE_TICKS = 50000000ull;

constexpr u32 OFF_CNT   = 0;
constexpr u32 OFF_TMO   = 32768;
constexpr u32 OFF_TICK  = 33024;
constexpr u32 OFF_AMAXL = 36864;
constexpr u32 OFF_AMAXG = 40960;
constexpr int NREP      = 8;
constexpr u32 OFF_FLAGA = 65536;
constexpr u32 OFF_FLAGB = 98304;
constexpr u32 OFF_FLAGO = 131072;
constexpr u32 OFF_DIAG  = 262144;
constexpr u32 PIECE     = 16384;
constexpr u32 OFF_A     = 1u << 20;
constexpr u32 OFF_B     = OFF_A + 64u * 16u * PIECE;
constexpr u32 MSG       = 4096;
constexpr u32 OFF_MSG   = OFF_B + 64u * 8u * PIECE;
constexpr u32 WS_USED   = OFF_MSG + 8u * 512u * MSG;

constexpr int LDS_W    = 0;
constexpr int LDS_STR  = 32768;
constexpr int LDS_MSK  = 65536;
constexpr int LDS_LUT  = 67584;
constexpr int LDS_RED  = 67712;
constexpr int LDS_ORD  = 67840;
constexpr int LDS_TAB  = 69632;
constexpr int LDS_HX   = 98304;
constexpr int LDS_SIZE = 114688;

struct Params {
    const char* x;
    const char* w;
    char* myws;
    char* out;
    int rank;
    int pad;
    char* wsr[NR];
};

#define RLX_SYS __ATOMIC_RELAXED, __HIP_MEMORY_SCOPE_SYSTEM
#define RLX_AGT __ATOMIC_RELAXED, __HIP_MEMORY_SCOPE_AGENT
#define WAIT_VM0() asm volatile("s_waitcnt vmcnt(0)" ::: "memory")
#define CBAR() asm volatile("" ::: "memory")
#define WAIT_LGKM0() do { CBAR(); __builtin_amdgcn_s_waitcnt(0xC07F); CBAR(); } while (0)
#define LDS_BARRIER() do { WAIT_LGKM0(); __builtin_amdgcn_s_barrier(); CBAR(); } while (0)
constexpr int AUX_SYS = 17;
constexpr int AUX_OUT = 16;
constexpr int NWL = 30;
constexpr int MEARLY = 5;
constexpr int MWIN = 2;
constexpr int DECW = 5;
constexpr int NBLK = 768;
constexpr u32 SEQ_SH = 17, SEQ_MASK = 0x7fffu;

__device__ __forceinline__ u64 now() { return __builtin_amdgcn_s_memrealtime(); }
__device__ __forceinline__ u32 rfl(u32 v) { return (u32)__builtin_amdgcn_readfirstlane((int)v); }
__device__ __forceinline__ u32 rdl(u32 v, int l) { return (u32)__builtin_amdgcn_readlane((int)v, l); }
__device__ __forceinline__ u32 perm(u32 a, u32 b, u32 s) { return __builtin_amdgcn_perm(a, b, s); }
__device__ __forceinline__ u32 bfi(u32 m, u32 a, u32 b) { return (a & m) | (b & ~m); }
__device__ __forceinline__ __amdgpu_buffer_rsrc_t mk_rsrc(const char* base, u32 bytes) {
    return __builtin_amdgcn_make_buffer_rsrc((void*)base, 0, bytes, 0x00020000);
}
template <int CTRL, int RMASK> __device__ __forceinline__ u32 dpp0(u32 v) {
    return (u32)__builtin_amdgcn_update_dpp(0, (int)v, CTRL, RMASK, 0xf, false);
}
__device__ __forceinline__ u32 wave_max(u32 v) {
    v = max(v, dpp0<0x111, 0xf>(v)); v = max(v, dpp0<0x112, 0xf>(v));
    v = max(v, dpp0<0x114, 0xf>(v)); v = max(v, dpp0<0x118, 0xf>(v));
    v = max(v, dpp0<0x142, 0xa>(v)); v = max(v, dpp0<0x143, 0xc>(v));
    return rdl(v, 63);
}
__device__ __forceinline__ u32 wave_scan(u32 v) {
    v += dpp0<0x111, 0xf>(v); v += dpp0<0x112, 0xf>(v);
    v += dpp0<0x114, 0xf>(v); v += dpp0<0x118, 0xf>(v);
    v += dpp0<0x142, 0xa>(v); v += dpp0<0x143, 0xc>(v);
    return v;
}

#define STAMP(i) do { } while (0)

__device__ __forceinline__ bool spin_fail(u64 t0, gu32* tmo, u32 call, u32& spins) {
    __builtin_amdgcn_s_sleep(1);
    ++spins;
    if ((spins & 63u) == 0u) {
        if (rfl(__hip_atomic_load(tmo, RLX_AGT)) == call) return true;
        if (now() - t0 > DEADLINE_TICKS) { __hip_atomic_store(tmo, call, RLX_AGT); return true; }
    }
    return false;
}

__device__ __forceinline__ u32 block_id(u32 nb, bool plain) {
    const u32 n = nb >> 2, m = plain ? n : n - 8u, g = (m * 2731u) >> 16, q = m - g * 24u, t = (q * 11u) >> 5;
    u32 ce = q == 3u * t ? (g + 1u) * 32u + t : g * 32u + 7u + q - t;
    if (n < 32u) ce = n + (n >= 24u ? 8u : 0u);
    if (n >= 176u) ce = 7u * 32u + n - 168u;
    if (plain) ce = g * 32u + q;
    return nb < (u32)NBLK ? (ce << 2) | (nb & 3u) : 0xffffffffu;
}
constexpr u32 M83 = 0x83838383u;
template <bool ISW> __device__ __forceinline__ void patch(u32 img, unsigned short ex, u64 hi, u32 vmask) {
    u32 ad, val, sh;
    asm volatile("v_lshrrev_b32 %0, %7, %3\n\tv_bfe_i32 %1, %3, %8, 5\n\tv_lshl_or_b32 %2, %3, %9, %10\n\t"
                 "v_and_or_b32 %0, %0, %6, %4\n\tv_lshlrev_b32 %1, %2, %1\n\t"
                 "s_mov_b64 exec, %5\n\tds_add_u32 %0, %1\n\ts_mov_b64 exec, -1"
                 : "=&v"(ad), "=&v"(val), "=&v"(sh)
                 : "v"(__builtin_bit_cast(_Float16, ex)), "s"(img), "s"(hi), "v"(vmask), "n"(ISW ? 4 : 5), "n"(ISW ? 1 : 2), "n"(ISW ? 4 : 3), "n"(ISW ? 10 : 2) : "memory");
}
__device__ __forceinline__ void raw_planes(u32 (&H)[8], const char* p) {
    u32x4 a, b;
    asm volatile("global_load_dwordx4 %0, %2, off offset:2048 sc0 sc1\n\tglobal_load_dwordx4 %1, %2, off offset:3072 sc0 sc1\n\ts_waitcnt vmcnt(0)"
                 : "=&v"(a), "=&v"(b) : "v"(p) : "memory");
    asm volatile("v_mov_b32 %0, %8\n\tv_mov_b32 %1, %9\n\tv_mov_b32 %2, %10\n\tv_mov_b32 %3, %11\n\t"
                 "v_mov_b32 %4, %12\n\tv_mov_b32 %5, %13\n\tv_mov_b32 %6, %14\n\tv_mov_b32 %7, %15"
                 : "+v"(H[0]), "+v"(H[1]), "+v"(H[2]), "+v"(H[3]), "+v"(H[4]), "+v"(H[5]), "+v"(H[6]), "+v"(H[7])
                 : "v"(a.x), "v"(a.y), "v"(a.z), "v"(a.w), "v"(b.x), "v"(b.y), "v"(b.z), "v"(b.w));
}
template <bool ISW> __device__ __forceinline__ void patch_more(u32 img, const char* p, u32 vmask) {
    unsigned short ex;
    asm volatile("global_load_ushort %0, %1, off offset:2944 sc0 sc1\n\ts_waitcnt vmcnt(0)" : "=&v"(ex) : "v"(p) : "memory");
    patch<ISW>(img, ex, ~0ull, vmask);
}

__global__ void __launch_bounds__(NTHR, 3) fused_kernel(Params P) {
    __shared__ __attribute__((aligned(32768))) char smem[LDS_SIZE];
    LAS char* lds = (LAS char*)smem;
    const int tid = threadIdx.x, lane = tid & 63, wave = (int)rfl((u32)tid >> 6);
    const int b = blockIdx.x, xcd = b & 7, slot = b >> 3;
    const int rank = P.rank;
    char* myws = P.myws;
    asm volatile("" :: "s"(P.x), "s"(P.w), "s"(P.wsr[slot < 16 ? slot & 7 : 7]));
    gu32* tmo = (gu32*)(myws + OFF_TMO);
    const u64 tstart = now();
    STAMP(0);
    auto get_call = [&]() __attribute__((always_inline)) {
        u32 ticket = 0;
        if (lane == 0) ticket = __hip_atomic_fetch_add((gu32*)(myws + OFF_CNT + b * 128 + wave * 8), 1u, RLX_AGT);
        return rfl(ticket) + 1u;
    };

    const bool comm = slot < 16;
    const int d = comm ? slot & 7 : 7;
    const int sw = (slot >> 3) * 8 + xcd;
    const int wl = wave * 16 + sw;
    const bool producer = d != 7 && wl < NWL;
    const int p = (rank + 1 + d) & 7;
    const int r = (rank - p) & 7;
    char* dws = P.wsr[d];
    __amdgpu_buffer_rsrc_t rs = mk_rsrc(dws, WS_USED);
    const int row0 = (p >> 2) * 2048, col0 = (p & 3) * 1024;
    const u32 xlane = (u32)(lane & 31) * 1024u + (u32)(lane >> 5) * 16u;
    const u32 wlane = (u32)((lane >> 5) * 8) * 8192u + (u32)(lane & 31) * 8u;
    u32 v[16];
    u32x2 vn[8];
    auto load_block = [&](int nb) __attribute__((always_inline)) {
        const int wb = nb & 3, c = nb >> 7, e = (nb >> 2) & 31;
        const bool isw = e < 8;
        const u32 so = isw ? (u32)((c * 64 + wb * 16) * 8192 + (col0 + ((e + r) & 7) * 128) * 2)
                           : (u32)((row0 + ((e - 8 + 2 * r) & 15) * 128 + wb * 32) * 1024 + c * 128);
        const u32 s1 = isw ? 8192u : 8u, s2 = isw ? 16384u : 32u;
        __amdgpu_buffer_rsrc_t ri = mk_rsrc(isw ? P.w : P.x, 4096u * 512u * 2u);
        const u32 vo = isw ? wlane : xlane;
#pragma unroll
        for (int k = 0; k < 8; ++k) vn[k] = __builtin_amdgcn_raw_buffer_load_b64(ri, vo, so + (k & 1) * s1 + (k >> 1) * s2, 0);
    };
    auto take_block = [&](u32 nb) __attribute__((always_inline)) {
        const u32 inc = (nb & 0x60u) == 0u ? 0xff7fff80u : 0x00800080u;
#pragma unroll
        for (int k = 0; k < 8; ++k) { v[2 * k] = vn[k].x + inc; v[2 * k + 1] = vn[k].y + inc; }
#pragma unroll
        for (int i = 0; i < 16; ++i) asm volatile("" : "+v"(v[i]));
    };
    u32x4 pk[4];
    LAS char* stg = lds + LDS_TAB + wave * 3072;
    u32 p_off = 0, p_foff = 0, p_mode = 0, p_lanes = 0;
    u32 p_flag = 0;
    auto pack = [&](int nb, bool early, u32 call) __attribute__((always_inline)) {
        const int wb = nb & 3, c = nb >> 7, e = (nb >> 2) & 31;
        const int step = rank * 8 + c;
        u32 selL, selH;
        if (e < 8) { const int j = (e + r) & 7;
            p_off = OFF_B + (u32)(step * 8 + j) * PIECE;  p_foff = OFF_FLAGB + (u32)((j * 64 + step) * 4 + wb) * 4u;
            selL = 0x06020400u; selH = 0x07030501u;
        } else { const int i = (e - 8 + 2 * r) & 15;
            p_off = OFF_A + (u32)(step * 16 + i) * PIECE; p_foff = OFF_FLAGA + (u32)((i * 64 + step) * 4 + wb) * 4u;
            selL = 0x06040200u; selH = 0x07050301u;
        }
        p_off += (u32)wb * 4096u;
        u32 H[8], D[8], mode, base, total;
        bool exact = false;
        for (;;) {
            f16x2 hi = __builtin_bit_cast(f16x2, v[0]), lo = hi;
#pragma unroll
            for (int i = 1; i < 16; ++i) {
                hi = __builtin_elementwise_maximum(hi, __builtin_bit_cast(f16x2, v[i]));
                lo = __builtin_elementwise_minimum(lo, __builtin_bit_cast(f16x2, v[i]));
            }
            const u32 h2 = __builtin_bit_cast(u32, __builtin_elementwise_maximum(hi, -lo)) & 0x7fff7fffu;
            u32 mag = max(h2 & 0xffffu, h2 >> 16);
            if (__builtin_expect(exact, 0)) {
                mag = 0;
#pragma unroll
                for (int i = 0; i < 16; ++i) mag = max(mag, max(v[i] & 0x7fffu, (v[i] >> 16) & 0x7fffu));
            }
#pragma unroll
            for (int i = 0; i < 8; ++i) {
                const u32 a = v[4 * (i >> 1) + (i & 1)], c2 = v[4 * (i >> 1) + (i & 1) + 2];
                pk[i >> 2][i & 3] = perm(c2, a, selL); H[i] = perm(c2, a, selH);
            }
            if (early) {
                STAMP(7);
                __builtin_amdgcn_raw_buffer_store_b128(pk[0], rs, (u32)lane * 16u, p_off, AUX_SYS);
                __builtin_amdgcn_raw_buffer_store_b128(pk[1], rs, (u32)lane * 16u + 1024u, p_off, AUX_SYS);
            }
            const u32 top = wave_max(mag >> 8);
            base = top > 3u ? top - 3u : 0u;
            const bool lower = base > 0u && __popcll(__ballot((mag >> 8) == top)) <= 8;
            base -= (u32)lower;
            const u32 base4 = base * 0x01010101u;
            u32 pm = 0;
#pragma unroll
            for (int i = 0; i < 8; ++i) {
                D[i] = (H[i] | 0x80808080u) - base4;
                pm |= ~(D[i] >> (7 - i)) & (0x01010101u << i);
            }
            if (lower) {
#pragma unroll
                for (int i = 0; i < 8; ++i) pm |= (D[i] >> 2 << i) & (0x01010101u << i);
            }
            *(LAS u32x4*)(stg + 1024 + lane * 16) = u32x4{D[0], D[1], D[2], D[3]};
            *(LAS u32x4*)(stg + 2048 + lane * 16) = u32x4{D[4], D[5], D[6], D[7]};
            *(LAS u32*)(stg + 768 + lane * 4) = (u32)lane * 0x01000100u + 0x00800000u;
            u32 dmin = 255;
            const u32 np = (u32)__popc(pm), npi = wave_scan(np);
            total = rdl(npi, 63);
            mode = total > 64u ? 2u : 0u;
            p_lanes = total > 64u ? 64u : total > 32u ? 56u : 52u;
            if (__builtin_expect(total <= 128u, 1)) {
                u32 E[8];
#pragma unroll
                for (int i = 0; i < 8; ++i) E[i] = bfi(0x80808080u, H[i], D[i]);
                LAS u32* cw = (LAS u32*)(stg + lane * 12);
                cw[0] = (E[0] & M83) | (__builtin_amdgcn_alignbit(E[1], E[1], 29) & 0x1c1c1c1cu) | ((E[6] << 5) & 0x60606060u);
                cw[1] = (E[2] & M83) | (__builtin_amdgcn_alignbit(E[3], E[3], 29) & 0x1c1c1c1cu) | ((E[7] << 5) & 0x60606060u);
                cw[2] = (E[4] & M83) | (__builtin_amdgcn_alignbit(E[5], E[5], 29) & 0x1c1c1c1cu) | ((E[6] >> 1) & 0x40404040u) | ((E[7] >> 2) & 0x20202020u);
                LAS unsigned short* lp = (LAS unsigned short*)(stg + 768) + (npi - np);
                while (__ballot(pm != 0u) != 0ull) {
                    if (pm != 0u) {
                        const u32 pb = (u32)__ffs((int)pm) - 1u, by = pb >> 3, pl = pb & 7u;
                        pm &= pm - 1u;
                        const u32 dw = (pl >> 2) * 256u + (u32)lane * 4u + (pl & 3u);
                        const u32 d = *(LAS unsigned char*)(stg + 1024 + dw * 4u + by);
                        const u32 d5 = (d >> 2) & 31u;
                        dmin = min(dmin, d);
                        const u32 ew = ((((pl & 1u) * 2u + (by >> 1)) << 8 | (u32)lane << 2 | pl >> 1) << 6) | (d5 << 1) | (by & 1u);
                        *lp++ = (unsigned short)(e < 8 ? ew : (dw << 7) | (d5 << 2) | by);
                    }
                }
                CBAR();
                pk[2] = *(LAS u32x4*)(stg + lane * 16);
            }
            const bool raw = total > 128u || __ballot(dmin < 64u) != 0ull;
            const bool odd = raw || top >= 0x7cu || base <= 2u || __ballot(dmin + base <= 129u) != 0ull;
            if (__builtin_expect(odd && !exact, 0)) {
                const u32 inc = e < 8 ? 0xff7fff80u : 0x00800080u;
                const float f = e < 8 ? 0.5f : 2.0f;
#pragma unroll
                for (int i = 0; i < 16; ++i) {
                    const u32 o = v[i] - inc;
                    const bf16x2 t = {(__bf16)(__uint_as_float(o << 16) * f), (__bf16)(__uint_as_float(o & 0xffff0000u) * f)};
                    v[i] = __builtin_bit_cast(u32, t);
                }
                exact = true;
                continue;
            }
            if (__builtin_expect(raw, 0)) {
                mode = 3; p_lanes = 64u;
                pk[2] = u32x4{H[0], H[1], H[2], H[3]};
            }
            break;
        }
        pk[3] = u32x4{H[4], H[5], H[6], H[7]};
        p_mode = mode;
        p_flag = (call << SEQ_SH) | (mode << 15) | (base << 8) | ((u32)step << 1) | (total > 32u && mode != 3u ? 1u : 0u);
    };

    if (__builtin_expect(slot < 16, 1)) {
        if (wave >= 8) {
            const int wi = (b << 2) | (wave & 3);
            const char* px = P.x + (size_t)(wi * 8 + (lane >> 3)) * 1024 + (lane & 7) * 16;
            const char* pw = P.w + (size_t)wi * 1024 + lane * 16;
            u32x4 t[3];
#pragma unroll 1
            for (int c = 0; c < 8; ++c) {
                t[0] = *(const gu32x4*)(pw + c * 524288);
                t[1] = *(const gu32x4*)(px + c * 128);
                asm volatile("" :: "v"(t[0]), "v"(t[1]));
                __builtin_amdgcn_s_sleep(64);
            }
            __amdgpu_buffer_rsrc_t rt = mk_rsrc(myws, WS_USED);
            const int mwt = (slot * 8 + xcd) * 4 + (wave & 3);
#pragma unroll 1
            for (int k = 6; k > 4; --k) {
                for (int i = 0; i < 1 + (wi & 3); ++i) __builtin_amdgcn_s_sleep(64);
                int dd = k + mwt % 7; if (dd >= 7) dd -= 7;
                const u32 off = OFF_MSG + (u32)(((rank - 1 - dd) & 7) * 512 + mwt) * MSG;
                t[0] = __builtin_amdgcn_raw_buffer_load_b128(rt, (u32)lane * 16u, off, AUX_SYS);
                t[1] = __builtin_amdgcn_raw_buffer_load_b128(rt, (u32)lane * 16u + 1024u, off, AUX_SYS);
                t[2] = __builtin_amdgcn_raw_buffer_load_b128(rt, (u32)(lane & 31) * 16u + 2048u, off, AUX_SYS);
#pragma unroll
                for (int u = 0; u < 3; ++u) asm volatile("" :: "v"(t[u]));
                {
                    const int q = (rank - 1 - dd) & 7, tl = slot * 8 + xcd;
                    const u32 oo = (u32)((q >> 2) * 2048 + (tl >> 3) * 128 + (wave & 3) * 32 + (lane & 31)) * 4096u + (u32)((q & 3) * 1024 + (tl & 7) * 128);
                    const u32 t1 = __builtin_amdgcn_raw_buffer_load_b32(mk_rsrc(P.out, (u32)MM * NN), oo + (lane >> 5) * 64u, 0, AUX_SYS);
                    asm volatile("" :: "v"(t1));
                }
            }
            return;
        }
        const u32 nA = block_id((u32)wl, false);
        if (producer) load_block((int)nA);
        CBAR();
        const u32 call = get_call();
        if (tid < 16) {
            u32 sel = 0, k = 0;
            for (int j = 0; j < 4; ++j) { const u32 on = (tid >> j) & 1; sel |= (on ? k : 0x0cu) << (8 * j); k += on; }
            *(LAS u32*)(lds + LDS_LUT + tid * 4) = sel;
        }
        gu32* tick = (gu32*)(myws + OFF_TICK + d * 128);
        const u32 tbase = (call - 1u) * (u32)(NBLK - 2 * NWL) - (u32)(3 * NWL);
        if (producer) {
            u32 nB = block_id((u32)(wl + NWL), false), nC = block_id((u32)(wl + 2 * NWL), false);
            take_block(nA);
            load_block((int)nB);
            pack((int)nA, true, call);
            const u32 vo = (u32)lane * 16u;
            char* fdst = nullptr;
            u32 flag = 0;
            const u32 zero = 0, one = 1;
            auto iter = [&](bool first) __attribute__((always_inline)) {
                const bool more = nB != 0xffffffffu, more2 = nC != 0xffffffffu;
                u32 tk = 0;
                if (more2) asm volatile("s_mov_b64 exec, 1\n\tglobal_atomic_add %0, %1, %2, %3 sc0\n\ts_mov_b64 exec, -1"
                                        : "=&v"(tk) : "v"(zero), "v"(one), "s"(tick) : "memory");
                if (!first && lane == 0) __hip_atomic_store((gu32*)fdst, flag, RLX_SYS);
                CBAR();
                if (!first) {
                    __builtin_amdgcn_raw_buffer_store_b128(pk[0], rs, vo, p_off, AUX_SYS);
                    __builtin_amdgcn_raw_buffer_store_b128(pk[1], rs, vo + 1024, p_off, AUX_SYS);
                }
                __builtin_amdgcn_raw_buffer_store_b128(pk[2], rs, (u32)lane >= p_lanes ? 0x80000000u : vo + 2048, p_off, AUX_SYS);
                if (p_mode == 3) __builtin_amdgcn_raw_buffer_store_b128(pk[3], rs, vo + 3072, p_off, AUX_SYS);
                CBAR();
                fdst = dws + p_foff;
                flag = p_flag;
                take_block(nB);
                if (more2) load_block((int)nC);
                CBAR();
                if (more) pack((int)nB, false, call);
                __builtin_amdgcn_s_waitcnt(0x0F70);
                asm volatile("s_waitcnt vmcnt(0)" : "+v"(tk) :: "memory");
                nB = nC;
                if (more2) nC = block_id(rfl(tk) - tbase, false);
                return more;
            };
            if (iter(true)) {
#pragma unroll 1
                while (iter(false)) { }
            }
            if (lane == 0) __hip_atomic_store((gu32*)fdst, flag, RLX_SYS);
        }
        STAMP(1);
        LDS_BARRIER();
        bool dead = false;
        if (b == 56 && wave == 0) {
            u32 spins = 0, m = 0;
            for (;;) {
                const u64 g0 = __hip_atomic_load((gu64*)(myws + OFF_AMAXL) + lane, RLX_AGT);
                const u64 g1 = __hip_atomic_load((gu64*)(myws + OFF_AMAXL) + 64 + lane, RLX_AGT);
                m = max((u32)g0, (u32)g1);
                if (__all((u32)(g0 >> 32) == call && (u32)(g1 >> 32) == call)) break;
                if (spin_fail(tstart, tmo, call, spins)) { dead = true; break; }
            }
            if (!dead) {
                m = wave_max(m);
                char* dst = P.wsr[lane & 7] + OFF_AMAXG + (lane >> 3) * 128 + rank * 8;
#pragma unroll
                for (int k = 0; k < NREP / 8; ++k)
                    __hip_atomic_store((gu64*)(dst + k * 1024), ((u64)call << 32) | m, RLX_SYS);
            }
            STAMP(2);
        }
        const int tile = slot * 8 + xcd, ti = tile >> 3, tj = tile & 7;
        const int wq = wave & 3, hf = wave >> 2;
        const int mw = tile * 4 + wq, rot = mw % 7;
        __amdgpu_buffer_rsrc_t rm = mk_rsrc(myws, WS_USED);
        __amdgpu_buffer_rsrc_t ro = mk_rsrc(P.out, (u32)MM * NN);
        gu64* fo = (gu64*)(myws + OFF_FLAGO + mw * 128) + (lane & 7);
        LAS char* str = lds + LDS_STR + wave * 4096;
        LAS char* ot = lds + LDS_W + wave * 2048;
        const int el = (lane & 32) + hf * 16 + (lane & 15), uh = (lane >> 4) & 1;
        u64 fl = 0;
#pragma unroll 1
        for (int k = 0; k < 7; ++k) {
            int dd = k + rot; if (dd >= 7) dd -= 7;
            const int q = (rank - 1 - dd) & 7;
            if (!dead && rdl((u32)(fl >> 32), q) != call) {
                u32 spins = 0;
                for (;;) {
                    fl = __hip_atomic_load(fo, RLX_SYS);
                    if (rdl((u32)(fl >> 32), q) == call) break;
                    if (spin_fail(tstart, tmo, call, spins)) { dead = true; break; }
                }
            }
            CBAR();
            const u32 ooff = (u32)((q >> 2) * 2048 + ti * 128 + wq * 32 + hf * 16 + (lane >> 3)) * 4096u
                           + (u32)((q & 3) * 1024 + tj * 128 + (lane & 7) * 16);
            if (dead) {
#pragma unroll
                for (int bb = 0; bb < 2; ++bb)
                    __builtin_amdgcn_raw_buffer_store_b128(u32x4{0x7f7f7f7fu, 0x7f7f7f7fu, 0x7f7f7f7fu, 0x7f7f7f7fu}, ro, ooff + bb * 32768u, 0, 0);
                continue;
            }
            const u32 fw = rdl((u32)fl, q);
            const u32 nch = fw & 255u;
            const u32 moff = OFF_MSG + (u32)(q * 512 + mw) * MSG;
            const u32x2 mk = __builtin_amdgcn_raw_buffer_load_b64(rm, (u32)lane * 8u, moff, AUX_SYS);
            const u32x4 c0 = __builtin_amdgcn_raw_buffer_load_b128(rm, 512u + (u32)lane * 16u, moff, AUX_SYS);
            const u32x4 c1 = __builtin_amdgcn_raw_buffer_load_b128(rm, 1536u + (u32)lane * 16u, moff, AUX_SYS);
            const u32 c_lo = (u32)__popc(mk.x), cnt = c_lo + (u32)__popc(mk.y);
            const u32 f0 = wave_scan(cnt) - cnt;
            const u32 ff = (u32)__builtin_amdgcn_ds_bpermute(el * 4, (int)(f0 | ((f0 + c_lo) << 16)));
            const u32 ma = (u32)__builtin_amdgcn_ds_bpermute(el * 4, (int)mk.x), mb = (u32)__builtin_amdgcn_ds_bpermute(el * 4, (int)mk.y);
            const u32 first = uh ? ff >> 16 : ff & 0xffffu, mm = uh ? mb : ma;
            *(LAS u32x4*)(str + lane * 16) = c0;
            *(LAS u32x4*)(str + 1024 + lane * 16) = c1;
            if (nch > 128u) {
                const u32x4 c2 = __builtin_amdgcn_raw_buffer_load_b128(rm, 2560u + (u32)lane * 16u, moff, AUX_SYS);
                *(LAS u32x4*)(str + 2048 + lane * 16) = c2;
                if (lane < 32) {
                    const u32x4 c3 = __builtin_amdgcn_raw_buffer_load_b128(rm, 3584u + (u32)lane * 16u, moff, AUX_SYS);
                    *(LAS u32x4*)(str + 3072 + lane * 16) = c3;
                }
            }
            WAIT_LGKM0();
            u32x4 o[2];
            u32 ex = 0;
            if (__builtin_expect((fw & (1u << 27)) == 0u, 1)) {
                ex = *(LAS unsigned short*)(str + ((fw >> 8) & 255u) * 16u + lane * 2);
                const u32 b8 = ((fw >> 23) & 15u) * 0x08080808u;
                u32 pos = first * 6u;
#pragma unroll
                for (int i = 0; i < 8; ++i) {
                    const int sh = 4 * i;
                    const u32 n4 = (sh ? (mm >> (sh - 2)) : (mm << 2)) & 0x3cu;
                    const u32 sel = *(LAS u32*)(lds + LDS_LUT + n4);
                    const LAS u32* wp = (const LAS u32*)(str + ((pos >> 5) << 2));
                    const u32 z = __builtin_amdgcn_alignbit(wp[1], wp[0], pos);
                    pos += 6u * (u32)__popc(n4);
                    const u32 y = bfi(0x00000fffu, z, z << 4);
                    const u32 x = (bfi(0x003f003fu, y, y << 2) & 0x3f3f3f3fu) + b8;
                    o[i >> 2][i & 3] = perm(x, x, sel);
                }
            } else {
                u32 pos = first * 7u;
#pragma unroll
                for (int i = 0; i < 8; ++i) {
                    const int sh = 4 * i;
                    const u32 n4 = (sh ? (mm >> (sh - 2)) : (mm << 2)) & 0x3cu;
                    const u32 sel = *(LAS u32*)(lds + LDS_LUT + n4);
                    const LAS u32* wp = (const LAS u32*)(str + ((pos >> 5) << 2));
                    const u32 z = __builtin_amdgcn_alignbit(wp[1], wp[0], pos);
                    pos += 7u * (u32)__popc(n4);
                    const u32 y = bfi(0x00003fffu, z, z << 2);
                    const u32 x = bfi(0x007f007fu, y, y << 1) & 0x7f7f7f7fu;
                    o[i >> 2][i & 3] = perm(x, x, sel);
                }
            }
            *(LAS u32x4*)(ot + (lane & 15) * 128 + uh * 64 + (lane >> 5) * 16) = o[0];
            *(LAS u32x4*)(ot + (lane & 15) * 128 + uh * 64 + (lane >> 5) * 16 + 32) = o[1];
            if ((u32)lane < ((fw >> 16) & 127u) && ((ex >> 11) & 1u) == (u32)hf)
                __hip_atomic_fetch_xor((LAS u32*)(ot + (ex & 0x7fcu)), 0x40u << (8u * (ex & 3u)), __ATOMIC_RELAXED, __HIP_MEMORY_SCOPE_WAVEFRONT);
            CBAR();
#pragma unroll
            for (int bb = 0; bb < 2; ++bb)
                __builtin_amdgcn_raw_buffer_store_b128(*(LAS u32x4*)(ot + bb * 1024 + lane * 16), ro, ooff + bb * 32768u, 0, AUX_OUT);
        }
        STAMP(6);
        return;
    }

    const int sl = slot - 16;
    const int ti = (xcd >> 1) * 4 + (sl >> 2), tj = (xcd & 1) * 4 + (sl & 3), tile = ti * 8 + tj;
    const bool helper = wave >= 4;
    const int wq = wave & 3;
    if (tid < 16) {
        u32 sel = 0x0c0c0c0cu, k = 0;
        for (int j = 0; j < 4; ++j) if ((tid >> j) & 1) { sel = (sel & ~(0xffu << (8 * k))) | ((u32)j << (8 * k)); ++k; }
        *(LAS u32*)(lds + LDS_LUT + tid * 4) = sel;
    }
    if (tid < 8) *(LAS u32*)(lds + LDS_RED + tid * 4) = 0;
    LAS char* str = lds + LDS_STR + wq * 4096;
    if (!helper) {
#pragma unroll
        for (int u = 0; u < 4; ++u) *(LAS u32x4*)(str + u * 1024 + lane * 16) = u32x4{0, 0, 0, 0};
    }
    u32 order = 0;
    for (int k = 0; k < 7; ++k) order |= (u32)((rank + ((ti >> 1) + 13 - k) % 7 + 1) & 7) << (4 * k);
    order = rfl(order);
    auto step_id = [&](u32 s) __attribute__((always_inline)) {
        const u32 m = s - 8u, c = (m * 37u) >> 8;
        return s < 8u ? (u32)rank * 8u + s : ((order >> ((m - c * 7u) * 4u)) & 7u) * 8u + c;
    };
    const u32 stepid = step_id((u32)lane);
    const u32 nown = block_id((u32)((b - 128) * 6 + wave), true);
    if (wave < 6) load_block((int)nown);
    CBAR();
    const u32 call = get_call();
    const u32 seq = call & SEQ_MASK;
    if (wave < 6) {
        take_block(nown);
        pack((int)nown, true, call);
        __builtin_amdgcn_raw_buffer_store_b128(pk[2], rs, (u32)lane >= p_lanes ? 0x80000000u : (u32)lane * 16u + 2048u, p_off, AUX_SYS);
        if (p_mode == 3) __builtin_amdgcn_raw_buffer_store_b128(pk[3], rs, (u32)lane * 16u + 3072u, p_off, AUX_SYS);
        WAIT_VM0();
        if (lane == 0) __hip_atomic_store((gu32*)(myws + p_foff), p_flag, RLX_SYS);
    }
    bool dead = false;
    const u32 xblk = OFF_A + (u32)ti * PIECE + (u32)wq * 4096u;
    LAS char* ord = lds + LDS_ORD;
#define SB() __builtin_amdgcn_sched_barrier(0)
    using T = std::true_type; using F = std::false_type;
    using C0 = std::integral_constant<int, 0>; using C1 = std::integral_constant<int, 1>;
    if (helper) {
        auto lanes = [&](u32 f) __attribute__((always_inline)) { return ((u64)((f & 1u) ? 0xffffffffu : 0u) << 32) | ((f & 0x8000u) ? 0u : 0xffffffffu); };
        auto planes = [&](u32 fhi, const u32x3 c, u32 blk, u32 (&H)[8]) __attribute__((always_inline)) {
            const u32 base4 = ((fhi >> 8) & 127u) * 0x01010101u;
#pragma unroll
            for (int q = 0; q < 3; ++q) { H[2 * q] = c[q] & M83; H[2 * q + 1] = __builtin_amdgcn_alignbit(c[q], c[q], 3) & M83; }
            H[6] = bfi(0x80808080u, c[2] << 1, c[0] >> 5) & M83;
            H[7] = bfi(0x80808080u, c[2] << 2, c[1] >> 5) & M83;
            SB();
#pragma unroll
            for (int q = 0; q < 8; ++q) H[q] += base4;
            if (__builtin_expect((fhi & 0x8000u) != 0u, 0)) raw_planes(H, myws + (blk + (u32)lane * 16u));
        };
        u32 vmask = wave >= 8 ? 0x7fcu : 0xffcu;
        asm volatile("" : "+v"(vmask));
        const u32 fo = OFF_FLAGA + (u32)ti * 1024u + stepid * 16u, go = OFF_FLAGB + (u32)tj * 1024u + stepid * 16u;
        u32x4 fv = {0, 0, 0, 0}, gv = {0, 0, 0, 0}, pv, pg;
        u64 todo = 0, done = 0;
        const u32 want = seq << SEQ_SH;
        auto take = [&](const u32x4 v, const u32x4 g) __attribute__((always_inline)) {
            fv = v; gv = g;
            const u32 df = ((v.x ^ want) | (v.y ^ want) | (v.z ^ want) | (v.w ^ want)) | ((g.x ^ want) | (g.y ^ want) | (g.z ^ want) | (g.w ^ want));
            todo = __ballot((df >> SEQ_SH) == 0u) & ~done;
        };
        auto pick = [&](int slot_) __attribute__((always_inline)) {
            const int id = __ffsll((long long)todo) - 1;
            done |= 1ull << id; todo &= todo - 1ull;
            if (lane == id) { *(LAS u32x4*)(ord + (slot_ & 3) * 32) = fv; *(LAS u32x4*)(ord + (slot_ & 3) * 32 + 16) = gv; }
        };
        auto peek = [&](int slot_) __attribute__((always_inline)) {
            bool later = false;
            if (wave == DECW) {
                later = todo == 0ull;
                if (later) { pv = __builtin_amdgcn_raw_buffer_load_b128(rs, fo, 0, AUX_SYS); pg = __builtin_amdgcn_raw_buffer_load_b128(rs, go, 0, AUX_SYS); }
                else pick(slot_);
            }
            SB();
            return later;
        };
        auto choose = [&](int slot_, bool peeked) __attribute__((always_inline)) {
            if (peeked) take(pv, pg);
            if (todo == 0ull) {
                u32 spins = 0;
                for (;;) {
                    const u32x4 v = __builtin_amdgcn_raw_buffer_load_b128(rs, fo, 0, AUX_SYS), g = __builtin_amdgcn_raw_buffer_load_b128(rs, go, 0, AUX_SYS);
                    take(v, g);
                    CBAR();
                    if (todo != 0ull) break;
                    if (spin_fail(tstart, tmo, call, spins)) { dead = true; fv = gv = u32x4{0, 0, 0, 0}; todo = ~done; break; }
                }
            }
            pick(slot_);
        };
        if (wave == DECW) { choose(0, false); choose(1, false); }
        LDS_BARRIER();
        if (wave >= 8) {
            struct XC { u32x3 c; unsigned short ex; u32 f, so; };
            XC x0, x1;
            const u32 xc_ = xblk + 2048u + (u32)lane * 12u, xe_ = xblk + 2816u + (u32)lane * 2u;
            u32 img = (u32)(uintptr_t)(lds + LDS_HX + wq * 2048);
            auto ldx = [&](int slot_, XC& x) __attribute__((always_inline)) {
                x.f = rfl(*(LAS u32*)(ord + (slot_ & 3) * 32 + wq * 4));
                x.so = ((x.f >> 1) & 63u) * (16u * PIECE);
                x.c = __builtin_amdgcn_raw_buffer_load_b96(rs, xc_, x.so, AUX_SYS);
                x.ex = __builtin_amdgcn_raw_buffer_load_b16(rs, xe_, x.so, AUX_SYS);
                SB();
            };
            auto unit = [&](int j, XC& x, XC& xn, bool ld) __attribute__((always_inline)) {
                if (ld) ldx(j + 1, xn);
                u32 H[8];
                planes(x.f, x.c, xblk + x.so, H);
                *(LAS u32x4*)(uintptr_t)(img + lane * 16) = u32x4{H[0], H[1], H[2], H[3]};
                *(LAS u32x4*)(uintptr_t)(img + 1024 + lane * 16) = u32x4{H[4], H[5], H[6], H[7]};
                patch<false>(img, x.ex, lanes(x.f), vmask);
                if (__builtin_expect((x.f & 0x18000u) == 0x10000u, 0)) patch_more<false>(img, myws + (xblk + (u32)lane * 2u + x.so), vmask);
                img ^= 8192u;
            };
            ldx(0, x0);
#pragma unroll 1
            for (int j = 0; j < 62; j += 2) {
                unit(j, x0, x1, true); LDS_BARRIER();
                unit(j + 1, x1, x0, true); LDS_BARRIER();
            }
            unit(62, x0, x1, true); LDS_BARRIER();
            unit(63, x1, x0, false);
            if (dead && lane == 0) *(LAS u32*)(lds + LDS_RED + 20) = 1;
            LDS_BARRIER();
            return;
        }
        struct WB { u32x4 l0, l1; u32x3 c; unsigned short ex; u32 f, so; };
        WB w0, w1;
        const u32 wblk = OFF_B + (u32)tj * PIECE + (u32)wq * 4096u;
        const u32 wl_ = wblk + (u32)lane * 16u, wc_ = wblk + 2048u + (u32)lane * 12u, we_ = wblk + 2816u + (u32)lane * 2u;
        u32 img = (u32)(uintptr_t)(lds + LDS_W + wq * 4096);
        auto unit = [&](int j, WB& w, WB& wn, bool ld, bool ch) __attribute__((always_inline)) {
            const bool pk_ = ch && peek(j + 2);
            u32 fn = 0;
            if (ld) fn = *(LAS u32*)(ord + ((j + 1) & 3) * 32 + 16 + wq * 4);
            SB();
            u32 H[8];
            planes(w.f, w.c, wblk + w.so, H);
            SB();
            if (ld) {
                wn.f = rfl(fn); wn.so = ((wn.f >> 1) & 63u) * (8u * PIECE);
                wn.c = __builtin_amdgcn_raw_buffer_load_b96(rs, wc_, wn.so, AUX_SYS);
                wn.l0 = __builtin_amdgcn_raw_buffer_load_b128(rs, wl_, wn.so, AUX_SYS);
            }
            SB();
            u32x4 c[4];
            LAS char* pw = (LAS char*)(uintptr_t)(img + lane * 16);
#pragma unroll
            for (int q = 0; q < 4; ++q) {
                const u32 sl_ = (q & 1) ? 0x07030602u : 0x05010400u;
                const int xy = q >> 1;
                c[q] = u32x4{perm(H[xy], w.l0[xy], sl_), perm(H[2 + xy], w.l0[2 + xy], sl_),
                             perm(H[4 + xy], w.l1[xy], sl_), perm(H[6 + xy], w.l1[2 + xy], sl_)};
                if (q) *(LAS u32x4*)(pw + (q - 1) * 1024) = c[q - 1];
                SB();
            }
            *(LAS u32x4*)(pw + 3 * 1024) = c[3];
            patch<true>(img, w.ex, lanes(w.f), vmask);
            if (__builtin_expect((w.f & 0x18000u) == 0x10000u, 0)) patch_more<true>(img, myws + (wblk + (u32)lane * 2u + w.so), vmask);
            img ^= 16384u;
            SB();
            if (ld) {
                wn.l1 = __builtin_amdgcn_raw_buffer_load_b128(rs, wl_ + 1024, wn.so, AUX_SYS);
                wn.ex = __builtin_amdgcn_raw_buffer_load_b16(rs, we_, wn.so, AUX_SYS);
            }
            if (pk_) choose(j + 2, true);
        };
        w0.f = rfl(*(LAS u32*)(ord + 16 + wq * 4)); w0.so = ((w0.f >> 1) & 63u) * (8u * PIECE);
        w0.c = __builtin_amdgcn_raw_buffer_load_b96(rs, wc_, w0.so, AUX_SYS);
        w0.l0 = __builtin_amdgcn_raw_buffer_load_b128(rs, wl_, w0.so, AUX_SYS);
        w0.l1 = __builtin_amdgcn_raw_buffer_load_b128(rs, wl_ + 1024, w0.so, AUX_SYS);
        w0.ex = __builtin_amdgcn_raw_buffer_load_b16(rs, we_, w0.so, AUX_SYS);
#pragma unroll 1
        for (int j = 0; j < 62; j += 2) {
            unit(j, w0, w1, true, true); LDS_BARRIER();
            unit(j + 1, w1, w0, true, true); LDS_BARRIER();
        }
        unit(62, w0, w1, true, false); LDS_BARRIER();
        unit(63, w1, w0, false, false);
        if (dead && lane == 0) *(LAS u32*)(lds + LDS_RED + 20) = 1;
        LDS_BARRIER();
        LDS_BARRIER();
        if (wave == 4 && lane == 0) {
            const u32x4 t = *(LAS u32x4*)(lds + LDS_RED);
            __hip_atomic_store((gu64*)(myws + OFF_AMAXL) + tile, ((u64)call << 32) | max(max(t.x, t.y), max(t.z, t.w)), RLX_AGT);
        }
        LDS_BARRIER();
        if (wave != 4) {
#pragma unroll 1
            for (int u = wave - 5; u < 28; u += 3) {
                if (u >= 4 * MEARLY)
                    while (__hip_atomic_load((LAS u64*)(lds + LDS_RED + 24), __ATOMIC_RELAXED, __HIP_MEMORY_SCOPE_WORKGROUP) == 0ull) __builtin_amdgcn_s_sleep(1);
                const int mwu = tile * 4 + (u & 3);
                int dd = (u >> 2) + mwu % 7; if (dd >= 7) dd -= 7;
                if (lane < 32) {
                    const u32x4 mk = *(LAS u32x4*)(lds + LDS_MSK + (u & 3) * 512 + lane * 16);
                    __builtin_amdgcn_raw_buffer_store_b128(mk, mk_rsrc(P.wsr[dd], WS_USED), (u32)lane * 16u,
                                                           OFF_MSG + (u32)(rank * 512 + mwu) * MSG, AUX_SYS);
                }
                asm volatile("s_waitcnt vmcnt(%0)" :: "n"(MWIN - 1) : "memory");
            }
            WAIT_VM0();
            __builtin_amdgcn_s_barrier();
            return;
        }
        u32 gb = 0;
        if (!dead) {
            gu64* g = lane == 8 ? (gu64*)tmo : (gu64*)(myws + OFF_AMAXG + (tile & (NREP - 1)) * 128) + (lane & 7);
            u32 spins = 0;
            for (;;) {
                const u64 a = __hip_atomic_load(g, RLX_SYS);
                gb = (u32)a;
                if ((__ballot((u32)(a >> 32) == call) & 0xffull) == 0xffull) break;
                if (spin_fail(tstart, tmo, call, spins)) { dead = true; break; }
            }
            if (rdl(gb, 8) == call) dead = true;
            gb = wave_max(lane == 8 ? 0u : gb);
        }
        if (*(LAS u32*)(lds + LDS_RED + 20) != 0u) dead = true;
        if (lane == 0) __hip_atomic_store((LAS u64*)(lds + LDS_RED + 24), ((u64)(dead ? 3u : 1u) << 32) | __float_as_uint(448.0f / __uint_as_float(gb)), __ATOMIC_RELEASE, __HIP_MEMORY_SCOPE_WORKGROUP);
        return;
    }
    bf16x8 xf[4], wfA[8], wfB[8];
    f32x16 acc[4];
#pragma unroll
    for (int j = 0; j < 4; ++j)
#pragma unroll
        for (int i = 0; i < 16; ++i) acc[j][i] = 0.f;
    u32x4 l0[2], l1[2], ha, hb;
    const u32 xl_ = xblk + (u32)lane * 16u;
    auto putX = [&](u32 h0, u32 h1, u32 la, u32 lb, bf16x8& xo) __attribute__((always_inline)) {
        u32x4 c = u32x4{perm(h0, la, 0x05010400u), perm(h1, lb, 0x05010400u), perm(h0, la, 0x07030602u), perm(h1, lb, 0x07030602u)};
        asm volatile("" : "+v"(c));
        xo = __builtin_bit_cast(bf16x8, c);
    };
    u32 so_l1 = 0;
    auto step = [&](int s, auto cur_c, auto dec_c, auto iss_c, auto is1_c) __attribute__((always_inline)) {
        constexpr int cur = decltype(cur_c)::value, nxt = cur ^ 1;
        constexpr bool DEC = decltype(dec_c)::value, ISS = decltype(iss_c)::value, IS1 = decltype(is1_c)::value;
        LAS char* pr = lds + LDS_W + cur * 16384 + 8192 + lane * 16;
#pragma unroll
        for (int q = 0; q < 8; ++q) {
            acc[q & 3] = __builtin_amdgcn_mfma_f32_32x32x16_bf16(wfA[q], xf[q >> 2], acc[q & 3], 0, 0, 0);
            SB();
            if (q < 4) { wfB[2 * q] = *(LAS bf16x8*)(pr + (2 * q) * 1024); wfB[2 * q + 1] = *(LAS bf16x8*)(pr + (2 * q + 1) * 1024); }
            if (q == 4) putX(hb.x, hb.y, l1[cur].x, l1[cur].y, xf[2]);
            if (q == 5) putX(hb.z, hb.w, l1[cur].z, l1[cur].w, xf[3]);
            if constexpr (IS1) { if (q == 6) l1[cur] = __builtin_amdgcn_raw_buffer_load_b128(rs, xl_ + 1024, so_l1, AUX_SYS); }
            SB();
        }
        if constexpr (DEC) LDS_BARRIER(); else WAIT_LGKM0();
        SB();
        pr = lds + LDS_W + nxt * 16384 + lane * 16;
        LAS char* px = lds + LDS_HX + nxt * 8192 + wq * 2048 + lane * 16;
        u32 idv = 0;
#pragma unroll
        for (int q = 0; q < 8; ++q) {
            acc[q & 3] = __builtin_amdgcn_mfma_f32_32x32x16_bf16(wfB[q], xf[2 + (q >> 2)], acc[q & 3], 0, 0, 0);
            SB();
            if constexpr (DEC) {
                if (q == 0) { ha = *(LAS u32x4*)px; hb = *(LAS u32x4*)(px + 1024); }
                if (q >= 1 && q < 5) { wfA[2 * q - 2] = *(LAS bf16x8*)(pr + (2 * q - 2) * 1024); wfA[2 * q - 1] = *(LAS bf16x8*)(pr + (2 * q - 1) * 1024); }
                if (q == 4) putX(ha.x, ha.y, l0[nxt].x, l0[nxt].y, xf[0]);
                if (q == 5) putX(ha.z, ha.w, l0[nxt].z, l0[nxt].w, xf[1]);
            }
            if constexpr (ISS) {
                if (q == 0) idv = *(LAS u32*)(ord + ((s + 3) & 3) * 32);
                if (q == 5) { so_l1 = ((rfl(idv) >> 1) & 63u) * (16u * PIECE); asm volatile("" : "+s"(so_l1)); }
                if (q == 6) l0[nxt] = __builtin_amdgcn_raw_buffer_load_b128(rs, xl_, so_l1, AUX_SYS);
            }
            if (q == 7) __builtin_amdgcn_s_waitcnt(0xC07F);
            SB();
        }
    };
    LDS_BARRIER();
#pragma unroll
    for (int k = 0; k < 2; ++k) {
        const u32 so = ((rfl(*(LAS u32*)(ord + k * 32)) >> 1) & 63u) * (16u * PIECE);
        l0[k] = __builtin_amdgcn_raw_buffer_load_b128(rs, xl_, so, AUX_SYS);
        l1[k] = __builtin_amdgcn_raw_buffer_load_b128(rs, xl_ + 1024, so, AUX_SYS);
    }
    LDS_BARRIER();
    ha = *(LAS u32x4*)(lds + LDS_HX + wq * 2048 + lane * 16); hb = *(LAS u32x4*)(lds + LDS_HX + wq * 2048 + 1024 + lane * 16);
#pragma unroll
    for (int i = 0; i < 8; ++i) wfA[i] = *(LAS bf16x8*)(lds + LDS_W + lane * 16 + i * 1024);
    putX(ha.x, ha.y, l0[0].x, l0[0].y, xf[0]);
    putX(ha.z, ha.w, l0[0].z, l0[0].w, xf[1]);
    so_l1 = ((rfl(*(LAS u32*)(ord + 64)) >> 1) & 63u) * (16u * PIECE);
    l0[0] = __builtin_amdgcn_raw_buffer_load_b128(rs, xl_, so_l1, AUX_SYS);
    WAIT_LGKM0();
    STAMP(1);
#pragma unroll 1
    for (int s = 0; s < 60; s += 2) {
        step(s, C0{}, T{}, T{}, T{});
        step(s + 1, C1{}, T{}, T{}, T{});
    }
    step(60, C0{}, T{}, T{}, T{});
    step(61, C1{}, T{}, F{}, T{});
    step(62, C0{}, T{}, F{}, F{});
    step(63, C1{}, F{}, F{}, F{});
    STAMP(5);

    float mx = 0.f;
#pragma unroll
    for (int j = 0; j < 4; ++j)
#pragma unroll
        for (int i = 0; i < 16; ++i) mx = fmaxf(mx, acc[j][i]);
    const u32 mb = wave_max(__float_as_uint(mx));
    if (lane == 0) *(LAS u32*)(lds + LDS_RED + wave * 4) = mb;
    if (dead && lane == 0) *(LAS u32*)(lds + LDS_RED + 20) = 1;
    LDS_BARRIER();
    u32 m0 = 0, m1 = 0;
#pragma unroll
    for (int i = 7; i >= 0; --i)
#pragma unroll
        for (int j = 3; j >= 0; --j) {
            m0 = __builtin_amdgcn_alignbit(m0, __float_as_uint(acc[j][i]), 31);
            m1 = __builtin_amdgcn_alignbit(m1, __float_as_uint(acc[j][i + 8]), 31);
        }
    m0 = ~m0; m1 = ~m1;
    const int mw = tile * 4 + wave, rot = mw % 7;
    const u32 moff = OFF_MSG + (u32)(rank * 512 + mw) * MSG;
    *(LAS u32x2*)(lds + LDS_MSK + wave * 512 + lane * 8) = u32x2{m0, m1};
    LDS_BARRIER();
    const u32 cnt = (u32)(__popc(m0) + __popc(m1));
    const u32 incl = wave_scan(cnt);
    const u32 total = rdl(incl, 63);
    u32 sel[16], pa[16], ps[16], bm[16];
    {
        u32 pp = (incl - cnt) * 6u;
#pragma unroll
        for (int i = 0; i < 16; ++i) {
            const u32 mm = i < 8 ? m0 : m1;
            const int sh = 4 * (i & 7);
            const u32 n4 = (sh ? (mm >> (sh - 2)) : (mm << 2)) & 0x3cu;
            sel[i] = *(LAS u32*)(lds + LDS_LUT + n4);
            bm[i] = __umul24(n4 << 4, 0x00204081u) & 0x40404040u;
            pa[i] = (pp >> 5) << 2; ps[i] = pp & 31u;
            pp += 6u * (u32)__popc(n4);
            asm volatile("" : "+v"(bm[i]), "+v"(pa[i]), "+v"(ps[i]));
        }
    }
#pragma unroll
    for (int j = 0; j < 4; ++j)
#pragma unroll
        for (int i = 0; i < 16; ++i) acc[j][i] = fmaxf(acc[j][i], 0.f);
#pragma unroll
    for (int j = 0; j < 4; ++j) asm volatile("" : "+v"(acc[j]));
    u64 gd;
    while ((gd = __hip_atomic_load((LAS u64*)(lds + LDS_RED + 24), __ATOMIC_ACQUIRE, __HIP_MEMORY_SCOPE_WORKGROUP)) == 0ull) __builtin_amdgcn_s_sleep(1);
    const float inv = __uint_as_float(rfl((u32)gd));
    dead = rfl((u32)(gd >> 32)) != 1u;
    STAMP(6);

#pragma unroll
    for (int j = 0; j < 4; ++j) acc[j] *= inv;
    const u32 etop = (rfl((u32)__builtin_amdgcn_cvt_pk_fp8_f32(__uint_as_float(mb) * inv, 0.f, 0, false)) >> 3) & 15u;
    u32 base = etop > 8u ? etop - 8u : 0u;
    u32x4 q[4];
    u32 bad0 = 0, bad1 = 0;
    {
        const u32 b8 = base * 0x08080808u;
#pragma unroll
        for (int i = 0; i < 16; ++i) {
            int w0 = __builtin_amdgcn_cvt_pk_fp8_f32(acc[0][i], acc[1][i], 0, false);
            w0 = __builtin_amdgcn_cvt_pk_fp8_f32(acc[2][i], acc[3][i], w0, true);
            q[i >> 2][i & 3] = (u32)w0 & 0x7f7f7f7fu;
            const u32 t = ((u32)w0 | 0x80808080u) - b8;
            const u32 n64 = __builtin_amdgcn_udot4(t & bm[i], 0x08040201u, 0u, false);
            const int sh = 4 * (i & 7) - 6;
            if (i < 8) bad0 |= sh < 0 ? n64 >> -sh : n64 << sh; else bad1 |= sh < 0 ? n64 >> -sh : n64 << sh;
            const u32 c = perm(t, t, sel[i]) & 0x3f3f3f3fu;
            const u32 y = bfi(0x003f003fu, c, c >> 2);
            const u32 z = bfi(0x00000fffu, y, y >> 4);
            const u64 sh64 = (u64)z << ps[i];
            LAS u32* wp = (LAS u32*)(str + pa[i]);
            __hip_atomic_fetch_or(wp, (u32)sh64, __ATOMIC_RELAXED, __HIP_MEMORY_SCOPE_WAVEFRONT);
            __hip_atomic_fetch_or(wp + 1, (u32)(sh64 >> 32), __ATOMIC_RELAXED, __HIP_MEMORY_SCOPE_WAVEFRONT);
        }
    }
    u32 nst = (total * 6u + 127u) >> 7;
    const u32 nb = (u32)(__popc(bad0) + __popc(bad1));
    const u32 nbi = wave_scan(nb);
    u32 nexc = rdl(nbi, 63), seven = 0;
    if (__builtin_expect(nexc <= 64u, 1)) {
        LAS unsigned short* lp = (LAS unsigned short*)(str + nst * 16u) + (nbi - nb);
        const u32 lb = (u32)(lane & 31) * 128u + (u32)(lane >> 5) * 16u;
        while (__ballot((bad0 | bad1) != 0u) != 0ull) {
            if ((bad0 | bad1) != 0u) {
                const u32 pb = bad0 ? (u32)__ffs((int)bad0) - 1u : (u32)__ffs((int)bad1) + 31u;
                if (bad0) bad0 &= bad0 - 1u; else bad1 &= bad1 - 1u;
                *lp++ = (unsigned short)(lb + ((pb >> 4) << 5) + (pb & 15u));
            }
        }
    } else {
        seven = 1; base = 0; nexc = 0;
        nst = (total * 7u + 127u) >> 7;
        WAIT_LGKM0();
#pragma unroll
        for (int u = 0; u < 4; ++u) *(LAS u32x4*)(str + u * 1024 + lane * 16) = u32x4{0, 0, 0, 0};
        u32 pp = (incl - cnt) * 7u;
#pragma unroll
        for (int i = 0; i < 16; ++i) {
            const u32 qq = q[i >> 2][i & 3];
            const u32 c = perm(qq, qq, sel[i]);
            const u32 y = bfi(0x007f007fu, c, c >> 1);
            const u32 z = bfi(0x00003fffu, y, y >> 2);
            const u64 sh64 = (u64)z << (pp & 31u);
            LAS u32* wp = (LAS u32*)(str + ((pp >> 5) << 2));
            __hip_atomic_fetch_or(wp, (u32)sh64, __ATOMIC_RELAXED, __HIP_MEMORY_SCOPE_WAVEFRONT);
            __hip_atomic_fetch_or(wp + 1, (u32)(sh64 >> 32), __ATOMIC_RELAXED, __HIP_MEMORY_SCOPE_WAVEFRONT);
            pp += 7u * (u32)__popc(bm[i]);
        }
    }
    const u32 nch = nst + ((nexc + 7u) >> 3);
    WAIT_LGKM0();
    u32x4 cs[4];
#pragma unroll
    for (int u = 0; u < 4; ++u) cs[u] = *(LAS u32x4*)(str + u * 1024 + lane * 16);
    STAMP(7);
    auto own_tile = [&]() __attribute__((always_inline)) {
        const u32 ooff = (u32)((rank >> 2) * 2048 + ti * 128 + wave * 32 + (lane >> 3)) * 4096u + (u32)((rank & 3) * 1024 + tj * 128 + (lane & 7) * 16);
        __amdgpu_buffer_rsrc_t ro = mk_rsrc(P.out, (u32)MM * NN);
        CBAR();
#pragma unroll
        for (int bb = 0; bb < 4; ++bb)
            *(LAS u32x4*)(str + (lane & 31) * 128 + bb * 32 + (lane >> 5) * 16) = dead ? u32x4{0x7f7f7f7fu, 0x7f7f7f7fu, 0x7f7f7f7fu, 0x7f7f7f7fu} : q[bb];
        CBAR();
#pragma unroll
        for (int bb = 0; bb < 4; ++bb)
            __builtin_amdgcn_raw_buffer_store_b128(*(LAS u32x4*)(str + bb * 1024 + lane * 16), ro, ooff + bb * 32768u, 0, AUX_OUT);
        CBAR();
    };
    if (!dead) {
        const u64 flag = ((u64)call << 32) | nch | (nst << 8) | (nexc << 16) | (base << 23) | (seven << 27);
        const u32 foff = OFF_FLAGO + (u32)mw * 128u + (u32)rank * 8u;
        if (__builtin_expect(nch <= 128u, 1)) {
            const u32 v0 = (u32)lane < nch ? 512u + (u32)lane * 16u : 0x80000000u, v1 = (u32)lane + 64u < nch ? 1536u + (u32)lane * 16u : 0x80000000u;
#pragma unroll
            for (int k = 0; k < 7; ++k) {
                int dd = k + rot; if (dd >= 7) dd -= 7;
                char* w0 = P.wsr[dd];
                __builtin_amdgcn_raw_buffer_store_b128(cs[0], mk_rsrc(w0, WS_USED), v0, moff, AUX_SYS);
                WAIT_VM0();
                __builtin_amdgcn_raw_buffer_store_b128(cs[1], mk_rsrc(w0, WS_USED), v1, moff, AUX_SYS);
                if (k == 0) { own_tile(); __builtin_amdgcn_s_barrier(); }
                WAIT_VM0();
                if (lane == 0) __hip_atomic_store((gu64*)(w0 + foff), flag, RLX_SYS);
            }
        } else {
            own_tile();
            __builtin_amdgcn_s_barrier();
#pragma unroll 1
            for (int k = 0; k < 7; ++k) {
                int dd = k + rot; if (dd >= 7) dd -= 7;
                char* w0 = P.wsr[dd];
#pragma unroll
                for (int u = 0; u < 4; ++u)
                    if ((u32)(u * 64 + lane) < nch) __builtin_amdgcn_raw_buffer_store_b128(cs[u], mk_rsrc(w0, WS_USED), 512u + u * 1024u + (u32)lane * 16u, moff, AUX_SYS);
                WAIT_VM0();
                if (lane == 0) __hip_atomic_store((gu64*)(w0 + foff), flag, RLX_SYS);
            }
        }
    } else {
        own_tile();
        __builtin_amdgcn_s_barrier();
    }
    STAMP(8);
}

extern "C" void kernel_launch(int rank, int nranks, ncclComm_t comm,
                              hipStream_t stream,
                              void* const* d_in, const int* in_sizes, int n_in,
                              void* d_out, int out_size,
                              const int* in_dtypes, int out_dtype,
                              void* const* d_out_all,
                              void* const* d_ws_all, size_t ws_bytes) {
    (void)comm; (void)in_sizes; (void)in_dtypes; (void)out_dtype; (void)d_out_all;
    if (nranks != NR || n_in < 2 || out_size != MM * NN || ws_bytes < WS_USED || (unsigned)rank >= (unsigned)NR) {
        fprintf(stderr, "unexpected problem shape\n");
        return;
    }
    Params P;
    P.x = (const char*)d_in[0];
    P.w = (const char*)d_in[1];
    for (int d = 0; d < NR; ++d) P.wsr[d] = (char*)d_ws_all[(rank + 1 + d) & 7];
    P.out = (char*)d_out;
    P.myws = (char*)d_ws_all[rank];
    P.rank = rank;
    P.pad = 0;
    fused_kernel<<<dim3(NGRID), dim3(NTHR), 0, stream>>>(P);
    HIP_CHECK(hipGetLastError());
}
```

```cpp
#include <hip/hip_runtime.h>
#include <rccl/rccl.h>
#include <cstdio>
#include <type_traits>

#define HIP_CHECK(expr) do { hipError_t _e = (expr); if (_e != hipSuccess) { \
    fprintf(stderr, "HIP error %s at %s:%d\n", hipGetErrorString(_e), __FILE__, __LINE__); \
    return; } } while (0)

typedef unsigned int u32;
typedef unsigned long long u64;
typedef u32 u32x4 __attribute__((ext_vector_type(4)));
typedef u32 u32x3 __attribute__((ext_vector_type(3)));
typedef u32 u32x2 __attribute__((ext_vector_type(2)));
typedef _Float16 f16x2 __attribute__((ext_vector_type(2)));
typedef float f32x16 __attribute__((ext_vector_type(16)));
typedef __bf16 bf16x8 __attribute__((ext_vector_type(8)));
typedef __bf16 bf16x2 __attribute__((ext_vector_type(2)));
#define GAS __attribute__((address_space(1)))
#define LAS __attribute__((address_space(3)))
typedef GAS u32 gu32;
typedef GAS u64 gu64;
typedef GAS u32x4 gu32x4;

constexpr int NR = 8;
constexpr int MM = 4096, NN = 4096;
constexpr int NGRID = 256, NTHR = 768;
constexpr u64 DEADLINE_TICKS = 50000000ull;

constexpr u32 OFF_CNT   = 0;
constexpr u32 OFF_TMO   = 32768;
constexpr u32 OFF_TICK  = 33024;
constexpr u32 OFF_AMAXL = 36864;
constexpr u32 OFF_AMAXG = 40960;
constexpr int NREP      = 8;
constexpr u32 OFF_FLAGA = 65536;
constexpr u32 OFF_FLAGB = 98304;
constexpr u32 OFF_FLAGO = 131072;
constexpr u32 OFF_DIAG  = 262144;
constexpr u32 PIECE     = 16384;
constexpr u32 OFF_A     = 1u << 20;
constexpr u32 OFF_B     = OFF_A + 64u * 16u * PIECE;
constexpr u32 MSG       = 4096;
constexpr u32 OFF_MSG   = OFF_B + 64u * 8u * PIECE;
constexpr u32 WS_USED   = OFF_MSG + 8u * 512u * MSG;

constexpr int LDS_W    = 0;
constexpr int LDS_STR  = 32768;
constexpr int LDS_MSK  = 65536;
constexpr int LDS_LUT  = 67584;
constexpr int LDS_RED  = 67712;
constexpr int LDS_ORD  = 67840;
constexpr int LDS_TAB  = 69632;
constexpr int LDS_HX   = 98304;
constexpr int LDS_SIZE = 114688;

struct Params {
    const char* x;
    const char* w;
    char* myws;
    char* out;
    int rank;
    int pad;
    char* wsr[NR];
};

#define RLX_SYS __ATOMIC_RELAXED, __HIP_MEMORY_SCOPE_SYSTEM
#define RLX_AGT __ATOMIC_RELAXED, __HIP_MEMORY_SCOPE_AGENT
#define WAIT_VM0() asm volatile("s_waitcnt vmcnt(0)" ::: "memory")
#define CBAR() asm volatile("" ::: "memory")
#define WAIT_LGKM0() do { CBAR(); __builtin_amdgcn_s_waitcnt(0xC07F); CBAR(); } while (0)
#define LDS_BARRIER() do { WAIT_LGKM0(); __builtin_amdgcn_s_barrier(); CBAR(); } while (0)
constexpr int AUX_SYS = 17;
constexpr int AUX_OUT = 16;
constexpr int NWL = 32;
constexpr u32 PER = 156;
constexpr u32 PER2 = 100;
constexpr int MEARLY = 5;
constexpr int MWIN = 2;
constexpr int DECW = 5;
constexpr int NBLK = 768;
constexpr u32 SEQ_SH = 17, SEQ_MASK = 0x7fffu;

__device__ __forceinline__ u64 now() { return __builtin_amdgcn_s_memrealtime(); }
__device__ __forceinline__ u32 rfl(u32 v) { return (u32)__builtin_amdgcn_readfirstlane((int)v); }
__device__ __forceinline__ u32 rdl(u32 v, int l) { return (u32)__builtin_amdgcn_readlane((int)v, l); }
__device__ __forceinline__ u32 perm(u32 a, u32 b, u32 s) { return __builtin_amdgcn_perm(a, b, s); }
__device__ __forceinline__ u32 bfi(u32 m, u32 a, u32 b) { return (a & m) | (b & ~m); }
__device__ __forceinline__ __amdgpu_buffer_rsrc_t mk_rsrc(const char* base, u32 bytes) {
    return __builtin_amdgcn_make_buffer_rsrc((void*)base, 0, bytes, 0x00020000);
}
template <int CTRL, int RMASK> __device__ __forceinline__ u32 dpp0(u32 v) {
    return (u32)__builtin_amdgcn_update_dpp(0, (int)v, CTRL, RMASK, 0xf, false);
}
__device__ __forceinline__ u32 wave_max(u32 v) {
    v = max(v, dpp0<0x111, 0xf>(v)); v = max(v, dpp0<0x112, 0xf>(v));
    v = max(v, dpp0<0x114, 0xf>(v)); v = max(v, dpp0<0x118, 0xf>(v));
    v = max(v, dpp0<0x142, 0xa>(v)); v = max(v, dpp0<0x143, 0xc>(v));
    return rdl(v, 63);
}
__device__ __forceinline__ u32 wave_scan(u32 v) {
    v += dpp0<0x111, 0xf>(v); v += dpp0<0x112, 0xf>(v);
    v += dpp0<0x114, 0xf>(v); v += dpp0<0x118, 0xf>(v);
    v += dpp0<0x142, 0xa>(v); v += dpp0<0x143, 0xc>(v);
    return v;
}

#define STAMP(i) do { } while (0)

__device__ __forceinline__ bool spin_fail(u64 t0, gu32* tmo, u32 call, u32& spins) {
    __builtin_amdgcn_s_sleep(1);
    ++spins;
    if ((spins & 63u) == 0u) {
        const u64 dt = now() - t0;
        if (dt > 100000ull) {
            if (rfl(__hip_atomic_load(tmo, RLX_AGT)) == call) return true;
            if (dt > DEADLINE_TICKS) { __hip_atomic_store(tmo, call, RLX_AGT); return true; }
        }
    }
    return false;
}

__device__ __forceinline__ void pace(u32& due, u32 per) {
    u32 t;
    while ((t = (u32)now()) - due > 0x80000000u) __builtin_amdgcn_s_sleep(1);
    due = t + per;
}

__device__ __forceinline__ u32 block_id(u32 nb, bool plain) {
    const u32 n = nb >> 2, m = plain ? n : n - 8u, g = (m * 2731u) >> 16, q = m - g * 24u, t = (q * 11u) >> 5;
    u32 ce = q == 3u * t ? (g + 1u) * 32u + t : g * 32u + 7u + q - t;
    if (n < 32u) ce = n + (n >= 24u ? 8u : 0u);
    if (n >= 176u) ce = 7u * 32u + n - 168u;
    if (plain) ce = g * 32u + q;
    return nb < (u32)NBLK ? (ce << 2) | (nb & 3u) : 0xffffffffu;
}
constexpr u32 M83 = 0x83838383u;
template <bool ISW> __device__ __forceinline__ void patch(u32 img, unsigned short ex, u64 hi, u32 vmask) {
    u32 ad, val, sh;
    asm volatile("v_lshrrev_b32 %0, %7, %3\n\tv_bfe_i32 %1, %3, %8, 5\n\tv_lshl_or_b32 %2, %3, %9, %10\n\t"
                 "v_and_or_b32 %0, %0, %6, %4\n\tv_lshlrev_b32 %1, %2, %1\n\t"
                 "s_mov_b64 exec, %5\n\tds_add_u32 %0, %1\n\ts_mov_b64 exec, -1"
                 : "=&v"(ad), "=&v"(val), "=&v"(sh)
                 : "v"(__builtin_bit_cast(_Float16, ex)), "s"(img), "s"(hi), "v"(vmask), "n"(ISW ? 4 : 5), "n"(ISW ? 1 : 2), "n"(ISW ? 4 : 3), "n"(ISW ? 10 : 2) : "memory");
}
__device__ __forceinline__ void raw_planes(u32 (&H)[8], const char* p) {
    u32x4 a, b;
    asm volatile("global_load_dwordx4 %0, %2, off offset:2048 sc0 sc1\n\tglobal_load_dwordx4 %1, %2, off offset:3072 sc0 sc1\n\ts_waitcnt vmcnt(0)"
                 : "=&v"(a), "=&v"(b) : "v"(p) : "memory");
    asm volatile("v_mov_b32 %0, %8\n\tv_mov_b32 %1, %9\n\tv_mov_b32 %2, %10\n\tv_mov_b32 %3, %11\n\t"
                 "v_mov_b32 %4, %12\n\tv_mov_b32 %5, %13\n\tv_mov_b32 %6, %14\n\tv_mov_b32 %7, %15"
                 : "+v"(H[0]), "+v"(H[1]), "+v"(H[2]), "+v"(H[3]), "+v"(H[4]), "+v"(H[5]), "+v"(H[6]), "+v"(H[7])
                 : "v"(a.x), "v"(a.y), "v"(a.z), "v"(a.w), "v"(b.x), "v"(b.y), "v"(b.z), "v"(b.w));
}
template <bool ISW> __device__ __forceinline__ void patch_more(u32 img, const char* p, u32 vmask) {
    unsigned short ex;
    asm volatile("global_load_ushort %0, %1, off offset:2944 sc0 sc1\n\ts_waitcnt vmcnt(0)" : "=&v"(ex) : "v"(p) : "memory");
    patch<ISW>(img, ex, ~0ull, vmask);
}

__global__ void __launch_bounds__(NTHR, 3) fused_kernel(Params P) {
    __shared__ __attribute__((aligned(32768))) char smem[LDS_SIZE];
    LAS char* lds = (LAS char*)smem;
    const int tid = threadIdx.x, lane = tid & 63, wave = (int)rfl((u32)tid >> 6);
    const int b = blockIdx.x, xcd = b & 7, slot = b >> 3;
    const int rank = P.rank;
    char* myws = P.myws;
    asm volatile("" :: "s"(P.x), "s"(P.w), "s"(P.wsr[slot < 16 ? slot & 7 : 7]));
    gu32* tmo = (gu32*)(myws + OFF_TMO);
    const u64 tstart = now();
    STAMP(0);
    auto get_call = [&]() __attribute__((always_inline)) {
        u32 ticket = 0;
        if (lane == 0) ticket = __hip_atomic_fetch_add((gu32*)(myws + OFF_CNT + b * 128 + wave * 8), 1u, RLX_AGT);
        return rfl(ticket) + 1u;
    };

    const bool comm = slot < 16;
    const int d = comm ? slot & 7 : 7;
    const int sw = (slot >> 3) * 8 + xcd;
    const int wl = wave * 16 + sw;
    const bool producer = d != 7 && wl < NWL;
    const int p = (rank + 1 + d) & 7;
    const int r = (rank - p) & 7;
    char* dws = P.wsr[d];
    __amdgpu_buffer_rsrc_t rs = mk_rsrc(dws, WS_USED);
    const int row0 = (p >> 2) * 2048, col0 = (p & 3) * 1024;
    const u32 xlane = (u32)(lane & 31) * 1024u + (u32)(lane >> 5) * 16u;
    const u32 wlane = (u32)((lane >> 5) * 8) * 8192u + (u32)(lane & 31) * 8u;
    u32 v[16];
    u32x2 vn[8];
    auto load_block = [&](int nb) __attribute__((always_inline)) {
        const int wb = nb & 3, c = nb >> 7, e = (nb >> 2) & 31;
        const bool isw = e < 8;
        const u32 so = isw ? (u32)((c * 64 + wb * 16) * 8192 + (col0 + ((e + r) & 7) * 128) * 2)
                           : (u32)((row0 + ((e - 8 + 2 * r) & 15) * 128 + wb * 32) * 1024 + c * 128);
        const u32 s1 = isw ? 8192u : 8u, s2 = isw ? 16384u : 32u;
        __amdgpu_buffer_rsrc_t ri = mk_rsrc(isw ? P.w : P.x, 4096u * 512u * 2u);
        const u32 vo = isw ? wlane : xlane;
#pragma unroll
        for (int k = 0; k < 8; ++k) vn[k] = __builtin_amdgcn_raw_buffer_load_b64(ri, vo, so + (k & 1) * s1 + (k >> 1) * s2, 0);
    };
    auto take_block = [&](u32 nb) __attribute__((always_inline)) {
        const u32 inc = (nb & 0x60u) == 0u ? 0xff7fff80u : 0x00800080u;
#pragma unroll
        for (int k = 0; k < 8; ++k) { v[2 * k] = vn[k].x + inc; v[2 * k + 1] = vn[k].y + inc; }
#pragma unroll
        for (int i = 0; i < 16; ++i) asm volatile("" : "+v"(v[i]));
    };
    u32x4 pk[4];
    LAS char* stg = lds + LDS_TAB + wave * 3072;
    u32 p_off = 0, p_foff = 0, p_mode = 0, p_lanes = 0;
    u32 p_flag = 0;
    auto pack = [&](int nb, bool early, u32 call) __attribute__((always_inline)) {
        const int wb = nb & 3, c = nb >> 7, e = (nb >> 2) & 31;
        const int step = rank * 8 + c;
        u32 selL, selH;
        if (e < 8) { const int j = (e + r) & 7;
            p_off = OFF_B + (u32)(step * 8 + j) * PIECE;  p_foff = OFF_FLAGB + (u32)((j * 64 + step) * 4 + wb) * 4u;
            selL = 0x06020400u; selH = 0x07030501u;
        } else { const int i = (e - 8 + 2 * r) & 15;
            p_off = OFF_A + (u32)(step * 16 + i) * PIECE; p_foff = OFF_FLAGA + (u32)((i * 64 + step) * 4 + wb) * 4u;
            selL = 0x06040200u; selH = 0x07050301u;
        }
        p_off += (u32)wb * 4096u;
        u32 H[8], D[8], mode, base, total;
        bool exact = false;
        for (;;) {
            f16x2 hi = __builtin_bit_cast(f16x2, v[0]), lo = hi;
#pragma unroll
            for (int i = 1; i < 16; ++i) {
                hi = __builtin_elementwise_maximum(hi, __builtin_bit_cast(f16x2, v[i]));
                lo = __builtin_elementwise_minimum(lo, __builtin_bit_cast(f16x2, v[i]));
            }
            const u32 h2 = __builtin_bit_cast(u32, __builtin_elementwise_maximum(hi, -lo)) & 0x7fff7fffu;
            u32 mag = max(h2 & 0xffffu, h2 >> 16);
            if (__builtin_expect(exact, 0)) {
                mag = 0;
#pragma unroll
                for (int i = 0; i < 16; ++i) mag = max(mag, max(v[i] & 0x7fffu, (v[i] >> 16) & 0x7fffu));
            }
#pragma unroll
            for (int i = 0; i < 8; ++i) {
                const u32 a = v[4 * (i >> 1) + (i & 1)], c2 = v[4 * (i >> 1) + (i & 1) + 2];
                pk[i >> 2][i & 3] = perm(c2, a, selL); H[i] = perm(c2, a, selH);
            }
            if (early) {
                STAMP(7);
                __builtin_amdgcn_raw_buffer_store_b128(pk[0], rs, (u32)lane * 16u, p_off, AUX_SYS);
                __builtin_amdgcn_raw_buffer_store_b128(pk[1], rs, (u32)lane * 16u + 1024u, p_off, AUX_SYS);
            }
            const u32 top = wave_max(mag >> 8);
            base = top > 3u ? top - 3u : 0u;
            const bool lower = base > 0u && __popcll(__ballot((mag >> 8) == top)) <= 8;
            base -= (u32)lower;
            const u32 base4 = base * 0x01010101u;
            u32 pm = 0;
#pragma unroll
            for (int i = 0; i < 8; ++i) {
                D[i] = (H[i] | 0x80808080u) - base4;
                pm |= ~(D[i] >> (7 - i)) & (0x01010101u << i);
            }
            if (lower) {
#pragma unroll
                for (int i = 0; i < 8; ++i) pm |= (D[i] >> 2 << i) & (0x01010101u << i);
            }
            *(LAS u32x4*)(stg + 1024 + lane * 16) = u32x4{D[0], D[1], D[2], D[3]};
            *(LAS u32x4*)(stg + 2048 + lane * 16) = u32x4{D[4], D[5], D[6], D[7]};
            *(LAS u32*)(stg + 768 + lane * 4) = (u32)lane * 0x01000100u + 0x00800000u;
            u32 dmin = 255;
            const u32 np = (u32)__popc(pm), npi = wave_scan(np);
            total = rdl(npi, 63);
            mode = total > 64u ? 2u : 0u;
            p_lanes = total > 64u ? 64u : total > 32u ? 56u : 52u;
            if (__builtin_expect(total <= 128u, 1)) {
                u32 E[8];
#pragma unroll
                for (int i = 0; i < 8; ++i) E[i] = bfi(0x80808080u, H[i], D[i]);
                LAS u32* cw = (LAS u32*)(stg + lane * 12);
                cw[0] = (E[0] & M83) | (__builtin_amdgcn_alignbit(E[1], E[1], 29) & 0x1c1c1c1cu) | ((E[6] << 5) & 0x60606060u);
                cw[1] = (E[2] & M83) | (__builtin_amdgcn_alignbit(E[3], E[3], 29) & 0x1c1c1c1cu) | ((E[7] << 5) & 0x60606060u);
                cw[2] = (E[4] & M83) | (__builtin_amdgcn_alignbit(E[5], E[5], 29) & 0x1c1c1c1cu) | ((E[6] >> 1) & 0x40404040u) | ((E[7] >> 2) & 0x20202020u);
                LAS unsigned short* lp = (LAS unsigned short*)(stg + 768) + (npi - np);
                while (__ballot(pm != 0u) != 0ull) {
                    if (pm != 0u) {
                        const u32 pb = (u32)__ffs((int)pm) - 1u, by = pb >> 3, pl = pb & 7u;
                        pm &= pm - 1u;
                        const u32 dw = (pl >> 2) * 256u + (u32)lane * 4u + (pl & 3u);
                        const u32 d = *(LAS unsigned char*)(stg + 1024 + dw * 4u + by);
                        const u32 d5 = (d >> 2) & 31u;
                        dmin = min(dmin, d);
                        const u32 ew = ((((pl & 1u) * 2u + (by >> 1)) << 8 | (u32)lane << 2 | pl >> 1) << 6) | (d5 << 1) | (by & 1u);
                        *lp++ = (unsigned short)(e < 8 ? ew : (dw << 7) | (d5 << 2) | by);
                    }
                }
                CBAR();
                pk[2] = *(LAS u32x4*)(stg + lane * 16);
            }
            const bool raw = total > 128u || __ballot(dmin < 64u) != 0ull;
            const bool odd = raw || top >= 0x7cu || base <= 2u || __ballot(dmin + base <= 129u) != 0ull;
            if (__builtin_expect(odd && !exact, 0)) {
                const u32 inc = e < 8 ? 0xff7fff80u : 0x00800080u;
                const float f = e < 8 ? 0.5f : 2.0f;
#pragma unroll
                for (int i = 0; i < 16; ++i) {
                    const u32 o = v[i] - inc;
                    const bf16x2 t = {(__bf16)(__uint_as_float(o << 16) * f), (__bf16)(__uint_as_float(o & 0xffff0000u) * f)};
                    v[i] = __builtin_bit_cast(u32, t);
                }
                exact = true;
                continue;
            }
            if (__builtin_expect(raw, 0)) {
                mode = 3; p_lanes = 64u;
                pk[2] = u32x4{H[0], H[1], H[2], H[3]};
            }
            break;
        }
        pk[3] = u32x4{H[4], H[5], H[6], H[7]};
        p_mode = mode;
        p_flag = (call << SEQ_SH) | (mode << 15) | (base << 8) | ((u32)step << 1) | (total > 32u && mode != 3u ? 1u : 0u);
    };

    if (__builtin_expect(slot < 16, 1)) {
        if (wave >= 8) {
            const int wi = (b << 2) | (wave & 3);
            const char* px = P.x + (size_t)(wi * 8 + (lane >> 3)) * 1024 + (lane & 7) * 16;
            const char* pw = P.w + (size_t)wi * 1024 + lane * 16;
            u32x4 t[3];
#pragma unroll 1
            for (int c = 0; c < 8; ++c) {
                t[0] = *(const gu32x4*)(pw + c * 524288);
                t[1] = *(const gu32x4*)(px + c * 128);
                asm volatile("" :: "v"(t[0]), "v"(t[1]));
                __builtin_amdgcn_s_sleep(64);
            }
            __amdgpu_buffer_rsrc_t rt = mk_rsrc(myws, WS_USED);
            const int mwt = (slot * 8 + xcd) * 4 + (wave & 3);
#pragma unroll 1
            for (int k = 6; k > 4; --k) {
                for (int i = 0; i < 1 + (wi & 3); ++i) __builtin_amdgcn_s_sleep(64);
                int dd = k + mwt % 7; if (dd >= 7) dd -= 7;
                const u32 off = OFF_MSG + (u32)(((rank - 1 - dd) & 7) * 512 + mwt) * MSG;
                t[0] = __builtin_amdgcn_raw_buffer_load_b128(rt, (u32)lane * 16u, off, AUX_SYS);
                t[1] = __builtin_amdgcn_raw_buffer_load_b128(rt, (u32)lane * 16u + 1024u, off, AUX_SYS);
                t[2] = __builtin_amdgcn_raw_buffer_load_b128(rt, (u32)(lane & 31) * 16u + 2048u, off, AUX_SYS);
#pragma unroll
                for (int u = 0; u < 3; ++u) asm volatile("" :: "v"(t[u]));
                {
                    const int q = (rank - 1 - dd) & 7, tl = slot * 8 + xcd;
                    const u32 oo = (u32)((q >> 2) * 2048 + (tl >> 3) * 128 + (wave & 3) * 32 + (lane & 31)) * 4096u + (u32)((q & 3) * 1024 + (tl & 7) * 128);
                    const u32 t1 = __builtin_amdgcn_raw_buffer_load_b32(mk_rsrc(P.out, (u32)MM * NN), oo + (lane >> 5) * 64u, 0, AUX_SYS);
                    asm volatile("" :: "v"(t1));
                }
            }
            return;
        }
        const u32 nA = block_id((u32)wl, false);
        if (producer) load_block((int)nA);
        CBAR();
        const u32 call = get_call();
        if (tid < 16) {
            u32 sel = 0, k = 0;
            for (int j = 0; j < 4; ++j) { const u32 on = (tid >> j) & 1; sel |= (on ? k : 0x0cu) << (8 * j); k += on; }
            *(LAS u32*)(lds + LDS_LUT + tid * 4) = sel;
        }
        gu32* tick = (gu32*)(myws + OFF_TICK + d * 128);
        const u32 tbase = (call - 1u) * (u32)(NBLK - 2 * NWL) - (u32)(3 * NWL);
        if (producer) {
            u32 nB = block_id((u32)(wl + NWL), false), nC = block_id((u32)(wl + 2 * NWL), false);
            take_block(nA);
            load_block((int)nB);
            pack((int)nA, true, call);
            const u32 vo = (u32)lane * 16u;
            char* fdst = nullptr;
            u32 flag = 0;
            const u32 zero = 0, one = 1;
            u32 due = (u32)tstart;
            auto iter = [&](bool first) __attribute__((always_inline)) {
                const bool more = nB != 0xffffffffu, more2 = nC != 0xffffffffu;
                u32 tk = 0;
                if (more2) asm volatile("s_mov_b64 exec, 1\n\tglobal_atomic_add %0, %1, %2, %3 sc0\n\ts_mov_b64 exec, -1"
                                        : "=&v"(tk) : "v"(zero), "v"(one), "s"(tick) : "memory");
                if (!first && lane == 0) __hip_atomic_store((gu32*)fdst, flag, RLX_SYS);
                CBAR();
                pace(due, PER);
                if (!first) {
                    __builtin_amdgcn_raw_buffer_store_b128(pk[0], rs, vo, p_off, AUX_SYS);
                    __builtin_amdgcn_raw_buffer_store_b128(pk[1], rs, vo + 1024, p_off, AUX_SYS);
                }
                __builtin_amdgcn_raw_buffer_store_b128(pk[2], rs, (u32)lane >= p_lanes ? 0x80000000u : vo + 2048, p_off, AUX_SYS);
                if (p_mode == 3) __builtin_amdgcn_raw_buffer_store_b128(pk[3], rs, vo + 3072, p_off, AUX_SYS);
                CBAR();
                fdst = dws + p_foff;
                flag = p_flag;
                take_block(nB);
                if (more2) load_block((int)nC);
                CBAR();
                if (more) pack((int)nB, false, call);
                __builtin_amdgcn_s_waitcnt(0x0F70);
                asm volatile("s_waitcnt vmcnt(0)" : "+v"(tk) :: "memory");
                nB = nC;
                if (more2) nC = block_id(rfl(tk) - tbase, false);
                return more;
            };
            if (iter(true)) {
#pragma unroll 1
                while (iter(false)) { }
            }
            if (lane == 0) __hip_atomic_store((gu32*)fdst, flag, RLX_SYS);
        }
        STAMP(1);
        LDS_BARRIER();
        bool dead = false;
        if (b == 56 && wave == 0) {
            u32 spins = 0, m = 0;
            for (;;) {
                const u64 g0 = __hip_atomic_load((gu64*)(myws + OFF_AMAXL) + lane, RLX_AGT);
                const u64 g1 = __hip_atomic_load((gu64*)(myws + OFF_AMAXL) + 64 + lane, RLX_AGT);
                m = max((u32)g0, (u32)g1);
                if (__all((u32)(g0 >> 32) == call && (u32)(g1 >> 32) == call)) break;
                if (spin_fail(tstart, tmo, call, spins)) { dead = true; break; }
            }
            if (!dead) {
                m = wave_max(m);
                char* dst = P.wsr[lane & 7] + OFF_AMAXG + (lane >> 3) * 128 + rank * 8;
#pragma unroll
                for (int k = 0; k < NREP / 8; ++k)
                    __hip_atomic_store((gu64*)(dst + k * 1024), ((u64)call << 32) | m, RLX_SYS);
            }
            STAMP(2);
        }
        const int tile = slot * 8 + xcd, ti = tile >> 3, tj = tile & 7;
        const int wq = wave & 3, hf = wave >> 2;
        const int mw = tile * 4 + wq, rot = mw % 7;
        __amdgpu_buffer_rsrc_t rm = mk_rsrc(myws, WS_USED);
        __amdgpu_buffer_rsrc_t ro = mk_rsrc(P.out, (u32)MM * NN);
        gu64* fo = (gu64*)(myws + OFF_FLAGO + mw * 128) + (lane & 7);
        LAS char* str = lds + LDS_STR + wave * 4096;
        LAS char* ot = lds + LDS_W + wave * 2048;
        const int el = (lane & 32) + hf * 16 + (lane & 15), uh = (lane >> 4) & 1;
        u64 fl = 0;
#pragma unroll 1
        for (int k = 0; k < 7; ++k) {
            int dd = k + rot; if (dd >= 7) dd -= 7;
            const int q = (rank - 1 - dd) & 7;
            if (!dead && rdl((u32)(fl >> 32), q) != call) {
                u32 spins = 0;
                for (;;) {
                    fl = __hip_atomic_load(fo, RLX_SYS);
                    if (rdl((u32)(fl >> 32), q) == call) break;
                    if (spin_fail(tstart, tmo, call, spins)) { dead = true; break; }
                }
            }
            CBAR();
            const u32 ooff = (u32)((q >> 2) * 2048 + ti * 128 + wq * 32 + hf * 16 + (lane >> 3)) * 4096u
                           + (u32)((q & 3) * 1024 + tj * 128 + (lane & 7) * 16);
            if (dead) {
#pragma unroll
                for (int bb = 0; bb < 2; ++bb)
                    __builtin_amdgcn_raw_buffer_store_b128(u32x4{0x7f7f7f7fu, 0x7f7f7f7fu, 0x7f7f7f7fu, 0x7f7f7f7fu}, ro, ooff + bb * 32768u, 0, 0);
                continue;
            }
            const u32 fw = rdl((u32)fl, q);
            const u32 nch = fw & 255u;
            const u32 moff = OFF_MSG + (u32)(q * 512 + mw) * MSG;
            const u32x2 mk = __builtin_amdgcn_raw_buffer_load_b64(rm, (u32)lane * 8u, moff, AUX_SYS);
            const u32x4 c0 = __builtin_amdgcn_raw_buffer_load_b128(rm, 512u + (u32)lane * 16u, moff, AUX_SYS);
            const u32x4 c1 = __builtin_amdgcn_raw_buffer_load_b128(rm, 1536u + (u32)lane * 16u, moff, AUX_SYS);
            const u32 c_lo = (u32)__popc(mk.x), cnt = c_lo + (u32)__popc(mk.y);
            const u32 f0 = wave_scan(cnt) - cnt;
            const u32 ff = (u32)__builtin_amdgcn_ds_bpermute(el * 4, (int)(f0 | ((f0 + c_lo) << 16)));
            const u32 ma = (u32)__builtin_amdgcn_ds_bpermute(el * 4, (int)mk.x), mb = (u32)__builtin_amdgcn_ds_bpermute(el * 4, (int)mk.y);
            const u32 first = uh ? ff >> 16 : ff & 0xffffu, mm = uh ? mb : ma;
            *(LAS u32x4*)(str + lane * 16) = c0;
            *(LAS u32x4*)(str + 1024 + lane * 16) = c1;
            if (nch > 128u) {
                const u32x4 c2 = __builtin_amdgcn_raw_buffer_load_b128(rm, 2560u + (u32)lane * 16u, moff, AUX_SYS);
                *(LAS u32x4*)(str + 2048 + lane * 16) = c2;
                if (lane < 32) {
                    const u32x4 c3 = __builtin_amdgcn_raw_buffer_load_b128(rm, 3584u + (u32)lane * 16u, moff, AUX_SYS);
                    *(LAS u32x4*)(str + 3072 + lane * 16) = c3;
                }
            }
            WAIT_LGKM0();
            u32x4 o[2];
            u32 ex = 0;
            if (__builtin_expect((fw & (1u << 27)) == 0u, 1)) {
                ex = *(LAS unsigned short*)(str + ((fw >> 8) & 255u) * 16u + lane * 2);
                const u32 b8 = ((fw >> 23) & 15u) * 0x08080808u;
                u32 pos = first * 6u;
#pragma unroll
                for (int i = 0; i < 8; ++i) {
                    const int sh = 4 * i;
                    const u32 n4 = (sh ? (mm >> (sh - 2)) : (mm << 2)) & 0x3cu;
                    const u32 sel = *(LAS u32*)(lds + LDS_LUT + n4);
                    const LAS u32* wp = (const LAS u32*)(str + ((pos >> 5) << 2));
                    const u32 z = __builtin_amdgcn_alignbit(wp[1], wp[0], pos);
                    pos += 6u * (u32)__popc(n4);
                    const u32 y = bfi(0x00000fffu, z, z << 4);
                    const u32 x = (bfi(0x003f003fu, y, y << 2) & 0x3f3f3f3fu) + b8;
                    o[i >> 2][i & 3] = perm(x, x, sel);
                }
            } else {
                u32 pos = first * 7u;
#pragma unroll
                for (int i = 0; i < 8; ++i) {
                    const int sh = 4 * i;
                    const u32 n4 = (sh ? (mm >> (sh - 2)) : (mm << 2)) & 0x3cu;
                    const u32 sel = *(LAS u32*)(lds + LDS_LUT + n4);
                    const LAS u32* wp = (const LAS u32*)(str + ((pos >> 5) << 2));
                    const u32 z = __builtin_amdgcn_alignbit(wp[1], wp[0], pos);
                    pos += 7u * (u32)__popc(n4);
                    const u32 y = bfi(0x00003fffu, z, z << 2);
                    const u32 x = bfi(0x007f007fu, y, y << 1) & 0x7f7f7f7fu;
                    o[i >> 2][i & 3] = perm(x, x, sel);
                }
            }
            *(LAS u32x4*)(ot + (lane & 15) * 128 + uh * 64 + (lane >> 5) * 16) = o[0];
            *(LAS u32x4*)(ot + (lane & 15) * 128 + uh * 64 + (lane >> 5) * 16 + 32) = o[1];
            if ((u32)lane < ((fw >> 16) & 127u) && ((ex >> 11) & 1u) == (u32)hf)
                __hip_atomic_fetch_xor((LAS u32*)(ot + (ex & 0x7fcu)), 0x40u << (8u * (ex & 3u)), __ATOMIC_RELAXED, __HIP_MEMORY_SCOPE_WAVEFRONT);
            CBAR();
#pragma unroll
            for (int bb = 0; bb < 2; ++bb)
                __builtin_amdgcn_raw_buffer_store_b128(*(LAS u32x4*)(ot + bb * 1024 + lane * 16), ro, ooff + bb * 32768u, 0, AUX_OUT);
        }
        STAMP(6);
        return;
    }

    const int sl = slot - 16;
    const int ti = (xcd >> 1) * 4 + (sl >> 2), tj = (xcd & 1) * 4 + (sl & 3), tile = ti * 8 + tj;
    const bool helper = wave >= 4;
    const int wq = wave & 3;
    if (tid < 16) {
        u32 sel = 0x0c0c0c0cu, k = 0;
        for (int j = 0; j < 4; ++j) if ((tid >> j) & 1) { sel = (sel & ~(0xffu << (8 * k))) | ((u32)j << (8 * k)); ++k; }
        *(LAS u32*)(lds + LDS_LUT + tid * 4) = sel;
    }
    if (tid < 8) *(LAS u32*)(lds + LDS_RED + tid * 4) = 0;
    LAS char* str = lds + LDS_STR + wq * 4096;
    if (!helper) {
#pragma unroll
        for (int u = 0; u < 4; ++u) *(LAS u32x4*)(str + u * 1024 + lane * 16) = u32x4{0, 0, 0, 0};
    }
    u32 order = 0;
    for (int k = 0; k < 7; ++k) order |= (u32)((rank + ((ti >> 1) + 13 - k) % 7 + 1) & 7) << (4 * k);
    order = rfl(order);
    auto step_id = [&](u32 s) __attribute__((always_inline)) {
        const u32 m = s - 8u, c = (m * 37u) >> 8;
        return s < 8u ? (u32)rank * 8u + s : ((order >> ((m - c * 7u) * 4u)) & 7u) * 8u + c;
    };
    const u32 stepid = step_id((u32)lane);
    const u32 nown = block_id((u32)((b - 128) * 6 + wave), true);
    if (wave < 6) load_block((int)nown);
    CBAR();
    const u32 call = get_call();
    const u32 seq = call & SEQ_MASK;
    if (wave < 6) {
        take_block(nown);
        pack((int)nown, true, call);
        __builtin_amdgcn_raw_buffer_store_b128(pk[2], rs, (u32)lane >= p_lanes ? 0x80000000u : (u32)lane * 16u + 2048u, p_off, AUX_SYS);
        if (p_mode == 3) __builtin_amdgcn_raw_buffer_store_b128(pk[3], rs, (u32)lane * 16u + 3072u, p_off, AUX_SYS);
        WAIT_VM0();
        if (lane == 0) __hip_atomic_store((gu32*)(myws + p_foff), p_flag, RLX_SYS);
    }
    bool dead = false;
    const u32 xblk = OFF_A + (u32)ti * PIECE + (u32)wq * 4096u;
    LAS char* ord = lds + LDS_ORD;
#define SB() __builtin_amdgcn_sched_barrier(0)
    using T = std::true_type; using F = std::false_type;
    using C0 = std::integral_constant<int, 0>; using C1 = std::integral_constant<int, 1>;
    if (helper) {
        auto lanes = [&](u32 f) __attribute__((always_inline)) { return ((u64)((f & 1u) ? 0xffffffffu : 0u) << 32) | ((f & 0x8000u) ? 0u : 0xffffffffu); };
        auto planes = [&](u32 fhi, const u32x3 c, u32 blk, u32 (&H)[8]) __attribute__((always_inline)) {
            const u32 base4 = ((fhi >> 8) & 127u) * 0x01010101u;
#pragma unroll
            for (int q = 0; q < 3; ++q) { H[2 * q] = c[q] & M83; H[2 * q + 1] = __builtin_amdgcn_alignbit(c[q], c[q], 3) & M83; }
            H[6] = bfi(0x80808080u, c[2] << 1, c[0] >> 5) & M83;
            H[7] = bfi(0x80808080u, c[2] << 2, c[1] >> 5) & M83;
            SB();
#pragma unroll
            for (int q = 0; q < 8; ++q) H[q] += base4;
            if (__builtin_expect((fhi & 0x8000u) != 0u, 0)) raw_planes(H, myws + (blk + (u32)lane * 16u));
        };
        u32 vmask = wave >= 8 ? 0x7fcu : 0xffcu;
        asm volatile("" : "+v"(vmask));
        const u32 fo = OFF_FLAGA + (u32)ti * 1024u + stepid * 16u, go = OFF_FLAGB + (u32)tj * 1024u + stepid * 16u;
        u32x4 fv = {0, 0, 0, 0}, gv = {0, 0, 0, 0}, pv, pg;
        u64 todo = 0, done = 0;
        const u32 want = seq << SEQ_SH;
        auto take = [&](const u32x4 v, const u32x4 g) __attribute__((always_inline)) {
            fv = v; gv = g;
            const u32 df = ((v.x ^ want) | (v.y ^ want) | (v.z ^ want) | (v.w ^ want)) | ((g.x ^ want) | (g.y ^ want) | (g.z ^ want) | (g.w ^ want));
            todo = __ballot((df >> SEQ_SH) == 0u) & ~done;
        };
        auto pick = [&](int slot_) __attribute__((always_inline)) {
            const int id = __ffsll((long long)todo) - 1;
            done |= 1ull << id; todo &= todo - 1ull;
            if (lane == id) { *(LAS u32x4*)(ord + (slot_ & 3) * 32) = fv; *(LAS u32x4*)(ord + (slot_ & 3) * 32 + 16) = gv; }
        };
        auto peek = [&](int slot_) __attribute__((always_inline)) {
            bool later = false;
            if (wave == DECW) {
                later = todo == 0ull;
                if (later) { pv = __builtin_amdgcn_raw_buffer_load_b128(rs, fo, 0, AUX_SYS); pg = __builtin_amdgcn_raw_buffer_load_b128(rs, go, 0, AUX_SYS); }
                else pick(slot_);
            }
            SB();
            return later;
        };
        auto choose = [&](int slot_, bool peeked) __attribute__((always_inline)) {
            if (peeked) take(pv, pg);
            if (todo == 0ull) {
                u32 spins = 0;
                for (;;) {
                    const u32x4 v = __builtin_amdgcn_raw_buffer_load_b128(rs, fo, 0, AUX_SYS), g = __builtin_amdgcn_raw_buffer_load_b128(rs, go, 0, AUX_SYS);
                    take(v, g);
                    CBAR();
                    if (todo != 0ull) break;
                    if (spin_fail(tstart, tmo, call, spins)) { dead = true; fv = gv = u32x4{0, 0, 0, 0}; todo = ~done; break; }
                }
            }
            pick(slot_);
        };
        if (wave == DECW) { choose(0, false); choose(1, false); }
        LDS_BARRIER();
        if (wave >= 8) {
            struct XC { u32x3 c; unsigned short ex; u32 f, so; };
            XC x0, x1;
            const u32 xc_ = xblk + 2048u + (u32)lane * 12u, xe_ = xblk + 2816u + (u32)lane * 2u;
            u32 img = (u32)(uintptr_t)(lds + LDS_HX + wq * 2048);
            auto ldx = [&](int slot_, XC& x) __attribute__((always_inline)) {
                x.f = rfl(*(LAS u32*)(ord + (slot_ & 3) * 32 + wq * 4));
                x.so = ((x.f >> 1) & 63u) * (16u * PIECE);
                x.c = __builtin_amdgcn_raw_buffer_load_b96(rs, xc_, x.so, AUX_SYS);
                x.ex = __builtin_amdgcn_raw_buffer_load_b16(rs, xe_, x.so, AUX_SYS);
                SB();
            };
            auto unit = [&](int j, XC& x, XC& xn, bool ld) __attribute__((always_inline)) {
                if (ld) ldx(j + 1, xn);
                u32 H[8];
                planes(x.f, x.c, xblk + x.so, H);
                *(LAS u32x4*)(uintptr_t)(img + lane * 16) = u32x4{H[0], H[1], H[2], H[3]};
                *(LAS u32x4*)(uintptr_t)(img + 1024 + lane * 16) = u32x4{H[4], H[5], H[6], H[7]};
                patch<false>(img, x.ex, lanes(x.f), vmask);
                if (__builtin_expect((x.f & 0x18000u) == 0x10000u, 0)) patch_more<false>(img, myws + (xblk + (u32)lane * 2u + x.so), vmask);
                img ^= 8192u;
            };
            ldx(0, x0);
#pragma unroll 1
            for (int j = 0; j < 62; j += 2) {
                unit(j, x0, x1, true); LDS_BARRIER();
                unit(j + 1, x1, x0, true); LDS_BARRIER();
            }
            unit(62, x0, x1, true); LDS_BARRIER();
            unit(63, x1, x0, false);
            if (dead && lane == 0) *(LAS u32*)(lds + LDS_RED + 20) = 1;
            LDS_BARRIER();
            return;
        }
        struct WB { u32x4 l0, l1; u32x3 c; unsigned short ex; u32 f, so; };
        WB w0, w1;
        const u32 wblk = OFF_B + (u32)tj * PIECE + (u32)wq * 4096u;
        const u32 wl_ = wblk + (u32)lane * 16u, wc_ = wblk + 2048u + (u32)lane * 12u, we_ = wblk + 2816u + (u32)lane * 2u;
        u32 img = (u32)(uintptr_t)(lds + LDS_W + wq * 4096);
        auto unit = [&](int j, WB& w, WB& wn, bool ld, bool ch) __attribute__((always_inline)) {
            const bool pk_ = ch && peek(j + 2);
            u32 fn = 0;
            if (ld) fn = *(LAS u32*)(ord + ((j + 1) & 3) * 32 + 16 + wq * 4);
            SB();
            u32 H[8];
            planes(w.f, w.c, wblk + w.so, H);
            SB();
            if (ld) {
                wn.f = rfl(fn); wn.so = ((wn.f >> 1) & 63u) * (8u * PIECE);
                wn.c = __builtin_amdgcn_raw_buffer_load_b96(rs, wc_, wn.so, AUX_SYS);
                wn.l0 = __builtin_amdgcn_raw_buffer_load_b128(rs, wl_, wn.so, AUX_SYS);
            }
            SB();
            u32x4 c[4];
            LAS char* pw = (LAS char*)(uintptr_t)(img + lane * 16);
#pragma unroll
            for (int q = 0; q < 4; ++q) {
                const u32 sl_ = (q & 1) ? 0x07030602u : 0x05010400u;
                const int xy = q >> 1;
                c[q] = u32x4{perm(H[xy], w.l0[xy], sl_), perm(H[2 + xy], w.l0[2 + xy], sl_),
                             perm(H[4 + xy], w.l1[xy], sl_), perm(H[6 + xy], w.l1[2 + xy], sl_)};
                if (q) *(LAS u32x4*)(pw + (q - 1) * 1024) = c[q - 1];
                SB();
            }
            *(LAS u32x4*)(pw + 3 * 1024) = c[3];
            patch<true>(img, w.ex, lanes(w.f), vmask);
            if (__builtin_expect((w.f & 0x18000u) == 0x10000u, 0)) patch_more<true>(img, myws + (wblk + (u32)lane * 2u + w.so), vmask);
            img ^= 16384u;
            SB();
            if (ld) {
                wn.l1 = __builtin_amdgcn_raw_buffer_load_b128(rs, wl_ + 1024, wn.so, AUX_SYS);
                wn.ex = __builtin_amdgcn_raw_buffer_load_b16(rs, we_, wn.so, AUX_SYS);
            }
            if (pk_) choose(j + 2, true);
        };
        w0.f = rfl(*(LAS u32*)(ord + 16 + wq * 4)); w0.so = ((w0.f >> 1) & 63u) * (8u * PIECE);
        w0.c = __builtin_amdgcn_raw_buffer_load_b96(rs, wc_, w0.so, AUX_SYS);
        w0.l0 = __builtin_amdgcn_raw_buffer_load_b128(rs, wl_, w0.so, AUX_SYS);
        w0.l1 = __builtin_amdgcn_raw_buffer_load_b128(rs, wl_ + 1024, w0.so, AUX_SYS);
        w0.ex = __builtin_amdgcn_raw_buffer_load_b16(rs, we_, w0.so, AUX_SYS);
#pragma unroll 1
        for (int j = 0; j < 62; j += 2) {
            unit(j, w0, w1, true, true); LDS_BARRIER();
            unit(j + 1, w1, w0, true, true); LDS_BARRIER();
        }
        unit(62, w0, w1, true, false); LDS_BARRIER();
        unit(63, w1, w0, false, false);
        if (dead && lane == 0) *(LAS u32*)(lds + LDS_RED + 20) = 1;
        LDS_BARRIER();
        LDS_BARRIER();
        if (wave == 4 && lane == 0) {
            const u32x4 t = *(LAS u32x4*)(lds + LDS_RED);
            __hip_atomic_store((gu64*)(myws + OFF_AMAXL) + tile, ((u64)call << 32) | max(max(t.x, t.y), max(t.z, t.w)), RLX_AGT);
        }
        LDS_BARRIER();
        if (wave != 4) {
#pragma unroll 1
            for (int u = wave - 5; u < 28; u += 3) {
                if (u >= 4 * MEARLY)
                    while (__hip_atomic_load((LAS u64*)(lds + LDS_RED + 24), __ATOMIC_RELAXED, __HIP_MEMORY_SCOPE_WORKGROUP) == 0ull) __builtin_amdgcn_s_sleep(1);
                const int mwu = tile * 4 + (u & 3);
                int dd = (u >> 2) + mwu % 7; if (dd >= 7) dd -= 7;
                if (lane < 32) {
                    const u32x4 mk = *(LAS u32x4*)(lds + LDS_MSK + (u & 3) * 512 + lane * 16);
                    __builtin_amdgcn_raw_buffer_store_b128(mk, mk_rsrc(P.wsr[dd], WS_USED), (u32)lane * 16u,
                                                           OFF_MSG + (u32)(rank * 512 + mwu) * MSG, AUX_SYS);
                }
                asm volatile("s_waitcnt vmcnt(%0)" :: "n"(MWIN - 1) : "memory");
            }
            WAIT_VM0();
            __builtin_amdgcn_s_barrier();
            return;
        }
        u32 gb = 0;
        if (!dead) {
            gu64* g = lane == 8 ? (gu64*)tmo : (gu64*)(myws + OFF_AMAXG + (tile & (NREP - 1)) * 128) + (lane & 7);
            u32 spins = 0;
            for (;;) {
                const u64 a = __hip_atomic_load(g, RLX_SYS);
                gb = (u32)a;
                if ((__ballot((u32)(a >> 32) == call) & 0xffull) == 0xffull) break;
                if (spin_fail(tstart, tmo, call, spins)) { dead = true; break; }
            }
            if (rdl(gb, 8) == call) dead = true;
            gb = wave_max(lane == 8 ? 0u : gb);
        }
        if (*(LAS u32*)(lds + LDS_RED + 20) != 0u) dead = true;
        if (lane == 0) __hip_atomic_store((LAS u64*)(lds + LDS_RED + 24), ((u64)(dead ? 3u : 1u) << 32) | __float_as_uint(448.0f / __uint_as_float(gb)), __ATOMIC_RELEASE, __HIP_MEMORY_SCOPE_WORKGROUP);
        return;
    }
    bf16x8 xf[4], wfA[8], wfB[8];
    f32x16 acc[4];
#pragma unroll
    for (int j = 0; j < 4; ++j)
#pragma unroll
        for (int i = 0; i < 16; ++i) acc[j][i] = 0.f;
    u32x4 l0[2], l1[2], ha, hb;
    const u32 xl_ = xblk + (u32)lane * 16u;
    auto putX = [&](u32 h0, u32 h1, u32 la, u32 lb, bf16x8& xo) __attribute__((always_inline)) {
        u32x4 c = u32x4{perm(h0, la, 0x05010400u), perm(h1, lb, 0x05010400u), perm(h0, la, 0x07030602u), perm(h1, lb, 0x07030602u)};
        asm volatile("" : "+v"(c));
        xo = __builtin_bit_cast(bf16x8, c);
    };
    u32 so_l1 = 0;
    auto step = [&](int s, auto cur_c, auto dec_c, auto iss_c, auto is1_c) __attribute__((always_inline)) {
        constexpr int cur = decltype(cur_c)::value, nxt = cur ^ 1;
        constexpr bool DEC = decltype(dec_c)::value, ISS = decltype(iss_c)::value, IS1 = decltype(is1_c)::value;
        LAS char* pr = lds + LDS_W + cur * 16384 + 8192 + lane * 16;
#pragma unroll
        for (int q = 0; q < 8; ++q) {
            acc[q & 3] = __builtin_amdgcn_mfma_f32_32x32x16_bf16(wfA[q], xf[q >> 2], acc[q & 3], 0, 0, 0);
            SB();
            if (q < 4) { wfB[2 * q] = *(LAS bf16x8*)(pr + (2 * q) * 1024); wfB[2 * q + 1] = *(LAS bf16x8*)(pr + (2 * q + 1) * 1024); }
            if (q == 4) putX(hb.x, hb.y, l1[cur].x, l1[cur].y, xf[2]);
            if (q == 5) putX(hb.z, hb.w, l1[cur].z, l1[cur].w, xf[3]);
            if constexpr (IS1) { if (q == 6) l1[cur] = __builtin_amdgcn_raw_buffer_load_b128(rs, xl_ + 1024, so_l1, AUX_SYS); }
            SB();
        }
        if constexpr (DEC) LDS_BARRIER(); else WAIT_LGKM0();
        SB();
        pr = lds + LDS_W + nxt * 16384 + lane * 16;
        LAS char* px = lds + LDS_HX + nxt * 8192 + wq * 2048 + lane * 16;
        u32 idv = 0;
#pragma unroll
        for (int q = 0; q < 8; ++q) {
            acc[q & 3] = __builtin_amdgcn_mfma_f32_32x32x16_bf16(wfB[q], xf[2 + (q >> 2)], acc[q & 3], 0, 0, 0);
            SB();
            if constexpr (DEC) {
                if (q == 0) { ha = *(LAS u32x4*)px; hb = *(LAS u32x4*)(px + 1024); }
                if (q >= 1 && q < 5) { wfA[2 * q - 2] = *(LAS bf16x8*)(pr + (2 * q - 2) * 1024); wfA[2 * q - 1] = *(LAS bf16x8*)(pr + (2 * q - 1) * 1024); }
                if (q == 4) putX(ha.x, ha.y, l0[nxt].x, l0[nxt].y, xf[0]);
                if (q == 5) putX(ha.z, ha.w, l0[nxt].z, l0[nxt].w, xf[1]);
            }
            if constexpr (ISS) {
                if (q == 0) idv = *(LAS u32*)(ord + ((s + 3) & 3) * 32);
                if (q == 5) { so_l1 = ((rfl(idv) >> 1) & 63u) * (16u * PIECE); asm volatile("" : "+s"(so_l1)); }
                if (q == 6) l0[nxt] = __builtin_amdgcn_raw_buffer_load_b128(rs, xl_, so_l1, AUX_SYS);
            }
            if (q == 7) __builtin_amdgcn_s_waitcnt(0xC07F);
            SB();
        }
    };
    LDS_BARRIER();
#pragma unroll
    for (int k = 0; k < 2; ++k) {
        const u32 so = ((rfl(*(LAS u32*)(ord + k * 32)) >> 1) & 63u) * (16u * PIECE);
        l0[k] = __builtin_amdgcn_raw_buffer_load_b128(rs, xl_, so, AUX_SYS);
        l1[k] = __builtin_amdgcn_raw_buffer_load_b128(rs, xl_ + 1024, so, AUX_SYS);
    }
    LDS_BARRIER();
    ha = *(LAS u32x4*)(lds + LDS_HX + wq * 2048 + lane * 16); hb = *(LAS u32x4*)(lds + LDS_HX + wq * 2048 + 1024 + lane * 16);
#pragma unroll
    for (int i = 0; i < 8; ++i) wfA[i] = *(LAS bf16x8*)(lds + LDS_W + lane * 16 + i * 1024);
    putX(ha.x, ha.y, l0[0].x, l0[0].y, xf[0]);
    putX(ha.z, ha.w, l0[0].z, l0[0].w, xf[1]);
    so_l1 = ((rfl(*(LAS u32*)(ord + 64)) >> 1) & 63u) * (16u * PIECE);
    l0[0] = __builtin_amdgcn_raw_buffer_load_b128(rs, xl_, so_l1, AUX_SYS);
    WAIT_LGKM0();
    STAMP(1);
#pragma unroll 1
    for (int s = 0; s < 60; s += 2) {
        step(s, C0{}, T{}, T{}, T{});
        step(s + 1, C1{}, T{}, T{}, T{});
    }
    step(60, C0{}, T{}, T{}, T{});
    step(61, C1{}, T{}, F{}, T{});
    step(62, C0{}, T{}, F{}, F{});
    step(63, C1{}, F{}, F{}, F{});
    STAMP(5);

    float mx = 0.f;
#pragma unroll
    for (int j = 0; j < 4; ++j)
#pragma unroll
        for (int i = 0; i < 16; ++i) mx = fmaxf(mx, acc[j][i]);
    const u32 mb = wave_max(__float_as_uint(mx));
    if (lane == 0) *(LAS u32*)(lds + LDS_RED + wave * 4) = mb;
    if (dead && lane == 0) *(LAS u32*)(lds + LDS_RED + 20) = 1;
    LDS_BARRIER();
    u32 m0 = 0, m1 = 0;
#pragma unroll
    for (int i = 7; i >= 0; --i)
#pragma unroll
        for (int j = 3; j >= 0; --j) {
            m0 = __builtin_amdgcn_alignbit(m0, __float_as_uint(acc[j][i]), 31);
            m1 = __builtin_amdgcn_alignbit(m1, __float_as_uint(acc[j][i + 8]), 31);
        }
    m0 = ~m0; m1 = ~m1;
    const int mw = tile * 4 + wave, rot = mw % 7;
    const u32 moff = OFF_MSG + (u32)(rank * 512 + mw) * MSG;
    *(LAS u32x2*)(lds + LDS_MSK + wave * 512 + lane * 8) = u32x2{m0, m1};
    LDS_BARRIER();
    const u32 cnt = (u32)(__popc(m0) + __popc(m1));
    const u32 incl = wave_scan(cnt);
    const u32 total = rdl(incl, 63);
    u32 sel[16], pa[16], ps[16], bm[16];
    {
        u32 pp = (incl - cnt) * 6u;
#pragma unroll
        for (int i = 0; i < 16; ++i) {
            const u32 mm = i < 8 ? m0 : m1;
            const int sh = 4 * (i & 7);
            const u32 n4 = (sh ? (mm >> (sh - 2)) : (mm << 2)) & 0x3cu;
            sel[i] = *(LAS u32*)(lds + LDS_LUT + n4);
            bm[i] = __umul24(n4 << 4, 0x00204081u) & 0x40404040u;
            pa[i] = (pp >> 5) << 2; ps[i] = pp & 31u;
            pp += 6u * (u32)__popc(n4);
            asm volatile("" : "+v"(bm[i]), "+v"(pa[i]), "+v"(ps[i]));
        }
    }
#pragma unroll
    for (int j = 0; j < 4; ++j)
#pragma unroll
        for (int i = 0; i < 16; ++i) acc[j][i] = fmaxf(acc[j][i], 0.f);
#pragma unroll
    for (int j = 0; j < 4; ++j) asm volatile("" : "+v"(acc[j]));
    u64 gd;
    while ((gd = __hip_atomic_load((LAS u64*)(lds + LDS_RED + 24), __ATOMIC_ACQUIRE, __HIP_MEMORY_SCOPE_WORKGROUP)) == 0ull) __builtin_amdgcn_s_sleep(1);
    const float inv = __uint_as_float(rfl((u32)gd));
    dead = rfl((u32)(gd >> 32)) != 1u;
    STAMP(6);

#pragma unroll
    for (int j = 0; j < 4; ++j) acc[j] *= inv;
    const u32 etop = (rfl((u32)__builtin_amdgcn_cvt_pk_fp8_f32(__uint_as_float(mb) * inv, 0.f, 0, false)) >> 3) & 15u;
    u32 base = etop > 8u ? etop - 8u : 0u;
    u32x4 q[4];
    u32 bad0 = 0, bad1 = 0;
    {
        const u32 b8 = base * 0x08080808u;
#pragma unroll
        for (int i = 0; i < 16; ++i) {
            int w0 = __builtin_amdgcn_cvt_pk_fp8_f32(acc[0][i], acc[1][i], 0, false);
            w0 = __builtin_amdgcn_cvt_pk_fp8_f32(acc[2][i], acc[3][i], w0, true);
            q[i >> 2][i & 3] = (u32)w0 & 0x7f7f7f7fu;
            const u32 t = ((u32)w0 | 0x80808080u) - b8;
            const u32 n64 = __builtin_amdgcn_udot4(t & bm[i], 0x08040201u, 0u, false);
            const int sh = 4 * (i & 7) - 6;
            if (i < 8) bad0 |= sh < 0 ? n64 >> -sh : n64 << sh; else bad1 |= sh < 0 ? n64 >> -sh : n64 << sh;
            const u32 c = perm(t, t, sel[i]) & 0x3f3f3f3fu;
            const u32 y = bfi(0x003f003fu, c, c >> 2);
            const u32 z = bfi(0x00000fffu, y, y >> 4);
            const u64 sh64 = (u64)z << ps[i];
            LAS u32* wp = (LAS u32*)(str + pa[i]);
            __hip_atomic_fetch_or(wp, (u32)sh64, __ATOMIC_RELAXED, __HIP_MEMORY_SCOPE_WAVEFRONT);
            __hip_atomic_fetch_or(wp + 1, (u32)(sh64 >> 32), __ATOMIC_RELAXED, __HIP_MEMORY_SCOPE_WAVEFRONT);
        }
    }
    u32 nst = (total * 6u + 127u) >> 7;
    const u32 nb = (u32)(__popc(bad0) + __popc(bad1));
    const u32 nbi = wave_scan(nb);
    u32 nexc = rdl(nbi, 63), seven = 0;
    if (__builtin_expect(nexc <= 64u, 1)) {
        LAS unsigned short* lp = (LAS unsigned short*)(str + nst * 16u) + (nbi - nb);
        const u32 lb = (u32)(lane & 31) * 128u + (u32)(lane >> 5) * 16u;
        while (__ballot((bad0 | bad1) != 0u) != 0ull) {
            if ((bad0 | bad1) != 0u) {
                const u32 pb = bad0 ? (u32)__ffs((int)bad0) - 1u : (u32)__ffs((int)bad1) + 31u;
                if (bad0) bad0 &= bad0 - 1u; else bad1 &= bad1 - 1u;
                *lp++ = (unsigned short)(lb + ((pb >> 4) << 5) + (pb & 15u));
            }
        }
    } else {
        seven = 1; base = 0; nexc = 0;
        nst = (total * 7u + 127u) >> 7;
        WAIT_LGKM0();
#pragma unroll
        for (int u = 0; u < 4; ++u) *(LAS u32x4*)(str + u * 1024 + lane * 16) = u32x4{0, 0, 0, 0};
        u32 pp = (incl - cnt) * 7u;
#pragma unroll
        for (int i = 0; i < 16; ++i) {
            const u32 qq = q[i >> 2][i & 3];
            const u32 c = perm(qq, qq, sel[i]);
            const u32 y = bfi(0x007f007fu, c, c >> 1);
            const u32 z = bfi(0x00003fffu, y, y >> 2);
            const u64 sh64 = (u64)z << (pp & 31u);
            LAS u32* wp = (LAS u32*)(str + ((pp >> 5) << 2));
            __hip_atomic_fetch_or(wp, (u32)sh64, __ATOMIC_RELAXED, __HIP_MEMORY_SCOPE_WAVEFRONT);
            __hip_atomic_fetch_or(wp + 1, (u32)(sh64 >> 32), __ATOMIC_RELAXED, __HIP_MEMORY_SCOPE_WAVEFRONT);
            pp += 7u * (u32)__popc(bm[i]);
        }
    }
    const u32 nch = nst + ((nexc + 7u) >> 3);
    WAIT_LGKM0();
    u32x4 cs[4];
#pragma unroll
    for (int u = 0; u < 4; ++u) cs[u] = *(LAS u32x4*)(str + u * 1024 + lane * 16);
    STAMP(7);
    auto own_tile = [&]() __attribute__((always_inline)) {
        const u32 ooff = (u32)((rank >> 2) * 2048 + ti * 128 + wave * 32 + (lane >> 3)) * 4096u + (u32)((rank & 3) * 1024 + tj * 128 + (lane & 7) * 16);
        __amdgpu_buffer_rsrc_t ro = mk_rsrc(P.out, (u32)MM * NN);
        CBAR();
#pragma unroll
        for (int bb = 0; bb < 4; ++bb)
            *(LAS u32x4*)(str + (lane & 31) * 128 + bb * 32 + (lane >> 5) * 16) = dead ? u32x4{0x7f7f7f7fu, 0x7f7f7f7fu, 0x7f7f7f7fu, 0x7f7f7f7fu} : q[bb];
        CBAR();
#pragma unroll
        for (int bb = 0; bb < 4; ++bb)
            __builtin_amdgcn_raw_buffer_store_b128(*(LAS u32x4*)(str + bb * 1024 + lane * 16), ro, ooff + bb * 32768u, 0, AUX_OUT);
        CBAR();
    };
    if (!dead) {
        const u64 flag = ((u64)call << 32) | nch | (nst << 8) | (nexc << 16) | (base << 23) | (seven << 27);
        const u32 foff = OFF_FLAGO + (u32)mw * 128u + (u32)rank * 8u;
        if (__builtin_expect(nch <= 128u, 1)) {
            const u32 v0 = (u32)lane < nch ? 512u + (u32)lane * 16u : 0x80000000u, v1 = (u32)lane + 64u < nch ? 1536u + (u32)lane * 16u : 0x80000000u;
            u32 due = (u32)now();
#pragma unroll
            for (int k = 0; k < 7; ++k) {
                int dd = k + rot; if (dd >= 7) dd -= 7;
                char* w0 = P.wsr[dd];
                pace(due, PER2);
                __builtin_amdgcn_raw_buffer_store_b128(cs[0], mk_rsrc(w0, WS_USED), v0, moff, AUX_SYS);
                pace(due, PER2);
                __builtin_amdgcn_raw_buffer_store_b128(cs[1], mk_rsrc(w0, WS_USED), v1, moff, AUX_SYS);
                if (k == 0) { own_tile(); __builtin_amdgcn_s_barrier(); }
                WAIT_VM0();
                if (lane == 0) __hip_atomic_store((gu64*)(w0 + foff), flag, RLX_SYS);
            }
        } else {
            own_tile();
            __builtin_amdgcn_s_barrier();
#pragma unroll 1
            for (int k = 0; k < 7; ++k) {
                int dd = k + rot; if (dd >= 7) dd -= 7;
                char* w0 = P.wsr[dd];
#pragma unroll
                for (int u = 0; u < 4; ++u)
                    if ((u32)(u * 64 + lane) < nch) __builtin_amdgcn_raw_buffer_store_b128(cs[u], mk_rsrc(w0, WS_USED), 512u + u * 1024u + (u32)lane * 16u, moff, AUX_SYS);
                WAIT_VM0();
                if (lane == 0) __hip_atomic_store((gu64*)(w0 + foff), flag, RLX_SYS);
            }
        }
    } else {
        own_tile();
        __builtin_amdgcn_s_barrier();
    }
    STAMP(8);
}

extern "C" void kernel_launch(int rank, int nranks, ncclComm_t comm,
                              hipStream_t stream,
                              void* const* d_in, const int* in_sizes, int n_in,
                              void* d_out, int out_size,
                              const int* in_dtypes, int out_dtype,
                              void* const* d_out_all,
                              void* const* d_ws_all, size_t ws_bytes) {
    (void)comm; (void)in_sizes; (void)in_dtypes; (void)out_dtype; (void)d_out_all;
    if (nranks != NR || n_in < 2 || out_size != MM * NN || ws_bytes < WS_USED || (unsigned)rank >= (unsigned)NR) {
        fprintf(stderr, "unexpected problem shape\n");
        return;
    }
    Params P;
    P.x = (const char*)d_in[0];
    P.w = (const char*)d_in[1];
    for (int d = 0; d < NR; ++d) P.wsr[d] = (char*)d_ws_all[(rank + 1 + d) & 7];
    P.out = (char*)d_out;
    P.myws = (char*)d_ws_all[rank];
    P.rank = rank;
    P.pad = 0;
    fused_kernel<<<dim3(NGRID), dim3(NTHR), 0, stream>>>(P);
    HIP_CHECK(hipGetLastError());
}
```

```cpp
#include <hip/hip_runtime.h>
#include <rccl/rccl.h>
#include <cstdio>
#include <type_traits>

#define HIP_CHECK(expr) do { hipError_t _e = (expr); if (_e != hipSuccess) { \
    fprintf(stderr, "HIP error %s at %s:%d\n", hipGetErrorString(_e), __FILE__, __LINE__); \
    return; } } while (0)

typedef unsigned int u32;
typedef unsigned long long u64;
typedef u32 u32x4 __attribute__((ext_vector_type(4)));
typedef u32 u32x3 __attribute__((ext_vector_type(3)));
typedef u32 u32x2 __attribute__((ext_vector_type(2)));
typedef _Float16 f16x2 __attribute__((ext_vector_type(2)));
typedef float f32x16 __attribute__((ext_vector_type(16)));
typedef __bf16 bf16x8 __attribute__((ext_vector_type(8)));
typedef __bf16 bf16x2 __attribute__((ext_vector_type(2)));
#define GAS __attribute__((address_space(1)))
#define LAS __attribute__((address_space(3)))
typedef GAS u32 gu32;
typedef GAS u64 gu64;
typedef GAS u32x4 gu32x4;

constexpr int NR = 8;
constexpr int MM = 4096, NN = 4096;
constexpr int NGRID = 256, NTHR = 768;
constexpr u64 DEADLINE_TICKS = 50000000ull;

constexpr u32 OFF_CNT   = 0;
constexpr u32 OFF_TMO   = 32768;
constexpr u32 OFF_TICK  = 33024;
constexpr u32 OFF_AMAXL = 36864;
constexpr u32 OFF_AMAXG = 40960;
constexpr int NREP      = 8;
constexpr u32 OFF_FLAGA = 65536;
constexpr u32 OFF_FLAGB = 98304;
constexpr u32 OFF_FLAGO = 131072;
constexpr u32 OFF_DIAG  = 262144;
constexpr u32 PIECE     = 16384;
constexpr u32 OFF_A     = 1u << 20;
constexpr u32 OFF_B     = OFF_A + 64u * 16u * PIECE;
constexpr u32 MSG       = 4096;
constexpr u32 OFF_MSG   = OFF_B + 64u * 8u * PIECE;
constexpr u32 WS_USED   = OFF_MSG + 8u * 512u * MSG;

constexpr int LDS_W    = 0;
constexpr int LDS_STR  = 32768;
constexpr int LDS_MSK  = 65536;
constexpr int LDS_LUT  = 67584;
constexpr int LDS_RED  = 67712;
constexpr int LDS_ORD  = 67840;
constexpr int LDS_TAB  = 69632;
constexpr int LDS_HX   = 98304;
constexpr int LDS_SIZE = 114688;

struct Params {
    const char* x;
    const char* w;
    char* myws;
    char* out;
    int rank;
    int pad;
    char* wsr[NR];
};

#define RLX_SYS __ATOMIC_RELAXED, __HIP_MEMORY_SCOPE_SYSTEM
#define RLX_AGT __ATOMIC_RELAXED, __HIP_MEMORY_SCOPE_AGENT
#define WAIT_VM0() asm volatile("s_waitcnt vmcnt(0)" ::: "memory")
#define CBAR() asm volatile("" ::: "memory")
#define WAIT_LGKM0() do { CBAR(); __builtin_amdgcn_s_waitcnt(0xC07F); CBAR(); } while (0)
#define LDS_BARRIER() do { WAIT_LGKM0(); __builtin_amdgcn_s_barrier(); CBAR(); } while (0)
constexpr int AUX_SYS = 17;
constexpr int AUX_OUT = 16;
constexpr int NWL = 32;
constexpr u32 PER = 156;
constexpr u32 PER2 = 100;
constexpr int MEARLY = 5;
constexpr int MWIN = 2;
constexpr int DECW = 5;
constexpr int NBLK = 768;
constexpr u32 SEQ_SH = 17, SEQ_MASK = 0x7fffu;

__device__ __forceinline__ u64 now() { return __builtin_amdgcn_s_memrealtime(); }
__device__ __forceinline__ u32 rfl(u32 v) { return (u32)__builtin_amdgcn_readfirstlane((int)v); }
__device__ __forceinline__ u32 rdl(u32 v, int l) { return (u32)__builtin_amdgcn_readlane((int)v, l); }
__device__ __forceinline__ u32 perm(u32 a, u32 b, u32 s) { return __builtin_amdgcn_perm(a, b, s); }
__device__ __forceinline__ u32 bfi(u32 m, u32 a, u32 b) { return (a & m) | (b & ~m); }
__device__ __forceinline__ __amdgpu_buffer_rsrc_t mk_rsrc(const char* base, u32 bytes) {
    return __builtin_amdgcn_make_buffer_rsrc((void*)base, 0, bytes, 0x00020000);
}
template <int CTRL, int RMASK> __device__ __forceinline__ u32 dpp0(u32 v) {
    return (u32)__builtin_amdgcn_update_dpp(0, (int)v, CTRL, RMASK, 0xf, false);
}
__device__ __forceinline__ u32 wave_max(u32 v) {
    v = max(v, dpp0<0x111, 0xf>(v)); v = max(v, dpp0<0x112, 0xf>(v));
    v = max(v, dpp0<0x114, 0xf>(v)); v = max(v, dpp0<0x118, 0xf>(v));
    v = max(v, dpp0<0x142, 0xa>(v)); v = max(v, dpp0<0x143, 0xc>(v));
    return rdl(v, 63);
}
__device__ __forceinline__ u32 wave_scan(u32 v) {
    v += dpp0<0x111, 0xf>(v); v += dpp0<0x112, 0xf>(v);
    v += dpp0<0x114, 0xf>(v); v += dpp0<0x118, 0xf>(v);
    v += dpp0<0x142, 0xa>(v); v += dpp0<0x143, 0xc>(v);
    return v;
}

#define STAMP(i) do { } while (0)

__device__ __forceinline__ bool spin_fail(u64 t0, gu32* tmo, u32 call, u32& spins) {
    __builtin_amdgcn_s_sleep(1);
    ++spins;
    if ((spins & 63u) == 0u) {
        const u64 dt = now() - t0;
        if (dt > 100000ull) {
            if (rfl(__hip_atomic_load(tmo, RLX_AGT)) == call) return true;
            if (dt > DEADLINE_TICKS) { __hip_atomic_store(tmo, call, RLX_AGT); return true; }
        }
    }
    return false;
}

__device__ __forceinline__ void pace(u32& due, u32 per) {
    u32 t;
    while ((t = (u32)now()) - due > 0x80000000u) __builtin_amdgcn_s_sleep(1);
    due = t + per;
}

__device__ __forceinline__ u32 block_id(u32 nb, bool plain) {
    const u32 n = nb >> 2, m = plain ? n : n - 8u, g = (m * 2731u) >> 16, q = m - g * 24u, t = (q * 11u) >> 5;
    u32 ce = q == 3u * t ? (g + 1u) * 32u + t : g * 32u + 7u + q - t;
    if (n < 32u) ce = n + (n >= 24u ? 8u : 0u);
    if (n >= 176u) ce = 7u * 32u + n - 168u;
    if (plain) ce = g * 32u + q;
    return nb < (u32)NBLK ? (ce << 2) | (nb & 3u) : 0xffffffffu;
}
constexpr u32 M83 = 0x83838383u;
template <bool ISW> __device__ __forceinline__ void patch(u32 img, unsigned short ex, u64 hi, u32 vmask) {
    u32 ad, val, sh;
    asm volatile("v_lshrrev_b32 %0, %7, %3\n\tv_bfe_i32 %1, %3, %8, 5\n\tv_lshl_or_b32 %2, %3, %9, %10\n\t"
                 "v_and_or_b32 %0, %0, %6, %4\n\tv_lshlrev_b32 %1, %2, %1\n\t"
                 "s_mov_b64 exec, %5\n\tds_add_u32 %0, %1\n\ts_mov_b64 exec, -1"
                 : "=&v"(ad), "=&v"(val), "=&v"(sh)
                 : "v"(__builtin_bit_cast(_Float16, ex)), "s"(img), "s"(hi), "v"(vmask), "n"(ISW ? 4 : 5), "n"(ISW ? 1 : 2), "n"(ISW ? 4 : 3), "n"(ISW ? 10 : 2) : "memory");
}
__device__ __forceinline__ void raw_planes(u32 (&H)[8], const char* p) {
    u32x4 a, b;
    asm volatile("global_load_dwordx4 %0, %2, off offset:2048 sc0 sc1\n\tglobal_load_dwordx4 %1, %2, off offset:3072 sc0 sc1\n\ts_waitcnt vmcnt(0)"
                 : "=&v"(a), "=&v"(b) : "v"(p) : "memory");
    asm volatile("v_mov_b32 %0, %8\n\tv_mov_b32 %1, %9\n\tv_mov_b32 %2, %10\n\tv_mov_b32 %3, %11\n\t"
                 "v_mov_b32 %4, %12\n\tv_mov_b32 %5, %13\n\tv_mov_b32 %6, %14\n\tv_mov_b32 %7, %15"
                 : "+v"(H[0]), "+v"(H[1]), "+v"(H[2]), "+v"(H[3]), "+v"(H[4]), "+v"(H[5]), "+v"(H[6]), "+v"(H[7])
                 : "v"(a.x), "v"(a.y), "v"(a.z), "v"(a.w), "v"(b.x), "v"(b.y), "v"(b.z), "v"(b.w));
}
template <bool ISW> __device__ __forceinline__ void patch_more(u32 img, const char* p, u32 vmask) {
    unsigned short ex;
    asm volatile("global_load_ushort %0, %1, off offset:2944 sc0 sc1\n\ts_waitcnt vmcnt(0)" : "=&v"(ex) : "v"(p) : "memory");
    patch<ISW>(img, ex, ~0ull, vmask);
}

__global__ void __launch_bounds__(NTHR, 3) fused_kernel(Params P) {
    __shared__ __attribute__((aligned(32768))) char smem[LDS_SIZE];
    LAS char* lds = (LAS char*)smem;
    const int tid = threadIdx.x, lane = tid & 63, wave = (int)rfl((u32)tid >> 6);
    const int b = blockIdx.x, xcd = b & 7, slot = b >> 3;
    const int rank = P.rank;
    char* myws = P.myws;
    asm volatile("" :: "s"(P.x), "s"(P.w), "s"(P.wsr[slot < 16 ? slot & 7 : 7]));
    gu32* tmo = (gu32*)(myws + OFF_TMO);
    const u64 tstart = now();
    STAMP(0);
    auto get_call = [&]() __attribute__((always_inline)) {
        u32 ticket = 0;
        if (lane == 0) ticket = __hip_atomic_fetch_add((gu32*)(myws + OFF_CNT + b * 128 + wave * 8), 1u, RLX_AGT);
        return rfl(ticket) + 1u;
    };

    const bool comm = slot < 16;
    const int d = comm ? slot & 7 : 7;
    const int sw = (slot >> 3) * 8 + xcd;
    const int wl = wave * 16 + sw;
    const bool producer = d != 7 && wl < NWL;
    const int p = (rank + 1 + d) & 7;
    const int r = (rank - p) & 7;
    char* dws = P.wsr[d];
    __amdgpu_buffer_rsrc_t rs = mk_rsrc(dws, WS_USED);
    const int row0 = (p >> 2) * 2048, col0 = (p & 3) * 1024;
    const u32 xlane = (u32)(lane & 31) * 1024u + (u32)(lane >> 5) * 16u;
    const u32 wlane = (u32)((lane >> 5) * 8) * 8192u + (u32)(lane & 31) * 8u;
    u32 v[16];
    u32x2 vn[8];
    auto load_block = [&](int nb) __attribute__((always_inline)) {
        const int wb = nb & 3, c = nb >> 7, e = (nb >> 2) & 31;
        const bool isw = e < 8;
        const u32 so = isw ? (u32)((c * 64 + wb * 16) * 8192 + (col0 + ((e + r) & 7) * 128) * 2)
                           : (u32)((row0 + ((e - 8 + 2 * r) & 15) * 128 + wb * 32) * 1024 + c * 128);
        const u32 s1 = isw ? 8192u : 8u, s2 = isw ? 16384u : 32u;
        __amdgpu_buffer_rsrc_t ri = mk_rsrc(isw ? P.w : P.x, 4096u * 512u * 2u);
        const u32 vo = isw ? wlane : xlane;
#pragma unroll
        for (int k = 0; k < 8; ++k) vn[k] = __builtin_amdgcn_raw_buffer_load_b64(ri, vo, so + (k & 1) * s1 + (k >> 1) * s2, 0);
    };
    auto take_block = [&](u32 nb) __attribute__((always_inline)) {
        const u32 inc = (nb & 0x60u) == 0u ? 0xff7fff80u : 0x00800080u;
#pragma unroll
        for (int k = 0; k < 8; ++k) { v[2 * k] = vn[k].x + inc; v[2 * k + 1] = vn[k].y + inc; }
#pragma unroll
        for (int i = 0; i < 16; ++i) asm volatile("" : "+v"(v[i]));
    };
    u32x4 pk[4];
    LAS char* stg = lds + LDS_TAB + wave * 3072;
    u32 p_off = 0, p_foff = 0, p_mode = 0, p_lanes = 0;
    u32 p_flag = 0;
    auto pack = [&](int nb, bool early, u32 call) __attribute__((always_inline)) {
        const int wb = nb & 3, c = nb >> 7, e = (nb >> 2) & 31;
        const int step = rank * 8 + c;
        u32 selL, selH;
        if (e < 8) { const int j = (e + r) & 7;
            p_off = OFF_B + (u32)(step * 8 + j) * PIECE;  p_foff = OFF_FLAGB + (u32)((j * 64 + step) * 4 + wb) * 4u;
            selL = 0x06020400u; selH = 0x07030501u;
        } else { const int i = (e - 8 + 2 * r) & 15;
            p_off = OFF_A + (u32)(step * 16 + i) * PIECE; p_foff = OFF_FLAGA + (u32)((i * 64 + step) * 4 + wb) * 4u;
            selL = 0x06040200u; selH = 0x07050301u;
        }
        p_off += (u32)wb * 4096u;
        u32 H[8], D[8], mode, base, total;
        bool exact = false;
        for (;;) {
            f16x2 hi = __builtin_bit_cast(f16x2, v[0]), lo = hi;
#pragma unroll
            for (int i = 1; i < 16; ++i) {
                hi = __builtin_elementwise_maximum(hi, __builtin_bit_cast(f16x2, v[i]));
                lo = __builtin_elementwise_minimum(lo, __builtin_bit_cast(f16x2, v[i]));
            }
            const u32 h2 = __builtin_bit_cast(u32, __builtin_elementwise_maximum(hi, -lo)) & 0x7fff7fffu;
            u32 mag = max(h2 & 0xffffu, h2 >> 16);
            if (__builtin_expect(exact, 0)) {
                mag = 0;
#pragma unroll
                for (int i = 0; i < 16; ++i) mag = max(mag, max(v[i] & 0x7fffu, (v[i] >> 16) & 0x7fffu));
            }
#pragma unroll
            for (int i = 0; i < 8; ++i) {
                const u32 a = v[4 * (i >> 1) + (i & 1)], c2 = v[4 * (i >> 1) + (i & 1) + 2];
                pk[i >> 2][i & 3] = perm(c2, a, selL); H[i] = perm(c2, a, selH);
            }
            if (early) {
                STAMP(7);
                __builtin_amdgcn_raw_buffer_store_b128(pk[0], rs, (u32)lane * 16u, p_off, AUX_SYS);
                __builtin_amdgcn_raw_buffer_store_b128(pk[1], rs, (u32)lane * 16u + 1024u, p_off, AUX_SYS);
            }
            const u32 top = wave_max(mag >> 8);
            base = top > 3u ? top - 3u : 0u;
            const bool lower = base > 0u && __popcll(__ballot((mag >> 8) == top)) <= 8;
            base -= (u32)lower;
            const u32 base4 = base * 0x01010101u;
            u32 pm = 0;
#pragma unroll
            for (int i = 0; i < 8; ++i) {
                D[i] = (H[i] | 0x80808080u) - base4;
                pm |= ~(D[i] >> (7 - i)) & (0x01010101u << i);
            }
            if (lower) {
#pragma unroll
                for (int i = 0; i < 8; ++i) pm |= (D[i] >> 2 << i) & (0x01010101u << i);
            }
            *(LAS u32x4*)(stg + 1024 + lane * 16) = u32x4{D[0], D[1], D[2], D[3]};
            *(LAS u32x4*)(stg + 2048 + lane * 16) = u32x4{D[4], D[5], D[6], D[7]};
            *(LAS u32*)(stg + 768 + lane * 4) = (u32)lane * 0x01000100u + 0x00800000u;
            u32 dmin = 255;
            const u32 np = (u32)__popc(pm), npi = wave_scan(np);
            total = rdl(npi, 63);
            mode = total > 64u ? 2u : 0u;
            p_lanes = total > 64u ? 64u : total > 32u ? 56u : 52u;
            if (__builtin_expect(total <= 128u, 1)) {
                u32 E[8];
#pragma unroll
                for (int i = 0; i < 8; ++i) E[i] = bfi(0x80808080u, H[i], D[i]);
                LAS u32* cw = (LAS u32*)(stg + lane * 12);
                cw[0] = (E[0] & M83) | (__builtin_amdgcn_alignbit(E[1], E[1], 29) & 0x1c1c1c1cu) | ((E[6] << 5) & 0x60606060u);
                cw[1] = (E[2] & M83) | (__builtin_amdgcn_alignbit(E[3], E[3], 29) & 0x1c1c1c1cu) | ((E[7] << 5) & 0x60606060u);
                cw[2] = (E[4] & M83) | (__builtin_amdgcn_alignbit(E[5], E[5], 29) & 0x1c1c1c1cu) | ((E[6] >> 1) & 0x40404040u) | ((E[7] >> 2) & 0x20202020u);
                LAS unsigned short* lp = (LAS unsigned short*)(stg + 768) + (npi - np);
                while (__ballot(pm != 0u) != 0ull) {
                    if (pm != 0u) {
                        const u32 pb = (u32)__ffs((int)pm) - 1u, by = pb >> 3, pl = pb & 7u;
                        pm &= pm - 1u;
                        const u32 dw = (pl >> 2) * 256u + (u32)lane * 4u + (pl & 3u);
                        const u32 d = *(LAS unsigned char*)(stg + 1024 + dw * 4u + by);
                        const u32 d5 = (d >> 2) & 31u;
                        dmin = min(dmin, d);
                        const u32 ew = ((((pl & 1u) * 2u + (by >> 1)) << 8 | (u32)lane << 2 | pl >> 1) << 6) | (d5 << 1) | (by & 1u);
                        *lp++ = (unsigned short)(e < 8 ? ew : (dw << 7) | (d5 << 2) | by);
                    }
                }
                CBAR();
                pk[2] = *(LAS u32x4*)(stg + lane * 16);
            }
            const bool raw = total > 128u || __ballot(dmin < 64u) != 0ull;
            const bool odd = raw || top >= 0x7cu || base <= 2u || __ballot(dmin + base <= 129u) != 0ull;
            if (__builtin_expect(odd && !exact, 0)) {
                const u32 inc = e < 8 ? 0xff7fff80u : 0x00800080u;
                const float f = e < 8 ? 0.5f : 2.0f;
#pragma unroll
                for (int i = 0; i < 16; ++i) {
                    const u32 o = v[i] - inc;
                    const bf16x2 t = {(__bf16)(__uint_as_float(o << 16) * f), (__bf16)(__uint_as_float(o & 0xffff0000u) * f)};
                    v[i] = __builtin_bit_cast(u32, t);
                }
                exact = true;
                continue;
            }
            if (__builtin_expect(raw, 0)) {
                mode = 3; p_lanes = 64u;
                pk[2] = u32x4{H[0], H[1], H[2], H[3]};
            }
            break;
        }
        pk[3] = u32x4{H[4], H[5], H[6], H[7]};
        p_mode = mode;
        p_flag = (call << SEQ_SH) | (mode << 15) | (base << 8) | ((u32)step << 1) | (total > 32u && mode != 3u ? 1u : 0u);
    };

    if (__builtin_expect(slot < 16, 1)) {
        if (wave >= 8) {
            const int wi = (b << 2) | (wave & 3);
            const char* px = P.x + (size_t)(wi * 8 + (lane >> 3)) * 1024 + (lane & 7) * 16;
            const char* pw = P.w + (size_t)wi * 1024 + lane * 16;
            u32x4 t[3];
#pragma unroll 1
            for (int c = 0; c < 8; ++c) {
                t[0] = *(const gu32x4*)(pw + c * 524288);
                t[1] = *(const gu32x4*)(px + c * 128);
                asm volatile("" :: "v"(t[0]), "v"(t[1]));
                __builtin_amdgcn_s_sleep(64);
            }
            __amdgpu_buffer_rsrc_t rt = mk_rsrc(myws, WS_USED);
#pragma unroll 1
            for (int c = 7; c > 5; --c) {
                const u32 t1 = __builtin_amdgcn_raw_buffer_load_b32(rt, (u32)min(lane, 22) * 128u,
                                                                    OFF_A + (u32)(((wi >> 6) * 8 + c) * 16) * PIECE + (u32)(wi & 63) * 4096u, AUX_SYS);
                asm volatile("" :: "v"(t1));
                __builtin_amdgcn_s_sleep(32);
            }
            const int mwt = (slot * 8 + xcd) * 4 + (wave & 3);
#pragma unroll 1
            for (int k = 6; k > 4; --k) {
                for (int i = 0; i < 1 + (wi & 3); ++i) __builtin_amdgcn_s_sleep(64);
                int dd = k + mwt % 7; if (dd >= 7) dd -= 7;
                const u32 off = OFF_MSG + (u32)(((rank - 1 - dd) & 7) * 512 + mwt) * MSG;
                t[0] = __builtin_amdgcn_raw_buffer_load_b128(rt, (u32)lane * 16u, off, AUX_SYS);
                t[1] = __builtin_amdgcn_raw_buffer_load_b128(rt, (u32)lane * 16u + 1024u, off, AUX_SYS);
                t[2] = __builtin_amdgcn_raw_buffer_load_b128(rt, (u32)(lane & 31) * 16u + 2048u, off, AUX_SYS);
#pragma unroll
                for (int u = 0; u < 3; ++u) asm volatile("" :: "v"(t[u]));
                {
                    const int q = (rank - 1 - dd) & 7, tl = slot * 8 + xcd;
                    const u32 oo = (u32)((q >> 2) * 2048 + (tl >> 3) * 128 + (wave & 3) * 32 + (lane & 31)) * 4096u + (u32)((q & 3) * 1024 + (tl & 7) * 128);
                    const u32 t1 = __builtin_amdgcn_raw_buffer_load_b32(mk_rsrc(P.out, (u32)MM * NN), oo + (lane >> 5) * 64u, 0, AUX_SYS);
                    asm volatile("" :: "v"(t1));
                }
            }
            return;
        }
        const u32 nA = block_id((u32)wl, false);
        if (producer) load_block((int)nA);
        CBAR();
        const u32 call = get_call();
        if (tid < 16) {
            u32 sel = 0, k = 0;
            for (int j = 0; j < 4; ++j) { const u32 on = (tid >> j) & 1; sel |= (on ? k : 0x0cu) << (8 * j); k += on; }
            *(LAS u32*)(lds + LDS_LUT + tid * 4) = sel;
        }
        gu32* tick = (gu32*)(myws + OFF_TICK + d * 128);
        const u32 tbase = (call - 1u) * (u32)(NBLK - 2 * NWL) - (u32)(3 * NWL);
        if (producer) {
            u32 nB = block_id((u32)(wl + NWL), false), nC = block_id((u32)(wl + 2 * NWL), false);
            take_block(nA);
            load_block((int)nB);
            pack((int)nA, true, call);
            const u32 vo = (u32)lane * 16u;
            char* fdst = nullptr;
            u32 flag = 0;
            const u32 zero = 0, one = 1;
            u32 due = (u32)tstart;
            auto iter = [&](bool first) __attribute__((always_inline)) {
                const bool more = nB != 0xffffffffu, more2 = nC != 0xffffffffu;
                u32 tk = 0;
                if (more2) asm volatile("s_mov_b64 exec, 1\n\tglobal_atomic_add %0, %1, %2, %3 sc0\n\ts_mov_b64 exec, -1"
                                        : "=&v"(tk) : "v"(zero), "v"(one), "s"(tick) : "memory");
                if (!first && lane == 0) __hip_atomic_store((gu32*)fdst, flag, RLX_SYS);
                CBAR();
                pace(due, PER);
                if (!first) {
                    __builtin_amdgcn_raw_buffer_store_b128(pk[0], rs, vo, p_off, AUX_SYS);
                    __builtin_amdgcn_raw_buffer_store_b128(pk[1], rs, vo + 1024, p_off, AUX_SYS);
                }
                __builtin_amdgcn_raw_buffer_store_b128(pk[2], rs, (u32)lane >= p_lanes ? 0x80000000u : vo + 2048, p_off, AUX_SYS);
                if (p_mode == 3) __builtin_amdgcn_raw_buffer_store_b128(pk[3], rs, vo + 3072, p_off, AUX_SYS);
                CBAR();
                fdst = dws + p_foff;
                flag = p_flag;
                take_block(nB);
                if (more2) load_block((int)nC);
                CBAR();
                if (more) pack((int)nB, false, call);
                __builtin_amdgcn_s_waitcnt(0x0F70);
                asm volatile("s_waitcnt vmcnt(0)" : "+v"(tk) :: "memory");
                nB = nC;
                if (more2) nC = block_id(rfl(tk) - tbase, false);
                return more;
            };
            if (iter(true)) {
#pragma unroll 1
                while (iter(false)) { }
            }
            if (lane == 0) __hip_atomic_store((gu32*)fdst, flag, RLX_SYS);
        }
        STAMP(1);
        LDS_BARRIER();
        bool dead = false;
        if (b == 56 && wave == 0) {
            u32 spins = 0, m = 0;
            for (;;) {
                const u64 g0 = __hip_atomic_load((gu64*)(myws + OFF_AMAXL) + lane, RLX_AGT);
                const u64 g1 = __hip_atomic_load((gu64*)(myws + OFF_AMAXL) + 64 + lane, RLX_AGT);
                m = max((u32)g0, (u32)g1);
                if (__all((u32)(g0 >> 32) == call && (u32)(g1 >> 32) == call)) break;
                if (spin_fail(tstart, tmo, call, spins)) { dead = true; break; }
            }
            if (!dead) {
                m = wave_max(m);
                char* dst = P.wsr[lane & 7] + OFF_AMAXG + (lane >> 3) * 128 + rank * 8;
#pragma unroll
                for (int k = 0; k < NREP / 8; ++k)
                    __hip_atomic_store((gu64*)(dst + k * 1024), ((u64)call << 32) | m, RLX_SYS);
            }
            STAMP(2);
        }
        const int tile = slot * 8 + xcd, ti = tile >> 3, tj = tile & 7;
        const int wq = wave & 3, hf = wave >> 2;
        const int mw = tile * 4 + wq, rot = mw % 7;
        __amdgpu_buffer_rsrc_t rm = mk_rsrc(myws, WS_USED);
        __amdgpu_buffer_rsrc_t ro = mk_rsrc(P.out, (u32)MM * NN);
        gu64* fo = (gu64*)(myws + OFF_FLAGO + mw * 128) + (lane & 7);
        LAS char* str = lds + LDS_STR + wave * 4096;
        LAS char* ot = lds + LDS_W + wave * 2048;
        const int el = (lane & 32) + hf * 16 + (lane & 15), uh = (lane >> 4) & 1;
        u64 fl = 0;
#pragma unroll 1
        for (int k = 0; k < 7; ++k) {
            int dd = k + rot; if (dd >= 7) dd -= 7;
            const int q = (rank - 1 - dd) & 7;
            if (!dead && rdl((u32)(fl >> 32), q) != call) {
                u32 spins = 0;
                for (;;) {
                    fl = __hip_atomic_load(fo, RLX_SYS);
                    if (rdl((u32)(fl >> 32), q) == call) break;
                    if (spin_fail(tstart, tmo, call, spins)) { dead = true; break; }
                }
            }
            CBAR();
            const u32 ooff = (u32)((q >> 2) * 2048 + ti * 128 + wq * 32 + hf * 16 + (lane >> 3)) * 4096u
                           + (u32)((q & 3) * 1024 + tj * 128 + (lane & 7) * 16);
            if (dead) {
#pragma unroll
                for (int bb = 0; bb < 2; ++bb)
                    __builtin_amdgcn_raw_buffer_store_b128(u32x4{0x7f7f7f7fu, 0x7f7f7f7fu, 0x7f7f7f7fu, 0x7f7f7f7fu}, ro, ooff + bb * 32768u, 0, 0);
                continue;
            }
            const u32 fw = rdl((u32)fl, q);
            const u32 nch = fw & 255u;
            const u32 moff = OFF_MSG + (u32)(q * 512 + mw) * MSG;
            const u32x2 mk = __builtin_amdgcn_raw_buffer_load_b64(rm, (u32)lane * 8u, moff, AUX_SYS);
            const u32x4 c0 = __builtin_amdgcn_raw_buffer_load_b128(rm, 512u + (u32)lane * 16u, moff, AUX_SYS);
            const u32x4 c1 = __builtin_amdgcn_raw_buffer_load_b128(rm, 1536u + (u32)lane * 16u, moff, AUX_SYS);
            const u32 c_lo = (u32)__popc(mk.x), cnt = c_lo + (u32)__popc(mk.y);
            const u32 f0 = wave_scan(cnt) - cnt;
            const u32 ff = (u32)__builtin_amdgcn_ds_bpermute(el * 4, (int)(f0 | ((f0 + c_lo) << 16)));
            const u32 ma = (u32)__builtin_amdgcn_ds_bpermute(el * 4, (int)mk.x), mb = (u32)__builtin_amdgcn_ds_bpermute(el * 4, (int)mk.y);
            const u32 first = uh ? ff >> 16 : ff & 0xffffu, mm = uh ? mb : ma;
            *(LAS u32x4*)(str + lane * 16) = c0;
            *(LAS u32x4*)(str + 1024 + lane * 16) = c1;
            if (nch > 128u) {
                const u32x4 c2 = __builtin_amdgcn_raw_buffer_load_b128(rm, 2560u + (u32)lane * 16u, moff, AUX_SYS);
                *(LAS u32x4*)(str + 2048 + lane * 16) = c2;
                if (lane < 32) {
                    const u32x4 c3 = __builtin_amdgcn_raw_buffer_load_b128(rm, 3584u + (u32)lane * 16u, moff, AUX_SYS);
                    *(LAS u32x4*)(str + 3072 + lane * 16) = c3;
                }
            }
            WAIT_LGKM0();
            u32x4 o[2];
            u32 ex = 0;
            if (__builtin_expect((fw & (1u << 27)) == 0u, 1)) {
                ex = *(LAS unsigned short*)(str + ((fw >> 8) & 255u) * 16u + lane * 2);
                const u32 b8 = ((fw >> 23) & 15u) * 0x08080808u;
                u32 pos = first * 6u;
#pragma unroll
                for (int i = 0; i < 8; ++i) {
                    const int sh = 4 * i;
                    const u32 n4 = (sh ? (mm >> (sh - 2)) : (mm << 2)) & 0x3cu;
                    const u32 sel = *(LAS u32*)(lds + LDS_LUT + n4);
                    const LAS u32* wp = (const LAS u32*)(str + ((pos >> 5) << 2));
                    const u32 z = __builtin_amdgcn_alignbit(wp[1], wp[0], pos);
                    pos += 6u * (u32)__popc(n4);
                    const u32 y = bfi(0x00000fffu, z, z << 4);
                    const u32 x = (bfi(0x003f003fu, y, y << 2) & 0x3f3f3f3fu) + b8;
                    o[i >> 2][i & 3] = perm(x, x, sel);
                }
            } else {
                u32 pos = first * 7u;
#pragma unroll
                for (int i = 0; i < 8; ++i) {
                    const int sh = 4 * i;
                    const u32 n4 = (sh ? (mm >> (sh - 2)) : (mm << 2)) & 0x3cu;
                    const u32 sel = *(LAS u32*)(lds + LDS_LUT + n4);
                    const LAS u32* wp = (const LAS u32*)(str + ((pos >> 5) << 2));
                    const u32 z = __builtin_amdgcn_alignbit(wp[1], wp[0], pos);
                    pos += 7u * (u32)__popc(n4);
                    const u32 y = bfi(0x00003fffu, z, z << 2);
                    const u32 x = bfi(0x007f007fu, y, y << 1) & 0x7f7f7f7fu;
                    o[i >> 2][i & 3] = perm(x, x, sel);
                }
            }
            *(LAS u32x4*)(ot + (lane & 15) * 128 + uh * 64 + (lane >> 5) * 16) = o[0];
            *(LAS u32x4*)(ot + (lane & 15) * 128 + uh * 64 + (lane >> 5) * 16 + 32) = o[1];
            if ((u32)lane < ((fw >> 16) & 127u) && ((ex >> 11) & 1u) == (u32)hf)
                __hip_atomic_fetch_xor((LAS u32*)(ot + (ex & 0x7fcu)), 0x40u << (8u * (ex & 3u)), __ATOMIC_RELAXED, __HIP_MEMORY_SCOPE_WAVEFRONT);
            CBAR();
#pragma unroll
            for (int bb = 0; bb < 2; ++bb)
                __builtin_amdgcn_raw_buffer_store_b128(*(LAS u32x4*)(ot + bb * 1024 + lane * 16), ro, ooff + bb * 32768u, 0, AUX_OUT);
        }
        STAMP(6);
        return;
    }

    const int sl = slot - 16;
    const int ti = (xcd >> 1) * 4 + (sl >> 2), tj = (xcd & 1) * 4 + (sl & 3), tile = ti * 8 + tj;
    const bool helper = wave >= 4;
    const int wq = wave & 3;
    if (tid < 16) {
        u32 sel = 0x0c0c0c0cu, k = 0;
        for (int j = 0; j < 4; ++j) if ((tid >> j) & 1) { sel = (sel & ~(0xffu << (8 * k))) | ((u32)j << (8 * k)); ++k; }
        *(LAS u32*)(lds + LDS_LUT + tid * 4) = sel;
    }
    if (tid < 8) *(LAS u32*)(lds + LDS_RED + tid * 4) = 0;
    LAS char* str = lds + LDS_STR + wq * 4096;
    if (!helper) {
#pragma unroll
        for (int u = 0; u < 4; ++u) *(LAS u32x4*)(str + u * 1024 + lane * 16) = u32x4{0, 0, 0, 0};
    }
    u32 order = 0;
    for (int k = 0; k < 7; ++k) order |= (u32)((rank + ((ti >> 1) + 13 - k) % 7 + 1) & 7) << (4 * k);
    order = rfl(order);
    auto step_id = [&](u32 s) __attribute__((always_inline)) {
        const u32 m = s - 8u, c = (m * 37u) >> 8;
        return s < 8u ? (u32)rank * 8u + s : ((order >> ((m - c * 7u) * 4u)) & 7u) * 8u + c;
    };
    const u32 stepid = step_id((u32)lane);
    const u32 nown = block_id((u32)((b - 128) * 6 + wave), true);
    if (wave < 6) load_block((int)nown);
    CBAR();
    const u32 call = get_call();
    const u32 seq = call & SEQ_MASK;
    if (wave < 6) {
        take_block(nown);
        pack((int)nown, true, call);
        __builtin_amdgcn_raw_buffer_store_b128(pk[2], rs, (u32)lane >= p_lanes ? 0x80000000u : (u32)lane * 16u + 2048u, p_off, AUX_SYS);
        if (p_mode == 3) __builtin_amdgcn_raw_buffer_store_b128(pk[3], rs, (u32)lane * 16u + 3072u, p_off, AUX_SYS);
        WAIT_VM0();
        if (lane == 0) __hip_atomic_store((gu32*)(myws + p_foff), p_flag, RLX_SYS);
    }
    bool dead = false;
    const u32 xblk = OFF_A + (u32)ti * PIECE + (u32)wq * 4096u;
    LAS char* ord = lds + LDS_ORD;
#define SB() __builtin_amdgcn_sched_barrier(0)
    using T = std::true_type; using F = std::false_type;
    using C0 = std::integral_constant<int, 0>; using C1 = std::integral_constant<int, 1>;
    if (helper) {
        auto lanes = [&](u32 f) __attribute__((always_inline)) { return ((u64)((f & 1u) ? 0xffffffffu : 0u) << 32) | ((f & 0x8000u) ? 0u : 0xffffffffu); };
        auto planes = [&](u32 fhi, const u32x3 c, u32 blk, u32 (&H)[8]) __attribute__((always_inline)) {
            const u32 base4 = ((fhi >> 8) & 127u) * 0x01010101u;
#pragma unroll
            for (int q = 0; q < 3; ++q) { H[2 * q] = c[q] & M83; H[2 * q + 1] = __builtin_amdgcn_alignbit(c[q], c[q], 3) & M83; }
            H[6] = bfi(0x80808080u, c[2] << 1, c[0] >> 5) & M83;
            H[7] = bfi(0x80808080u, c[2] << 2, c[1] >> 5) & M83;
            SB();
#pragma unroll
            for (int q = 0; q < 8; ++q) H[q] += base4;
            if (__builtin_expect((fhi & 0x8000u) != 0u, 0)) raw_planes(H, myws + (blk + (u32)lane * 16u));
        };
        u32 vmask = wave >= 8 ? 0x7fcu : 0xffcu;
        asm volatile("" : "+v"(vmask));
        const u32 fo = OFF_FLAGA + (u32)ti * 1024u + stepid * 16u, go = OFF_FLAGB + (u32)tj * 1024u + stepid * 16u;
        u32x4 fv = {0, 0, 0, 0}, gv = {0, 0, 0, 0}, pv, pg;
        u64 todo = 0, done = 0;
        const u32 want = seq << SEQ_SH;
        auto take = [&](const u32x4 v, const u32x4 g) __attribute__((always_inline)) {
            fv = v; gv = g;
            const u32 df = ((v.x ^ want) | (v.y ^ want) | (v.z ^ want) | (v.w ^ want)) | ((g.x ^ want) | (g.y ^ want) | (g.z ^ want) | (g.w ^ want));
            todo = __ballot((df >> SEQ_SH) == 0u) & ~done;
        };
        auto pick = [&](int slot_) __attribute__((always_inline)) {
            const int id = __ffsll((long long)todo) - 1;
            done |= 1ull << id; todo &= todo - 1ull;
            if (lane == id) { *(LAS u32x4*)(ord + (slot_ & 3) * 32) = fv; *(LAS u32x4*)(ord + (slot_ & 3) * 32 + 16) = gv; }
        };
        auto peek = [&](int slot_) __attribute__((always_inline)) {
            bool later = false;
            if (wave == DECW) {
                later = todo == 0ull;
                if (later) { pv = __builtin_amdgcn_raw_buffer_load_b128(rs, fo, 0, AUX_SYS); pg = __builtin_amdgcn_raw_buffer_load_b128(rs, go, 0, AUX_SYS); }
                else pick(slot_);
            }
            SB();
            return later;
        };
        auto choose = [&](int slot_, bool peeked) __attribute__((always_inline)) {
            if (peeked) take(pv, pg);
            if (todo == 0ull) {
                u32 spins = 0;
                for (;;) {
                    const u32x4 v = __builtin_amdgcn_raw_buffer_load_b128(rs, fo, 0, AUX_SYS), g = __builtin_amdgcn_raw_buffer_load_b128(rs, go, 0, AUX_SYS);
                    take(v, g);
                    CBAR();
                    if (todo != 0ull) break;
                    if (spin_fail(tstart, tmo, call, spins)) { dead = true; fv = gv = u32x4{0, 0, 0, 0}; todo = ~done; break; }
                }
            }
            pick(slot_);
        };
        if (wave == DECW) { choose(0, false); choose(1, false); }
        LDS_BARRIER();
        if (wave >= 8) {
            struct XC { u32x3 c; unsigned short ex; u32 f, so; };
            XC x0, x1;
            const u32 xc_ = xblk + 2048u + (u32)lane * 12u, xe_ = xblk + 2816u + (u32)lane * 2u;
            u32 img = (u32)(uintptr_t)(lds + LDS_HX + wq * 2048);
            auto ldx = [&](int slot_, XC& x) __attribute__((always_inline)) {
                x.f = rfl(*(LAS u32*)(ord + (slot_ & 3) * 32 + wq * 4));
                x.so = ((x.f >> 1) & 63u) * (16u * PIECE);
                x.c = __builtin_amdgcn_raw_buffer_load_b96(rs, xc_, x.so, AUX_SYS);
                x.ex = __builtin_amdgcn_raw_buffer_load_b16(rs, xe_, x.so, AUX_SYS);
                SB();
            };
            auto unit = [&](int j, XC& x, XC& xn, bool ld) __attribute__((always_inline)) {
                if (ld) ldx(j + 1, xn);
                u32 H[8];
                planes(x.f, x.c, xblk + x.so, H);
                *(LAS u32x4*)(uintptr_t)(img + lane * 16) = u32x4{H[0], H[1], H[2], H[3]};
                *(LAS u32x4*)(uintptr_t)(img + 1024 + lane * 16) = u32x4{H[4], H[5], H[6], H[7]};
                patch<false>(img, x.ex, lanes(x.f), vmask);
                if (__builtin_expect((x.f & 0x18000u) == 0x10000u, 0)) patch_more<false>(img, myws + (xblk + (u32)lane * 2u + x.so), vmask);
                img ^= 8192u;
            };
            ldx(0, x0);
#pragma unroll 1
            for (int j = 0; j < 62; j += 2) {
                unit(j, x0, x1, true); LDS_BARRIER();
                unit(j + 1, x1, x0, true); LDS_BARRIER();
            }
            unit(62, x0, x1, true); LDS_BARRIER();
            unit(63, x1, x0, false);
            if (dead && lane == 0) *(LAS u32*)(lds + LDS_RED + 20) = 1;
            LDS_BARRIER();
            return;
        }
        struct WB { u32x4 l0, l1; u32x3 c; unsigned short ex; u32 f, so; };
        WB w0, w1;
        const u32 wblk = OFF_B + (u32)tj * PIECE + (u32)wq * 4096u;
        const u32 wl_ = wblk + (u32)lane * 16u, wc_ = wblk + 2048u + (u32)lane * 12u, we_ = wblk + 2816u + (u32)lane * 2u;
        u32 img = (u32)(uintptr_t)(lds + LDS_W + wq * 4096);
        auto unit = [&](int j, WB& w, WB& wn, bool ld, bool ch) __attribute__((always_inline)) {
            const bool pk_ = ch && peek(j + 2);
            u32 fn = 0;
            if (ld) fn = *(LAS u32*)(ord + ((j + 1) & 3) * 32 + 16 + wq * 4);
            SB();
            u32 H[8];
            planes(w.f, w.c, wblk + w.so, H);
            SB();
            if (ld) {
                wn.f = rfl(fn); wn.so = ((wn.f >> 1) & 63u) * (8u * PIECE);
                wn.c = __builtin_amdgcn_raw_buffer_load_b96(rs, wc_, wn.so, AUX_SYS);
                wn.l0 = __builtin_amdgcn_raw_buffer_load_b128(rs, wl_, wn.so, AUX_SYS);
            }
            SB();
            u32x4 c[4];
            LAS char* pw = (LAS char*)(uintptr_t)(img + lane * 16);
#pragma unroll
            for (int q = 0; q < 4; ++q) {
                const u32 sl_ = (q & 1) ? 0x07030602u : 0x05010400u;
                const int xy = q >> 1;
                c[q] = u32x4{perm(H[xy], w.l0[xy], sl_), perm(H[2 + xy], w.l0[2 + xy], sl_),
                             perm(H[4 + xy], w.l1[xy], sl_), perm(H[6 + xy], w.l1[2 + xy], sl_)};
                if (q) *(LAS u32x4*)(pw + (q - 1) * 1024) = c[q - 1];
                SB();
            }
            *(LAS u32x4*)(pw + 3 * 1024) = c[3];
            patch<true>(img, w.ex, lanes(w.f), vmask);
            if (__builtin_expect((w.f & 0x18000u) == 0x10000u, 0)) patch_more<true>(img, myws + (wblk + (u32)lane * 2u + w.so), vmask);
            img ^= 16384u;
            SB();
            if (ld) {
                wn.l1 = __builtin_amdgcn_raw_buffer_load_b128(rs, wl_ + 1024, wn.so, AUX_SYS);
                wn.ex = __builtin_amdgcn_raw_buffer_load_b16(rs, we_, wn.so, AUX_SYS);
            }
            if (pk_) choose(j + 2, true);
        };
        w0.f = rfl(*(LAS u32*)(ord + 16 + wq * 4)); w0.so = ((w0.f >> 1) & 63u) * (8u * PIECE);
        w0.c = __builtin_amdgcn_raw_buffer_load_b96(rs, wc_, w0.so, AUX_SYS);
        w0.l0 = __builtin_amdgcn_raw_buffer_load_b128(rs, wl_, w0.so, AUX_SYS);
        w0.l1 = __builtin_amdgcn_raw_buffer_load_b128(rs, wl_ + 1024, w0.so, AUX_SYS);
        w0.ex = __builtin_amdgcn_raw_buffer_load_b16(rs, we_, w0.so, AUX_SYS);
#pragma unroll 1
        for (int j = 0; j < 62; j += 2) {
            unit(j, w0, w1, true, true); LDS_BARRIER();
            unit(j + 1, w1, w0, true, true); LDS_BARRIER();
        }
        unit(62, w0, w1, true, false); LDS_BARRIER();
        unit(63, w1, w0, false, false);
        if (dead && lane == 0) *(LAS u32*)(lds + LDS_RED + 20) = 1;
        LDS_BARRIER();
        LDS_BARRIER();
        if (wave == 4 && lane == 0) {
            const u32x4 t = *(LAS u32x4*)(lds + LDS_RED);
            __hip_atomic_store((gu64*)(myws + OFF_AMAXL) + tile, ((u64)call << 32) | max(max(t.x, t.y), max(t.z, t.w)), RLX_AGT);
        }
        LDS_BARRIER();
        if (wave != 4) {
#pragma unroll 1
            for (int u = wave - 5; u < 28; u += 3) {
                if (u >= 4 * MEARLY)
                    while (__hip_atomic_load((LAS u64*)(lds + LDS_RED + 24), __ATOMIC_RELAXED, __HIP_MEMORY_SCOPE_WORKGROUP) == 0ull) __builtin_amdgcn_s_sleep(1);
                const int mwu = tile * 4 + (u & 3);
                int dd = (u >> 2) + mwu % 7; if (dd >= 7) dd -= 7;
                if (lane < 32) {
                    const u32x4 mk = *(LAS u32x4*)(lds + LDS_MSK + (u & 3) * 512 + lane * 16);
                    __builtin_amdgcn_raw_buffer_store_b128(mk, mk_rsrc(P.wsr[dd], WS_USED), (u32)lane * 16u,
                                                           OFF_MSG + (u32)(rank * 512 + mwu) * MSG, AUX_SYS);
                }
                asm volatile("s_waitcnt vmcnt(%0)" :: "n"(MWIN - 1) : "memory");
            }
            WAIT_VM0();
            __builtin_amdgcn_s_barrier();
            return;
        }
        u32 gb = 0;
        if (!dead) {
            gu64* g = lane == 8 ? (gu64*)tmo : (gu64*)(myws + OFF_AMAXG + (tile & (NREP - 1)) * 128) + (lane & 7);
            u32 spins = 0;
            for (;;) {
                const u64 a = __hip_atomic_load(g, RLX_SYS);
                gb = (u32)a;
                if ((__ballot((u32)(a >> 32) == call) & 0xffull) == 0xffull) break;
                if (spin_fail(tstart, tmo, call, spins)) { dead = true; break; }
            }
            if (rdl(gb, 8) == call) dead = true;
            gb = wave_max(lane == 8 ? 0u : gb);
        }
        if (*(LAS u32*)(lds + LDS_RED + 20) != 0u) dead = true;
        if (lane == 0) __hip_atomic_store((LAS u64*)(lds + LDS_RED + 24), ((u64)(dead ? 3u : 1u) << 32) | __float_as_uint(448.0f / __uint_as_float(gb)), __ATOMIC_RELEASE, __HIP_MEMORY_SCOPE_WORKGROUP);
        return;
    }
    bf16x8 xf[4], wfA[8], wfB[8];
    f32x16 acc[4];
#pragma unroll
    for (int j = 0; j < 4; ++j)
#pragma unroll
        for (int i = 0; i < 16; ++i) acc[j][i] = 0.f;
    u32x4 l0[2], l1[2], ha, hb;
    const u32 xl_ = xblk + (u32)lane * 16u;
    auto putX = [&](u32 h0, u32 h1, u32 la, u32 lb, bf16x8& xo) __attribute__((always_inline)) {
        u32x4 c = u32x4{perm(h0, la, 0x05010400u), perm(h1, lb, 0x05010400u), perm(h0, la, 0x07030602u), perm(h1, lb, 0x07030602u)};
        asm volatile("" : "+v"(c));
        xo = __builtin_bit_cast(bf16x8, c);
    };
    u32 so_l1 = 0;
    auto step = [&](int s, auto cur_c, auto dec_c, auto iss_c, auto is1_c) __attribute__((always_inline)) {
        constexpr int cur = decltype(cur_c)::value, nxt = cur ^ 1;
        constexpr bool DEC = decltype(dec_c)::value, ISS = decltype(iss_c)::value, IS1 = decltype(is1_c)::value;
        LAS char* pr = lds + LDS_W + cur * 16384 + 8192 + lane * 16;
#pragma unroll
        for (int q = 0; q < 8; ++q) {
            acc[q & 3] = __builtin_amdgcn_mfma_f32_32x32x16_bf16(wfA[q], xf[q >> 2], acc[q & 3], 0, 0, 0);
            SB();
            if (q < 4) { wfB[2 * q] = *(LAS bf16x8*)(pr + (2 * q) * 1024); wfB[2 * q + 1] = *(LAS bf16x8*)(pr + (2 * q + 1) * 1024); }
            if (q == 4) putX(hb.x, hb.y, l1[cur].x, l1[cur].y, xf[2]);
            if (q == 5) putX(hb.z, hb.w, l1[cur].z, l1[cur].w, xf[3]);
            if constexpr (IS1) { if (q == 6) l1[cur] = __builtin_amdgcn_raw_buffer_load_b128(rs, xl_ + 1024, so_l1, AUX_SYS); }
            SB();
        }
        if constexpr (DEC) LDS_BARRIER(); else WAIT_LGKM0();
        SB();
        pr = lds + LDS_W + nxt * 16384 + lane * 16;
        LAS char* px = lds + LDS_HX + nxt * 8192 + wq * 2048 + lane * 16;
        u32 idv = 0;
#pragma unroll
        for (int q = 0; q < 8; ++q) {
            acc[q & 3] = __builtin_amdgcn_mfma_f32_32x32x16_bf16(wfB[q], xf[2 + (q >> 2)], acc[q & 3], 0, 0, 0);
            SB();
            if constexpr (DEC) {
                if (q == 0) { ha = *(LAS u32x4*)px; hb = *(LAS u32x4*)(px + 1024); }
                if (q >= 1 && q < 5) { wfA[2 * q - 2] = *(LAS bf16x8*)(pr + (2 * q - 2) * 1024); wfA[2 * q - 1] = *(LAS bf16x8*)(pr + (2 * q - 1) * 1024); }
                if (q == 4) putX(ha.x, ha.y, l0[nxt].x, l0[nxt].y, xf[0]);
                if (q == 5) putX(ha.z, ha.w, l0[nxt].z, l0[nxt].w, xf[1]);
            }
            if constexpr (ISS) {
                if (q == 0) idv = *(LAS u32*)(ord + ((s + 3) & 3) * 32);
                if (q == 5) { so_l1 = ((rfl(idv) >> 1) & 63u) * (16u * PIECE); asm volatile("" : "+s"(so_l1)); }
                if (q == 6) l0[nxt] = __builtin_amdgcn_raw_buffer_load_b128(rs, xl_, so_l1, AUX_SYS);
            }
            if (q == 7) __builtin_amdgcn_s_waitcnt(0xC07F);
            SB();
        }
    };
    LDS_BARRIER();
#pragma unroll
    for (int k = 0; k < 2; ++k) {
        const u32 so = ((rfl(*(LAS u32*)(ord + k * 32)) >> 1) & 63u) * (16u * PIECE);
        l0[k] = __builtin_amdgcn_raw_buffer_load_b128(rs, xl_, so, AUX_SYS);
        l1[k] = __builtin_amdgcn_raw_buffer_load_b128(rs, xl_ + 1024, so, AUX_SYS);
    }
    LDS_BARRIER();
    ha = *(LAS u32x4*)(lds + LDS_HX + wq * 2048 + lane * 16); hb = *(LAS u32x4*)(lds + LDS_HX + wq * 2048 + 1024 + lane * 16);
#pragma unroll
    for (int i = 0; i < 8; ++i) wfA[i] = *(LAS bf16x8*)(lds + LDS_W + lane * 16 + i * 1024);
    putX(ha.x, ha.y, l0[0].x, l0[0].y, xf[0]);
    putX(ha.z, ha.w, l0[0].z, l0[0].w, xf[1]);
    so_l1 = ((rfl(*(LAS u32*)(ord + 64)) >> 1) & 63u) * (16u * PIECE);
    l0[0] = __builtin_amdgcn_raw_buffer_load_b128(rs, xl_, so_l1, AUX_SYS);
    WAIT_LGKM0();
    STAMP(1);
#pragma unroll 1
    for (int s = 0; s < 60; s += 2) {
        step(s, C0{}, T{}, T{}, T{});
        step(s + 1, C1{}, T{}, T{}, T{});
    }
    step(60, C0{}, T{}, T{}, T{});
    step(61, C1{}, T{}, F{}, T{});
    step(62, C0{}, T{}, F{}, F{});
    step(63, C1{}, F{}, F{}, F{});
    STAMP(5);

    float mx = 0.f;
#pragma unroll
    for (int j = 0; j < 4; ++j)
#pragma unroll
        for (int i = 0; i < 16; ++i) mx = fmaxf(mx, acc[j][i]);
    const u32 mb = wave_max(__float_as_uint(mx));
    if (lane == 0) *(LAS u32*)(lds + LDS_RED + wave * 4) = mb;
    if (dead && lane == 0) *(LAS u32*)(lds + LDS_RED + 20) = 1;
    LDS_BARRIER();
    u32 m0 = 0, m1 = 0;
#pragma unroll
    for (int i = 7; i >= 0; --i)
#pragma unroll
        for (int j = 3; j >= 0; --j) {
            m0 = __builtin_amdgcn_alignbit(m0, __float_as_uint(acc[j][i]), 31);
            m1 = __builtin_amdgcn_alignbit(m1, __float_as_uint(acc[j][i + 8]), 31);
        }
    m0 = ~m0; m1 = ~m1;
    const int mw = tile * 4 + wave, rot = mw % 7;
    const u32 moff = OFF_MSG + (u32)(rank * 512 + mw) * MSG;
    *(LAS u32x2*)(lds + LDS_MSK + wave * 512 + lane * 8) = u32x2{m0, m1};
    LDS_BARRIER();
    const u32 cnt = (u32)(__popc(m0) + __popc(m1));
    const u32 incl = wave_scan(cnt);
    const u32 total = rdl(incl, 63);
    u32 sel[16], pa[16], ps[16], bm[16];
    {
        u32 pp = (incl - cnt) * 6u;
#pragma unroll
        for (int i = 0; i < 16; ++i) {
            const u32 mm = i < 8 ? m0 : m1;
            const int sh = 4 * (i & 7);
            const u32 n4 = (sh ? (mm >> (sh - 2)) : (mm << 2)) & 0x3cu;
            sel[i] = *(LAS u32*)(lds + LDS_LUT + n4);
            bm[i] = __umul24(n4 << 4, 0x00204081u) & 0x40404040u;
            pa[i] = (pp >> 5) << 2; ps[i] = pp & 31u;
            pp += 6u * (u32)__popc(n4);
            asm volatile("" : "+v"(bm[i]), "+v"(pa[i]), "+v"(ps[i]));
        }
    }
#pragma unroll
    for (int j = 0; j < 4; ++j)
#pragma unroll
        for (int i = 0; i < 16; ++i) acc[j][i] = fmaxf(acc[j][i], 0.f);
#pragma unroll
    for (int j = 0; j < 4; ++j) asm volatile("" : "+v"(acc[j]));
    u64 gd;
    while ((gd = __hip_atomic_load((LAS u64*)(lds + LDS_RED + 24), __ATOMIC_ACQUIRE, __HIP_MEMORY_SCOPE_WORKGROUP)) == 0ull) __builtin_amdgcn_s_sleep(1);
    const float inv = __uint_as_float(rfl((u32)gd));
    dead = rfl((u32)(gd >> 32)) != 1u;
    STAMP(6);

#pragma unroll
    for (int j = 0; j < 4; ++j) acc[j] *= inv;
    const u32 etop = (rfl((u32)__builtin_amdgcn_cvt_pk_fp8_f32(__uint_as_float(mb) * inv, 0.f, 0, false)) >> 3) & 15u;
    u32 base = etop > 8u ? etop - 8u : 0u;
    u32x4 q[4];
    u32 bad0 = 0, bad1 = 0;
    {
        const u32 b8 = base * 0x08080808u;
#pragma unroll
        for (int i = 0; i < 16; ++i) {
            int w0 = __builtin_amdgcn_cvt_pk_fp8_f32(acc[0][i], acc[1][i], 0, false);
            w0 = __builtin_amdgcn_cvt_pk_fp8_f32(acc[2][i], acc[3][i], w0, true);
            q[i >> 2][i & 3] = (u32)w0 & 0x7f7f7f7fu;
            const u32 t = ((u32)w0 | 0x80808080u) - b8;
            const u32 n64 = __builtin_amdgcn_udot4(t & bm[i], 0x08040201u, 0u, false);
            const int sh = 4 * (i & 7) - 6;
            if (i < 8) bad0 |= sh < 0 ? n64 >> -sh : n64 << sh; else bad1 |= sh < 0 ? n64 >> -sh : n64 << sh;
            const u32 c = perm(t, t, sel[i]) & 0x3f3f3f3fu;
            const u32 y = bfi(0x003f003fu, c, c >> 2);
            const u32 z = bfi(0x00000fffu, y, y >> 4);
            const u64 sh64 = (u64)z << ps[i];
            LAS u32* wp = (LAS u32*)(str + pa[i]);
            __hip_atomic_fetch_or(wp, (u32)sh64, __ATOMIC_RELAXED, __HIP_MEMORY_SCOPE_WAVEFRONT);
            __hip_atomic_fetch_or(wp + 1, (u32)(sh64 >> 32), __ATOMIC_RELAXED, __HIP_MEMORY_SCOPE_WAVEFRONT);
        }
    }
    u32 nst = (total * 6u + 127u) >> 7;
    const u32 nb = (u32)(__popc(bad0) + __popc(bad1));
    const u32 nbi = wave_scan(nb);
    u32 nexc = rdl(nbi, 63), seven = 0;
    if (__builtin_expect(nexc <= 64u, 1)) {
        LAS unsigned short* lp = (LAS unsigned short*)(str + nst * 16u) + (nbi - nb);
        const u32 lb = (u32)(lane & 31) * 128u + (u32)(lane >> 5) * 16u;
        while (__ballot((bad0 | bad1) != 0u) != 0ull) {
            if ((bad0 | bad1) != 0u) {
                const u32 pb = bad0 ? (u32)__ffs((int)bad0) - 1u : (u32)__ffs((int)bad1) + 31u;
                if (bad0) bad0 &= bad0 - 1u; else bad1 &= bad1 - 1u;
                *lp++ = (unsigned short)(lb + ((pb >> 4) << 5) + (pb & 15u));
            }
        }
    } else {
        seven = 1; base = 0; nexc = 0;
        nst = (total * 7u + 127u) >> 7;
        WAIT_LGKM0();
#pragma unroll
        for (int u = 0; u < 4; ++u) *(LAS u32x4*)(str + u * 1024 + lane * 16) = u32x4{0, 0, 0, 0};
        u32 pp = (incl - cnt) * 7u;
#pragma unroll
        for (int i = 0; i < 16; ++i) {
            const u32 qq = q[i >> 2][i & 3];
            const u32 c = perm(qq, qq, sel[i]);
            const u32 y = bfi(0x007f007fu, c, c >> 1);
            const u32 z = bfi(0x00003fffu, y, y >> 2);
            const u64 sh64 = (u64)z << (pp & 31u);
            LAS u32* wp = (LAS u32*)(str + ((pp >> 5) << 2));
            __hip_atomic_fetch_or(wp, (u32)sh64, __ATOMIC_RELAXED, __HIP_MEMORY_SCOPE_WAVEFRONT);
            __hip_atomic_fetch_or(wp + 1, (u32)(sh64 >> 32), __ATOMIC_RELAXED, __HIP_MEMORY_SCOPE_WAVEFRONT);
            pp += 7u * (u32)__popc(bm[i]);
        }
    }
    const u32 nch = nst + ((nexc + 7u) >> 3);
    WAIT_LGKM0();
    u32x4 cs[4];
#pragma unroll
    for (int u = 0; u < 4; ++u) cs[u] = *(LAS u32x4*)(str + u * 1024 + lane * 16);
    STAMP(7);
    auto own_tile = [&]() __attribute__((always_inline)) {
        const u32 ooff = (u32)((rank >> 2) * 2048 + ti * 128 + wave * 32 + (lane >> 3)) * 4096u + (u32)((rank & 3) * 1024 + tj * 128 + (lane & 7) * 16);
        __amdgpu_buffer_rsrc_t ro = mk_rsrc(P.out, (u32)MM * NN);
        CBAR();
#pragma unroll
        for (int bb = 0; bb < 4; ++bb)
            *(LAS u32x4*)(str + (lane & 31) * 128 + bb * 32 + (lane >> 5) * 16) = dead ? u32x4{0x7f7f7f7fu, 0x7f7f7f7fu, 0x7f7f7f7fu, 0x7f7f7f7fu} : q[bb];
        CBAR();
#pragma unroll
        for (int bb = 0; bb < 4; ++bb)
            __builtin_amdgcn_raw_buffer_store_b128(*(LAS u32x4*)(str + bb * 1024 + lane * 16), ro, ooff + bb * 32768u, 0, AUX_OUT);
        CBAR();
    };
    if (!dead) {
        const u64 flag = ((u64)call << 32) | nch | (nst << 8) | (nexc << 16) | (base << 23) | (seven << 27);
        const u32 foff = OFF_FLAGO + (u32)mw * 128u + (u32)rank * 8u;
        if (__builtin_expect(nch <= 128u, 1)) {
            const u32 v0 = (u32)lane < nch ? 512u + (u32)lane * 16u : 0x80000000u, v1 = (u32)lane + 64u < nch ? 1536u + (u32)lane * 16u : 0x80000000u;
            u32 due = (u32)now();
#pragma unroll
            for (int k = 0; k < 7; ++k) {
                int dd = k + rot; if (dd >= 7) dd -= 7;
                char* w0 = P.wsr[dd];
                pace(due, PER2);
                __builtin_amdgcn_raw_buffer_store_b128(cs[0], mk_rsrc(w0, WS_USED), v0, moff, AUX_SYS);
                pace(due, PER2);
                __builtin_amdgcn_raw_buffer_store_b128(cs[1], mk_rsrc(w0, WS_USED), v1, moff, AUX_SYS);
                if (k == 0) { own_tile(); __builtin_amdgcn_s_barrier(); }
                WAIT_VM0();
                if (lane == 0) __hip_atomic_store((gu64*)(w0 + foff), flag, RLX_SYS);
            }
        } else {
            own_tile();
            __builtin_amdgcn_s_barrier();
#pragma unroll 1
            for (int k = 0; k < 7; ++k) {
                int dd = k + rot; if (dd >= 7) dd -= 7;
                char* w0 = P.wsr[dd];
#pragma unroll
                for (int u = 0; u < 4; ++u)
                    if ((u32)(u * 64 + lane) < nch) __builtin_amdgcn_raw_buffer_store_b128(cs[u], mk_rsrc(w0, WS_USED), 512u + u * 1024u + (u32)lane * 16u, moff, AUX_SYS);
                WAIT_VM0();
                if (lane == 0) __hip_atomic_store((gu64*)(w0 + foff), flag, RLX_SYS);
            }
        }
    } else {
        own_tile();
        __builtin_amdgcn_s_barrier();
    }
    STAMP(8);
}

extern "C" void kernel_launch(int rank, int nranks, ncclComm_t comm,
                              hipStream_t stream,
                              void* const* d_in, const int* in_sizes, int n_in,
                              void* d_out, int out_size,
                              const int* in_dtypes, int out_dtype,
                              void* const* d_out_all,
                              void* const* d_ws_all, size_t ws_bytes) {
    (void)comm; (void)in_sizes; (void)in_dtypes; (void)out_dtype; (void)d_out_all;
    if (nranks != NR || n_in < 2 || out_size != MM * NN || ws_bytes < WS_USED || (unsigned)rank >= (unsigned)NR) {
        fprintf(stderr, "unexpected problem shape\n");
        return;
    }
    Params P;
    P.x = (const char*)d_in[0];
    P.w = (const char*)d_in[1];
    for (int d = 0; d < NR; ++d) P.wsr[d] = (char*)d_ws_all[(rank + 1 + d) & 7];
    P.out = (char*)d_out;
    P.myws = (char*)d_ws_all[rank];
    P.rank = rank;
    P.pad = 0;
    fused_kernel<<<dim3(NGRID), dim3(NTHR), 0, stream>>>(P);
    HIP_CHECK(hipGetLastError());
}
```
